```python
import numpy as np

kernel_name = "gpu_gemm_ar_m4096_k4096_n4096_e4m3e5m2_dequant"


ABSMAX_RTOL = 0.05

COMPUTE_DTYPE = "float32"
EPILOGUE = "dequant"
IN_QUANT = None
OUT_QUANT = None
IN_FP8 = "e4m3e5m2"
IN_DTYPES = {"x": "e4m3", "w_mat": "e5m2",
             "scale_x": "float32", "scale_w": "float32"}
WORLD_SIZE = 8

def _snap_fp8(x, mant_bits, max_abs, min_normal):
    a = np.abs(x.astype(np.float64))
    _m, _e = np.frexp(a)
    q = np.where(a >= min_normal, np.ldexp(1.0, _e - 1 - mant_bits),
                 min_normal * 2.0 ** -mant_bits)
    snapped = np.minimum(np.round(a / q) * q, max_abs)
    return (np.sign(x) * snapped).astype(np.float32)

def setup_inputs():

    x = _snap_fp8(np.random.randn(4096, 4096).astype(np.float32), 3, 448.0, 2.0 ** -6)
    w_mat = _snap_fp8(np.random.randn(4096, 4096).astype(np.float32), 2, 57344.0, 2.0 ** -14)
    scale_x = (0.02 * (1.0 + np.random.rand(1))).astype(np.float32)
    scale_w = (0.02 * (1.0 + np.random.rand(1))).astype(np.float32)
    return {"x": x, "w_mat": w_mat, "scale_x": scale_x, "scale_w": scale_w}

def reference(x, w_mat, scale_x, scale_w):
    acc = (x @ w_mat).astype(np.float32)
    y = (acc * (scale_x * scale_w)).astype(np.float32)
    return y
```

```cpp
#include <hip/hip_runtime.h>
#include <rccl/rccl.h>
#include <cstdio>

#define HIP_CHECK(expr) do { hipError_t _e = (expr); if (_e != hipSuccess) { \
    fprintf(stderr, "HIP error %s at %s:%d\n", hipGetErrorString(_e), __FILE__, __LINE__); \
    return; } } while (0)

#define GAS __attribute__((address_space(1)))
#define LAS __attribute__((address_space(3)))
typedef unsigned u32;
typedef unsigned long long u64;
typedef unsigned u32x4 __attribute__((ext_vector_type(4)));
typedef int i32x4 __attribute__((ext_vector_type(4)));
typedef int i32x8 __attribute__((ext_vector_type(8)));
typedef float f32x4 __attribute__((ext_vector_type(4)));
typedef float f32x16 __attribute__((ext_vector_type(16)));

constexpr int NR = 8;
constexpr int MN = 4096;
constexpr int KS = 512;
constexpr int NWG = 256;
constexpr int NTHREADS = 768;
constexpr int ROUNDS = 32;
constexpr int NSTAGE = 64;
constexpr u32 UNIT = 4096;
constexpr size_t PANEL_BYTES = 1u << 20;
constexpr size_t IMG_BYTES = 32 * PANEL_BYTES;
constexpr size_t WS_CNT = 0;
constexpr size_t WS_TMO = 48 << 10;
constexpr size_t WS_FLAG = 64 << 10;
constexpr size_t WS_DBG = 256 << 10;
constexpr size_t WS_IMG = 1 << 20;
constexpr size_t WS_NEED = WS_IMG + 2 * IMG_BYTES;
constexpr u64 DEADLINE = 50000000ull;
constexpr u32 LDS_A = 0, LDS_B = 65536, LDS_CTL = 131072;
constexpr u32 LDS_ERR = LDS_CTL + 128;
constexpr u32 LDS_BYTES = LDS_CTL + 256;

struct Params {
    const unsigned char* x;
    const unsigned char* w;
    const float* sx;
    const float* sw;
    float* out;
    char* ws[NR];
    int rank;
    int pad;
};

__device__ __forceinline__ u64 rt() { return __builtin_amdgcn_s_memrealtime(); }
#define WAIT_VM0() do { __builtin_amdgcn_s_waitcnt(0x0F70); asm volatile("s_waitcnt vmcnt(0)" ::: "memory"); } while (0)

__device__ __forceinline__ void st_sys(char* p, u32x4 v) {
    asm volatile("global_store_dwordx4 %0, %1, off sc0 sc1\n\ts_nop 1" :: "v"(p), "v"(v) : "memory");
}

#define STAMP(i) do {} while (0)

__device__ __forceinline__ i32x8 ld_frag(LAS unsigned char* p) {
    i32x4 lo = *(LAS i32x4*)p, hi = *(LAS i32x4*)(p + UNIT);
    return __builtin_shufflevector(lo, hi, 0, 1, 2, 3, 4, 5, 6, 7);
}
#define MFMA(c, a, b) asm volatile("v_mfma_f32_32x32x64_f8f6f4 %0, %1, %2, %0 cbsz:0 blgp:1" : "+v"(c) : "v"(a), "v"(b))
#define SB() __builtin_amdgcn_sched_barrier(0)

#define DMA_BITS " sc0 sc1"
#define DMA4(v) "s_nop 0\n\t" \
                "global_load_lds_dwordx4 " v ", %5" DMA_BITS "\n\t" \
                "global_load_lds_dwordx4 " v ", %5 offset:1024" DMA_BITS "\n\t" \
                "global_load_lds_dwordx4 " v ", %5 offset:2048" DMA_BITS "\n\t" \
                "global_load_lds_dwordx4 " v ", %5 offset:3072" DMA_BITS "\n\t"
__device__ __forceinline__ void dma16(u32 v0, u32 v1, u32 v2, u32 v3, const char* g, u32 l) {
    u32 keep;
    asm volatile("s_mov_b32 %0, m0\n\ts_mov_b32 m0, %6\n\t"
                 DMA4("%1") "s_add_u32 m0, m0, 0x1000\n\t"
                 DMA4("%2") "s_add_u32 m0, m0, 0x1000\n\t"
                 DMA4("%3") "s_add_u32 m0, m0, 0x1000\n\t"
                 DMA4("%4") "s_mov_b32 m0, %0"
                 : "=&s"(keep) : "v"(v0), "v"(v1), "v"(v2), "v"(v3), "s"(g), "s"(l) : "memory", "scc");
}

__device__ __forceinline__ u32 perm(u32 hi, u32 lo, u32 sel) { return __builtin_amdgcn_perm(hi, lo, sel); }

__global__ void __launch_bounds__(NTHREADS, 3) ag_gemm(Params P) {
    __shared__ __attribute__((aligned(16))) unsigned char lds_raw[LDS_BYTES];
    LAS unsigned char* L = (LAS unsigned char*)lds_raw;
    const int wave = __builtin_amdgcn_readfirstlane(threadIdx.x >> 6);
    const int lane = threadIdx.x & 63;
    const int bid = blockIdx.x;
    const int rank = P.rank;
    char* myws = P.ws[rank];
    const int xg = bid & 7, idx = bid >> 3;
    const int tm = (xg >> 1) * 4 + (idx >> 3), tn = (xg & 1) * 8 + (idx & 7);

    u32 call = 0;
    if (wave >= 8) {
        call = __hip_atomic_load((GAS u32*)(myws + WS_CNT + bid * 128), __ATOMIC_RELAXED, __HIP_MEMORY_SCOPE_AGENT) + 1;
        call = __builtin_amdgcn_readfirstlane(call);
    }
    if (wave == 0) { ((LAS u32*)(L + LDS_CTL))[lane] = 0; STAMP(0); }
    __syncthreads();

    if (wave < 8) {
        const int g = wave >> 1, nh = wave & 1;
        LAS unsigned char* pa = L + LDS_A + (lane >> 5) * 8192 + (g * 64 + (lane & 31)) * 16;
        LAS unsigned char* pb = L + LDS_B + (lane >> 5) * 8192 + (nh * 32 + (lane & 31)) * 16;
        volatile LAS u32* ctl = (volatile LAS u32*)(L + LDS_CTL + wave * 16);
        f32x16 c00 = {}, c01 = {}, c02 = {}, c03 = {}, c10 = {}, c11 = {}, c12 = {}, c13 = {};
        auto wait_full = [&](u32 need) {
            for (u32 spin = 0; spin < (1u << 25); ++spin) {
                u32 fa = ctl[0], fb = ctl[1];
                u32 m = __builtin_amdgcn_readfirstlane(fa < fb ? fa : fb);
                if (m >= need) break;
                __builtin_amdgcn_s_sleep(1);
            }
            asm volatile("" ::: "memory");
        };
        wait_full(1);
        i32x8 bx = ld_frag(pb); SB();
        i32x8 a0 = ld_frag(pa); SB();
        i32x8 a1 = ld_frag(pa + 512); SB();
        i32x8 by = ld_frag(pb + 1024); SB();
        for (int s4 = 0; s4 < NSTAGE; s4 += 4) {
#pragma unroll
            for (int q = 0; q < 4; ++q) {
                const u32 so = q * 16384, sn = ((q + 1) & 3) * 16384;
                const u32 s = s4 + q;
                SB();
                MFMA(c00, a0, bx);
                MFMA(c10, a1, bx);
                SB();
                u32 fa = ctl[0], fb = ctl[1];
                bx = ld_frag(pb + so + 2048);
                SB();
                MFMA(c01, a0, by);
                MFMA(c11, a1, by);
                SB();
                by = ld_frag(pb + so + 3072);
                SB();
                MFMA(c02, a0, bx);
                SB();
                u32 m = __builtin_amdgcn_readfirstlane(fa < fb ? fa : fb);
                SB();
                MFMA(c12, a1, bx);
                SB();
                u32 need = s + 2 > NSTAGE ? NSTAGE : s + 2;
                if (__builtin_expect(m < need, 0)) wait_full(need);
                asm volatile("" ::: "memory");
                bx = ld_frag(pb + sn);
                SB();
                MFMA(c03, a0, by);
                SB();
                ctl[2] = s + 1;
                a0 = ld_frag(pa + sn);
                SB();
                MFMA(c13, a1, by);
                SB();
                a1 = ld_frag(pa + sn + 512);
                by = ld_frag(pb + sn + 1024);
            }
        }
        SB();
        asm volatile("s_nop 15\n\ts_nop 7" : "+v"(c00), "+v"(c01), "+v"(c02), "+v"(c03), "+v"(c10), "+v"(c11), "+v"(c12), "+v"(c13));
        if (wave == 0) STAMP(1);
        int tz = threadIdx.x; asm volatile("" : "+v"(tz));
        const int ln = tz & 63;
        float sc = P.sx[0] * P.sw[0];
        if (*(volatile LAS u32*)(L + LDS_ERR) != 0) sc = __builtin_nanf("");
        GAS float* o = (GAS float*)P.out + (size_t)(tm * 256 + g * 64 + 4 * (ln >> 5)) * MN + tn * 256 + nh * 128 + 4 * (ln & 31);
#pragma unroll
        for (int r = 0; r < 16; ++r) {
            const int row = (r & 3) + 8 * (r >> 2);
            f32x4 v = {c00[r] * sc, c01[r] * sc, c02[r] * sc, c03[r] * sc};
            *(GAS f32x4*)(o + (size_t)row * MN) = v;
        }
#pragma unroll
        for (int r = 0; r < 16; ++r) {
            const int row = 32 + (r & 3) + 8 * (r >> 2);
            f32x4 v = {c10[r] * sc, c11[r] * sc, c12[r] * sc, c13[r] * sc};
            *(GAS f32x4*)(o + (size_t)row * MN) = v;
        }
    } else if (wave < 10) {
        const int isB = wave - 8;
        const int panel = isB ? 16 + tn : tm;
        const char* img = myws + WS_IMG + (call & 1) * IMG_BYTES + (size_t)panel * PANEL_BYTES;
        const u32 ldsb = (u32)(size_t)L + (isB ? LDS_B : LDS_A);
        volatile LAS u32* ctl = (volatile LAS u32*)(L + LDS_CTL + (lane & 7) * 16);
        GAS u32* flag = (GAS u32*)(myws + WS_FLAG + (panel * 8 + ((rank + 1 + lane) & 7)) * 128);
        GAS u32* tmo = (GAS u32*)(myws + WS_TMO);
        const u32 fbase = call * 64;
        const u32 voff = lane * 16;
        u32 ready = 0, issued = 0, pub = 0;
        u64 t0 = rt();
        for (u32 spin = 0; pub < NSTAGE && spin < (1u << 25); ++spin) {
            bool can = issued < ready;
            if (can && issued >= 4) {
                u32 d = ctl[2];
                can = __builtin_amdgcn_ballot_w64(d + 3 < issued) == 0;
            }
            if (can) {
                const char* gp = img + (size_t)issued * 16384;
                const u32 lp = ldsb + (issued & 3) * 16384;
                dma16(voff, voff + 4096, voff + 8192, voff + 12288, gp, lp);
                ++issued;
                if (issued - pub == 3) {
                    asm volatile("s_waitcnt vmcnt(32)" ::: "memory");
                    ++pub;
                    ctl[isB] = pub;
                }
            } else if (issued - pub == 2) {
                asm volatile("s_waitcnt vmcnt(16)" ::: "memory");
                ++pub;
                ctl[isB] = pub;
            } else if (issued < ready || issued == NSTAGE) {
                if (pub < issued) {
                    asm volatile("s_waitcnt vmcnt(0)" ::: "memory");
                    pub = issued;
                    ctl[isB] = pub;
                } else {
                    __builtin_amdgcn_s_sleep(1);
                }
            } else {
                u32 f = 0xffffffffu, tm_ = 0;
                if (lane < 8) f = __hip_atomic_load(flag, __ATOMIC_RELAXED, __HIP_MEMORY_SCOPE_SYSTEM);
                if (lane == 8) tm_ = __hip_atomic_load(tmo, __ATOMIC_RELAXED, __HIP_MEMORY_SCOPE_AGENT);
                asm volatile("s_waitcnt vmcnt(0)" ::: "memory");
                if (pub < issued) { pub = issued; ctl[isB] = pub; }
                u32 r = f < fbase ? 0 : (f - fbase > ROUNDS ? ROUNDS : f - fbase);
                u32 un = lane == 7 ? (r < ROUNDS ? r : 256u) : 32 + 7 * r + lane;
                u32 mn = 256;
#pragma unroll
                for (int j = 0; j < 8; ++j) { u32 v = __builtin_amdgcn_readlane(un, j); mn = v < mn ? v : mn; }
                u32 nr = mn >> 2;
                if (nr <= ready) {
                    if (__builtin_amdgcn_readlane(tm_, 8) == call || rt() - t0 > DEADLINE) {
                        if (lane == 0) __hip_atomic_store(tmo, call, __ATOMIC_RELAXED, __HIP_MEMORY_SCOPE_AGENT);
                        *(volatile LAS u32*)(L + LDS_ERR) = 1;
                        asm volatile("s_waitcnt vmcnt(0) lgkmcnt(0)" ::: "memory");
                        nr = NSTAGE;
                    } else {
                        __builtin_amdgcn_s_sleep(2);
                    }
                }
                ready = nr > ready ? nr : ready;
            }
        }
        STAMP(3 + isB);
    } else if (wave == 10) {
        const int panel = idx;
        const u32 dst = (rank + xg) & 7;
        char* dws = P.ws[dst];
        const int j = (rank - dst - 1) & 7;
        const u32 u0 = j == 7 ? 0 : 32 + j, ustep = j == 7 ? 1 : 7;
        char* dp = dws + WS_IMG + (call & 1) * IMG_BYTES + (size_t)panel * PANEL_BYTES + (size_t)u0 * UNIT + lane * 16;
        GAS u32* flag = (GAS u32*)(dws + WS_FLAG + (panel * 8 + rank) * 128);
        const u32 fbase = call * 64;
        u32x4 e[4];
        if (panel < 16) {
            const GAS unsigned char* src = (const GAS unsigned char*)P.x + (size_t)(panel * 256 + lane) * KS;
            u32x4 ra[4], rb[4];
            auto ld = [&](u32x4 (&d)[4], int r) {
#pragma unroll
                for (int i = 0; i < 4; ++i) d[i] = *(const GAS u32x4*)(src + i * 64 * KS + r * 16);
            };
            auto step = [&](u32x4 (&d)[4], int r) {
#pragma unroll
                for (int i = 0; i < 4; ++i) e[i] = d[i];
                WAIT_VM0();
                if (r > 0 && lane == 0) __hip_atomic_store(flag, fbase + r, __ATOMIC_RELAXED, __HIP_MEMORY_SCOPE_SYSTEM);
#pragma unroll
                for (int i = 0; i < 4; ++i) st_sys(dp + (size_t)r * ustep * UNIT + i * 1024, e[i]);
                ld(d, r + 2 < ROUNDS ? r + 2 : r);
            };
            ld(ra, 0); ld(rb, 1);
            WAIT_VM0();
            for (int r = 0; r < ROUNDS; r += 2) { step(ra, r); step(rb, r + 1); }
        } else {
            const GAS unsigned char* src = (const GAS unsigned char*)P.w + (panel - 16) * 256 + lane * 4;
            u32 ra[16], rb[16];
            auto ld = [&](u32 (&d)[16], int r) {
#pragma unroll
                for (int k = 0; k < 16; ++k) d[k] = *(const GAS u32*)(src + (size_t)(r * 16 + k) * MN);
            };
            auto step = [&](u32 (&d)[16], int r) {
#pragma unroll
                for (int q = 0; q < 4; ++q) {
                    u32 t0_ = perm(d[4 * q + 1], d[4 * q], 0x05010400), t1_ = perm(d[4 * q + 1], d[4 * q], 0x07030602);
                    u32 t2_ = perm(d[4 * q + 3], d[4 * q + 2], 0x05010400), t3_ = perm(d[4 * q + 3], d[4 * q + 2], 0x07030602);
                    e[0][q] = perm(t2_, t0_, 0x05040100); e[1][q] = perm(t2_, t0_, 0x07060302);
                    e[2][q] = perm(t3_, t1_, 0x05040100); e[3][q] = perm(t3_, t1_, 0x07060302);
                }
                WAIT_VM0();
                if (r > 0 && lane == 0) __hip_atomic_store(flag, fbase + r, __ATOMIC_RELAXED, __HIP_MEMORY_SCOPE_SYSTEM);
#pragma unroll
                for (int i = 0; i < 4; ++i) st_sys(dp + (size_t)r * ustep * UNIT + i * 1024, e[i]);
                ld(d, r + 2 < ROUNDS ? r + 2 : r);
            };
            ld(ra, 0); ld(rb, 1);
            WAIT_VM0();
            for (int r = 0; r < ROUNDS; r += 2) { step(ra, r); step(rb, r + 1); }
        }
        WAIT_VM0();
        if (lane == 0) __hip_atomic_store(flag, fbase + ROUNDS, __ATOMIC_RELAXED, __HIP_MEMORY_SCOPE_SYSTEM);
        STAMP(5);
    } else {
        if (lane == 0) __hip_atomic_store((GAS u32*)(myws + WS_CNT + bid * 128), call, __ATOMIC_RELAXED, __HIP_MEMORY_SCOPE_AGENT);
    }
}

extern "C" void kernel_launch(int rank, int nranks, ncclComm_t comm,
                              hipStream_t stream,
                              void* const* d_in, const int* in_sizes, int n_in,
                              void* d_out, int out_size,
                              const int* in_dtypes, int out_dtype,
                              void* const* d_out_all,
                              void* const* d_ws_all, size_t ws_bytes) {
    (void)comm; (void)in_sizes; (void)n_in; (void)out_size; (void)in_dtypes; (void)out_dtype; (void)d_out_all;
    if (nranks != NR || (unsigned)rank >= (unsigned)NR || ws_bytes < WS_NEED) {
        fprintf(stderr, "kernel_launch: unexpected configuration\n");
        return;
    }
    Params p;
    p.x = (const unsigned char*)d_in[0];
    p.w = (const unsigned char*)d_in[1];
    p.sx = (const float*)d_in[2];
    p.sw = (const float*)d_in[3];
    p.out = (float*)d_out;
    for (int i = 0; i < NR; ++i) p.ws[i] = (char*)d_ws_all[i];
    p.rank = rank;
    p.pad = 0;
    hipLaunchKernelGGL(ag_gemm, dim3(NWG), dim3(NTHREADS), 0, stream, p);
    HIP_CHECK(hipGetLastError());
}
```

```cpp
#include <hip/hip_runtime.h>
#include <rccl/rccl.h>
#include <cstdio>

#define HIP_CHECK(expr) do { hipError_t _e = (expr); if (_e != hipSuccess) { \
    fprintf(stderr, "HIP error %s at %s:%d\n", hipGetErrorString(_e), __FILE__, __LINE__); \
    return; } } while (0)

#define MSLEEP 1
#define GAS __attribute__((address_space(1)))
#define LAS __attribute__((address_space(3)))
typedef unsigned u32;
typedef unsigned short u16;
typedef unsigned long long u64;
typedef unsigned u32x2 __attribute__((ext_vector_type(2)));
typedef unsigned u32x4 __attribute__((ext_vector_type(4)));
typedef int i32x4 __attribute__((ext_vector_type(4)));
typedef int i32x8 __attribute__((ext_vector_type(8)));
typedef float f32x4 __attribute__((ext_vector_type(4)));
typedef float f32x16 __attribute__((ext_vector_type(16)));

constexpr int NR = 8;
constexpr int MN = 4096;
constexpr int KS = 512;
constexpr int NWG = 256;
constexpr int NTHREADS = 768;
constexpr u32 ROUNDS = 32;
constexpr u32 NSTAGE = 64;
constexpr u32 UNIT = 4096;
constexpr size_t IMG_BYTES = 32u << 20;
constexpr size_t CB_BYTES = 4u << 20;
__host__ __device__ constexpr u32 unit_off(u32 panel, u32 u) { return (u >> 2) * 524288u + panel * 16384u + (u & 3) * UNIT; }
constexpr size_t WS_CNT = 0;
constexpr size_t WS_TMO = 48 << 10;
constexpr size_t WS_UF = 64 << 10;
constexpr size_t WS_RDY = 128 << 10;
constexpr size_t WS_JOB = 192 << 10;
constexpr size_t WS_DBG = 256 << 10;
constexpr size_t WS_IMG = 1 << 20;
constexpr size_t WS_CB = WS_IMG + 2 * IMG_BYTES;
constexpr size_t WS_NEED = WS_CB + 2 * CB_BYTES;
constexpr u64 DEADLINE = 50000000ull;
constexpr u32 SPINS = 1u << 25;
constexpr u32 LDS_A = 0, LDS_B = 65536, LDS_CTL = 131072;
constexpr u32 LDS_ERR = LDS_CTL + 128, LDS_FET = LDS_CTL + 132, LDS_CON = LDS_CTL + 136;
constexpr u32 LDS_RAWF = LDS_CTL + 144;
constexpr u32 LDS_JOB = LDS_CTL + 256, LDS_LEN = LDS_CTL + 272;
constexpr u32 LDS_RING = LDS_CTL + 512;
constexpr u32 LDS_XB = LDS_RING + 16384;
constexpr u32 LDS_LUT = LDS_XB + 1024;
constexpr u32 LDS_BYTES = LDS_LUT + 2048;

struct Params {
    const unsigned char* x;
    const unsigned char* w;
    const float* sx;
    const float* sw;
    float* out;
    char* ws[NR];
    int rank;
    int pad;
    u32 srcmap;
    u32 pad2;
};

__device__ __forceinline__ u64 rt() { return __builtin_amdgcn_s_memrealtime(); }
#define WAIT_VM(n) asm volatile("s_waitcnt vmcnt(" #n ")" ::: "memory")
#define WAIT_VM0() do { __builtin_amdgcn_s_waitcnt(0x0F70); WAIT_VM(0); } while (0)
#define FENCE() asm volatile("" ::: "memory")
#define SB() __builtin_amdgcn_sched_barrier(0)
#define XCD_ID() (__builtin_amdgcn_s_getreg((3 << 11) | 20) & 7u)
#define UNI(x) __builtin_amdgcn_readfirstlane(x)

#define STAMP(i) do {} while (0)

__device__ __forceinline__ void st_sys(char* p, u32x4 v) {
    asm volatile("global_store_dwordx4 %0, %1, off sc0 sc1\n\ts_nop 1" :: "v"(p), "v"(v) : "memory");
}
__device__ __forceinline__ void st_sys_s(u32 voff, const void* base, u32x4 v) {
    asm volatile("global_store_dwordx4 %0, %1, %2 sc0 sc1\n\ts_nop 1" :: "v"(voff), "v"(v), "s"(base) : "memory");
}
__device__ __forceinline__ void st_flag(const void* base, u32 v, int lane) {
    if (lane == 0) asm volatile("global_store_dword %0, %1, %2 sc0 sc1" :: "v"(0u), "v"(v), "s"(base) : "memory");
}

#define DMA4(v) "s_nop 0\n\t" \
                "global_load_lds_dwordx4 " v ", %5 sc0 sc1\n\t" \
                "global_load_lds_dwordx4 " v ", %5 offset:1024 sc0 sc1\n\t" \
                "global_load_lds_dwordx4 " v ", %5 offset:2048 sc0 sc1\n\t" \
                "global_load_lds_dwordx4 " v ", %5 offset:3072 sc0 sc1\n\t"
__device__ __forceinline__ void dma16(u32 v0, u32 v1, u32 v2, u32 v3, const char* g, u32 l) {
    u32 keep;
    asm volatile("s_mov_b32 %0, m0\n\ts_mov_b32 m0, %6\n\t"
                 DMA4("%1") "s_add_u32 m0, m0, 0x1000\n\t"
                 DMA4("%2") "s_add_u32 m0, m0, 0x1000\n\t"
                 DMA4("%3") "s_add_u32 m0, m0, 0x1000\n\t"
                 DMA4("%4") "s_mov_b32 m0, %0"
                 : "=&s"(keep) : "v"(v0), "v"(v1), "v"(v2), "v"(v3), "s"(g), "s"(l) : "memory", "scc");
}
__device__ __forceinline__ void dma1(u32 v, const void* g, u32 l) {
    u32 keep;
    asm volatile("s_mov_b32 %0, m0\n\ts_mov_b32 m0, %3\n\ts_nop 0\n\t"
                 "global_load_lds_dwordx4 %1, %2 sc0 sc1\n\ts_mov_b32 m0, %0"
                 : "=&s"(keep) : "v"(v), "s"(g), "s"(l) : "memory");
}

__device__ __forceinline__ i32x8 ld_frag(LAS unsigned char* p) {
    i32x4 lo = *(LAS i32x4*)p, hi = *(LAS i32x4*)(p + UNIT);
    return __builtin_shufflevector(lo, hi, 0, 1, 2, 3, 4, 5, 6, 7);
}
#define MFMA(c, a, b) asm volatile("v_mfma_f32_32x32x64_f8f6f4 %0, %1, %2, %0 cbsz:0 blgp:1" : "+v"(c) : "v"(a), "v"(b))

__device__ __forceinline__ u32 perm(u32 hi, u32 lo, u32 sel) { return __builtin_amdgcn_perm(hi, lo, sel); }
__device__ __forceinline__ u32 bfi(u32 m, u32 a, u32 b) { return (a & m) | (b & ~m); }
template <int S> __device__ __forceinline__ u32 shl(u32 v) { return S >= 0 ? v << (S >= 0 ? S : 0) : v >> (S < 0 ? -S : 0); }
__device__ __forceinline__ u32 scan64(u32 v) {
    v += __builtin_amdgcn_update_dpp(0u, v, 0x111, 0xf, 0xf, true);
    v += __builtin_amdgcn_update_dpp(0u, v, 0x112, 0xf, 0xf, true);
    v += __builtin_amdgcn_update_dpp(0u, v, 0x114, 0xf, 0xf, true);
    v += __builtin_amdgcn_update_dpp(0u, v, 0x118, 0xf, 0xf, true);
    v += __builtin_amdgcn_update_dpp(0u, v, 0x142, 0xa, 0xf, false);
    v += __builtin_amdgcn_update_dpp(0u, v, 0x143, 0xc, 0xf, false);
    return v;
}
__device__ __forceinline__ void lds_or(LAS unsigned char* p, u32 v) {
    __hip_atomic_fetch_or((LAS u32*)p, v, __ATOMIC_RELAXED, __HIP_MEMORY_SCOPE_WORKGROUP);
}

template <bool W> __device__ __forceinline__ void load_unit(u32 (&x)[16], const Params& P, int panel, int r, int lane) {
    if (!W) {
        const GAS unsigned char* b = (const GAS unsigned char*)P.x + (size_t)panel * 256 * KS + r * 16;
#pragma unroll
        for (int i = 0; i < 4; ++i) {
            u32x4 v = *(const GAS u32x4*)(b + (u32)((i * 64 + lane) * KS));
            x[4 * i] = v[0]; x[4 * i + 1] = v[1]; x[4 * i + 2] = v[2]; x[4 * i + 3] = v[3];
        }
    } else {
        const GAS unsigned char* b = (const GAS unsigned char*)P.w + (size_t)r * 16 * MN + (panel - 16) * 256;
        u32 d[16];
#pragma unroll
        for (int k = 0; k < 16; ++k) d[k] = *(const GAS u32*)(b + (u32)(k * MN + lane * 4));
#pragma unroll
        for (int q = 0; q < 4; ++q) {
            u32 t0 = perm(d[4 * q + 1], d[4 * q], 0x05010400), t1 = perm(d[4 * q + 1], d[4 * q], 0x07030602);
            u32 t2 = perm(d[4 * q + 3], d[4 * q + 2], 0x05010400), t3 = perm(d[4 * q + 3], d[4 * q + 2], 0x07030602);
            x[q] = perm(t2, t0, 0x05040100); x[4 + q] = perm(t2, t0, 0x07060302);
            x[8 + q] = perm(t3, t1, 0x05040100); x[12 + q] = perm(t3, t1, 0x07060302);
        }
    }
}

template <int MB> struct Fmt {
    static constexpr u32 LOW = MB == 3 ? 0x3F3F3F3Fu : 0x1F1F1F1Fu;
    static constexpr u32 HM = 0x01010101u << (MB + 2);
    static constexpr u32 K3 = (3u << MB) * 0x01010101u;
    static constexpr u32 HEAD = MB == 3 ? 3072 : 2048;
    static constexpr u32 OFF_F = MB == 3 ? 0 : 512;
    static constexpr u32 OFF_H = OFF_F + 128, OFF_N = OFF_H + 4;
};

template <int MB> __device__ __forceinline__ u32 encode_unit(const u32 (&x)[16], LAS unsigned char* T, char* dst, int lane) {
    typedef Fmt<MB> F;
    *(LAS u32x4*)(T + lane * 16) = u32x4{0, 0, 0, 0};
    *(LAS u32x4*)(T + 1024 + lane * 16) = u32x4{0, 0, 0, 0};
    u32 sum = 0;
#pragma unroll
    for (int j = 0; j < 16; ++j) sum = __builtin_amdgcn_udot4(x[j] & 0x7F7F7F7Fu, 0x01010101u, sum, false);
    const int tot = __builtin_amdgcn_readlane(scan64(sum), 63);
    const int e0 = (tot - (5038 << MB)) >> (12 + MB);
    const u32 b1 = ((u32)(e0 - 3) << MB) & 0x7Fu;
    const u32 B0 = (((u32)e0 << MB) & 0x7Fu) * 0x01010101u, B1 = b1 * 0x01010101u;
    u32 c[16], m[4], h[4];
#define ENC1(q, jj) { const int j = 4 * q + jj; const u32 xo = x[j] | 0x80808080u; c[j] = xo - B0; \
        const u32 xp = ((((xo - B1) & F::LOW) + B1) ^ x[j]); \
        m[q] = jj == 0 ? (xp >> 5) & 0x03030303u : bfi(0x03030303u << (2 * jj), shl<2 * jj - 5>(xp), m[q]); \
        h[q] = jj == 0 ? (c[j] >> (MB + 2)) & 0x01010101u : bfi(0x01010101u << jj, c[j] >> (MB + 2 - jj), h[q]); }
#define ENC4(q) ENC1(q, 0) ENC1(q, 1) ENC1(q, 2) ENC1(q, 3)
    ENC4(0) ENC4(1) ENC4(2) ENC4(3)
    u32 s[2] = {0, 0}, pos[2] = {0, 0}, flags = 0;
#pragma unroll
    for (int n = 0; n < 16; ++n) {
        const u32 nib = (h[n >> 2] >> (8 * (n & 3))) & 15u, fl = nib < 1u ? nib : 1u;
        s[n >> 3] |= nib << (pos[n >> 3] & 31u);
        pos[n >> 3] += fl * 4;
        flags |= fl << n;
    }
    u32 cnt = 0;
#pragma unroll
    for (int q = 0; q < 4; ++q) cnt += __builtin_popcount((m[q] | (m[q] >> 1)) & 0x55555555u);
    const u32 mine = ((pos[0] + pos[1]) >> 2) | (cnt << 16);
    const u32 incl = scan64(mine), all = __builtin_amdgcn_readlane(incl, 63), excl = incl - mine;
    const u32 off_e = F::OFF_N + (((all & 0xFFFFu) + 7) >> 3) * 4;
    const u32 tail = off_e + 2 * (all >> 16);
    const u32 len64 = (F::HEAD + tail + 63) >> 6;
    if (len64 >= 64) return 64;
    FENCE();
    {
        const u64 S = (u64)s[0] | ((u64)s[1] << pos[0]);
        const u32 off = excl & 0xFFFFu, sh = (off & 7) * 4;
        LAS unsigned char* p = T + F::OFF_N + (off >> 3) * 4;
        lds_or(p, (u32)S << sh);
        lds_or(p + 4, (u32)((S << sh) >> 32));
        lds_or(p + 8, (u32)(((S >> 32) << sh) >> 32));
    }
    *(LAS u16*)(T + F::OFF_F + lane * 2) = (u16)flags;
    if (lane == 0) *(LAS u32*)(T + F::OFF_H) = b1 | ((all >> 16) << 8);
    FENCE();
    {
        LAS unsigned char* p = T + off_e + 2 * (excl >> 16);
#pragma unroll
        for (int q = 0; q < 4; ++q) {
            u32 mm = m[q];
            while (mm) {
                const u32 b = __builtin_ctz(mm) & 30u, pat = (mm >> b) & 3u;
                mm ^= pat << b;
                *(LAS u16*)p = (u16)(((lane * 4 + q) << 6) | (b << 1) | pat);
                p += 2;
            }
        }
    }
    if (MB == 3) {
        u32x4 o[3];
#pragma unroll
        for (int q = 0; q < 4; ++q) {
            const u32 g3 = c[4 * q + 3];
            o[0][q] = bfi(0x60606060u, g3 << 5, bfi(0x80808080u, x[4 * q], c[4 * q]));
            o[1][q] = bfi(0x60606060u, g3 << 3, bfi(0x80808080u, x[4 * q + 1], c[4 * q + 1]));
            o[2][q] = bfi(0x60606060u, bfi(0x20202020u, g3 << 1, x[4 * q + 3] >> 1), bfi(0x80808080u, x[4 * q + 2], c[4 * q + 2]));
        }
#pragma unroll
        for (int i = 0; i < 3; ++i) st_sys_s(lane * 16, dst + i * 1024, o[i]);
    } else {
        u32x4 o[2];
        u32x2 sg;
#pragma unroll
        for (int p = 0; p < 8; ++p) o[p >> 2][p & 3] = bfi(0x0F0F0F0Fu, c[2 * p], c[2 * p + 1] << 4);
#pragma unroll
        for (int k = 0; k < 2; ++k) {
            u32 v = x[8 * k];
#pragma unroll
            for (int j = 1; j < 8; ++j) v = bfi(0x80808080u >> j, x[8 * k + j] >> j, v);
            sg[k] = v;
        }
        *(LAS u32x2*)(T + lane * 8) = sg;
#pragma unroll
        for (int i = 0; i < 2; ++i) st_sys_s(lane * 16, dst + i * 1024, o[i]);
    }
    FENCE();
    const u32 tl = len64 * 64 - F::HEAD;
    if (lane * 16 < tl) st_sys_s(lane * 16, dst + F::HEAD, *(LAS u32x4*)(T + lane * 16));
    if (MB == 2 && 1024 + lane * 16 < tl) st_sys_s(lane * 16, dst + F::HEAD + 1024, *(LAS u32x4*)(T + 1024 + lane * 16));
    return len64;
}

template <int MB> __device__ __forceinline__ void decode_unit(u32x4 (&e)[4], LAS unsigned char* S, LAS unsigned char* L, int lane) {
    typedef Fmt<MB> F;
    LAS unsigned char* T = S + F::HEAD;
    *(LAS u32x4*)(L + LDS_XB + lane * 16) = u32x4{0, 0, 0, 0};
    const u32 flags = *(LAS u16*)(T + F::OFF_F + lane * 2);
    const u32 hdr = UNI(*(LAS u32*)(T + F::OFF_H));
    const u32x2 l0 = *(LAS u32x2*)(L + LDS_LUT + (flags & 255u) * 8), l1 = *(LAS u32x2*)(L + LDS_LUT + (flags >> 8) * 8);
    const u32 n = __builtin_popcount(flags), incl = scan64(n), off = incl - n;
    const u32 off_e = F::OFF_N + ((__builtin_amdgcn_readlane(incl, 63) + 7) >> 3) * 4;
    LAS u32* w = (LAS u32*)(T + F::OFF_N + (off >> 3) * 4);
    const u32 w0 = w[0], w1 = w[1], w2 = w[2];
    FENCE();
    const u32 nexc = (hdr >> 8) & 1023u;
    for (u32 i = lane; i < nexc; i += 64) {
        const u32 en = *(LAS u16*)(T + off_e + 2 * i);
        lds_or(L + LDS_XB + ((en >> 6) & 255u) * 4, (en & 3u) << ((en >> 1) & 30u));
    }
    FENCE();
    const u32x4 m = *(LAS u32x4*)(L + LDS_XB + lane * 16);
    const u32 sh = (off & 7) * 4;
    const u32 s0 = __builtin_amdgcn_alignbit(w1, w0, sh), shi = __builtin_amdgcn_alignbit(w2, w1, sh);
    const u32 s1 = (u32)((((u64)shi << 32) | s0) >> (4 * __builtin_popcount(flags & 255u)));
    u32 h[4];
    {
        const u32 a = s0 & 0x0F0F0F0Fu, b = (s0 >> 4) & 0x0F0F0F0Fu, c = s1 & 0x0F0F0F0Fu, d = (s1 >> 4) & 0x0F0F0F0Fu;
        h[0] = perm(b, a, l0[0]); h[1] = perm(b, a, l0[1]); h[2] = perm(d, c, l1[0]); h[3] = perm(d, c, l1[1]);
    }
    const u32 B1 = (hdr & 0x7Fu) * 0x01010101u;
    u32 lo[16], sg[16];
    if (MB == 3) {
        const u32x4 o0 = *(LAS u32x4*)(S + lane * 16), o1 = *(LAS u32x4*)(S + 1024 + lane * 16), o2 = *(LAS u32x4*)(S + 2048 + lane * 16);
#pragma unroll
        for (int q = 0; q < 4; ++q) {
            lo[4 * q] = o0[q] & 0x1F1F1F1Fu; sg[4 * q] = o0[q];
            lo[4 * q + 1] = o1[q] & 0x1F1F1F1Fu; sg[4 * q + 1] = o1[q];
            lo[4 * q + 2] = o2[q] & 0x1F1F1F1Fu; sg[4 * q + 2] = o2[q];
            lo[4 * q + 3] = ((o0[q] >> 5) & 0x03030303u) | ((o1[q] >> 3) & 0x0C0C0C0Cu) | ((o2[q] >> 1) & 0x10101010u);
            sg[4 * q + 3] = o2[q] << 1;
        }
    } else {
        const u32x4 o0 = *(LAS u32x4*)(S + lane * 16), o1 = *(LAS u32x4*)(S + 1024 + lane * 16);
        const u32x2 s2 = *(LAS u32x2*)(T + lane * 8);
#pragma unroll
        for (int p = 0; p < 8; ++p) {
            const u32 v = p < 4 ? o0[p & 3] : o1[p & 3];
            lo[2 * p] = v & 0x0F0F0F0Fu; lo[2 * p + 1] = (v >> 4) & 0x0F0F0F0Fu;
        }
#pragma unroll
        for (int j = 0; j < 16; ++j) sg[j] = s2[j >> 3] << (j & 7);
    }
#define DEC1(q, jj) { const int j = 4 * q + jj; \
        const u32 z = ((lo[j] + ((h[q] << (MB + 2 - jj)) & F::HM) + F::K3) & F::LOW) + B1; \
        e[q][jj] = bfi(0x7F7F7F7Fu, z, sg[j]) ^ (shl<5 - 2 * jj>(m[q]) & 0x60606060u); }
#define DEC4(q) DEC1(q, 0) DEC1(q, 1) DEC1(q, 2) DEC1(q, 3)
    DEC4(0) DEC4(1) DEC4(2) DEC4(3)
}

template <bool W> __device__ __forceinline__ void prologue_unit(const Params& P, LAS unsigned char* L, char* myws, u32 callv,
                                                                 int idx, int xg, int wave, int lane) {
    u32 x[16];
    const int u = xg * 4 + (wave & 3);
    load_unit<W>(x, P, idx, u, lane);
    const u32 par = UNI(callv) & 1;
    if (wave < 4) {
        u32 len64 = encode_unit<W ? 2 : 3>(x, L + wave * 2048, myws + WS_CB + par * CB_BYTES + unit_off(idx, u), lane);
        if (lane == 0) *(volatile LAS u32*)(L + LDS_LEN + wave * 4) = len64;
    } else {
        const char* dp = myws + WS_IMG + par * IMG_BYTES + unit_off(idx, u);
#pragma unroll
        for (int i = 0; i < 4; ++i) st_sys_s(lane * 16, dp + i * 1024, u32x4{x[4 * i], x[4 * i + 1], x[4 * i + 2], x[4 * i + 3]});
    }
    WAIT_VM0();
}

__global__ void __launch_bounds__(NTHREADS, 3) ag_gemm(Params P) {
    __shared__ __attribute__((aligned(16))) unsigned char lds_raw[LDS_BYTES];
    LAS unsigned char* L = (LAS unsigned char*)lds_raw;
    const int wave = UNI(threadIdx.x >> 6);
    const int lane = threadIdx.x & 63;
    const int bid = blockIdx.x;
    const int rank = P.rank;
    char* myws = P.ws[rank];
    const int xg = bid & 7, idx = bid >> 3;
    const int tm = (xg >> 1) * 4 + (idx >> 3), tn = (xg & 1) * 8 + (idx & 7);

    u32 call = __hip_atomic_load((GAS u32*)(myws + WS_CNT + bid * 128), __ATOMIC_RELAXED, __HIP_MEMORY_SCOPE_AGENT) + 1;
    char* peer = nullptr;
    if (wave < 8) {
        if (idx < 16) prologue_unit<false>(P, L, myws, call, idx, xg, wave, lane);
        else prologue_unit<true>(P, L, myws, call, idx, xg, wave, lane);
    } else if (wave == 8) {
        ((LAS u32*)(L + LDS_CTL))[lane] = 0;
        peer = P.ws[lane & 7];
        STAMP(0);
        WAIT_VM0();
    } else if (wave == 10) {
        if (lane == 0) {
            u32 j = XCD_ID() * 32 + idx;
            for (int k = 0; k < NWG; ++k, j = (j + 1) & (NWG - 1))
                if (__hip_atomic_fetch_max((GAS u32*)(myws + WS_JOB + j * 128), call, __ATOMIC_RELAXED, __HIP_MEMORY_SCOPE_AGENT) < call) break;
            *(volatile LAS u32*)(L + LDS_JOB) = j;
        }
    } else if (wave == 11) {
#pragma unroll
        for (int k = 0; k < 4; ++k) {
            const u32 f = k * 64 + lane;
            u32x2 sel = {0, 0};
#pragma unroll
            for (int n = 0; n < 8; ++n) {
                const u32 i = __builtin_popcount(f & ((1u << n) - 1));
                sel[n >> 2] |= (((f >> n) & 1) ? (i >> 1) + 4 * (i & 1) : 12u) << (8 * (n & 3));
            }
            *(LAS u32x2*)(L + LDS_LUT + f * 8) = sel;
        }
    }
    call = UNI(call);
    if (wave >= 8) __builtin_amdgcn_s_setprio(3);
    __syncthreads();
    if (wave == 8) {
        const u32x4 ln = *(LAS u32x4*)(L + LDS_LEN);
        if (lane < NR) {
            if (lane != rank) st_sys(peer + WS_RDY + (call & 1) * 32768 + (idx * 8 + rank) * 128 + xg * 16, ln | (call << 7));
            else st_sys(peer + WS_UF + idx * 1024 + xg * 16, u32x4{call, call, call, call});
        }
        if (lane == 0) __hip_atomic_store((GAS u32*)(myws + WS_CNT + bid * 128), call, __ATOMIC_RELAXED, __HIP_MEMORY_SCOPE_AGENT);
        STAMP(7);
    }

    if (wave < 8) {
        const int g = wave >> 1, nh = wave & 1;
        LAS unsigned char* pa = L + LDS_A + (lane >> 5) * 8192 + (g * 64 + (lane & 31)) * 16;
        LAS unsigned char* pb = L + LDS_B + (lane >> 5) * 8192 + (nh * 32 + (lane & 31)) * 16;
        volatile LAS u32* ctl = (volatile LAS u32*)(L + LDS_CTL + wave * 16);
        f32x16 c00 = {}, c01 = {}, c02 = {}, c03 = {}, c10 = {}, c11 = {}, c12 = {}, c13 = {};
        auto wait_full = [&](u32 need) {
            for (u32 spin = 0; spin < SPINS; ++spin) {
                u32 fa = ctl[0], fb = ctl[1];
                u32 m = UNI(fa < fb ? fa : fb);
                if (m >= need) break;
                __builtin_amdgcn_s_sleep(MSLEEP);
            }
            FENCE();
        };
        wait_full(1);
        i32x8 bx = ld_frag(pb); SB();
        i32x8 a0 = ld_frag(pa); SB();
        i32x8 a1 = ld_frag(pa + 512); SB();
        i32x8 by = ld_frag(pb + 1024); SB();
        for (u32 s4 = 0; s4 < NSTAGE; s4 += 4) {
#pragma unroll
            for (int q = 0; q < 4; ++q) {
                const u32 so = q * 16384, sn = ((q + 1) & 3) * 16384;
                const u32 s = s4 + q;
                SB();
                MFMA(c00, a0, bx);
                MFMA(c10, a1, bx);
                SB();
                u32 fa = ctl[0], fb = ctl[1];
                bx = ld_frag(pb + so + 2048);
                SB();
                MFMA(c01, a0, by);
                MFMA(c11, a1, by);
                SB();
                by = ld_frag(pb + so + 3072);
                SB();
                MFMA(c02, a0, bx);
                SB();
                u32 m = UNI(fa < fb ? fa : fb);
                SB();
                MFMA(c12, a1, bx);
                SB();
                u32 need = s + 2 > NSTAGE ? NSTAGE : s + 2;
                if (__builtin_expect(m < need, 0)) wait_full(need);
                FENCE();
                bx = ld_frag(pb + sn);
                SB();
                MFMA(c03, a0, by);
                SB();
                ctl[2] = s + 1;
                a0 = ld_frag(pa + sn);
                SB();
                MFMA(c13, a1, by);
                SB();
                a1 = ld_frag(pa + sn + 512);
                by = ld_frag(pb + sn + 1024);
            }
        }
        SB();
        asm volatile("s_nop 15\n\ts_nop 7" : "+v"(c00), "+v"(c01), "+v"(c02), "+v"(c03), "+v"(c10), "+v"(c11), "+v"(c12), "+v"(c13));
        int tz = threadIdx.x; asm volatile("" : "+v"(tz));
        const int ln = tz & 63;
        float sc = P.sx[0] * P.sw[0];
        if (*(volatile LAS u32*)(L + LDS_ERR) != 0) sc = __builtin_nanf("");
        GAS float* o = (GAS float*)P.out + (size_t)(tm * 256 + g * 64 + 4 * (ln >> 5)) * MN + tn * 256 + nh * 128 + 4 * (ln & 31);
#pragma unroll
        for (int r = 0; r < 16; ++r) {
            const int row = (r & 3) + 8 * (r >> 2);
            f32x4 v = {c00[r] * sc, c01[r] * sc, c02[r] * sc, c03[r] * sc};
            *(GAS f32x4*)(o + (size_t)row * MN) = v;
        }
#pragma unroll
        for (int r = 0; r < 16; ++r) {
            const int row = 32 + (r & 3) + 8 * (r >> 2);
            f32x4 v = {c10[r] * sc, c11[r] * sc, c12[r] * sc, c13[r] * sc};
            *(GAS f32x4*)(o + (size_t)row * MN) = v;
        }
    } else if (wave < 10) {
        const int isB = wave - 8;
        const int panel = isB ? 16 + tn : tm;
        const char* img = myws + WS_IMG + (call & 1) * IMG_BYTES + unit_off(panel, 0);
        const u32 ldsb = (u32)(size_t)L + (isB ? LDS_B : LDS_A);
        volatile LAS u32* ctl = (volatile LAS u32*)(L + LDS_CTL + (lane & 7) * 16);
        GAS u32* uf = (GAS u32*)(myws + WS_UF + panel * 1024);
        GAS u32* tmo = (GAS u32*)(myws + WS_TMO);
        const u32 voff = lane * 16;
        u32 ready = 0, issued = 0, pub = 0;
        const u64 t0 = rt();
        WAIT_VM0();
        for (u32 spin = 0; pub < NSTAGE && spin < SPINS; ++spin) {
            bool can = issued < ready;
            if (can && issued >= 4) {
                u32 d = ctl[2];
                can = __builtin_amdgcn_ballot_w64(d + 3 < issued) == 0;
            }
            if (can) {
                dma16(voff, voff + 4096, voff + 8192, voff + 12288, img + (size_t)issued * 524288, ldsb + (issued & 3) * 16384);
                ++issued;
                if (issued - pub == 3) {
                    WAIT_VM(32);
                    ++pub;
                    ctl[isB] = pub;
                }
            } else if (issued - pub == 2) {
                WAIT_VM(16);
                ++pub;
                ctl[isB] = pub;
            } else if (issued < ready || issued == NSTAGE) {
                if (pub < issued) {
                    WAIT_VM(0);
                    pub = issued;
                    ctl[isB] = pub;
                } else {
                    __builtin_amdgcn_s_sleep(1);
                }
            } else {
                const u32 u = ready * 4 + (lane & 31);
                const u32 f = __hip_atomic_load(uf + (u < 256 ? u : 255), __ATOMIC_RELAXED, __HIP_MEMORY_SCOPE_AGENT);
                WAIT_VM(0);
                if (pub < issued) { pub = issued; ctl[isB] = pub; }
                const u32 miss = (u32)__builtin_amdgcn_ballot_w64(f < call);
                u32 nr = ready + (miss ? (u32)__builtin_ctz(miss) >> 2 : 8u);
                nr = nr < NSTAGE ? nr : NSTAGE;
                if (nr == ready) {
                    if ((spin & 63) == 63 && (UNI(__hip_atomic_load(tmo, __ATOMIC_RELAXED, __HIP_MEMORY_SCOPE_AGENT)) == call || rt() - t0 > DEADLINE)) {
                        if (lane == 0) __hip_atomic_store(tmo, call, __ATOMIC_RELAXED, __HIP_MEMORY_SCOPE_AGENT);
                        *(volatile LAS u32*)(L + LDS_ERR) = 1;
                        asm volatile("s_waitcnt vmcnt(0) lgkmcnt(0)" ::: "memory");
                        nr = NSTAGE;
                    } else {
                        __builtin_amdgcn_s_sleep(2);
                    }
                }
                ready = nr;
            }
        }
        STAMP(3 + isB);
    } else {
        const u32 job = UNI(*(volatile LAS u32*)(L + LDS_JOB));
        const u32 src = (P.srcmap >> (4 * ((job >> 5) & 7))) & 7u, fp = job & 31;
        if (src == (u32)rank) return;
        volatile LAS u32* fet = (volatile LAS u32*)(L + LDS_FET);
        volatile LAS u32* con = (volatile LAS u32*)(L + LDS_CON);
        volatile LAS u32* rawf = (volatile LAS u32*)(L + LDS_RAWF);
        const u32 vo = lane * 16;
        if (wave == 10) {
            const char* sraw = P.ws[src] + WS_IMG + (call & 1) * IMG_BYTES;
            const char* scb = P.ws[src] + WS_CB + (call & 1) * CB_BYTES;
            GAS u32* rdy = (GAS u32*)(myws + WS_RDY + (call & 1) * 32768 + ((fp ^ ((lane & 1) << 4)) * 8 + src) * 128) + (lane & 31);
            GAS u32* tmo = (GAS u32*)(myws + WS_TMO);
            const u32 lr = (u32)(size_t)L + LDS_RING;
            u32 avail = 0, f = 0;
            const u64 t0 = rt();
            for (u32 r = 0; r < ROUNDS; ++r) {
                if (avail <= r) {
                    WAIT_VM(0);
                    *fet = r;
                    while (true) {
                        f = __hip_atomic_load(rdy, __ATOMIC_RELAXED, __HIP_MEMORY_SCOPE_SYSTEM);
                        const u32 miss = (u32)__builtin_amdgcn_ballot_w64((f >> 7) < call);
                        avail = miss ? __builtin_ctz(miss) : ROUNDS;
                        if (avail > r) break;
                        if (UNI(__hip_atomic_load(tmo, __ATOMIC_RELAXED, __HIP_MEMORY_SCOPE_AGENT)) == call || rt() - t0 > DEADLINE) {
                            if (lane == 0) __hip_atomic_store(tmo, call, __ATOMIC_RELAXED, __HIP_MEMORY_SCOPE_AGENT);
                            *fet = 0x80000000u;
                            return;
                        }
                        __builtin_amdgcn_s_sleep(1);
                    }
                    WAIT_VM0();
                }
                if (r >= 4 && UNI(*con) + 4 <= r) {
                    WAIT_VM(0);
                    *fet = r;
                    for (u32 spin = 0; spin < SPINS && UNI(*con) + 4 <= r; ++spin) __builtin_amdgcn_s_sleep(1);
                }
                u32 len64 = __builtin_amdgcn_readlane(f, r) & 127u;
                len64 = len64 < 64 ? len64 : 64;
                const u32 len = len64 * 64;
                rawf[r & 3] = len64 >> 6;
                const char* q = (len64 == 64 ? sraw : scb) + unit_off(fp ^ ((r & 1) << 4), r);
                const u32 l = lr + (r & 3) * UNIT;
                for (u32 o = 0; o < len; o += 512) {
                    WAIT_VM(2);
                    if (o == 1024) *fet = r;
                    if (lane < 32 && o + vo < len) dma1(vo, q + o, l + o);
                }
            }
            WAIT_VM(0);
            *fet = ROUNDS;
            STAMP(5);
        } else {
            const char* dp = myws + WS_IMG + (call & 1) * IMG_BYTES;
            const char* uf = myws + WS_UF;
            const u32 u0 = 32 + ((src - rank - 1) & 7);
            const char* pend = nullptr;
            for (u32 r = 0; r < ROUNDS; ++r) {
                u32 f = UNI(*fet);
                if (f <= r) {
                    if (pend) { WAIT_VM(0); st_flag(pend, call, lane); pend = nullptr; }
                    for (u32 spin = 0; spin < 2 * SPINS; ++spin) {
                        f = UNI(*fet);
                        if (f > r) break;
                        __builtin_amdgcn_s_sleep(1);
                    }
                }
                if (f <= r || f > ROUNDS) return;
                FENCE();
                const u32 panel = fp ^ ((r & 1) << 4);
                LAS unsigned char* sl = L + LDS_RING + (r & 3) * UNIT;
                u32x4 e[4];
                if (UNI(rawf[r & 3])) {
#pragma unroll
                    for (int i = 0; i < 4; ++i) e[i] = *(LAS u32x4*)(sl + i * 1024 + lane * 16);
                } else if (panel < 16) {
                    decode_unit<3>(e, sl, L, lane);
                } else {
                    decode_unit<2>(e, sl, L, lane);
                }
                asm volatile("" : "+v"(e[0]), "+v"(e[1]), "+v"(e[2]), "+v"(e[3]) :: "memory");
                *con = r + 1;
                if (pend) { WAIT_VM(0); st_flag(pend, call, lane); }
                const u32 u = u0 + 7 * r;
                const char* q = dp + unit_off(panel, u);
#pragma unroll
                for (int i = 0; i < 4; ++i) st_sys_s(vo, q + i * 1024, e[i]);
                pend = uf + panel * 1024 + u * 4;
            }
            WAIT_VM(0);
            st_flag(pend, call, lane);
            STAMP(17);
        }
    }
}

extern "C" void kernel_launch(int rank, int nranks, ncclComm_t comm,
                              hipStream_t stream,
                              void* const* d_in, const int* in_sizes, int n_in,
                              void* d_out, int out_size,
                              const int* in_dtypes, int out_dtype,
                              void* const* d_out_all,
                              void* const* d_ws_all, size_t ws_bytes) {
    (void)comm; (void)in_sizes; (void)n_in; (void)out_size; (void)in_dtypes; (void)out_dtype; (void)d_out_all;
    if (nranks != NR || (unsigned)rank >= (unsigned)NR || ws_bytes < WS_NEED) {
        fprintf(stderr, "kernel_launch: unexpected configuration\n");
        return;
    }
    Params p;
    p.x = (const unsigned char*)d_in[0];
    p.w = (const unsigned char*)d_in[1];
    p.sx = (const float*)d_in[2];
    p.sw = (const float*)d_in[3];
    p.out = (float*)d_out;
    for (int i = 0; i < NR; ++i) p.ws[i] = (char*)d_ws_all[i];
    p.rank = rank;
    p.pad = 0;
    p.pad2 = 0;
    {
        const unsigned grp = 0x4D;
        u32 m = (u32)rank; int a = 1, b = 4;
        for (int k = 1; k < NR; ++k) {
            const int q = (rank + k) & 7;
            const bool same = ((grp >> q) & 1) == ((grp >> rank) & 1);
            m |= (u32)q << (4 * (same ? a++ : b++));
        }
        p.srcmap = m;
    }
    hipLaunchKernelGGL(ag_gemm, dim3(NWG), dim3(NTHREADS), 0, stream, p);
    HIP_CHECK(hipGetLastError());
}
```

```cpp
#include <hip/hip_runtime.h>
#include <rccl/rccl.h>
#include <cstdio>

#define HIP_CHECK(expr) do { hipError_t _e = (expr); if (_e != hipSuccess) { \
    fprintf(stderr, "HIP error %s at %s:%d\n", hipGetErrorString(_e), __FILE__, __LINE__); \
    return; } } while (0)

#define MSLEEP 1
#define GAS __attribute__((address_space(1)))
#define LAS __attribute__((address_space(3)))
typedef unsigned u32;
typedef unsigned short u16;
typedef unsigned long long u64;
typedef unsigned u32x2 __attribute__((ext_vector_type(2)));
typedef unsigned u32x4 __attribute__((ext_vector_type(4)));
typedef int i32x4 __attribute__((ext_vector_type(4)));
typedef int i32x8 __attribute__((ext_vector_type(8)));
typedef float f32x4 __attribute__((ext_vector_type(4)));
typedef float f32x16 __attribute__((ext_vector_type(16)));

constexpr int NR = 8;
constexpr int MN = 4096;
constexpr int KS = 512;
constexpr int NWG = 256;
constexpr int NTHREADS = 768;
constexpr u32 ROUNDS = 32;
constexpr u32 NSTAGE = 64;
#define NPUSH 1
#define WN 3
#define SLOWEND 48
#define FASTM 0xFFFFu
#define RAWTAIL 1
#define NSPLIT 8
constexpr u32 S0 = NSTAGE - NSPLIT;
constexpr u32 NQ = NSTAGE + NSPLIT;
constexpr u32 R0 = NSPLIT ? ((S0 * 4 - 32 + 13) / 14) * 2 : ROUNDS;
constexpr u32 US = 32 + 7 * R0;
constexpr u32 NPAIR = (ROUNDS - R0) / 2;
constexpr u32 T1 = (R0 - NPUSH) * 32, T2 = T1 + NPAIR * 48, NT = T2 + NPAIR * 16;
constexpr u32 UNIT = 4096;
constexpr size_t IMG_BYTES = 32u << 20;
constexpr size_t CB_BYTES = 4u << 20;
__host__ __device__ constexpr u32 unit_off(u32 panel, u32 u) { return (u >> 2) * 524288u + panel * 16384u + (u & 3) * UNIT; }
constexpr size_t WS_CNT = 0;
constexpr size_t WS_TMO = 48 << 10;
constexpr size_t WS_TIK = 52 << 10;
constexpr size_t WS_UF = 64 << 10;
constexpr size_t WS_RDY = 128 << 10;
constexpr size_t WS_JOB = 192 << 10;
constexpr size_t WS_DBG = 256 << 10;
constexpr size_t WS_IMG = 1 << 20;
constexpr size_t WS_CB = WS_IMG + 2 * IMG_BYTES;
constexpr size_t WS_NEED = WS_CB + 2 * CB_BYTES;
constexpr u64 DEADLINE = 50000000ull;
constexpr u32 SPINS = 1u << 25;
constexpr u32 LDS_A = 0, LDS_B = 65536, LDS_CTL = 131072;
constexpr u32 LDS_ERR = LDS_CTL + 128, LDS_FET = LDS_CTL + 132, LDS_CON = LDS_CTL + 136;
constexpr u32 LDS_PRO = LDS_CTL + 140, LDS_META = LDS_CTL + 144;
constexpr u32 LDS_LUTF = LDS_CTL + 164;
constexpr u32 LDS_JOB = LDS_CTL + 160, LDS_LEN = LDS_CTL + 176;
constexpr u32 LDS_RING = LDS_CTL + 512;
constexpr u32 LDS_XB = LDS_RING + 16384;
constexpr u32 LDS_LUT = LDS_XB + 1024;
constexpr u32 LDS_BYTES = LDS_LUT + 2048;

struct Params {
    const unsigned char* x;
    const unsigned char* w;
    const float* sx;
    const float* sw;
    float* out;
    char* myws;
    char* ws[NR];
    int rank;
    int pad;
    u32 srcmap;
    u32 pad2;
};

__device__ __forceinline__ u64 rt() { return __builtin_amdgcn_s_memrealtime(); }
#define WAIT_VM(n) asm volatile("s_waitcnt vmcnt(" #n ")" ::: "memory")
#define WAIT_VM0() do { __builtin_amdgcn_s_waitcnt(0x0F70); WAIT_VM(0); } while (0)
#define FENCE() asm volatile("" ::: "memory")
#define SB() __builtin_amdgcn_sched_barrier(0)
#define XCD_ID() (__builtin_amdgcn_s_getreg((3 << 11) | 20) & 7u)
#define UNI(x) __builtin_amdgcn_readfirstlane(x)

#define STAMP(i) do {} while (0)
#define STAMP2(i) do {} while (0)
#define STAMP3(i) do {} while (0)

#define OB0 ""
#define OB1 ""
#define PIECE 512
#define WINP 2
#define WAIT_VMX(n) WAIT_VM(n)
__device__ __forceinline__ void st_sys(char* p, u32x4 v) {
    asm volatile("global_store_dwordx4 %0, %1, off sc0 sc1\n\ts_nop 1" :: "v"(p), "v"(v) : "memory");
}
__device__ __forceinline__ void st_sys_s(u32 voff, const void* base, u32x4 v) {
    asm volatile("global_store_dwordx4 %0, %1, %2 sc0 sc1\n\ts_nop 1" :: "v"(voff), "v"(v), "s"(base) : "memory");
}
__device__ __forceinline__ void st_flag(const void* base, u32 v, int lane) {
    if (lane == 0) asm volatile("global_store_dword %0, %1, %2 sc0 sc1" :: "v"(0u), "v"(v), "s"(base) : "memory");
}

#define DBITS " sc0 sc1"
#define UBITS " sc0 sc1"
#define FBITS " sc0 sc1"
#define STR2(x) #x
#define STR(x) STR2(x)
#define CAP 60
#define F2 true
#define ISLP 2
#define GAPS 4
#define GAPF 1
#define DSLP ""
#define LS(i) do {} while (0)
struct Ld {
    u32 nops, ub, part, rmask, lim, nf, pidx, pb, pw;
    u64 fifo;
};
#define GL(o) "global_load_lds_dwordx4 %[vo], vcc offset:" #o DBITS "\n\t"
#define NOPS(x) "s_add_u32 %[nops], %[nops], " x "\n\t"
#define GLU ".if (%c[hf]&1)\n\t" GL(0) DSLP GL(1024) ".else\n\t" GL(0) DSLP GL(1024) DSLP GL(2048) DSLP GL(3072) ".endif\n\t"
#define GLF ".if (%c[hf]&1)\n\t" GL(0) GL(1024) ".else\n\t" GL(0) GL(1024) GL(2048) GL(3072) ".endif\n\t"
#define GLN "s_add_u32 m0, m0, 0x1000\n\ts_add_u32 vcc_lo, vcc_lo, 0x1000\n\ts_addc_u32 vcc_hi, vcc_hi, 0\n\t"
template <bool HALF, bool FAST>
__device__ __forceinline__ u32 load_run(Ld& S, u32 end, u32 glo, u32 ghi, u32 lds, u32 orm, u32 call, const void* uf,
                                        u32 vo, u32 vf, u32 vctl, u32 vfull) {
    u32 a, b, land, rc, keep;
    u64 t;
    asm volatile(
        "s_mov_b32 %[keep], m0\n\t"
        "s_mov_b32 %[rc], 0\n\t"
        "s_or_b32 %[rmask], %[rmask], %[orm]\n\t"
        ".LL%=:\n\t"
        "s_getreg_b32 %[a], hwreg(HW_REG_IB_STS)\n\t"
        "s_bfe_u32 %[b], %[a], 0x20016\n\t"
        "s_and_b32 %[a], %[a], 15\n\t"
        "s_lshl4_add_u32 %[a], %[b], %[a]\n\t"
        "s_sub_u32 %[land], %[nops], %[a]\n\t"
        "s_cmp_eq_u32 %[nf], 0\n\t"
        "s_cbranch_scc1 .LN%=\n\t"
        "s_mov_b64 vcc, %[fifo]\n\t"
        "s_sub_u32 %[a], %[land], vcc_lo\n\t"
        "s_bitcmp1_b32 %[a], 15\n\t"
        "s_cbranch_scc1 .LP%=\n\t"
        "v_add_u32 v41, 1, v41\n\t"
        "s_lshr_b64 %[fifo], %[fifo], 16\n\t"
        "ds_write_b32 %[vfull], v41\n\t"
        "s_sub_u32 %[nf], %[nf], 1\n\t"
        "s_branch .LP%=\n\t"
        ".LN%=:\n\t"
        "s_cmp_eq_u32 %[end], 0\n\t"
        "s_cbranch_scc1 .LX%=\n\t"
        ".LP%=:\n\t"
        "s_sub_u32 %[a], %[land], %[pidx]\n\t"
        "s_cmp_lt_i32 %[a], 0\n\t"
        "s_cbranch_scc1 .LQ%=\n\t"
        "v_cmp_le_u32 vcc, %[call], v40\n\t"
        "s_sub_u32 %[a], %[ub], %[pb]\n\t"
        "s_lshr_b32 %[a], %[a], 1\n\t"
        "s_add_u32 %[pidx], %[nops], 0x40000000\n\t"
        "s_lshr_b32 %[b], vcc_lo, %[a]\n\t"
        "s_lshr_b32 %[b], %[b], %[a]\n\t"
        "s_or_b32 %[rmask], %[rmask], %[b]\n\t"
        ".if (%c[hf]&2)\n\ts_mov_b32 %[pw], " STR(GAPF) "\n\t.else\n\ts_mov_b32 %[pw], " STR(GAPS) "\n\t.endif\n\t"
        ".LQ%=:\n\t"
        "s_min_u32 %[a], %[lim], %[end]\n\t"
        "s_max_u32 %[a], %[a], %[ub]\n\t"
        "s_sub_u32 %[a], %[a], %[ub]\n\t"
        "s_bfm_b32 %[a], %[a], 0\n\t"
        "s_andn2_b32 %[b], %[rmask], %[part]\n\t"
        "s_and_b32 %[b], %[b], %[a]\n\t"
        "s_cbranch_scc1 .LI%=\n\t"
        "s_add_u32 %[rc], %[rc], 0x100000\n\t"
        "s_cbranch_scc1 .LT%=\n\t"
        "s_andn2_b32 %[a], 0xffff, %[rmask]\n\t"
        "s_cbranch_scc0 .LR%=\n\t"
        "s_sub_u32 %[pw], %[pw], 1\n\t"
        "s_cmp_lg_u32 %[pw], 0\n\t"
        "s_cbranch_scc1 .LR%=\n\t"
        "s_lshl_b32 %[b], %[ub], 2\n\t"
        "v_add_u32 v42, %[b], %[vf]\n\t"
        "s_add_u32 %[nops], %[nops], 1\n\t"
        "global_load_dword v40, v42, %[uf] sc0 sc1\n\t"
        "s_mov_b32 %[pidx], %[nops]\n\t"
        "s_mov_b32 %[pb], %[ub]\n\t"
        ".LR%=:\n\t"
        ".if (%c[hf]&2) == 0\n\t"
        "s_sleep " STR(ISLP) "\n\t"
        ".endif\n\t"
        "ds_read_b32 v42, %[vctl] offset:8\n\t"
        "s_waitcnt lgkmcnt(0)\n\t"
        "s_nop 0\n\t"
        "v_min_u32_dpp v42, v42, v42 row_shr:1 row_mask:0xf bank_mask:0xf\n\t"
        "s_nop 1\n\t"
        "v_min_u32_dpp v42, v42, v42 row_shr:2 row_mask:0xf bank_mask:0xf\n\t"
        "s_nop 1\n\t"
        "v_min_u32_dpp v42, v42, v42 row_shr:4 row_mask:0xf bank_mask:0xf\n\t"
        "s_nop 1\n\t"
        "v_readlane_b32 %[a], v42, 7\n\t"
        "s_lshl2_add_u32 %[lim], %[a], 16\n\t"
        "s_branch .LL%=\n\t"
        ".LI%=:\n\t"
        ".if (%c[hf]&2) == 0\n\t"
        "s_sub_u32 %[a], %[nops], %[land]\n\t"
        "s_cmp_gt_u32 %[a], " STR(CAP) "\n\t"
        "s_cbranch_scc1 .LL%=\n\t"
        ".endif\n\t"
        "s_ff1_i32_b32 %[a], %[b]\n\t"
        "s_bitset1_b32 %[part], %[a]\n\t"
        "s_add_u32 %[a], %[a], %[ub]\n\t"
        "s_and_b32 %[land], %[a], 15\n\t"
        "s_lshl_b32 %[land], %[land], 12\n\t"
        "s_add_u32 m0, %[land], %[lds]\n\t"
        "s_and_b32 %[land], %[land], 0x3000\n\t"
        "s_lshr_b32 %[a], %[a], 2\n\t"
        "s_lshl_b32 %[a], %[a], 19\n\t"
        "s_or_b32 %[a], %[a], %[land]\n\t"
        "s_add_u32 vcc_lo, %[glo], %[a]\n\t"
        "s_addc_u32 vcc_hi, %[ghi], 0\n\t"
        ".if (%c[hf]&2)\n\t"
        "s_andn2_b32 %[b], 15, %[b]\n\t"
        "s_cbranch_scc1 .LU%=\n\t"
        GLF GLN GLF GLN GLF GLN
        "s_or_b32 %[part], %[part], 15\n\t"
        NOPS("12>>(%c[hf]&1)")
        ".LU%=:\n\t"
        GLF
        ".else\n\t"
        GLU
        ".endif\n\t"
        NOPS("4>>(%c[hf]&1)")
        ".LS%=:\n\t"
        "s_andn2_b32 %[a], 15, %[part]\n\t"
        "s_cbranch_scc1 .LL%=\n\t"
        "s_and_b32 vcc_lo, %[nops], 0xffff\n\t"
        "s_mov_b32 vcc_hi, 0\n\t"
        "s_lshl_b32 %[a], %[nf], 4\n\t"
        "s_lshl_b64 %[t], vcc, %[a]\n\t"
        "s_or_b64 %[fifo], %[fifo], %[t]\n\t"
        "s_add_u32 %[nf], %[nf], 1\n\t"
        "s_lshr_b32 %[part], %[part], 4\n\t"
        "s_ashr_i32 %[a], %[orm], 31\n\t"
        "s_lshr_b32 %[rmask], %[rmask], 4\n\t"
        "s_or_b32 %[rmask], %[rmask], %[a]\n\t"
        "s_add_u32 %[ub], %[ub], 4\n\t"
        "s_cmp_lt_u32 %[ub], %[end]\n\t"
        "s_cbranch_scc1 .LS%=\n\t"
        ".LX%=:\n\t"
        "s_mov_b32 %[rc], 0\n\t"
        "s_branch .LE%=\n\t"
        ".LT%=:\n\t"
        "s_mov_b32 %[rc], 1\n\t"
        ".LE%=:\n\t"
        "s_mov_b32 m0, %[keep]"
        : [nops] "+s"(S.nops), [ub] "+s"(S.ub), [part] "+s"(S.part), [rmask] "+s"(S.rmask), [lim] "+s"(S.lim), [nf] "+s"(S.nf),
          [pidx] "+s"(S.pidx), [pb] "+s"(S.pb), [pw] "+s"(S.pw), [fifo] "+s"(S.fifo),
          [a] "=&s"(a), [b] "=&s"(b), [land] "=&s"(land), [rc] "=&s"(rc), [keep] "=&s"(keep), [t] "=&s"(t)
        : [end] "s"(end), [glo] "s"(glo), [ghi] "s"(ghi), [lds] "s"(lds), [orm] "s"(orm), [call] "s"(call), [uf] "s"(uf),
          [vo] "v"(vo), [vf] "v"(vf), [vctl] "v"(vctl), [vfull] "v"(vfull), [hf] "i"((HALF ? 1 : 0) + (FAST ? 2 : 0))
        : "memory", "scc", "vcc", "v40", "v41", "v42");
    return rc;
}
#define VPC "v44", "v45", "v46", "v47", "v48", "v49", "v50", "v51", "v52", "v53", "v54", "v55", "v56", "v57", "v58", "v59", \
            "v60", "v61", "v62", "v63", "v64", "v65", "v66", "v67", "v68", "v69", "v70", "v71", "v72", "v73", "v74", "v75"
#define PL(i, g) asm volatile("global_load_dwordx4 v[44+4*" #i ":47+4*" #i "], %0, %1 offset:512*" #i FBITS :: "v"(vo), "s"(g) : "memory", VPC)
#define PW(i, l) asm volatile("ds_write_b128 %0, v[44+4*" #i ":47+4*" #i "] offset:512*" #i :: "v"(l) : "memory", VPC)
template <int N> __device__ __forceinline__ void wait_vm() { asm volatile("s_waitcnt vmcnt(%0)" :: "i"(N) : "memory"); }

__device__ __forceinline__ void st_img(u32 voff, const void* base, u32x4 v) {
    asm volatile("global_store_dwordx4 %0, %1, %2" UBITS "\n\ts_nop 1" :: "v"(voff), "v"(v), "s"(base) : "memory");
}
__device__ __forceinline__ u32 perm(u32 hi, u32 lo, u32 sel) { return __builtin_amdgcn_perm(hi, lo, sel); }
__device__ __forceinline__ u32 bfi(u32 m, u32 a, u32 b) { return __builtin_amdgcn_bitop3_b32(m, a, b, 0xCA); }
__device__ __forceinline__ u32 and_or(u32 a, u32 m, u32 c) { return __builtin_amdgcn_bitop3_b32(a, m, c, 0xEA); }
__device__ __forceinline__ u32 xor_and(u32 a, u32 b, u32 m) { return __builtin_amdgcn_bitop3_b32(a, b, m, 0x78); }
#define BAR() asm volatile("s_waitcnt lgkmcnt(0)\n\ts_barrier" ::: "memory")
template <int S> __device__ __forceinline__ u32 shl(u32 v) { return S >= 0 ? v << (S >= 0 ? S : 0) : v >> (S < 0 ? -S : 0); }
__device__ __forceinline__ u32 scan64(u32 v) {
    v += __builtin_amdgcn_update_dpp(0u, v, 0x111, 0xf, 0xf, true);
    v += __builtin_amdgcn_update_dpp(0u, v, 0x112, 0xf, 0xf, true);
    v += __builtin_amdgcn_update_dpp(0u, v, 0x114, 0xf, 0xf, true);
    v += __builtin_amdgcn_update_dpp(0u, v, 0x118, 0xf, 0xf, true);
    v += __builtin_amdgcn_update_dpp(0u, v, 0x142, 0xa, 0xf, false);
    v += __builtin_amdgcn_update_dpp(0u, v, 0x143, 0xc, 0xf, false);
    return v;
}
__device__ __forceinline__ void lds_or(LAS unsigned char* p, u32 v) {
    __hip_atomic_fetch_or((LAS u32*)p, v, __ATOMIC_RELAXED, __HIP_MEMORY_SCOPE_WORKGROUP);
}

template <bool W> __device__ __forceinline__ void load_unit(u32 (&d)[16], const Params& P, int panel, int r, int lane, bool half) {
    if (!W) {
        const GAS unsigned char* b = (const GAS unsigned char*)P.x + (size_t)panel * 256 * KS;
        b += half ? (r & 1) * 128 * KS + (r & ~1) * 16 : r * 16;
        const u32 hi = half ? 16 : 128 * KS;
#pragma unroll
        for (int i = 0; i < 4; ++i) {
            u32x4 v = *(const GAS u32x4*)(b + (u32)(((i & 1) * 64 + lane) * KS + (i >> 1) * hi));
            d[4 * i] = v[0]; d[4 * i + 1] = v[1]; d[4 * i + 2] = v[2]; d[4 * i + 3] = v[3];
        }
    } else {
        const GAS unsigned char* b = (const GAS unsigned char*)P.w + (size_t)r * 16 * MN + (panel - 16) * 256;
#pragma unroll
        for (int k = 0; k < 16; ++k) d[k] = *(const GAS u32*)(b + (u32)(k * MN + lane * 4));
    }
}
template <bool W> __device__ __forceinline__ void order_unit(u32 (&x)[16], const u32 (&d)[16]) {
    if (!W) {
#pragma unroll
        for (int k = 0; k < 16; ++k) x[k] = d[k];
    } else {
#pragma unroll
        for (int q = 0; q < 4; ++q) {
            u32 t0 = perm(d[4 * q + 1], d[4 * q], 0x05010400), t1 = perm(d[4 * q + 1], d[4 * q], 0x07030602);
            u32 t2 = perm(d[4 * q + 3], d[4 * q + 2], 0x05010400), t3 = perm(d[4 * q + 3], d[4 * q + 2], 0x07030602);
            x[q] = perm(t2, t0, 0x05040100); x[4 + q] = perm(t2, t0, 0x07060302);
            x[8 + q] = perm(t3, t1, 0x05040100); x[12 + q] = perm(t3, t1, 0x07060302);
        }
    }
}
__device__ __forceinline__ u32x4 piece(const u32 (&x)[16], int i) { return u32x4{x[4 * i], x[4 * i + 1], x[4 * i + 2], x[4 * i + 3]}; }

template <int MB> struct Fmt {
    static constexpr u32 LOW = MB == 3 ? 0x3F3F3F3Fu : 0x1F1F1F1Fu;
    static constexpr u32 HM = 0x01010101u << (MB + 2);
    static constexpr u32 K3 = (3u << MB) * 0x01010101u;
    static constexpr u32 HEAD = MB == 3 ? 3072 : 2048;
    static constexpr u32 OFF_F = MB == 3 ? 0 : 512;
    static constexpr u32 OFF_H = OFF_F + 128, OFF_N = OFF_H + 4;
};

template <int MB> __device__ __forceinline__ u32 encode_unit(const u32 (&x)[16], LAS unsigned char* T, char* dst, int lane, bool force_raw) {
    typedef Fmt<MB> F;
    *(LAS u32x4*)(T + lane * 16) = u32x4{0, 0, 0, 0};
    *(LAS u32x4*)(T + 1024 + lane * 16) = u32x4{0, 0, 0, 0};
    u32 sum = 0;
#pragma unroll
    for (int j = 0; j < 16; ++j) sum = __builtin_amdgcn_udot4(x[j] & 0x7F7F7F7Fu, 0x01010101u, sum, false);
    const int tot = __builtin_amdgcn_readlane(scan64(sum), 63);
    const int e0 = (tot - (5038 << MB)) >> (12 + MB);
    const u32 b1 = ((u32)(e0 - 3) << MB) & 0x7Fu;
    const u32 B0 = (((u32)e0 << MB) & 0x7Fu) * 0x01010101u, B1 = b1 * 0x01010101u;
    u32 c[16], m[4], h[4];
#define ENC1(q, jj) { const int j = 4 * q + jj; const u32 xo = x[j] | 0x80808080u; c[j] = xo - B0; \
        const u32 xp = ((((xo - B1) & F::LOW) + B1) ^ x[j]); \
        m[q] = jj == 0 ? (xp >> 5) & 0x03030303u : bfi(0x03030303u << (2 * jj), shl<2 * jj - 5>(xp), m[q]); \
        h[q] = jj == 0 ? (c[j] >> (MB + 2)) & 0x01010101u : bfi(0x01010101u << jj, c[j] >> (MB + 2 - jj), h[q]); }
#define ENC4(q) ENC1(q, 0) ENC1(q, 1) ENC1(q, 2) ENC1(q, 3)
    ENC4(0) ENC4(1) ENC4(2) ENC4(3)
    u32 s[2] = {0, 0}, pos[2] = {0, 0}, flags = 0;
#pragma unroll
    for (int n = 0; n < 16; ++n) {
        const u32 nib = (h[n >> 2] >> (8 * (n & 3))) & 15u, fl = nib < 1u ? nib : 1u;
        s[n >> 3] |= nib << (pos[n >> 3] & 31u);
        pos[n >> 3] += fl * 4;
        flags |= fl << n;
    }
    u32 cnt = 0;
#pragma unroll
    for (int q = 0; q < 4; ++q) cnt += __builtin_popcount((m[q] | (m[q] >> 1)) & 0x55555555u);
    const u32 mine = ((pos[0] + pos[1]) >> 2) | (cnt << 16);
    const u32 incl = scan64(mine), all = __builtin_amdgcn_readlane(incl, 63), excl = incl - mine;
    const u32 off_e = F::OFF_N + (((all & 0xFFFFu) + 7) >> 3) * 4;
    const u32 tail = off_e + 2 * (all >> 16);
    const u32 len64 = (F::HEAD + tail + 63) >> 6;
    if (len64 >= 64 || force_raw) {
#pragma unroll
        for (int i = 0; i < 4; ++i) st_sys_s(lane * 16, dst + i * 1024, piece(x, i));
        return 64;
    }
    FENCE();
    {
        const u64 S = (u64)s[0] | ((u64)s[1] << pos[0]);
        const u32 off = excl & 0xFFFFu, sh = (off & 7) * 4;
        LAS unsigned char* p = T + F::OFF_N + (off >> 3) * 4;
        lds_or(p, (u32)S << sh);
        lds_or(p + 4, (u32)((S << sh) >> 32));
        lds_or(p + 8, (u32)(((S >> 32) << sh) >> 32));
    }
    *(LAS u16*)(T + F::OFF_F + lane * 2) = (u16)flags;
    if (lane == 0) *(LAS u32*)(T + F::OFF_H) = b1 | ((all >> 16) << 8);
    FENCE();
    {
        LAS unsigned char* p = T + off_e + 2 * (excl >> 16);
#pragma unroll
        for (int q = 0; q < 4; ++q) {
            u32 mm = m[q];
            while (mm) {
                const u32 b = __builtin_ctz(mm) & 30u, pat = (mm >> b) & 3u;
                mm ^= pat << b;
                *(LAS u16*)p = (u16)(((lane * 4 + q) << 6) | (b << 1) | pat);
                p += 2;
            }
        }
    }
    if (MB == 3) {
        u32x4 o[3];
#pragma unroll
        for (int q = 0; q < 4; ++q) {
            const u32 g3 = c[4 * q + 3];
            o[0][q] = bfi(0x60606060u, g3 << 5, bfi(0x80808080u, x[4 * q], c[4 * q]));
            o[1][q] = bfi(0x60606060u, g3 << 3, bfi(0x80808080u, x[4 * q + 1], c[4 * q + 1]));
            o[2][q] = bfi(0x60606060u, bfi(0x20202020u, g3 << 1, x[4 * q + 3] >> 1), bfi(0x80808080u, x[4 * q + 2], c[4 * q + 2]));
        }
#pragma unroll
        for (int i = 0; i < 3; ++i) st_sys_s(lane * 16, dst + i * 1024, o[i]);
    } else {
        u32x4 o[2];
        u32x2 sg;
#pragma unroll
        for (int p = 0; p < 8; ++p) o[p >> 2][p & 3] = bfi(0x0F0F0F0Fu, c[2 * p], c[2 * p + 1] << 4);
#pragma unroll
        for (int k = 0; k < 2; ++k) {
            u32 v = x[8 * k];
#pragma unroll
            for (int j = 1; j < 8; ++j) v = bfi(0x80808080u >> j, x[8 * k + j] >> j, v);
            sg[k] = v;
        }
        *(LAS u32x2*)(T + lane * 8) = sg;
#pragma unroll
        for (int i = 0; i < 2; ++i) st_sys_s(lane * 16, dst + i * 1024, o[i]);
    }
    FENCE();
    const u32 tl = len64 * 64 - F::HEAD;
    if (lane * 16 < tl) st_sys_s(lane * 16, dst + F::HEAD, *(LAS u32x4*)(T + lane * 16));
    if (MB == 2 && 1024 + lane * 16 < tl) st_sys_s(lane * 16, dst + F::HEAD + 1024, *(LAS u32x4*)(T + 1024 + lane * 16));
    return len64;
}

template <int MB> __device__ __forceinline__ void decode_unit(u32x4 (&e)[4], LAS unsigned char* S, LAS unsigned char* L, int lane) {
    typedef Fmt<MB> F;
    LAS unsigned char* T = S + F::HEAD;
    *(LAS u32x4*)(L + LDS_XB + lane * 16) = u32x4{0, 0, 0, 0};
    const u32 flags = *(LAS u16*)(T + F::OFF_F + lane * 2);
    const u32 hdr = UNI(*(LAS u32*)(T + F::OFF_H));
    const u32x2 l0 = *(LAS u32x2*)(L + LDS_LUT + (flags & 255u) * 8), l1 = *(LAS u32x2*)(L + LDS_LUT + (flags >> 8) * 8);
    const u32 n = __builtin_popcount(flags), incl = scan64(n), off = incl - n;
    const u32 off_e = F::OFF_N + ((__builtin_amdgcn_readlane(incl, 63) + 7) >> 3) * 4;
    LAS u32* w = (LAS u32*)(T + F::OFF_N + (off >> 3) * 4);
    const u32 w0 = w[0], w1 = w[1], w2 = w[2];
    FENCE();
    const u32 nexc = (hdr >> 8) & 1023u;
    for (u32 i = lane; i < nexc; i += 64) {
        const u32 en = *(LAS u16*)(T + off_e + 2 * i);
        lds_or(L + LDS_XB + ((en >> 6) & 255u) * 4, (en & 3u) << ((en >> 1) & 30u));
    }
    FENCE();
    const u32x4 m = *(LAS u32x4*)(L + LDS_XB + lane * 16);
    const u32 sh = (off & 7) * 4;
    const u32 s0 = __builtin_amdgcn_alignbit(w1, w0, sh), shi = __builtin_amdgcn_alignbit(w2, w1, sh);
    const u32 s1 = (u32)((((u64)shi << 32) | s0) >> (4 * __builtin_popcount(flags & 255u)));
    u32 h[4];
    {
        const u32 a = s0 & 0x0F0F0F0Fu, b = (s0 >> 4) & 0x0F0F0F0Fu, c = s1 & 0x0F0F0F0Fu, d = (s1 >> 4) & 0x0F0F0F0Fu;
        h[0] = perm(b, a, l0[0]); h[1] = perm(b, a, l0[1]); h[2] = perm(d, c, l1[0]); h[3] = perm(d, c, l1[1]);
    }
    const u32 B1 = (hdr & 0x7Fu) * 0x01010101u;
    u32 lo[16], sg[16];
    if (MB == 3) {
        const u32x4 o0 = *(LAS u32x4*)(S + lane * 16), o1 = *(LAS u32x4*)(S + 1024 + lane * 16), o2 = *(LAS u32x4*)(S + 2048 + lane * 16);
#pragma unroll
        for (int q = 0; q < 4; ++q) {
            lo[4 * q] = o0[q]; sg[4 * q] = o0[q];
            lo[4 * q + 1] = o1[q]; sg[4 * q + 1] = o1[q];
            lo[4 * q + 2] = o2[q]; sg[4 * q + 2] = o2[q];
            lo[4 * q + 3] = bfi(0x03030303u, o0[q] >> 5, bfi(0x0C0C0C0Cu, o1[q] >> 3, o2[q] >> 1));
            sg[4 * q + 3] = o2[q] << 1;
        }
    } else {
        const u32x4 o0 = *(LAS u32x4*)(S + lane * 16), o1 = *(LAS u32x4*)(S + 1024 + lane * 16);
        const u32x2 s2 = *(LAS u32x2*)(T + lane * 8);
#pragma unroll
        for (int p = 0; p < 8; ++p) {
            const u32 v = p < 4 ? o0[p & 3] : o1[p & 3];
            lo[2 * p] = v; lo[2 * p + 1] = v >> 4;
        }
#pragma unroll
        for (int j = 0; j < 16; ++j) sg[j] = s2[j >> 3] << (j & 7);
    }
#define DEC1(q, jj) { const int j = 4 * q + jj; \
        const u32 z = ((and_or(lo[j], F::LOW ^ F::HM, (h[q] << (MB + 2 - jj)) & F::HM) + F::K3) & F::LOW) + B1; \
        e[q][jj] = xor_and(bfi(0x7F7F7F7Fu, z, sg[j]), shl<5 - 2 * jj>(m[q]), 0x60606060u); }
#define DEC4(q) DEC1(q, 0) DEC1(q, 1) DEC1(q, 2) DEC1(q, 3)
    DEC4(0) DEC4(1) DEC4(2) DEC4(3)
}

#define TS(t)
#define TC(t)
#define V10(a) "v" #a "0", "v" #a "1", "v" #a "2", "v" #a "3", "v" #a "4", "v" #a "5", "v" #a "6", "v" #a "7", "v" #a "8", "v" #a "9"
__device__ __forceinline__ void mfma_role(u32 sa, u32 sb, u32 sc, u32 olo, u32 ohi, u32 scale,
                                          u64& ta, u64& tb, u64& tc, u64& td, u64& ca, u64& cb) {
    u32 need, m, spin, scl;
    asm volatile(
        ".macro AG_FRAG d, a, o\n\t"
        "ds_read_b128 v[\\d:\\d+3], \\a offset:\\o\n\t"
        "ds_read_b128 v[\\d+4:\\d+7], \\a offset:\\o+4096\n\t"
        ".endm\n\t"
        ".macro AG_MM c, a, b\n\t"
        "v_mfma_f32_32x32x64_f8f6f4 v[\\c:\\c+15], v[\\a:\\a+7], v[\\b:\\b+7], v[\\c:\\c+15] cbsz:0 blgp:1\n\t"
        ".endm\n\t"
        ".macro AG_MIN\n\t"
        "v_min_u32 v164, v164, v165\n\t"
        "s_nop 0\n\t"
        "v_readfirstlane_b32 %[m], v164\n\t"
        ".endm\n\t"
        ".macro AG_SPIN\n\t"
        "s_mov_b32 %[spin], 0\n\t"
        ".LAG_S\\@:\n\t"
        "ds_read_b64 v[164:165], v162\n\t"
        "s_add_u32 %[spin], %[spin], 1\n\t"
        "s_waitcnt lgkmcnt(0)\n\t"
        "AG_MIN\n\t"
        "s_cmp_ge_u32 %[m], %[need]\n\t"
        "s_cbranch_scc1 .LAG_E\\@\n\t"
        "s_sleep " STR(MSLEEP) "\n\t"
        "s_cmp_lt_u32 %[spin], 0x2000000\n\t"
        "s_cbranch_scc1 .LAG_S\\@\n\t"
        ".LAG_E\\@:\n\t"
        ".endm\n\t"
        ".macro AG_DONE\n\t"
        "v_add_u32 v163, 1, v163\n\t"
        "ds_write_b32 v162, v163 offset:8\n\t"
        ".endm\n\t"
        ".macro AG_FULL so, sn\n\t"
        "s_waitcnt lgkmcnt(4)\n\t"
        "AG_MM 0, 128, 144\n\t"
        "s_waitcnt lgkmcnt(2)\n\t"
        "AG_MM 64, 136, 144\n\t"
        "ds_read_b64 v[164:165], v162\n\t"
        "AG_FRAG 144, v161, \\so+2048\n\t"
        "s_add_u32 %[need], %[need], 1\n\t"
        "s_waitcnt lgkmcnt(3)\n\t"
        "AG_MM 16, 128, 152\n\t"
        "AG_MM 80, 136, 152\n\t"
        "AG_FRAG 152, v161, \\so+3072\n\t"
        "s_waitcnt lgkmcnt(2)\n\t"
        "AG_MM 32, 128, 144\n\t"
        "AG_MIN\n\t"
        "AG_MM 96, 136, 144\n\t"
        "s_cmp_ge_u32 %[m], %[need]\n\t"
        "s_cbranch_scc1 .LAG_F\\@\n\t"
        "s_waitcnt lgkmcnt(0)\n\t"
        "AG_MM 48, 128, 152\n\t"
        "AG_DONE\n\t"
        "AG_MM 112, 136, 152\n\t"
        "AG_SPIN\n\t"
        "AG_FRAG 144, v161, \\sn\n\t"
        "AG_FRAG 128, v160, \\sn\n\t"
        "s_branch .LAG_J\\@\n\t"
        ".LAG_F\\@:\n\t"
        "AG_FRAG 144, v161, \\sn\n\t"
        "s_waitcnt lgkmcnt(2)\n\t"
        "AG_MM 48, 128, 152\n\t"
        "AG_DONE\n\t"
        "AG_FRAG 128, v160, \\sn\n\t"
        "AG_MM 112, 136, 152\n\t"
        ".LAG_J\\@:\n\t"
        "AG_FRAG 136, v160, \\sn+2048\n\t"
        "AG_FRAG 152, v161, \\sn+1024\n\t"
        ".endm\n\t"
        ".macro AG_PREP c, b\n\t"
        ".irp j, 0, 16, 32, 48\n\t"
        ".irp i, 0, 1, 2, 3\n\t"
        "v_mul_f32 v[\\c+\\j+4*\\b+\\i], %[scl], v[\\c+\\j+4*\\b+\\i]\n\t"
        ".endr\n\t"
        ".endr\n\t"
        "v_swap_b32 v[\\c+4*\\b+1], v[\\c+16+4*\\b]\n\t"
        "v_swap_b32 v[\\c+4*\\b+2], v[\\c+32+4*\\b]\n\t"
        "v_swap_b32 v[\\c+4*\\b+3], v[\\c+48+4*\\b]\n\t"
        "v_swap_b32 v[\\c+16+4*\\b+2], v[\\c+32+4*\\b+1]\n\t"
        "v_swap_b32 v[\\c+16+4*\\b+3], v[\\c+48+4*\\b+1]\n\t"
        "v_swap_b32 v[\\c+32+4*\\b+3], v[\\c+48+4*\\b+2]\n\t"
        ".endm\n\t"
        ".macro AG_ST c, r\n\t"
        ".if (\\r) >= 0\n\t"
        ".if ((\\r) & 3) == 0\n\t"
        "AG_PREP \\c, (\\r)/4\n\t"
        ".endif\n\t"
        ".if \\c\n\t"
        "global_store_dwordx4 v166, v[\\c+16*((\\r)&3)+((\\r)&12):\\c+16*((\\r)&3)+((\\r)&12)+3], vcc" OB1 "\n\t"
        ".else\n\t"
        "global_store_dwordx4 v166, v[\\c+16*((\\r)&3)+((\\r)&12):\\c+16*((\\r)&3)+((\\r)&12)+3], vcc" OB0 "\n\t"
        ".endif\n\t"
        ".if ((\\r) & 3) == 3\n\t"
        "s_add_u32 vcc_lo, vcc_lo, 5*16384\n\t"
        ".else\n\t"
        "s_add_u32 vcc_lo, vcc_lo, 16384\n\t"
        ".endif\n\t"
        "s_addc_u32 vcc_hi, vcc_hi, 0\n\t"
        ".endif\n\t"
        ".endm\n\t"
        ".macro AG_HALF c, q, ao, last, r0, r1\n\t"
        "s_waitcnt lgkmcnt(2)\n\t"
        "AG_MM \\c, 128+8*((\\q)&1), 144\n\t"
        "ds_read_b64 v[164:165], v162\n\t"
        "AG_FRAG 144, v161, ((\\q)&3)*16384+2048\n\t"
        "s_add_u32 %[need], %[need], 1\n\t"
        "s_waitcnt lgkmcnt(3)\n\t"
        "AG_MM \\c+16, 128+8*((\\q)&1), 152\n\t"
        "AG_FRAG 152, v161, ((\\q)&3)*16384+3072\n\t"
        "AG_ST 0, \\r0\n\t"
        "s_waitcnt lgkmcnt(2)\n\t"
        "AG_MM \\c+32, 128+8*((\\q)&1), 144\n\t"
        ".if \\last\n\t"
        "s_waitcnt lgkmcnt(0)\n\t"
        "AG_MM \\c+48, 128+8*((\\q)&1), 152\n\t"
        ".else\n\t"
        "AG_MIN\n\t"
        "s_cmp_ge_u32 %[m], %[need]\n\t"
        "s_cbranch_scc1 .LAG_F\\@\n\t"
        "s_waitcnt lgkmcnt(0)\n\t"
        "AG_MM \\c+48, 128+8*((\\q)&1), 152\n\t"
        "AG_DONE\n\t"
        "AG_SPIN\n\t"
        "AG_FRAG 136-8*((\\q)&1), v160, (((\\q)+1)&3)*16384+\\ao\n\t"
        "AG_FRAG 144, v161, (((\\q)+1)&3)*16384\n\t"
        "s_branch .LAG_J\\@\n\t"
        ".LAG_F\\@:\n\t"
        "AG_FRAG 136-8*((\\q)&1), v160, (((\\q)+1)&3)*16384+\\ao\n\t"
        "AG_FRAG 144, v161, (((\\q)+1)&3)*16384\n\t"
        "s_waitcnt lgkmcnt(4)\n\t"
        "AG_MM \\c+48, 128+8*((\\q)&1), 152\n\t"
        "AG_DONE\n\t"
        ".LAG_J\\@:\n\t"
        "AG_FRAG 152, v161, (((\\q)+1)&3)*16384+1024\n\t"
        ".endif\n\t"
        "AG_ST 0, \\r1\n\t"
        ".endm\n\t"
        ".macro AG_SCALE\n\t"
        "ds_read_b32 v167, v162 offset:12\n\t"
        "s_nop 15\n\t"
        "s_nop 7\n\t"
        "s_waitcnt lgkmcnt(0)\n\t"
        "v_readfirstlane_b32 %[m], v167\n\t"
        "s_cmp_eq_u32 %[m], 0\n\t"
        "s_cselect_b32 %[scl], %[scale], 0x7fc00000\n\t"
        ".endm\n\t"

        "v_mbcnt_lo_u32_b32 v167, -1, 0\n\t"
        "v_mbcnt_hi_u32_b32 v167, -1, v167\n\t"
        "v_lshrrev_b32 v165, 5, v167\n\t"
        "v_and_b32 v167, 31, v167\n\t"
        "v_lshlrev_b32 v167, 4, v167\n\t"
        "v_lshl_add_u32 v166, v165, 16, v167\n\t"
        "v_lshl_add_u32 v167, v165, 13, v167\n\t"
        "v_add_u32 v160, %[sa], v167\n\t"
        "v_add_u32 v161, %[sb], v167\n\t"
        "v_mov_b32 v162, %[sc]\n\t"
        "v_mov_b32 v163, 0\n\t"
        ".set AG_I, 0\n\t"
        ".rept 128\n\t"
        "v_mov_b32 v[AG_I], 0\n\t"
        ".set AG_I, AG_I+1\n\t"
        ".endr\n\t"
        "s_mov_b32 vcc_lo, %[olo]\n\t"
        "s_mov_b32 vcc_hi, %[ohi]\n\t"
        "s_mov_b32 %[need], 1\n\t"
        "AG_SPIN\n\t"
        TS(ta) TC(ca)
        "AG_FRAG 144, v161, 0\n\t"
        "AG_FRAG 128, v160, 0\n\t"
        "AG_FRAG 136, v160, 2048\n\t"
        "AG_FRAG 152, v161, 1024\n\t"
        ".LAG_LOOP:\n\t"
        "AG_FULL 0, 16384\n\t"
        "AG_FULL 16384, 32768\n\t"
        "AG_FULL 32768, 49152\n\t"
        "AG_FULL 49152, 0\n\t"
        "s_cmp_le_u32 %[need], 64-" STR(NSPLIT) "\n\t"
        "s_cbranch_scc1 .LAG_LOOP\n\t"
        TS(td)
        ".if " STR(NSPLIT) "\n\t"
        ".set AG_I, 0\n\t"
        ".rept " STR(NSPLIT) "-1\n\t"
        "AG_HALF 0, AG_I, 0, 0, -1, -1\n\t"
        ".set AG_I, AG_I+1\n\t"
        ".endr\n\t"
        "AG_HALF 0, AG_I, 2048, 0, -1, -1\n\t"
        "AG_SCALE\n\t"
        TS(tb)
        ".set AG_I, 0\n\t"
        ".rept " STR(NSPLIT) "\n\t"
        ".set AG_R0, (AG_I*16+" STR(NSPLIT) "-1)/" STR(NSPLIT) "\n\t"
        ".set AG_R1, (AG_I*16+16+" STR(NSPLIT) "-1)/" STR(NSPLIT) "-1\n\t"
        ".if AG_R1 == AG_R0\n\t"
        ".set AG_R1, -1\n\t"
        ".endif\n\t"
        "AG_HALF 64, AG_I, 2048, (AG_I==" STR(NSPLIT) "-1), AG_R0, AG_R1\n\t"
        ".set AG_I, AG_I+1\n\t"
        ".endr\n\t"
        "AG_SCALE\n\t"
        TS(tc) TC(cb)
        ".else\n\t"
        "AG_SCALE\n\t"
        TS(tc) TC(cb)
        ".irp r, 0, 1, 2, 3, 4, 5, 6, 7, 8, 9, 10, 11, 12, 13, 14, 15\n\t"
        "AG_ST 0, \\r\n\t"
        ".endr\n\t"
        ".endif\n\t"
        "s_add_u32 vcc_lo, vcc_lo, 96*16384\n\t"
        "s_addc_u32 vcc_hi, vcc_hi, 0\n\t"
        ".irp r, 0, 1, 2, 3, 4, 5, 6, 7, 8, 9, 10, 11, 12, 13, 14, 15\n\t"
        "AG_ST 64, \\r\n\t"
        ".endr\n\t"
        "s_nop 1\n\t"
        : [need] "=&s"(need), [m] "=&s"(m), [spin] "=&s"(spin), [scl] "=&s"(scl)
        : [sa] "s"(sa), [sb] "s"(sb), [sc] "s"(sc), [olo] "s"(olo), [ohi] "s"(ohi), [scale] "s"(scale)
        : "memory", "scc", "vcc", V10(), V10(1), V10(2), V10(3), V10(4), V10(5), V10(6), V10(7), V10(8), V10(9),
          V10(10), V10(11), V10(12), V10(13), V10(14), V10(15),
          "v160", "v161", "v162", "v163", "v164", "v165", "v166", "v167");
}

__device__ __forceinline__ void arrive(LAS unsigned char* L, int lane) {
    if (lane == 0) __hip_atomic_fetch_add((LAS u32*)(L + LDS_PRO), 1u, __ATOMIC_RELAXED, __HIP_MEMORY_SCOPE_WORKGROUP);
}

template <bool W> __device__ __forceinline__ void prologue_unit(const Params& P, LAS unsigned char* L, char* myws, u32 callv,
                                                                 int idx, int xg, int wave, int lane) {
    u32 d[16], x[16];
    const int u = xg * 4 + wave;
    const bool half = !W && u >= (int)R0;
    load_unit<W>(d, P, idx, u, lane, half);
    BAR();
    WAIT_VM0();
    order_unit<W>(x, d);
    const u32 par = (UNI(callv) + 1) & 1;
    bool raw = RAWTAIL == 1 ? u == (int)ROUNDS - 1 && (!W || !NSPLIT)
             : RAWTAIL >= 2 && (W ? u == (int)ROUNDS - 1 : RAWTAIL == 3 && u == (int)ROUNDS - 2);
    const char* qa = myws + WS_IMG + par * IMG_BYTES + unit_off(idx, half ? u & ~1 : u) + (half ? (u & 1) * 2048 : 0);
    const char* qb = qa + (half ? 4096 : 2048);
    st_sys_s(lane * 16, qa, piece(x, 0)); st_sys_s(lane * 16, qa + 1024, piece(x, 1));
    st_sys_s(lane * 16, qb, piece(x, 2)); st_sys_s(lane * 16, qb + 1024, piece(x, 3));
    u32 len64 = 64;
    if (u >= NPUSH) len64 = encode_unit<W ? 2 : 3>(x, L + wave * 2048, myws + WS_CB + par * CB_BYTES + unit_off(idx, u), lane, raw);
    if (lane == 0) *(volatile LAS u32*)(L + LDS_LEN + wave * 4) = len64;
    WAIT_VM0();
    arrive(L, lane);
}

__global__ void __launch_bounds__(NTHREADS, 3) ag_gemm(Params P) {
    __shared__ __attribute__((aligned(16))) unsigned char lds_raw[LDS_BYTES];
    LAS unsigned char* L = (LAS unsigned char*)lds_raw;
    const int hw = UNI(threadIdx.x >> 6);
    const int wave = hw < 4 ? 11 - hw : hw - 4;
    const int lane = threadIdx.x & 63;
    const int bid = blockIdx.x;
    const int rank = P.rank;
    char* myws = P.myws;
    const int xg = bid & 7, idx = bid >> 3;
    const int tm = (xg >> 1) * 4 + (idx >> 3), tn = (xg & 1) * 8 + (idx & 7);
    GAS u32* tmo = (GAS u32*)(myws + WS_TMO);

    u32 call = __hip_atomic_load((GAS u32*)(myws + WS_CNT + bid * 128), __ATOMIC_RELAXED, __HIP_MEMORY_SCOPE_AGENT);

    if (wave < 8) {
        if (wave >= 4) {
            BAR();
            const u32 f = (wave - 4) * 64 + lane;
            u32x2 sel = {0, 0};
#pragma unroll
            for (int n = 0; n < 8; ++n) {
                const u32 i = __builtin_popcount(f & ((1u << n) - 1));
                sel[n >> 2] |= (((f >> n) & 1) ? (i >> 1) + 4 * (i & 1) : 12u) << (8 * (n & 3));
            }
            *(LAS u32x2*)(L + LDS_LUT + f * 8) = sel;
            FENCE();
            if (lane == 0) __hip_atomic_fetch_add((LAS u32*)(L + LDS_LUTF), 1u, __ATOMIC_RELAXED, __HIP_MEMORY_SCOPE_WORKGROUP);
        } else if (idx < 16) prologue_unit<false>(P, L, myws, call, idx, xg, wave, lane);
        else prologue_unit<true>(P, L, myws, call, idx, xg, wave, lane);
        const u32 scb = UNI(__builtin_bit_cast(u32, P.sx[0] * P.sw[0]));
        const int g = wave >> 1, nh = wave & 1;
        const u64 ob = (u64)P.out + ((size_t)(tm * 256 + g * 32) * MN + tn * 256 + nh * 128) * 4;
        u32 olo = (u32)ob, ohi = (u32)(ob >> 32);
        u64 ta = 0, tb = 0, tc = 0, td = 0, ca = 0, cb = 0;
        mfma_role((u32)(size_t)L + LDS_A + g * 512, (u32)(size_t)L + LDS_B + nh * 512, (u32)(size_t)L + LDS_CTL + wave * 16,
                  olo, ohi, scb, ta, tb, tc, td, ca, cb);
    } else if (wave < 10) {
        __builtin_amdgcn_s_setprio(3);
        if (wave == 8) {
            ((LAS u32*)(L + LDS_CTL))[lane] = 0;
            char* peer = P.ws[lane & 7];
            BAR();
            call = UNI(call) + 1;
            STAMP(0);
            WAIT_VM0();
            for (u32 spin = 0; spin < SPINS && UNI(*(volatile LAS u32*)(L + LDS_PRO)) < 7; ++spin) __builtin_amdgcn_s_sleep(1);
            FENCE();
            const u32x4 ln = *(LAS u32x4*)(L + LDS_LEN);
            if (lane < NR) {
                if (lane != rank) st_sys(peer + WS_RDY + (call & 1) * 32768 + (rank * 32 + idx) * 128 + xg * 16, ln | (call << 7));
                else st_sys(peer + WS_UF + idx * 2048 + xg * 16, u32x4{call, call, call, call});
            }
            if (lane == 0) __hip_atomic_store((GAS u32*)(myws + WS_CNT + bid * 128), call, __ATOMIC_RELAXED, __HIP_MEMORY_SCOPE_AGENT);
            STAMP(7);
        } else {
            BAR();
            call = UNI(call) + 1;
            arrive(L, lane);
            u32 job = XCD_ID() * 32 + idx;
#pragma unroll 1
            for (int k = 0; k < NWG; ++k, job = (job + 1) & (NWG - 1)) {
                u32 old = 0;
                if (lane == 0) {
                    __hip_atomic_fetch_max((GAS u32*)(myws + WS_TIK + (job >> 5) * 128), call * 2048, __ATOMIC_RELAXED, __HIP_MEMORY_SCOPE_AGENT);
                    FENCE();
                    old = __hip_atomic_fetch_max((GAS u32*)(myws + WS_JOB + job * 128), call, __ATOMIC_RELAXED, __HIP_MEMORY_SCOPE_AGENT);
                }
                WAIT_VM0();
                if (UNI(old) < call) break;
            }
            *(volatile LAS u32*)(L + LDS_JOB) = job + 1;
        }
        const int isB = wave - 8;
        const int panel = isB ? 16 + tn : tm;
        const char* img = myws + WS_IMG + (call & 1) * IMG_BYTES + unit_off(panel, 0);
        const u32 ldsb = (u32)(size_t)L + (isB ? LDS_B : LDS_A);
        volatile LAS u32* ctl = (volatile LAS u32*)(L + LDS_CTL + (lane & 7) * 16);
        const char* uf = myws + WS_UF + panel * 2048;
        const u32 voff = lane * 16;
        Ld S = {0, 0, 0, 0, 16, 0, 0, 0, 0, 0};
        u32 err = 0;
        const u32 vctl = (u32)(size_t)ctl, vfull = vctl + isB * 4, vf = (lane & 31) * 4;
        const u64 t0 = rt();
        WAIT_VM0();
        asm volatile("v_mov_b32 v40, 0\n\tv_mov_b32 v41, 0" ::: "v40", "v41");
        auto late = [&]() {
            if (UNI(__hip_atomic_load(tmo, __ATOMIC_RELAXED, __HIP_MEMORY_SCOPE_AGENT)) == call || rt() - t0 > DEADLINE) {
                if (lane == 0) __hip_atomic_store(tmo, call, __ATOMIC_RELAXED, __HIP_MEMORY_SCOPE_AGENT);
                ctl[3] = 1;
                err = ~0u;
            }
            asm volatile("s_waitcnt vmcnt(0) lgkmcnt(0)" ::: "memory");
            return rt() - t0 < 3 * DEADLINE;
        };
#define RUN(H, F, end, g, l, orm) do { const u64 g_ = (u64)(g); u32 o_ = (orm); \
            while (load_run<H, F>(S, end, (u32)g_, (u32)(g_ >> 32), l, o_ | err, call, uf, voff, vf, vctl, vfull) && late()) o_ &= (int)o_ >> 31; } while (0)
        RUN(false, false, 4 * SLOWEND, img, ldsb, 0);
        LS(9 + isB);
        if (isB) {
            RUN(false, F2, 4 * NSTAGE, img, ldsb, 0);
            LS(14);
            if (NSPLIT) RUN(false, true, 4 * NQ, img - (size_t)NSPLIT * 524288, ldsb, ~0u);
        } else {
            if (SLOWEND < S0) RUN(false, F2, 4 * S0, img, ldsb, 0);
            if (NSPLIT) {
                RUN(true, F2, 4 * NSTAGE, img, ldsb, 0);
                LS(15);
                RUN(true, true, 4 * NQ, img - (size_t)NSPLIT * 524288 + 2048, ldsb + 2048, (1u << (US - 4 * S0)) - 1);
                LS(12);
            }
        }
        RUN(false, false, 0, img, ldsb, 0);
        ctl[isB] = 0xFFFFu;
        WAIT_VM(0);
        STAMP(3 + isB);
    } else {
        __builtin_amdgcn_s_setprio(3);
        volatile LAS u32* fet = (volatile LAS u32*)(L + LDS_FET);
        volatile LAS u32* con = (volatile LAS u32*)(L + LDS_CON);
        volatile LAS u32* meta = (volatile LAS u32*)(L + LDS_META);
        constexpr u32 FIN = 0x40000000u;
        const u32 vo = lane * 16;
        if (wave == 10) {
#define FS(i) do {} while (0)
            BAR();
            call = UNI(call) + 1;
            arrive(L, lane);
            u32 job = 0;
            for (u32 spin = 0; spin < SPINS && job == 0; ++spin) job = UNI(*(volatile LAS u32*)(L + LDS_JOB));
            job = (job - 1) & (NWG - 1);
            const u32 slot = job >> 5, fp = job & 31, src = (P.srcmap >> (4 * slot)) & 7u;
            if (src == (u32)rank || (P.pad & 1)) return;
            GAS u32* tik = (GAS u32*)(myws + WS_TIK + slot * 128);
            const char* scb = P.ws[src] + WS_CB + (call & 1) * CB_BYTES;
            const char* rdy = myws + WS_RDY + (call & 1) * 32768 + src * 4096;
            const u32 lr = (u32)(size_t)L + LDS_RING;
            auto unit_of = [](u32 t) -> u32 {
                auto px = [](u32 k) { return 4 * (k & 3) + (k >> 2); };
                auto pw = [](u32 k) { return 16 + 8 * (k & 1) + (k >> 1); };
                if (t < T1) return ((t & 1) ? pw((t >> 1) & 15) : px((t >> 1) & 15)) * 32 + NPUSH + (t >> 5);
                if (t < T2) {
                    const u32 pr = (t - T1) / 48, i = (t - T1) % 48;
                    return ((i & 16) ? px(i & 15) : pw(i & 15)) * 32 + R0 + 2 * pr + (i >> 5);
                }
                return t < NT ? px((t - T2) & 15) * 32 + R0 + 2 * ((t - T2) >> 4) + 1 : 0u;
            };
            u32 t = fp, k = 0, un = unit_of(t), nv = 0, tv = call * 2048 + fp - 32;
            const u64 t0 = rt();
            constexpr int W = WN - 1;
            u32 lp = lr + 3 * UNIT + vo;
            for (; t < NT; ++k) {
                const char* q = scb + unit_off(un >> 5, un & 31);
                const u32 l = lr + (k & 3) * UNIT + vo;
                wait_vm<W>();
                asm volatile("" : "+v"(nv) :: "memory");
                u32 rv = UNI(nv);
                if (k == 0) FS(3);
                if ((rv >> 7) < call) {
                    if (k) { WAIT_VM(0); if (lane < 32) { PW(2, lp); PW(3, lp); PW(4, lp); PW(5, lp); PW(6, lp); PW(7, lp); } *fet = k; }
                    for (u32 n = 1;; ++n) {
                        rv = UNI(__hip_atomic_load((GAS u32*)(rdy + un * 4), __ATOMIC_RELAXED, __HIP_MEMORY_SCOPE_SYSTEM));
                        WAIT_VM(0);
                        if ((rv >> 7) >= call) break;
                        if ((n & 31) == 0 && (UNI(__hip_atomic_load(tmo, __ATOMIC_RELAXED, __HIP_MEMORY_SCOPE_AGENT)) == call || rt() - t0 > DEADLINE)) {
                            if (lane == 0) __hip_atomic_store(tmo, call, __ATOMIC_RELAXED, __HIP_MEMORY_SCOPE_AGENT);
                            *fet = k | FIN;
                            return;
                        }
                        __builtin_amdgcn_s_sleep(2);
                    }
                }
                if (k == 0) FS(4);
                if (k == 1) FS(5);
                if (k == 2) FS(6);
                if (k == 6) FS(7);
                u32 len64 = rv & 127u;
                len64 = len64 < 41 ? 41 : (len64 > 64 ? 64 : len64);
                const u32 len = len64 * 64, mt = un | ((len64 >> 6) << 10);
                if (lane < 32) { PL(0, q); PW(6, lp); PW(7, lp); if (W < 3) PW(3, lp); else PW(2, lp); }
                wait_vm<W>();
                if (lane < 32) { PL(1, q); if (W < 3) PW(4, lp); else PW(3, lp); }
                wait_vm<W>();
                if (lane < 32) { PL(2, q); if (W < 3) PW(5, lp); else PW(4, lp); }
                if (W < 3) *fet = k;
                asm volatile("" : "+v"(tv) :: "memory");
                t = UNI(tv) - call * 2048 + 64;
                un = unit_of(t);
                const bool last = t >= NT;
                if (W == 3) { wait_vm<W>(); if (lane < 32) PW(5, lp); *fet = k; }
                if (len > 3072) {
                    if (!last) wait_vm<W>();
                    if (lane < 32 && 3072 + vo < len) PL(6, q);
                    if (len > 3584) {
                        if (!last) wait_vm<W>();
                        if (lane < 32 && 3584 + vo < len) PL(7, q);
                    }
                }
                if (k >= 4)
                    for (u32 spin = 0; spin < SPINS && UNI(*con) + 4 <= k; ++spin) __builtin_amdgcn_s_sleep(1);
                meta[k & 3] = mt;
                if (!last) wait_vm<W>();
                {
                    u64 ex;
                    asm volatile("s_mov_b64 %2, exec\n\ts_mov_b64 exec, 1\n\tglobal_load_dword %1, %3, %6 sc0 sc1\n\t"
                                 "global_atomic_add %0, %3, %4, %5 sc0\n\ts_mov_b64 exec, %2"
                                 : "=&v"(tv), "=&v"(nv), "=&s"(ex) : "v"(0u), "v"(1u), "s"(tik), "s"(rdy + un * 4) : "memory");
                }
                if (lane < 32) { PL(3, q); if (W < 3) PW(0, l); }
                if (!last) wait_vm<W + 2>();
                if (lane < 32) { PL(4, q); if (W < 3) PW(1, l); else PW(0, l); }
                if (!last) wait_vm<(W < 2 ? W : W + 2)>();
                if (lane < 32 && 2560 + vo < len) PL(5, q);
                if (lane < 32) { if (W < 3) PW(2, l); else PW(1, l); }
                lp = l;
            }
            asm volatile("s_waitcnt vmcnt(0)" : "+v"(nv), "+v"(tv) :: "memory");
            if (lane < 32) { PW(0, lp); PW(1, lp); PW(2, lp); PW(3, lp); PW(4, lp); PW(5, lp); PW(6, lp); PW(7, lp); }
            *fet = k | FIN;
            STAMP(5);
        } else {
            {
                u32 d[16], x[16];
                const u32 pdst = (P.srcmap >> (4 * xg)) & 7u;
                const bool push = NPUSH && pdst != (u32)rank && !(P.pad & 1);
                if (push) { if (idx < 16) load_unit<false>(d, P, idx, 0, lane, false); else load_unit<true>(d, P, idx, 0, lane, false); }
                const u32 up = 32 + ((rank - pdst - 1) & 7);
                const char* pw = P.ws[pdst];
                const char* q0 = pw + WS_IMG + unit_off(idx, up);
                asm volatile("" : "+s"(q0));
                BAR();
                WAIT_VM0();
                call = UNI(call) + 1;
                if (push) {
                    if (idx < 16) order_unit<false>(x, d); else order_unit<true>(x, d);
                    q0 += (call & 1) * IMG_BYTES;
                    STAMP2(20);
                    st_sys_s(vo, q0, piece(x, 0)); st_sys_s(vo, q0 + 1024, piece(x, 1));
                    arrive(L, lane);
                    WAIT_VM(1); st_sys_s(vo, q0 + 2048, piece(x, 2));
                    WAIT_VM(1); st_sys_s(vo, q0 + 3072, piece(x, 3));
                    WAIT_VM(0); st_flag(pw + WS_UF + idx * 2048 + up * 4, call, lane);
                    STAMP(22);
                    STAMP3(20);
                } else {
                    arrive(L, lane);
                }
            }
            if (P.pad & 1) return;
            for (u32 spin = 0; spin < SPINS && UNI(*(volatile LAS u32*)(L + LDS_LUTF)) < 4; ++spin) {}
            u32 job = 0;
            for (u32 spin = 0; spin < SPINS && job == 0; ++spin) job = UNI(*(volatile LAS u32*)(L + LDS_JOB));
            const u32 src = (P.srcmap >> (4 * (((job - 1) >> 5) & 7))) & 7u;
            if (src == (u32)rank) return;
            const char* dp = myws + WS_IMG + (call & 1) * IMG_BYTES;
            const char* uf = myws + WS_UF;
            const u32 u0 = 32 + ((src - rank - 1) & 7);
            const char* pend = nullptr;
            u32 pn = 1;
            auto publish = [&]() {
                WAIT_VM(0);
                if ((u32)lane < pn) asm volatile("global_store_dword %0, %1, %2 sc0 sc1" :: "v"(lane * 28), "v"(call), "s"(pend) : "memory");
                pend = nullptr;
            };
            for (u32 k = 0;; ++k) {
                u32 f = UNI(*fet);
                if ((f & ~FIN) <= k) {
                    if (pend) publish();
                    for (u32 spin = 0; spin < 2 * SPINS && f <= k; ++spin) { __builtin_amdgcn_s_sleep(1); f = UNI(*fet); }
                    if ((f & ~FIN) <= k) break;
                }
                FENCE();
                const u32 mt = UNI(meta[k & 3]), panel = (mt >> 5) & 31, r = mt & 31;
                LAS unsigned char* sl = L + LDS_RING + (k & 3) * UNIT;
                u32x4 e[4];
                if (mt >> 10) {
#pragma unroll
                    for (int i = 0; i < 4; ++i) e[i] = *(LAS u32x4*)(sl + i * 1024 + lane * 16);
                } else if (panel < 16) {
                    decode_unit<3>(e, sl, L, lane);
                } else {
                    decode_unit<2>(e, sl, L, lane);
                }
                asm volatile("" : "+v"(e[0]), "+v"(e[1]), "+v"(e[2]), "+v"(e[3]) :: "memory");
                *con = k + 1;
                if (pend) publish();
                const bool half = panel < 16 && r >= R0;
                const u32 u = u0 + 7 * (half ? r & ~1u : r);
                const char* qa = dp + unit_off(panel, u) + (half ? (r & 1) * 2048 : 0);
                const char* qb = half ? dp + unit_off(panel, u + 7) + (r & 1) * 2048 : qa + 2048;
                st_img(vo, qa, e[0]); st_img(vo, qa + 1024, e[1]);
                st_img(vo, qb, e[2]); st_img(vo, qb + 1024, e[3]);
                pend = uf + panel * 2048 + (u + (half && (r & 1) ? 4 * NSPLIT : 0)) * 4;
                pn = half ? 2 : 1;
            }
            STAMP(17);
        }
    }
}

extern "C" void kernel_launch(int rank, int nranks, ncclComm_t comm,
                              hipStream_t stream,
                              void* const* d_in, const int* in_sizes, int n_in,
                              void* d_out, int out_size,
                              const int* in_dtypes, int out_dtype,
                              void* const* d_out_all,
                              void* const* d_ws_all, size_t ws_bytes) {
    (void)comm; (void)in_sizes; (void)n_in; (void)out_size; (void)in_dtypes; (void)out_dtype; (void)d_out_all;
    if (nranks != NR || (unsigned)rank >= (unsigned)NR || ws_bytes < WS_NEED) {
        fprintf(stderr, "kernel_launch: unexpected configuration\n");
        return;
    }
    Params p;
    p.x = (const unsigned char*)d_in[0];
    p.w = (const unsigned char*)d_in[1];
    p.sx = (const float*)d_in[2];
    p.sw = (const float*)d_in[3];
    p.out = (float*)d_out;
    for (int i = 0; i < NR; ++i) p.ws[i] = (char*)d_ws_all[i];
    p.myws = p.ws[rank];
    p.rank = rank;
    p.pad = 0;
    p.pad2 = 0;
    {
        const unsigned grp = 0x4D;
        u32 m = (u32)rank; int a = 1, b = 4;
        for (int k = 1; k < NR; ++k) {
            const int q = (rank + k) & 7;
            const bool same = ((grp >> q) & 1) == ((grp >> rank) & 1);
            m |= (u32)q << (4 * (same ? a++ : b++));
        }
        p.srcmap = m;
    }
    hipLaunchKernelGGL(ag_gemm, dim3(NWG), dim3(NTHREADS), 0, stream, p);
    HIP_CHECK(hipGetLastError());
}
```

```cpp
#include <hip/hip_runtime.h>
#include <rccl/rccl.h>
#include <cstdio>

#define HIP_CHECK(expr) do { hipError_t _e = (expr); if (_e != hipSuccess) { \
    fprintf(stderr, "HIP error %s at %s:%d\n", hipGetErrorString(_e), __FILE__, __LINE__); \
    return; } } while (0)

#define MSLEEP 1
#define GAS __attribute__((address_space(1)))
#define LAS __attribute__((address_space(3)))
typedef unsigned u32;
typedef unsigned short u16;
typedef unsigned long long u64;
typedef unsigned u32x2 __attribute__((ext_vector_type(2)));
typedef unsigned u32x4 __attribute__((ext_vector_type(4)));
typedef int i32x4 __attribute__((ext_vector_type(4)));
typedef int i32x8 __attribute__((ext_vector_type(8)));
typedef float f32x4 __attribute__((ext_vector_type(4)));
typedef float f32x16 __attribute__((ext_vector_type(16)));

constexpr int NR = 8;
constexpr int MN = 4096;
constexpr int KS = 512;
constexpr int NWG = 256;
constexpr int NTHREADS = 768;
constexpr u32 ROUNDS = 32;
constexpr u32 NSTAGE = 64;
#define NPUSH 1
#define WN 3
#define SLOWEND 52
#define FASTM 0xFFFFu
#define RAWTAIL 1
#define NSPLIT 8
constexpr u32 S0 = NSTAGE - NSPLIT;
constexpr u32 NQ = NSTAGE + NSPLIT;
constexpr u32 R0 = NSPLIT ? ((S0 * 4 - 32 + 13) / 14) * 2 : ROUNDS;
constexpr u32 US = 32 + 7 * R0;
constexpr u32 NPAIR = (ROUNDS - R0) / 2;
constexpr u32 T1 = (R0 - NPUSH) * 32, T2 = T1 + NPAIR * 48, NT = T2 + NPAIR * 16;
constexpr u32 UNIT = 4096;
constexpr size_t IMG_BYTES = 32u << 20;
constexpr size_t CB_BYTES = 4u << 20;
__host__ __device__ constexpr u32 unit_off(u32 panel, u32 u) { return (u >> 2) * 524288u + panel * 16384u + (u & 3) * UNIT; }
constexpr size_t WS_CNT = 0;
constexpr size_t WS_TMO = 48 << 10;
constexpr size_t WS_TIK = 52 << 10;
constexpr size_t WS_UF = 64 << 10;
constexpr size_t WS_RDY = 128 << 10;
constexpr size_t WS_JOB = 192 << 10;
constexpr size_t WS_DBG = 256 << 10;
constexpr size_t WS_IMG = 1 << 20;
constexpr size_t WS_CB = WS_IMG + 2 * IMG_BYTES;
constexpr size_t WS_NEED = WS_CB + 2 * CB_BYTES;
constexpr u64 DEADLINE = 50000000ull;
constexpr u32 SPINS = 1u << 25;
constexpr u32 LDS_CTL = 0;
constexpr u32 LDS_FET = LDS_CTL + 132, LDS_CON = LDS_CTL + 136;
constexpr u32 LDS_PRO = LDS_CTL + 140, LDS_META = LDS_CTL + 144;
constexpr u32 LDS_JOB = LDS_CTL + 160, LDS_LEN = LDS_CTL + 176;
constexpr u32 LDS_DL = 512, LDS_CL = LDS_DL + 1024;
constexpr u32 LDS_SC = LDS_CL + 1024, LDS_SB = LDS_SC + 1024;
constexpr u32 LDS_RING = LDS_SB + 2048;
constexpr u32 LDS_A = LDS_RING + 16384, LDS_B = LDS_A + 65536;
constexpr u32 LDS_BYTES = LDS_B + 65536;

struct Params {
    const unsigned char* x;
    const unsigned char* w;
    const float* sx;
    const float* sw;
    float* out;
    char* myws;
    char* ws[NR];
    int rank;
    int pad;
    u32 srcmap;
    u32 pad2;
};

__device__ __forceinline__ u64 rt() { return __builtin_amdgcn_s_memrealtime(); }
#define WAIT_VM(n) asm volatile("s_waitcnt vmcnt(" #n ")" ::: "memory")
#define WAIT_VM0() do { __builtin_amdgcn_s_waitcnt(0x0F70); WAIT_VM(0); } while (0)
#define FENCE() asm volatile("" ::: "memory")
#define SB() __builtin_amdgcn_sched_barrier(0)
#define XCD_ID() (__builtin_amdgcn_s_getreg((3 << 11) | 20) & 7u)
#define UNI(x) __builtin_amdgcn_readfirstlane(x)

#define STAMP(i) do {} while (0)
#define STAMP2(i) do {} while (0)
#define STAMP3(i) do {} while (0)

#define OB0 ""
#define OB1 ""
#define PIECE 512
#define WINP 2
#define WAIT_VMX(n) WAIT_VM(n)
__device__ __forceinline__ void st_sys(char* p, u32x4 v) {
    asm volatile("global_store_dwordx4 %0, %1, off sc0 sc1\n\ts_nop 1" :: "v"(p), "v"(v) : "memory");
}
__device__ __forceinline__ void st_sys_s(u32 voff, const void* base, u32x4 v) {
    asm volatile("global_store_dwordx4 %0, %1, %2 sc0 sc1\n\ts_nop 1" :: "v"(voff), "v"(v), "s"(base) : "memory");
}
__device__ __forceinline__ void st_flag(const void* base, u32 v, int lane) {
    if (lane == 0) asm volatile("global_store_dword %0, %1, %2 sc0 sc1" :: "v"(0u), "v"(v), "s"(base) : "memory");
}

#define DBITS " sc0 sc1"
#define UBITS " sc0 sc1"
#define FBITS " sc0 sc1"
#define STR2(x) #x
#define STR(x) STR2(x)
#define CAP 60
#define F2 true
#define ISLP 2
#define LEAN 1
#define NOBEH 0
#define GAPS 1
#define GAPF 1
#define DSLP ""
#define LS(i) do {} while (0)
struct Ld {
    u32 nops, ub, part, rmask;
    u32 lim, nf, pidx, pb, pw;
    u64 fifo;
};
#define POLL "s_mov_b32 exec_hi, 0\n\tglobal_atomic_or v40, v43, v47, %[uf] sc0 sc1\n\ts_mov_b32 exec_hi, -1\n\t"
#define GL(o) "global_load_lds_dwordx4 %[vo], vcc offset:" #o DBITS "\n\t"
#define GLU ".if (%c[md]&3) == 1\n\t" GL(0) GL(1024) ".elseif (%c[md]&3) == 2\n\t" GL(0) GL(1024) \
            "s_add_u32 m0, m0, %[land]\n\ts_nop 0\n\t" GL(2048) GL(3072) "s_sub_u32 m0, m0, %[land]\n\t" \
            ".else\n\t" GL(0) GL(1024) GL(2048) GL(3072) ".endif\n\t"
#define GLN "s_add_u32 m0, m0, 0x1000\n\ts_add_u32 vcc_lo, vcc_lo, 0x1000\n\ts_addc_u32 vcc_hi, vcc_hi, 0\n\t"
template <int MODE, bool FAST>
__device__ __forceinline__ u32 load_run(Ld& S, u32 end, u32 glo, u32 ghi, u32 lds, u32 err, u32 call, const void* uf,
                                        u32 vo, u32 vf, u32 vctl, u32 vfull) {
    u32 a, b, c, land, rc, keep;
    asm volatile(
        "s_mov_b32 %[keep], m0\n\t"
        "s_mov_b32 %[rc], 0\n\t"
        ".LL%=:\n\t"
        "s_getreg_b32 %[a], hwreg(HW_REG_IB_STS)\n\t"
        "s_bfe_u32 %[b], %[a], 0x20016\n\t"
        "s_and_b32 %[a], %[a], 15\n\t"
        "s_lshl4_add_u32 %[a], %[b], %[a]\n\t"
        "s_sub_u32 %[land], %[nops], %[a]\n\t"
        "s_cmp_eq_u32 %[nf], 0\n\t"
        "s_cbranch_scc1 .LN%=\n\t"
        "s_mov_b64 vcc, %[fifo]\n\t"
        "s_sub_u32 %[a], %[land], vcc_lo\n\t"
        "s_bitcmp1_b32 %[a], 11\n\t"
        "s_cbranch_scc1 .LP%=\n\t"
        "v_add_u32 v41, 1, v41\n\t"
        "s_lshr_b64 %[fifo], %[fifo], 12\n\t"
        "ds_write_b32 %[vfull], v41\n\t"
        "s_sub_u32 %[nf], %[nf], 1\n\t"
        "s_branch .LL%=\n\t"
        ".LN%=:\n\t"
        "s_cmp_eq_u32 %[end], 0\n\t"
        "s_cbranch_scc1 .LX%=\n\t"
        ".LP%=:\n\t"
        "s_sub_u32 %[a], %[land], %[pidx]\n\t"
        "s_cmp_lt_i32 %[a], 0\n\t"
        "s_cbranch_scc1 .LQ%=\n\t"
        "v_cmp_le_u32 vcc, %[call], v40\n\t"
        "s_sub_u32 %[a], %[ub], %[pb]\n\t"
        "s_lshr_b32 %[a], %[a], 1\n\t"
        "s_add_u32 %[pidx], %[nops], 0x40000000\n\t"
        "s_lshr_b32 %[b], vcc_lo, %[a]\n\t"
        "s_lshr_b32 %[b], %[b], %[a]\n\t"
        "s_or_b32 %[rmask], %[rmask], %[b]\n\t"
        ".if (%c[md]&4)\n\ts_mov_b32 %[pw], " STR(GAPF) "\n\t.else\n\ts_mov_b32 %[pw], " STR(GAPS) "\n\t.endif\n\t"
        ".LQ%=:\n\t"
        "s_or_b32 %[rmask], %[rmask], %[err]\n\t"
        "s_min_u32 %[a], %[lim], %[end]\n\t"
        "s_max_u32 %[a], %[a], %[ub]\n\t"
        "s_sub_u32 %[a], %[a], %[ub]\n\t"
        "s_bfm_b32 %[a], %[a], 0\n\t"
        "s_andn2_b32 %[b], %[rmask], %[part]\n\t"
        "s_and_b32 %[b], %[b], %[a]\n\t"
        "s_cbranch_scc1 .LI%=\n\t"
        "ds_read_b32 v42, %[vctl] offset:8\n\t"
        "s_add_u32 %[rc], %[rc], 0x100000\n\t"
        "s_cbranch_scc1 .LT%=\n\t"
        "s_andn2_b32 %[a], 0xfffff, %[rmask]\n\t"
        "s_cbranch_scc0 .LR%=\n\t"
        "s_sub_u32 %[pw], %[pw], 1\n\t"
        "s_cmp_lg_u32 %[pw], 0\n\t"
        "s_cbranch_scc1 .LR%=\n\t"
        "s_lshl_b32 %[b], %[ub], 2\n\t"
        "v_add_u32 v43, %[b], %[vf]\n\t"
        ".if (%c[md]&3)\n\t"
        "v_subrev_u32 v40, 0x400, v43\n\t"
        "v_cmp_gt_u32 vcc, %[whole], v40\n\t"
        "v_subrev_u32 v40, 16*" STR(NSPLIT) ", v43\n\t"
        "s_nop 0\n\t"
        "v_cndmask_b32 v43, v43, v40, vcc\n\t"
        ".endif\n\t"
        "s_add_u32 %[nops], %[nops], 1\n\t"
        POLL
        "s_mov_b32 %[pidx], %[nops]\n\t"
        "s_mov_b32 %[pb], %[ub]\n\t"
        ".LR%=:\n\t"
        ".if (%c[md]&4) == 0\n\t"
        "s_sleep " STR(ISLP) "\n\t"
        ".endif\n\t"
        "s_waitcnt lgkmcnt(0)\n\t"
        "s_nop 0\n\t"
        "v_min_u32_dpp v42, v42, v42 row_shr:1 row_mask:0xf bank_mask:0xf\n\t"
        "s_nop 1\n\t"
        "v_min_u32_dpp v42, v42, v42 row_shr:2 row_mask:0xf bank_mask:0xf\n\t"
        "s_nop 1\n\t"
        "v_min_u32_dpp v42, v42, v42 row_shr:4 row_mask:0xf bank_mask:0xf\n\t"
        "s_nop 1\n\t"
        "v_readlane_b32 %[a], v42, 7\n\t"
        ".if (%c[md]&3)\n\t"
        "s_cmp_ge_u32 %[a], 64-" STR(NSPLIT) "\n\t"
        "s_addc_u32 %[a], %[a], 0\n\t"
        ".endif\n\t"
        "s_lshl2_add_u32 %[lim], %[a], 16\n\t"
        "s_branch .LL%=\n\t"
        ".LI%=:\n\t"
        ".if (%c[md]&4) == 0\n\t"
        "s_sub_u32 %[a], %[nops], %[land]\n\t"
        "s_cmp_gt_u32 %[a], " STR(CAP) "\n\t"
        "s_cbranch_scc1 .LL%=\n\t"
        ".endif\n\t"
        ".if " STR(NOBEH) "\n\t"
        "s_sub_u32 %[a], %[pidx], %[nops]\n\t"
        "s_cmp_le_i32 %[a], 0\n\t"
        "s_cbranch_scc1 .LL%=\n\t"
        ".endif\n\t"
        "s_ff1_i32_b32 %[c], %[b]\n\t"
        "s_add_u32 %[a], %[c], %[ub]\n\t"
        "s_lshr_b32 %[a], %[a], 2\n\t"
        ".if (%c[md]&3) == 0\n\t"
        "s_and_b32 %[land], %[a], 3\n\t"
        "s_lshl_b32 m0, %[land], 14\n\t"
        "s_lshl_b32 %[a], %[a], 19\n\t"
        ".else\n\t"
        "s_sub_u32 %[land], %[a], 64-" STR(NSPLIT) "\n\t"
        "s_mul_i32 vcc_lo, %[land], 52\n\t"
        "s_lshr_b32 vcc_lo, vcc_lo, 8\n\t"
        "s_mul_i32 vcc_lo, vcc_lo, 5\n\t"
        "s_sub_u32 %[land], %[land], vcc_lo\n\t"
        "s_lshl_b32 m0, %[land], 14\n\t"
        "s_cmp_eq_u32 %[land], 4\n\t"
        ".if (%c[md]&3) == 1\n\t"
        "s_cselect_b32 m0, 2*16384+2048, m0\n\t"
        ".else\n\t"
        "s_cselect_b32 m0, 2048-65536, m0\n\t"
        "s_cselect_b32 %[land], 16384-2048, 0\n\t"
        ".endif\n\t"
        "s_cmp_ge_u32 %[a], 64\n\t"
        ".if (%c[md]&3) == 1\n\t"
        "s_cselect_b32 vcc_lo, 2048-" STR(NSPLIT) "*524288, 0\n\t"
        ".else\n\t"
        "s_cselect_b32 vcc_lo, -" STR(NSPLIT) "*524288, 0\n\t"
        ".endif\n\t"
        "s_lshl_b32 %[a], %[a], 19\n\t"
        "s_add_u32 %[a], %[a], vcc_lo\n\t"
        ".endif\n\t"
        "s_and_b32 vcc_hi, %[c], 3\n\t"
        "s_lshl_b32 vcc_hi, vcc_hi, 12\n\t"
        "s_add_u32 m0, m0, vcc_hi\n\t"
        "s_add_u32 m0, m0, %[lds]\n\t"
        "s_add_u32 %[a], %[a], vcc_hi\n\t"
        "s_add_u32 vcc_lo, %[glo], %[a]\n\t"
        "s_addc_u32 vcc_hi, %[ghi], 0\n\t"
        "s_bitset1_b32 %[part], %[c]\n\t"
        ".if (%c[md]&4)\n\t"
        "s_and_b32 %[c], %[c], 28\n\t"
        "s_lshl_b32 %[c], 15, %[c]\n\t"
        "s_andn2_b32 %[b], %[c], %[b]\n\t"
        "s_cbranch_scc1 .LU%=\n\t"
        GLU GLN GLU GLN GLU GLN
        "s_or_b32 %[part], %[part], %[c]\n\t"
        "s_add_u32 %[nops], %[nops], 12>>(%c[md]&1)\n\t"
        ".LU%=:\n\t"
        ".endif\n\t"
        GLU
        ".if (%c[md]&4) == 0\n\t" DSLP ".endif\n\t"
        "s_add_u32 %[nops], %[nops], 4>>(%c[md]&1)\n\t"
        ".LS%=:\n\t"
        "s_andn2_b32 %[a], 15, %[part]\n\t"
        "s_cbranch_scc1 .LL%=\n\t"
        "s_and_b32 vcc_lo, %[nops], 0xfff\n\t"
        "s_mov_b32 vcc_hi, 0\n\t"
        "s_mul_i32 %[a], %[nf], 12\n\t"
        "s_lshl_b64 vcc, vcc, %[a]\n\t"
        "s_or_b64 %[fifo], %[fifo], vcc\n\t"
        "s_add_u32 %[nf], %[nf], 1\n\t"
        "s_lshr_b32 %[part], %[part], 4\n\t"
        "s_lshr_b32 %[rmask], %[rmask], 4\n\t"
        "s_add_u32 %[ub], %[ub], 4\n\t"
        ".if (%c[md]&3) && " STR(LEAN) "\n\t"
        "s_cmp_lt_u32 %[ub], 256\n\t"
        "s_cbranch_scc1 .LS%=\n\t"
        "s_cmp_lg_u32 %[part], 0\n\t"
        ".else\n\t"
        "s_cmp_lt_u32 %[ub], %[end]\n\t"
        ".endif\n\t"
        "s_cbranch_scc1 .LS%=\n\t"
        ".LX%=:\n\t"
        "s_mov_b32 %[rc], 0\n\t"
        "s_branch .LE%=\n\t"
        ".LT%=:\n\t"
        "s_mov_b32 %[rc], 1\n\t"
        "s_waitcnt lgkmcnt(0)\n\t"
        ".LE%=:\n\t"
        "s_mov_b32 m0, %[keep]"
        : [nops] "+s"(S.nops), [ub] "+s"(S.ub), [part] "+s"(S.part), [rmask] "+s"(S.rmask), [lim] "+s"(S.lim), [nf] "+s"(S.nf),
          [pidx] "+s"(S.pidx), [pb] "+s"(S.pb), [pw] "+s"(S.pw), [fifo] "+s"(S.fifo),
          [a] "=&s"(a), [b] "=&s"(b), [c] "=&s"(c), [land] "=&s"(land), [rc] "=&s"(rc), [keep] "=&s"(keep)
        : [end] "s"(end), [glo] "s"(glo), [ghi] "s"(ghi), [lds] "s"(lds), [err] "s"(err), [call] "s"(call), [uf] "s"(uf),
          [vo] "v"(vo), [vf] "v"(vf), [vctl] "v"(vctl), [vfull] "v"(vfull), [md] "i"(MODE + (FAST ? 4 : 0)),
          [whole] "i"(MODE == 2 ? 1024 : 4 * (US - 4 * S0))
        : "memory", "scc", "vcc", "v40", "v41", "v42", "v43", "v47");
    return rc;
}
#define G2(v, s_, o) "global_load_lds_dwordx4 " v ", " s_ " offset:" #o DBITS "\n\tglobal_load_lds_dwordx4 " v ", " s_ " offset:" #o "+1024" DBITS "\n\t"
template <int MODE>
__device__ __forceinline__ u32 load_lean(Ld& S, u64& gx, u64& gw, u32& slot, u32 lds0, u32 err, u32 call, const void* uf,
                                         u32 vo, u32 vf, u32 vctl, u32 vfull) {
    u32 a, b, land, rc, keep;
    asm volatile(
        "s_mov_b32 %[keep], m0\n\t"
        "s_mov_b32 %[rc], 0\n\t"
        "v_add_u32 v44, 0x1000, %[vo]\n\t"
        "v_add_u32 v45, 0x2000, %[vo]\n\t"
        "v_add_u32 v46, 0x3000, %[vo]\n\t"
        ".LL%=:\n\t"
        "ds_read_b32 v42, %[vctl] offset:8\n\t"
        "s_getreg_b32 %[a], hwreg(HW_REG_IB_STS)\n\t"
        "s_bfe_u32 %[b], %[a], 0x20016\n\t"
        "s_and_b32 %[a], %[a], 15\n\t"
        "s_lshl4_add_u32 %[a], %[b], %[a]\n\t"
        "s_sub_u32 %[land], %[nops], %[a]\n\t"
        ".LM%=:\n\t"
        "s_cmp_eq_u32 %[nf], 0\n\t"
        "s_cbranch_scc1 .LP%=\n\t"
        "s_mov_b64 vcc, %[fifo]\n\t"
        "s_sub_u32 %[a], %[land], vcc_lo\n\t"
        "s_bitcmp1_b32 %[a], 11\n\t"
        "s_cbranch_scc1 .LP%=\n\t"
        "v_add_u32 v41, 1, v41\n\t"
        "s_lshr_b64 %[fifo], %[fifo], 12\n\t"
        "ds_write_b32 %[vfull], v41\n\t"
        "s_sub_u32 %[nf], %[nf], 1\n\t"
        "s_branch .LM%=\n\t"
        ".LP%=:\n\t"
        "s_add_u32 %[rc], %[rc], 0x100000\n\t"
        "s_cbranch_scc1 .LT%=\n\t"
        ".if %c[md] == 1\n\t"
        "s_or_b32 %[rmask], %[rmask], %[err]\n\t"
        "s_andn2_b32 %[a], 0xfffff, %[rmask]\n\t"
        "s_cbranch_scc0 .LQ%=\n\t"
        "s_sub_u32 %[a], %[land], %[pidx]\n\t"
        "s_cmp_lt_i32 %[a], 0\n\t"
        "s_cbranch_scc1 .LO%=\n\t"
        "v_cmp_le_u32 vcc, %[call], v40\n\t"
        "s_lshl_b32 %[b], %[ub], 2\n\t"
        "v_add_u32 v43, %[b], %[vf]\n\t"
        "s_sub_u32 %[a], %[ub], %[pb]\n\t"
        "s_lshr_b32 %[a], %[a], 1\n\t"
        "s_min_u32 %[a], %[a], 16\n\t"
        "s_lshr_b32 %[b], vcc_lo, %[a]\n\t"
        "s_lshr_b32 %[b], %[b], %[a]\n\t"
        "s_or_b32 %[rmask], %[rmask], %[b]\n\t"
        "v_subrev_u32 v40, 0x400, v43\n\t"
        "v_cmp_gt_u32 vcc, %[whole], v40\n\t"
        "v_subrev_u32 v40, 16*" STR(NSPLIT) ", v43\n\t"
        "s_add_u32 %[nops], %[nops], 1\n\t"
        "v_cndmask_b32 v43, v43, v40, vcc\n\t"
        "s_mov_b32 %[pidx], %[nops]\n\t"
        POLL
        "s_mov_b32 %[pb], %[ub]\n\t"
        ".LO%=:\n\t"
        "s_andn2_b32 %[a], 15, %[rmask]\n\t"
        "s_cbranch_scc1 .LL%=\n\t"
        ".LQ%=:\n\t"
        ".endif\n\t"
        "s_lshr_b32 %[a], %[ub], 2\n\t"
        "s_sub_u32 %[a], %[a], 4\n\t"
        "s_waitcnt lgkmcnt(0)\n\t"
        "v_cmp_gt_u32 vcc, %[a], v42\n\t"
        "s_lshl_b32 %[a], %[slot], 14\n\t"
        "s_add_u32 %[b], %[a], 0x10000\n\t"
        "s_cmp_eq_u32 %[slot], 4\n\t"
        "s_cselect_b32 %[a], 2*16384+2048, %[a]\n\t"
        "s_cselect_b32 %[b], 2048, %[b]\n\t"
        "s_cselect_b32 %[land], 16384-2048, 0\n\t"
        "s_cbranch_vccnz .LL%=\n\t"
        ".if %c[md] == 1\n\t"
        "s_add_u32 m0, %[a], %[lds]\n\t"
        "s_add_u32 %[b], %[b], %[lds]\n\t"
        G2("%[vo]", "%[gx]", 0) "s_add_u32 m0, m0, 0x1000\n\ts_add_u32 %[b], %[b], 0x3000\n\t"
        G2("v44", "%[gx]", 0) "s_add_u32 m0, m0, 0x1000\n\ts_add_u32 %[nops], %[nops], 12\n\t"
        G2("v45", "%[gx]", 0) "s_add_u32 m0, m0, 0x1000\n\ts_lshr_b32 %[rmask], %[rmask], 4\n\t"
        G2("v46", "%[gx]", 0) "s_mov_b32 m0, %[b]\n\ts_mov_b64 vcc, %[gx]\n\t"
        G2("v46", "%[gw]", 0) "s_add_u32 m0, m0, %[land]\n\ts_add_u32 vcc_lo, vcc_lo, 0x80000\n\t"
        G2("v46", "%[gw]", 2048)
        "s_addc_u32 vcc_hi, vcc_hi, 0\n\t"
        "s_mov_b64 %[gx], vcc\n\t"
        ".else\n\t"
        "s_add_u32 m0, %[b], %[lds]\n\t"
        "s_sub_u32 %[a], 0x1000, %[land]\n\t"
        G2("%[vo]", "%[gw]", 0) "s_add_u32 m0, m0, %[land]\n\ts_add_u32 %[nops], %[nops], 12\n\t"
        G2("%[vo]", "%[gw]", 2048) "s_add_u32 m0, m0, %[a]\n\ts_nop 0\n\t"
        G2("v44", "%[gw]", 0) "s_add_u32 m0, m0, %[land]\n\ts_nop 0\n\t"
        G2("v44", "%[gw]", 2048) "s_add_u32 m0, m0, %[a]\n\ts_nop 0\n\t"
        G2("v45", "%[gw]", 0) "s_add_u32 m0, m0, %[land]\n\ts_nop 0\n\t"
        G2("v45", "%[gw]", 2048)
        ".endif\n\t"
        "s_mov_b64 vcc, %[gw]\n\t"
        "s_add_u32 vcc_lo, vcc_lo, 0x80000\n\t"
        "s_addc_u32 vcc_hi, vcc_hi, 0\n\t"
        "s_mov_b64 %[gw], vcc\n\t"
        "s_and_b32 vcc_lo, %[nops], 0xfff\n\t"
        "s_mov_b32 vcc_hi, 0\n\t"
        "s_mul_i32 %[a], %[nf], 12\n\t"
        "s_lshl_b64 vcc, vcc, %[a]\n\t"
        "s_or_b64 %[fifo], %[fifo], vcc\n\t"
        "s_add_u32 %[nf], %[nf], 1\n\t"
        "s_add_u32 %[slot], %[slot], 1\n\t"
        "s_cmp_eq_u32 %[slot], 5\n\t"
        "s_cselect_b32 %[slot], 0, %[slot]\n\t"
        "s_add_u32 %[ub], %[ub], 4\n\t"
        "s_cmp_lt_u32 %[ub], 256+4*" STR(NSPLIT) "\n\t"
        "s_cbranch_scc1 .LL%=\n\t"
        "s_mov_b32 %[rc], 0\n\t"
        "s_branch .LE%=\n\t"
        ".LT%=:\n\t"
        "s_mov_b32 %[rc], 1\n\t"
        "s_waitcnt lgkmcnt(0)\n\t"
        ".LE%=:\n\t"
        "s_mov_b32 m0, %[keep]"
        : [nops] "+s"(S.nops), [ub] "+s"(S.ub), [rmask] "+s"(S.rmask), [nf] "+s"(S.nf), [pidx] "+s"(S.pidx), [pb] "+s"(S.pb),
          [fifo] "+s"(S.fifo), [gx] "+s"(gx), [gw] "+s"(gw), [slot] "+s"(slot),
          [a] "=&s"(a), [b] "=&s"(b), [land] "=&s"(land), [rc] "=&s"(rc), [keep] "=&s"(keep)
        : [lds] "s"(lds0), [err] "s"(err), [call] "s"(call), [uf] "s"(uf),
          [vo] "v"(vo), [vf] "v"(vf), [vctl] "v"(vctl), [vfull] "v"(vfull), [md] "i"(MODE), [whole] "i"(4 * (US - 4 * S0))
        : "memory", "scc", "vcc", "v40", "v41", "v42", "v43", "v44", "v45", "v46", "v47");
    return rc;
}
#define VPC "v44", "v45", "v46", "v47", "v48", "v49", "v50", "v51", "v52", "v53", "v54", "v55", "v56", "v57", "v58", "v59", \
            "v60", "v61", "v62", "v63", "v64", "v65", "v66", "v67", "v68", "v69", "v70", "v71", "v72", "v73", "v74", "v75"
#define PL(i, g) asm volatile("global_load_dwordx4 v[44+4*" #i ":47+4*" #i "], %0, %1 offset:512*" #i FBITS :: "v"(vo), "s"(g) : "memory", VPC)
#define PW(i, l) asm volatile("ds_write_b128 %0, v[44+4*" #i ":47+4*" #i "] offset:512*" #i :: "v"(l) : "memory", VPC)
template <int N> __device__ __forceinline__ void wait_vm() { asm volatile("s_waitcnt vmcnt(%0)" :: "i"(N) : "memory"); }

__device__ __forceinline__ void st_img(u32 voff, const void* base, u32x4 v) {
    asm volatile("global_store_dwordx4 %0, %1, %2" UBITS "\n\ts_nop 1" :: "v"(voff), "v"(v), "s"(base) : "memory");
}
__device__ __forceinline__ u32 perm(u32 hi, u32 lo, u32 sel) { return __builtin_amdgcn_perm(hi, lo, sel); }
__device__ __forceinline__ u32 bfi(u32 m, u32 a, u32 b) { return __builtin_amdgcn_bitop3_b32(m, a, b, 0xCA); }
__device__ __forceinline__ u32 and_or(u32 a, u32 m, u32 c) { return __builtin_amdgcn_bitop3_b32(a, m, c, 0xEA); }
__device__ __forceinline__ u32 xor_and(u32 a, u32 b, u32 m) { return __builtin_amdgcn_bitop3_b32(a, b, m, 0x78); }
#define BAR() asm volatile("s_waitcnt lgkmcnt(0)\n\ts_barrier" ::: "memory")
template <int S> __device__ __forceinline__ u32 shl(u32 v) { return S >= 0 ? v << (S >= 0 ? S : 0) : v >> (S < 0 ? -S : 0); }
__device__ __forceinline__ u32 scan64(u32 v) {
    v += __builtin_amdgcn_update_dpp(0u, v, 0x111, 0xf, 0xf, true);
    v += __builtin_amdgcn_update_dpp(0u, v, 0x112, 0xf, 0xf, true);
    v += __builtin_amdgcn_update_dpp(0u, v, 0x114, 0xf, 0xf, true);
    v += __builtin_amdgcn_update_dpp(0u, v, 0x118, 0xf, 0xf, true);
    v += __builtin_amdgcn_update_dpp(0u, v, 0x142, 0xa, 0xf, false);
    v += __builtin_amdgcn_update_dpp(0u, v, 0x143, 0xc, 0xf, false);
    return v;
}
__device__ __forceinline__ void lds_or(LAS unsigned char* p, u32 v) {
    __hip_atomic_fetch_or((LAS u32*)p, v, __ATOMIC_RELAXED, __HIP_MEMORY_SCOPE_WORKGROUP);
}

template <bool W> __device__ __forceinline__ void load_unit(u32 (&d)[16], const Params& P, int panel, int r, int lane, bool half) {
    if (!W) {
        const GAS unsigned char* b = (const GAS unsigned char*)P.x + (size_t)panel * 256 * KS;
        b += half ? (r & 1) * 128 * KS + (r & ~1) * 16 : r * 16;
        const u32 hi = half ? 16 : 128 * KS;
#pragma unroll
        for (int i = 0; i < 4; ++i) {
            u32x4 v = *(const GAS u32x4*)(b + (u32)(((i & 1) * 64 + lane) * KS + (i >> 1) * hi));
            d[4 * i] = v[0]; d[4 * i + 1] = v[1]; d[4 * i + 2] = v[2]; d[4 * i + 3] = v[3];
        }
    } else {
        const GAS unsigned char* b = (const GAS unsigned char*)P.w + (size_t)r * 16 * MN + (panel - 16) * 256;
#pragma unroll
        for (int k = 0; k < 16; ++k) d[k] = *(const GAS u32*)(b + (u32)(k * MN + lane * 4));
    }
}
template <bool W> __device__ __forceinline__ void order_unit(u32 (&x)[16], const u32 (&d)[16]) {
    if (!W) {
#pragma unroll
        for (int k = 0; k < 16; ++k) x[k] = d[k];
    } else {
#pragma unroll
        for (int q = 0; q < 4; ++q) {
            u32 t0 = perm(d[4 * q + 1], d[4 * q], 0x05010400), t1 = perm(d[4 * q + 1], d[4 * q], 0x07030602);
            u32 t2 = perm(d[4 * q + 3], d[4 * q + 2], 0x05010400), t3 = perm(d[4 * q + 3], d[4 * q + 2], 0x07030602);
            x[q] = perm(t2, t0, 0x05040100); x[4 + q] = perm(t2, t0, 0x07060302);
            x[8 + q] = perm(t3, t1, 0x05040100); x[12 + q] = perm(t3, t1, 0x07060302);
        }
    }
}
__device__ __forceinline__ u32x4 piece(const u32 (&x)[16], int i) { return u32x4{x[4 * i], x[4 * i + 1], x[4 * i + 2], x[4 * i + 3]}; }

#define ES(i) do {} while (0)
template <int MB> struct Fmt {
    static constexpr u32 EM = (MB == 3 ? 0x0Fu : 0x1Fu) * 0x01010101u;
    static constexpr u32 HEAD = MB == 3 ? 3072 : 2048;
    static constexpr u32 OFF_H = MB == 3 ? 0 : 512, OFF_2 = OFF_H + 4;
};
__device__ __forceinline__ u32 zero2(u32 w) { return __builtin_amdgcn_bitop3_b32(w, w >> 1, 0x55555555u, 0x02); }
__device__ __forceinline__ u32 code_of(u32 hi, u32 lo, u32 v) {
    const u32 r = perm(hi, lo, v);
    return __builtin_amdgcn_bitop3_b32(r, r >> 2, 0x03030303u, 0x28);
}
__device__ __forceinline__ void lookup4(u32* sel, u32 w, LAS unsigned char* tab) {
#pragma unroll
    for (int j = 0; j < 4; ++j) sel[j] = *(LAS u32*)(tab + ((w >> (8 * j)) & 255u) * 4);
}
__device__ __forceinline__ void gather4(const u32* v, const u32* sel, u32 z, u32 a) {
#pragma unroll
    for (int j = 0; j < 4; ++j) {
        const u32 o = j ? a + __builtin_popcount(z & ((1u << (8 * j)) - 1)) : a;
        const u32 c = perm(0u, v[j], sel[j]);
        LAS unsigned char* p = (LAS unsigned char*)(size_t)(o & ~3u);
        lds_or(p, __builtin_amdgcn_alignbyte(c, 0u, ~o));
        lds_or(p + 4, __builtin_amdgcn_alignbyte(0u, c, ~o));
    }
}
__device__ __forceinline__ void window4(u32* o, u32x2* q, u32 z, u32 a) {
#pragma unroll
    for (int j = 0; j < 4; ++j) {
        o[j] = j ? a + __builtin_popcount(z & ((1u << (8 * j)) - 1)) : a;
        LAS u32* p = (LAS u32*)(size_t)(o[j] & ~3u);
        q[j] = u32x2{p[0], p[1]};
    }
}
__device__ __forceinline__ u32 spread(u32 tab, u32 o, u32x2 q, u32 sel) {
    return perm(tab, __builtin_amdgcn_alignbyte(q[1], q[0], o), sel);
}
__device__ __forceinline__ u32 pack4(const u32* c) {
    u32 w = __builtin_amdgcn_udot4(c[0], 0x40100401u, 0u, false);
#pragma unroll
    for (int j = 1; j < 4; ++j) w |= __builtin_amdgcn_udot4(c[j], 0x40100401u, 0u, false) << (8 * j);
    return w;
}
__device__ __forceinline__ void encode_clear(LAS unsigned char* T, int lane) {
#pragma unroll
    for (int i = 2; i < 5; ++i) *(LAS u32x4*)(T + i * 1024 + lane * 16) = u32x4{0, 0, 0, 0};
}

template <int MB> __device__ __forceinline__ u32 encode_unit(const u32 (&x)[16], LAS unsigned char* T, LAS unsigned char* L,
                                                             char* dst, int lane, bool raw) {
    typedef Fmt<MB> F;
    u32 sum = 0;
#pragma unroll
    for (int j = 0; j < 16; j += 5) sum = __builtin_amdgcn_udot4(x[j] & 0x7F7F7F7Fu, 0x01010101u, sum, false);
    const int tot = __builtin_amdgcn_readlane(scan64(sum), 63);
    const u32 eb = (u32)((tot - (1516 << MB)) >> (10 + MB)) & (F::EM & 31u);
    const u32 B = (eb << MB) * 0x01010101u;
    u32 v[16], c[16];
    u32x4 w, z;
#pragma unroll
    for (int j = 0; j < 16; ++j) {
        v[j] = (((x[j] | 0x80808080u) - B) >> MB) & F::EM;
        c[j] = code_of(0u, 0x03020100u, v[j]);
    }
    u32 cnt[5] = {0};
#pragma unroll
    for (int q = 0; q < 4; ++q) {
        w[q] = pack4(c + 4 * q); z[q] = zero2(w[q]); cnt[q + 1] = cnt[q] + __builtin_popcount(z[q]);
        lookup4(c + 4 * q, w[q], L + LDS_CL);
    }
    FENCE();
    const u32 incl = scan64(cnt[4]), n1 = __builtin_amdgcn_readlane(incl, 63);
    ES(0);
    u32 n2 = 0, n3 = 0, len64 = 64;
    raw = raw || n1 > 2040;
    const u32 off3 = F::OFF_2 + ((n1 + 15) >> 4) * 4;
    if (!raw) {
        const u32 a1 = incl - cnt[4] + ((u32)(size_t)T + 2047);
#pragma unroll
        for (int q = 0; q < 4; ++q) gather4(v + 4 * q, c + 4 * q, z[q], a1 + cnt[q]);
        ES(1);
        st_sys_s(lane * 16, dst, w);
        if (MB == 3) {
            u32x4 o[2];
#pragma unroll
            for (int p = 0; p < 8; ++p) o[p >> 2][p & 3] = bfi(0x87878787u, x[2 * p], bfi(0xF0F0F0F0u, x[2 * p + 1] << 4, x[2 * p + 1] >> 4));
            st_sys_s(lane * 16, dst + 1024, o[0]);
            st_sys_s(lane * 16, dst + 2048, o[1]);
        } else {
            u32x4 o;
            u32x2 sg;
#pragma unroll
            for (int q = 0; q < 4; ++q)
                o[q] = bfi(0x0F0F0F0Fu, bfi(0x03030303u, x[4 * q], x[4 * q + 1] << 2), bfi(0x03030303u, x[4 * q + 2], x[4 * q + 3] << 2) << 4);
#pragma unroll
            for (int k = 0; k < 2; ++k) {
                u32 t = x[8 * k];
#pragma unroll
                for (int j = 1; j < 8; ++j) t = bfi(0x80808080u >> j, x[8 * k + j] >> j, t);
                sg[k] = t;
            }
            *(LAS u32x2*)(T + lane * 8) = sg;
            st_sys_s(lane * 16, dst + 1024, o);
        }
        FENCE();
        ES(2);
        for (u32 b = 0; b < n1; b += 1024) {
            const u32x4 y = *(LAS u32x4*)(T + 2048 + b + lane * 16);
            u32 u[4], c2[4];
#pragma unroll
            for (int j = 0; j < 4; ++j) {
                c2[j] = code_of(0x00000300u, 0x00000102u, (y[j] + 0x01010101u) & F::EM);
                u[j] = (y[j] + 0x05050505u) & F::EM;
            }
            const u32 w2 = pack4(c2);
            lookup4(c2, w2, L + LDS_CL);
            FENCE();
            const u32 z2 = zero2(w2), k2 = __builtin_popcount(z2), i2 = scan64(k2);
            if (b + lane * 16 < n1) *(LAS u32*)(T + F::OFF_2 + (b >> 2) + lane * 4) = w2;
            const u32 all = n2 + __builtin_amdgcn_readlane(i2, 63);
            if (all <= 1016 && k2) gather4(u, c2, z2, n2 + i2 - k2 + ((u32)(size_t)T + 4095));
            n2 = all;
        }
        raw = n2 > 1016;
        ES(3);
    }
    if (!raw) {
        FENCE();
        const u32x4 y = *(LAS u32x4*)(T + 4096 + lane * 16);
        u32 k3 = 0, m[4];
#pragma unroll
        for (int j = 0; j < 4; ++j) { m[j] = ((y[j] & 0x1C1C1C1Cu) + 0x1C1C1C1Cu) & 0x20202020u; k3 += __builtin_popcount(m[j]); }
        const u32 i3 = scan64(k3);
        n3 = __builtin_amdgcn_readlane(i3, 63);
        const u32 offp = off3 + ((n2 + 15) >> 4) * 4;
        len64 = (F::HEAD + offp + 2 * n3 + 63) >> 6;
        raw = len64 >= 64 || n3 > 63;
        if (!raw) {
            if (lane * 16 < n2)
                *(LAS u32*)(T + off3 + lane * 4) = bfi(0x0F0F0F0Fu, bfi(0x03030303u, y[0], y[1] << 2), bfi(0x03030303u, y[2], y[3] << 2) << 4);
            if (lane == 0) *(LAS u32*)(T + F::OFF_H) = eb | (n1 << 5) | (n2 << 16) | (n3 << 26);
            LAS unsigned char* p = T + offp + 2 * (i3 - k3);
#pragma unroll
            for (int j = 0; j < 4; ++j) {
                u32 mm = m[j];
                while (mm) {
                    const u32 b = __builtin_ctz(mm) & 24u;
                    mm &= mm - 1;
                    *(LAS u16*)p = (u16)(((lane * 16 + j * 4 + (b >> 3)) << 5) | ((y[j] >> b) & 31u));
                    p += 2;
                }
            }
        }
    }
    if (raw) {
        WAIT_VM(0);
#pragma unroll
        for (int i = 0; i < 4; ++i) st_sys_s(lane * 16, dst + i * 1024, piece(x, i));
        return 64;
    }
    FENCE();
    const u32 tl = len64 * 64 - F::HEAD;
    if (lane * 16 < tl) st_sys_s(lane * 16, dst + F::HEAD, *(LAS u32x4*)(T + lane * 16));
    if (MB == 2 && 1024 + lane * 16 < tl) st_sys_s(lane * 16, dst + F::HEAD + 1024, *(LAS u32x4*)(T + 1024 + lane * 16));
    return len64;
}

template <int MB> __device__ __forceinline__ void decode_unit(u32x4 (&e)[4], u32 hdr, u32x4 w, u32 w2,
                                                              LAS unsigned char* S, LAS unsigned char* L, int lane) {
    typedef Fmt<MB> F;
    LAS unsigned char* T = S + F::HEAD;
    const u32 eb = hdr & 31u, n1 = (hdr >> 5) & 2047u, n2 = (hdr >> 16) & 1023u, n3 = hdr >> 26;
    const u32 off3 = F::OFF_2 + ((n1 + 15) >> 4) * 4, offp = off3 + ((n2 + 15) >> 4) * 4;
    const u32 w3 = *(LAS u32*)(T + off3 + lane * 4);
    const u32 en = *(LAS u16*)(T + offp + lane * 2);
    u32 s1[16], s2[4];
    lookup4(s2, w2, L + LDS_DL);
    const u32x4 o0 = *(LAS u32x4*)(S + 1024 + lane * 16);
    u32x4 o1;
    if (MB == 3) o1 = *(LAS u32x4*)(S + 2048 + lane * 16);
    else { const u32x2 sg = *(LAS u32x2*)(T + lane * 8); o1 = u32x4{sg[0], sg[1], 0, 0}; }
    FENCE(); SB();
    const u32 eb4 = eb * 0x01010101u;
    const u32 t1 = ((eb4 + 0x03020100u) & F::EM) << MB, t2 = ((eb4 + 0x041F0000u) & F::EM) << MB, t3 = ((eb4 + 0x1E1D1C1Bu) & F::EM) << MB;
    u32x4 z;
    u32 cnt[5] = {0};
#pragma unroll
    for (int q = 0; q < 4; ++q) { z[q] = zero2(w[q]); cnt[q + 1] = cnt[q] + __builtin_popcount(z[q]); }
    u32 z2 = zero2(w2), k2 = __builtin_popcount(z2);
    const u32 both = cnt[4] | (k2 << 16), incl = scan64(both);
    const u32 excl = incl - both + ((u32)(size_t)L + LDS_SB) + (((u32)(size_t)L + LDS_SC) << 16);
    SB();
    {
        u32x4 y;
#pragma unroll
        for (int j = 0; j < 4; ++j) y[j] = perm(0u, t3, (w3 >> (2 * j)) & 0x03030303u);
        *(LAS u32x4*)(L + LDS_SC + lane * 16) = y;
        FENCE();
        if ((u32)lane < n3) *(L + LDS_SC + ((en >> 5) & 1023u)) = (unsigned char)(((en + eb + 27u) & (F::EM & 31u)) << MB);
        FENCE();
    }
    u32 o[16];
    u32x2 q[16];
    window4(o, q, z2, excl >> 16);
    SB();
#pragma unroll
    for (int i = 0; i < 4; ++i) lookup4(s1 + 4 * i, w[i], L + LDS_DL);
    FENCE(); SB();
    *(LAS u32x4*)(L + LDS_SB + lane * 16) =
        u32x4{spread(t2, o[0], q[0], s2[0]), spread(t2, o[1], q[1], s2[1]), spread(t2, o[2], q[2], s2[2]), spread(t2, o[3], q[3], s2[3])};
    if (n1 > 1024) {
        w2 = *(LAS u32*)(T + F::OFF_2 + 256 + lane * 4);
        lookup4(s2, w2, L + LDS_DL);
        z2 = zero2(w2); k2 = __builtin_popcount(z2);
        window4(o, q, z2, (__builtin_amdgcn_readlane(incl, 63) >> 16) + (u32)(size_t)L + LDS_SC + scan64(k2) - k2);
        *(LAS u32x4*)(L + LDS_SB + 1024 + lane * 16) =
            u32x4{spread(t2, o[0], q[0], s2[0]), spread(t2, o[1], q[1], s2[1]), spread(t2, o[2], q[2], s2[2]), spread(t2, o[3], q[3], s2[3])};
    }
    FENCE(); SB();
#pragma unroll
    for (int i = 0; i < 4; ++i) window4(o + 4 * i, q + 4 * i, z[i], (excl & 0xFFFFu) + cnt[i]);
    SB();
#pragma unroll
    for (int j = 0; j < 16; ++j) {
        const u32 ex = spread(t1, o[j], q[j], s1[j]);
        if (MB == 3) {
            const u32 v = j < 8 ? o0[(j >> 1) & 3] : o1[(j >> 1) & 3];
            e[j >> 2][j & 3] = (j & 1) ? and_or(v >> 4, 0x07070707u, and_or(v << 4, 0x80808080u, ex)) : and_or(v, 0x87878787u, ex);
        } else {
            e[j >> 2][j & 3] = and_or(o0[j >> 2] >> (2 * (j & 3)), 0x03030303u, and_or(o1[j >> 3] << (j & 7), 0x80808080u, ex));
        }
    }
}

#define TS(t)
#define TC(t)
#define V10(a) "v" #a "0", "v" #a "1", "v" #a "2", "v" #a "3", "v" #a "4", "v" #a "5", "v" #a "6", "v" #a "7", "v" #a "8", "v" #a "9"
__device__ __forceinline__ void mfma_role(u32 sa, u32 sb, u32 sc, u32 olo, u32 ohi, u32 scale,
                                          u64& ta, u64& tb, u64& tc, u64& td, u64& ca, u64& cb, u64& cc) {
    u32 need, m, spin, scl;
    asm volatile(
        ".macro AG_FRAG d, a, o\n\t"
        "ds_read_b128 v[\\d:\\d+3], \\a offset:\\o\n\t"
        "ds_read_b128 v[\\d+4:\\d+7], \\a offset:\\o+4096\n\t"
        ".endm\n\t"
        ".macro AG_MM c, a, b\n\t"
        "v_mfma_f32_32x32x64_f8f6f4 v[\\c:\\c+15], v[\\a:\\a+7], v[\\b:\\b+7], v[\\c:\\c+15] cbsz:0 blgp:1\n\t"
        ".endm\n\t"
        ".macro AG_MIN\n\t"
        "v_min_u32 v164, v164, v165\n\t"
        "s_nop 0\n\t"
        "v_readfirstlane_b32 %[m], v164\n\t"
        ".endm\n\t"
        ".macro AG_SPIN\n\t"
        "s_mov_b32 %[spin], 0\n\t"
        ".LAG_S\\@:\n\t"
        "ds_read_b64 v[164:165], v162\n\t"
        "s_add_u32 %[spin], %[spin], 1\n\t"
        "s_waitcnt lgkmcnt(0)\n\t"
        "AG_MIN\n\t"
        "s_cmp_ge_u32 %[m], %[need]\n\t"
        "s_cbranch_scc1 .LAG_E\\@\n\t"
        "s_sleep " STR(MSLEEP) "\n\t"
        "s_cmp_lt_u32 %[spin], 0x2000000\n\t"
        "s_cbranch_scc1 .LAG_S\\@\n\t"
        ".LAG_E\\@:\n\t"
        ".endm\n\t"
        ".macro AG_DONE\n\t"
        "v_add_u32 v163, 1, v163\n\t"
        "ds_write_b32 v162, v163 offset:8\n\t"
        ".endm\n\t"
        ".macro AG_FULL so, sn\n\t"
        "s_waitcnt lgkmcnt(4)\n\t"
        "AG_MM 0, 128, 144\n\t"
        "s_waitcnt lgkmcnt(2)\n\t"
        "AG_MM 64, 136, 144\n\t"
        "ds_read_b64 v[164:165], v162\n\t"
        "AG_FRAG 144, v161, \\so+2048\n\t"
        "s_add_u32 %[need], %[need], 1\n\t"
        "s_waitcnt lgkmcnt(3)\n\t"
        "AG_MM 16, 128, 152\n\t"
        "AG_MM 80, 136, 152\n\t"
        "AG_FRAG 152, v161, \\so+3072\n\t"
        "s_waitcnt lgkmcnt(2)\n\t"
        "AG_MM 32, 128, 144\n\t"
        "AG_MIN\n\t"
        "AG_MM 96, 136, 144\n\t"
        "s_cmp_ge_u32 %[m], %[need]\n\t"
        "s_cbranch_scc1 .LAG_F\\@\n\t"
        "s_waitcnt lgkmcnt(0)\n\t"
        "AG_MM 48, 128, 152\n\t"
        "AG_DONE\n\t"
        "AG_MM 112, 136, 152\n\t"
        "AG_SPIN\n\t"
        "AG_FRAG 144, v161, \\sn\n\t"
        "AG_FRAG 128, v160, \\sn\n\t"
        "s_branch .LAG_J\\@\n\t"
        ".LAG_F\\@:\n\t"
        "AG_FRAG 144, v161, \\sn\n\t"
        "s_waitcnt lgkmcnt(2)\n\t"
        "AG_MM 48, 128, 152\n\t"
        "AG_DONE\n\t"
        "AG_FRAG 128, v160, \\sn\n\t"
        "AG_MM 112, 136, 152\n\t"
        ".LAG_J\\@:\n\t"
        "AG_FRAG 136, v160, \\sn+2048\n\t"
        "AG_FRAG 152, v161, \\sn+1024\n\t"
        ".endm\n\t"
        ".macro AG_PREP c, b\n\t"
        ".irp j, 0, 16, 32, 48\n\t"
        ".irp i, 0, 1, 2, 3\n\t"
        "v_mul_f32 v[\\c+\\j+4*\\b+\\i], %[scl], v[\\c+\\j+4*\\b+\\i]\n\t"
        ".endr\n\t"
        ".endr\n\t"
        "v_swap_b32 v[\\c+4*\\b+1], v[\\c+16+4*\\b]\n\t"
        "v_swap_b32 v[\\c+4*\\b+2], v[\\c+32+4*\\b]\n\t"
        "v_swap_b32 v[\\c+4*\\b+3], v[\\c+48+4*\\b]\n\t"
        "v_swap_b32 v[\\c+16+4*\\b+2], v[\\c+32+4*\\b+1]\n\t"
        "v_swap_b32 v[\\c+16+4*\\b+3], v[\\c+48+4*\\b+1]\n\t"
        "v_swap_b32 v[\\c+32+4*\\b+3], v[\\c+48+4*\\b+2]\n\t"
        ".endm\n\t"
        ".macro AG_ST c, r\n\t"
        ".if (\\r) >= 0\n\t"
        ".if ((\\r) & 3) == 0\n\t"
        "AG_PREP \\c, (\\r)/4\n\t"
        ".endif\n\t"
        ".if \\c\n\t"
        "global_store_dwordx4 v166, v[\\c+16*((\\r)&3)+((\\r)&12):\\c+16*((\\r)&3)+((\\r)&12)+3], vcc" OB1 "\n\t"
        ".else\n\t"
        "global_store_dwordx4 v166, v[\\c+16*((\\r)&3)+((\\r)&12):\\c+16*((\\r)&3)+((\\r)&12)+3], vcc" OB0 "\n\t"
        ".endif\n\t"
        ".if ((\\r) & 3) == 3\n\t"
        "s_add_u32 vcc_lo, vcc_lo, 5*16384\n\t"
        ".else\n\t"
        "s_add_u32 vcc_lo, vcc_lo, 16384\n\t"
        ".endif\n\t"
        "s_addc_u32 vcc_hi, vcc_hi, 0\n\t"
        ".endif\n\t"
        ".endm\n\t"
        ".macro AG_XF d, q\n\t"
        ".if ((\\q) %% 5) == 4\n\t"
        "AG_FRAG \\d, v160, 2*16384+2048\n\t"
        ".else\n\t"
        "AG_FRAG \\d, v160, ((\\q) %% 5)*16384\n\t"
        ".endif\n\t"
        ".endm\n\t"
        ".macro AG_WF d, q, i\n\t"
        ".if ((\\q) %% 5) == 4\n\t"
        "AG_FRAG \\d, v167, (\\i/2)*16384+(\\i&1)*1024\n\t"
        ".else\n\t"
        "AG_FRAG \\d, v161, ((\\q) %% 5)*16384+\\i*1024\n\t"
        ".endif\n\t"
        ".endm\n\t"
        ".macro AG_HALF c, q, last, r0, r1\n\t"
        ".set AG_B0, 136+8*(((\\q)+1) %% 3)\n\t"
        ".set AG_B1, 136+8*(((\\q)+2) %% 3)\n\t"
        ".set AG_B2, 136+8*((\\q) %% 3)\n\t"
        "s_waitcnt lgkmcnt(2)\n\t"
        "AG_MM \\c, 128, AG_B0\n\t"
        "AG_WF AG_B0, \\q, 3\n\t"
        "s_add_u32 %[need], %[need], 1\n\t"
        "AG_MIN\n\t"
        "AG_MM \\c+16, 128, AG_B1\n\t"
        ".if \\last == 0\n\t"
        "s_cmp_ge_u32 %[m], %[need]\n\t"
        "s_cbranch_scc1 .LAG_F\\@\n\t"
        ".endif\n\t"
        "AG_ST 0, \\r0\n\t"
        "s_waitcnt lgkmcnt(2)\n\t"
        "AG_MM \\c+32, 128, AG_B2\n\t"
        "s_waitcnt lgkmcnt(0)\n\t"
        "AG_MM \\c+48, 128, AG_B0\n\t"
        ".if \\last == 0\n\t"
        "AG_DONE\n\t"
        "AG_SPIN\n\t"
        "AG_WF AG_B1, (\\q)+1, 0\n\t"
        "AG_WF AG_B2, (\\q)+1, 1\n\t"
        "s_branch .LAG_J\\@\n\t"
        ".LAG_F\\@:\n\t"
        "AG_WF AG_B1, (\\q)+1, 0\n\t"
        "AG_ST 0, \\r0\n\t"
        "s_waitcnt lgkmcnt(4)\n\t"
        "AG_MM \\c+32, 128, AG_B2\n\t"
        "AG_WF AG_B2, (\\q)+1, 1\n\t"
        "s_waitcnt lgkmcnt(4)\n\t"
        "AG_MM \\c+48, 128, AG_B0\n\t"
        "AG_DONE\n\t"
        ".LAG_J\\@:\n\t"
        "AG_XF 128, (\\q)+1\n\t"
        "ds_read_b64 v[164:165], v162\n\t"
        "AG_WF AG_B0, (\\q)+1, 2\n\t"
        ".endif\n\t"
        "AG_ST 0, \\r1\n\t"
        ".endm\n\t"
        ".macro AG_SCALE\n\t"
        "ds_read_b32 v164, v162 offset:12\n\t"
        "s_nop 15\n\t"
        "s_nop 7\n\t"
        "s_waitcnt lgkmcnt(0)\n\t"
        "v_readfirstlane_b32 %[m], v164\n\t"
        "s_cmp_eq_u32 %[m], 0\n\t"
        "s_cselect_b32 %[scl], %[scale], 0x7fc00000\n\t"
        "ds_read_b64 v[164:165], v162\n\t"
        "s_waitcnt lgkmcnt(0)\n\t"
        ".endm\n\t"

        "v_mbcnt_lo_u32_b32 v167, -1, 0\n\t"
        "v_mbcnt_hi_u32_b32 v167, -1, v167\n\t"
        "v_lshrrev_b32 v165, 5, v167\n\t"
        "v_and_b32 v167, 31, v167\n\t"
        "v_lshlrev_b32 v167, 4, v167\n\t"
        "v_lshl_add_u32 v166, v165, 16, v167\n\t"
        "v_lshl_add_u32 v167, v165, 13, v167\n\t"
        "v_add_u32 v160, %[sa], v167\n\t"
        "v_add_u32 v161, %[sb], v167\n\t"
        "v_subrev_u32 v167, 65536-2048, v161\n\t"
        "v_mov_b32 v162, %[sc]\n\t"
        "v_mov_b32 v163, 0\n\t"
        ".set AG_I, 0\n\t"
        ".rept 128\n\t"
        "v_mov_b32 v[AG_I], 0\n\t"
        ".set AG_I, AG_I+1\n\t"
        ".endr\n\t"
        "s_mov_b32 vcc_lo, %[olo]\n\t"
        "s_mov_b32 vcc_hi, %[ohi]\n\t"
        "s_mov_b32 %[need], 1\n\t"
        "AG_SPIN\n\t"
        TS(ta) TC(ca)
        "AG_FRAG 144, v161, 0\n\t"
        "AG_FRAG 128, v160, 0\n\t"
        "AG_FRAG 136, v160, 2048\n\t"
        "AG_FRAG 152, v161, 1024\n\t"
        ".LAG_LOOP:\n\t"
        "AG_FULL 0, 16384\n\t"
        "AG_FULL 16384, 32768\n\t"
        "AG_FULL 32768, 49152\n\t"
        "AG_FULL 49152, 0\n\t"
        "s_cmp_le_u32 %[need], 64-" STR(NSPLIT) "\n\t"
        "s_cbranch_scc1 .LAG_LOOP\n\t"
        TS(td)
        ".if " STR(NSPLIT) "\n\t"
        "ds_read_b64 v[164:165], v162\n\t"
        "AG_WF 136, 0, 2\n\t"
        ".set AG_I, 0\n\t"
        ".rept " STR(NSPLIT) "\n\t"
        "AG_HALF 0, AG_I, 0, -1, -1\n\t"
        ".set AG_I, AG_I+1\n\t"
        ".endr\n\t"
        "AG_SCALE\n\t"
        TS(tb) TC(cc)
        ".set AG_I, 0\n\t"
        ".rept " STR(NSPLIT) "\n\t"
        ".set AG_R0, (AG_I*16+" STR(NSPLIT) "-1)/" STR(NSPLIT) "\n\t"
        ".set AG_R1, (AG_I*16+16+" STR(NSPLIT) "-1)/" STR(NSPLIT) "-1\n\t"
        ".if AG_R1 == AG_R0\n\t"
        ".set AG_R1, -1\n\t"
        ".endif\n\t"
        "AG_HALF 64, AG_I+" STR(NSPLIT) ", (AG_I==" STR(NSPLIT) "-1), AG_R0, AG_R1\n\t"
        ".set AG_I, AG_I+1\n\t"
        ".endr\n\t"
        "AG_SCALE\n\t"
        TS(tc) TC(cb)
        ".else\n\t"
        "AG_SCALE\n\t"
        TS(tc) TC(cb)
        ".irp r, 0, 1, 2, 3, 4, 5, 6, 7, 8, 9, 10, 11, 12, 13, 14, 15\n\t"
        "AG_ST 0, \\r\n\t"
        ".endr\n\t"
        ".endif\n\t"
        "s_add_u32 vcc_lo, vcc_lo, 96*16384\n\t"
        "s_addc_u32 vcc_hi, vcc_hi, 0\n\t"
        ".irp r, 0, 1, 2, 3, 4, 5, 6, 7, 8, 9, 10, 11, 12, 13, 14, 15\n\t"
        "AG_ST 64, \\r\n\t"
        ".endr\n\t"
        "s_nop 1\n\t"
        : [need] "=&s"(need), [m] "=&s"(m), [spin] "=&s"(spin), [scl] "=&s"(scl)
        : [sa] "s"(sa), [sb] "s"(sb), [sc] "s"(sc), [olo] "s"(olo), [ohi] "s"(ohi), [scale] "s"(scale)
        : "memory", "scc", "vcc", V10(), V10(1), V10(2), V10(3), V10(4), V10(5), V10(6), V10(7), V10(8), V10(9),
          V10(10), V10(11), V10(12), V10(13), V10(14), V10(15),
          "v160", "v161", "v162", "v163", "v164", "v165", "v166", "v167");
}

__device__ __forceinline__ void arrive(LAS unsigned char* L, int lane) {
    if (lane == 0) __hip_atomic_fetch_add((LAS u32*)(L + LDS_PRO), 1u, __ATOMIC_RELAXED, __HIP_MEMORY_SCOPE_WORKGROUP);
}

template <bool W> __device__ __forceinline__ void prologue_unit(const Params& P, LAS unsigned char* L, char* myws, u32 callv,
                                                                 int idx, int xg, int wave, int lane) {
    u32 d[16], x[16];
    __builtin_amdgcn_s_setprio(3);
    const int u = xg * 4 + wave;
    const bool half = !W && u >= (int)R0;
    load_unit<W>(d, P, idx, u, lane, half);
    encode_clear(L + LDS_A + wave * 8192, lane);
    char* peer = P.ws[lane & 7];
    BAR();
    WAIT_VM0();
    order_unit<W>(x, d);
    const u32 par = (UNI(callv) + 1) & 1;
    bool raw = RAWTAIL == 1 ? u == (int)ROUNDS - 1 && (!W || !NSPLIT)
             : RAWTAIL >= 2 && (W ? u == (int)ROUNDS - 1 : RAWTAIL == 3 && u == (int)ROUNDS - 2);
    const char* qa = myws + WS_IMG + par * IMG_BYTES + unit_off(idx, half ? u & ~1 : u) + (half ? (u & 1) * 2048 : 0);
    const char* qb = qa + (half ? 4096 : 2048);
    st_sys_s(lane * 16, qa, piece(x, 0)); st_sys_s(lane * 16, qa + 1024, piece(x, 1));
    st_sys_s(lane * 16, qb, piece(x, 2)); st_sys_s(lane * 16, qb + 1024, piece(x, 3));
    u32 len64 = 64;
    if (u >= NPUSH) len64 = encode_unit<W ? 2 : 3>(x, L + LDS_A + wave * 8192, L, myws + WS_CB + par * CB_BYTES + unit_off(idx, u), lane, raw);
    if (lane == 0) *(volatile LAS u32*)(L + LDS_LEN + wave * 4) = len64;
    WAIT_VM0();
    arrive(L, lane);
    if (xg == 0 && u >= NPUSH && lane < NR && lane != P.rank)
        asm volatile("global_store_dword %0, %1, off sc0 sc1" :: "v"(peer + WS_RDY + par * 32768 + (P.rank * 32 + idx) * 128 + u * 4),
                     "v"(((UNI(callv) + 1) << 7) | len64) : "memory");
    __builtin_amdgcn_s_setprio(0);
}

__global__ void __launch_bounds__(NTHREADS, 3) ag_gemm(Params P) {
    __shared__ __attribute__((aligned(16))) unsigned char lds_raw[LDS_BYTES];
    LAS unsigned char* L = (LAS unsigned char*)lds_raw;
    const int hw = UNI(threadIdx.x >> 6);
    const int wave = hw < 4 ? 11 - hw : hw - 4;
    const int lane = threadIdx.x & 63;
    const int bid = blockIdx.x;
    const int rank = P.rank;
    char* myws = P.myws;
    const int xg = bid & 7, idx = bid >> 3;
    const int rg = (xg + 5) & 7;
    const int tm = (xg >> 1) * 4 + (idx >> 3), tn = (xg & 1) * 8 + (idx & 7);
    GAS u32* tmo = (GAS u32*)(myws + WS_TMO);

    u32 call = __hip_atomic_load((GAS u32*)(myws + WS_CNT + bid * 128), __ATOMIC_RELAXED, __HIP_MEMORY_SCOPE_AGENT);

    if (wave < 8) {
        if (wave >= 4) {
            const u32 f = (wave - 4) * 64 + lane;
            u32 dl = 0, cl = 0x0C0C0C0Cu, k = 0;
#pragma unroll
            for (u32 i = 0; i < 4; ++i) {
                const u32 c = (f >> (2 * i)) & 3u;
                dl |= (c ? 4 + c : k) << (8 * i);
                cl ^= c ? 0u : (12u ^ i) << (8 * k);
                k += c == 0;
            }
            *(LAS u32*)(L + LDS_DL + f * 4) = dl;
            *(LAS u32*)(L + LDS_CL + f * 4) = cl;
            BAR();
        } else if (idx < 16) prologue_unit<false>(P, L, myws, call, idx, rg, wave, lane);
        else prologue_unit<true>(P, L, myws, call, idx, rg, wave, lane);
        const u32 scb = UNI(__builtin_bit_cast(u32, P.sx[0] * P.sw[0]));
        const int g = wave >> 1, nh = wave & 1;
        const u64 ob = (u64)P.out + ((size_t)(tm * 256 + g * 32) * MN + tn * 256 + nh * 128) * 4;
        u32 olo = (u32)ob, ohi = (u32)(ob >> 32);
        u64 ta = 0, tb = 0, tc = 0, td = 0, ca = 0, cb = 0, cc = 0;
        mfma_role((u32)(size_t)L + LDS_A + g * 512, (u32)(size_t)L + LDS_B + nh * 512, (u32)(size_t)L + LDS_CTL + wave * 16,
                  olo, ohi, scb, ta, tb, tc, td, ca, cb, cc);
    } else if (wave < 10) {
        __builtin_amdgcn_s_setprio(2);
        if (wave == 8) {
            ((LAS u32*)(L + LDS_CTL))[lane] = 0;
            char* peer = P.ws[lane & 7];
            BAR();
            call = UNI(call) + 1;
            STAMP(0);
            WAIT_VM0();
            for (u32 spin = 0; spin < SPINS && UNI(*(volatile LAS u32*)(L + LDS_PRO)) < 7; ++spin) __builtin_amdgcn_s_sleep(1);
            FENCE();
            const u32x4 ln = *(LAS u32x4*)(L + LDS_LEN);
            if (lane < NR) {
                if (lane != rank) { if (rg) st_sys(peer + WS_RDY + (call & 1) * 32768 + (rank * 32 + idx) * 128 + rg * 16, ln | (call << 7)); }
                else st_sys(peer + WS_UF + idx * 2048 + rg * 16, u32x4{call, call, call, call});
            }
            if (lane == 0) __hip_atomic_store((GAS u32*)(myws + WS_CNT + bid * 128), call, __ATOMIC_RELAXED, __HIP_MEMORY_SCOPE_AGENT);
            STAMP(7);
        } else {
            BAR();
            call = UNI(call) + 1;
            arrive(L, lane);
            u32 job = XCD_ID() * 32 + idx;
#pragma unroll 1
            for (int k = 0; k < NWG; ++k, job = (job + 1) & (NWG - 1)) {
                u32 old = 0;
                if (lane == 0) {
                    __hip_atomic_fetch_max((GAS u32*)(myws + WS_TIK + (job >> 5) * 128), call * 2048, __ATOMIC_RELAXED, __HIP_MEMORY_SCOPE_AGENT);
                    FENCE();
                    old = __hip_atomic_fetch_max((GAS u32*)(myws + WS_JOB + job * 128), call, __ATOMIC_RELAXED, __HIP_MEMORY_SCOPE_AGENT);
                }
                WAIT_VM0();
                if (UNI(old) < call) break;
            }
            *(volatile LAS u32*)(L + LDS_JOB) = job + 1;
        }
        const int isB = wave - 8;
        const int panel = isB ? 16 + tn : tm;
        const char* img = myws + WS_IMG + (call & 1) * IMG_BYTES + unit_off(panel, 0);
        const u32 ldsb = (u32)(size_t)L + (isB ? LDS_B : LDS_A);
        volatile LAS u32* ctl = (volatile LAS u32*)(L + LDS_CTL + (lane & 7) * 16);
        const char* uf = myws + WS_UF + panel * 2048;
        const u32 voff = lane * 16;
        Ld S = {0, 0, 0, 0, 16, 0, 0, 0, 0, 0};
        u32 err = 0;
        const u32 vctl = (u32)(size_t)ctl, vfull = vctl + isB * 4, vf = (lane & 31) * 4;
        const u64 t0 = rt();
        WAIT_VM0();
        asm volatile("v_mov_b32 v40, 0\n\tv_mov_b32 v41, 0\n\tv_mov_b32 v47, 0" ::: "v40", "v41", "v47");
        auto late = [&]() {
            if (UNI(__hip_atomic_load(tmo, __ATOMIC_RELAXED, __HIP_MEMORY_SCOPE_AGENT)) == call || rt() - t0 > DEADLINE) {
                if (lane == 0) __hip_atomic_store(tmo, call, __ATOMIC_RELAXED, __HIP_MEMORY_SCOPE_AGENT);
                ctl[3] = 1;
                err = ~0u;
            }
            asm volatile("s_waitcnt vmcnt(0) lgkmcnt(0)" ::: "memory");
            return rt() - t0 < 3 * DEADLINE;
        };
#define RUN(M, F, end) while (load_run<M, F>(S, end, (u32)(u64)img, (u32)((u64)img >> 32), ldsb, err, call, uf, voff, vf, vctl, vfull) && late()) {}
        constexpr u32 SE = SLOWEND < S0 ? SLOWEND : S0;
        RUN(0, false, 4 * SE);
        LS(9 + isB);
        if (SE < S0) RUN(0, true, 4 * S0);
        if (NSPLIT) {
            constexpr u32 LQ = LEAN ? NSTAGE : NQ;
            if (isB) RUN(2, F2, 4 * NQ) else RUN(1, F2, 4 * LQ)
            if (LEAN && S.ub < 4 * NQ) {
                const u32 q = S.ub >> 2;
                const char* ib = myws + WS_IMG + (call & 1) * IMG_BYTES + (size_t)(q - NSPLIT) * 524288;
                u64 gx = (u64)(ib + tm * 16384 + 2048), gw = (u64)(ib + (16 + tn) * 16384);
                u32 slot = (q - S0) % 5;
                if ((int)(S.pidx - S.nops) > 0) { S.pidx = S.nops; S.pb = S.ub - 32; }
                if (isB) { while (load_lean<2>(S, gx, gw, slot, (u32)(size_t)L + LDS_A, err, call, uf, voff, vf, vctl, vfull) && late()) {} }
                else { while (load_lean<1>(S, gx, gw, slot, (u32)(size_t)L + LDS_A, err, call, uf, voff, vf, vctl, vfull) && late()) {} }
            }
        }
        LS(14 + isB);
        RUN(0, true, 0);
        ctl[isB] = 0xFFFFu;
        WAIT_VM(0);
        STAMP(3 + isB);
    } else {
        __builtin_amdgcn_s_setprio(2);
        volatile LAS u32* fet = (volatile LAS u32*)(L + LDS_FET);
        volatile LAS u32* con = (volatile LAS u32*)(L + LDS_CON);
        volatile LAS u32* meta = (volatile LAS u32*)(L + LDS_META);
        constexpr u32 FIN = 0x40000000u;
        const u32 vo = lane * 16;
        if (wave == 10) {
#define FS(i) do {} while (0)
            BAR();
            call = UNI(call) + 1;
            arrive(L, lane);
            u32 job = 0;
            for (u32 spin = 0; spin < SPINS && job == 0; ++spin) { __builtin_amdgcn_s_sleep(1); job = UNI(*(volatile LAS u32*)(L + LDS_JOB)); }
            job = (job - 1) & (NWG - 1);
            const u32 slot = job >> 5, fp = job & 31, src = (P.srcmap >> (4 * slot)) & 7u;
            if (src == (u32)rank || (P.pad & 1)) return;
            GAS u32* tik = (GAS u32*)(myws + WS_TIK + slot * 128);
            const char* scb = P.ws[src] + WS_CB + (call & 1) * CB_BYTES;
            const char* rdy = myws + WS_RDY + (call & 1) * 32768 + src * 4096;
            const u32 lr = (u32)(size_t)L + LDS_RING;
            auto unit_of = [](u32 t) -> u32 {
                auto px = [](u32 k) { return 4 * (k & 3) + (k >> 2); };
                auto pw = [](u32 k) { return 16 + 8 * (k & 1) + (k >> 1); };
                if (t < T1) return ((t & 1) ? pw((t >> 1) & 15) : px((t >> 1) & 15)) * 32 + NPUSH + (t >> 5);
                if (t < T2) {
                    const u32 pr = (t - T1) / 48, i = (t - T1) % 48;
                    return ((i & 16) ? px(i & 15) : pw(i & 15)) * 32 + R0 + 2 * pr + (i >> 5);
                }
                return t < NT ? px((t - T2) & 15) * 32 + R0 + 2 * ((t - T2) >> 4) + 1 : 0u;
            };
            u32 t = fp, k = 0, un = unit_of(t), nv = 0, tv = call * 2048 + fp - 32;
            const u64 t0 = rt();
            constexpr int W = WN - 1;
            u32 lp = lr + 3 * UNIT + vo;
            for (; t < NT; ++k) {
                const char* q = scb + unit_off(un >> 5, un & 31);
                const u32 l = lr + (k & 3) * UNIT + vo;
                wait_vm<W>();
                asm volatile("" : "+v"(nv) :: "memory");
                u32 rv = UNI(nv);
                if (k == 0) FS(3);
                if ((rv >> 7) < call) {
                    if (k) { WAIT_VM(0); if (lane < 32) { PW(2, lp); PW(3, lp); PW(4, lp); PW(5, lp); PW(6, lp); PW(7, lp); } *fet = k; }
                    for (u32 n = 1;; ++n) {
                        rv = UNI(__hip_atomic_load((GAS u32*)(rdy + un * 4), __ATOMIC_RELAXED, __HIP_MEMORY_SCOPE_SYSTEM));
                        WAIT_VM(0);
                        if ((rv >> 7) >= call) break;
                        if ((n & 31) == 0 && (UNI(__hip_atomic_load(tmo, __ATOMIC_RELAXED, __HIP_MEMORY_SCOPE_AGENT)) == call || rt() - t0 > DEADLINE)) {
                            if (lane == 0) __hip_atomic_store(tmo, call, __ATOMIC_RELAXED, __HIP_MEMORY_SCOPE_AGENT);
                            *fet = k | FIN;
                            return;
                        }
                        __builtin_amdgcn_s_sleep(2);
                    }
                }
                if (k == 0) FS(4);
                if (k == 1) FS(5);
                if (k == 2) FS(6);
                if (k == 6) FS(7);
                u32 len64 = rv & 127u;
                len64 = len64 < 41 ? 41 : (len64 > 64 ? 64 : len64);
                const u32 len = len64 * 64, mt = un | ((len64 >> 6) << 10);
                if (lane < 32) { PL(0, q); PW(6, lp); PW(7, lp); if (W < 3) PW(3, lp); else PW(2, lp); }
                wait_vm<W>();
                if (lane < 32) { PL(1, q); if (W < 3) PW(4, lp); else PW(3, lp); }
                wait_vm<W>();
                if (lane < 32) { PL(2, q); if (W < 3) PW(5, lp); else PW(4, lp); }
                if (W < 3) *fet = k;
                asm volatile("" : "+v"(tv) :: "memory");
                t = UNI(tv) - call * 2048 + 64;
                un = unit_of(t);
                const bool last = t >= NT;
                if (W == 3) { wait_vm<W>(); if (lane < 32) PW(5, lp); *fet = k; }
                if (len > 3072) {
                    if (!last) wait_vm<W>();
                    if (lane < 32 && 3072 + vo < len) PL(6, q);
                    if (len > 3584) {
                        if (!last) wait_vm<W>();
                        if (lane < 32 && 3584 + vo < len) PL(7, q);
                    }
                }
                if (k >= 4)
                    for (u32 spin = 0; spin < SPINS && UNI(*con) + 4 <= k; ++spin) __builtin_amdgcn_s_sleep(1);
                meta[k & 3] = mt;
                if (!last) wait_vm<W>();
                {
                    u64 ex;
                    asm volatile("s_mov_b64 %2, exec\n\ts_mov_b64 exec, 1\n\tglobal_load_dword %1, %3, %6 sc0 sc1\n\t"
                                 "global_atomic_add %0, %3, %4, %5 sc0\n\ts_mov_b64 exec, %2"
                                 : "=&v"(tv), "=&v"(nv), "=&s"(ex) : "v"(0u), "v"(1u), "s"(tik), "s"(rdy + un * 4) : "memory");
                }
                if (lane < 32) { PL(3, q); if (W < 3) PW(0, l); }
                if (!last) wait_vm<W + 2>();
                if (lane < 32) { PL(4, q); if (W < 3) PW(1, l); else PW(0, l); }
                if (!last) wait_vm<(W < 2 ? W : W + 2)>();
                if (lane < 32 && 2560 + vo < len) PL(5, q);
                if (lane < 32) { if (W < 3) PW(2, l); else PW(1, l); }
                lp = l;
            }
            asm volatile("s_waitcnt vmcnt(0)" : "+v"(nv), "+v"(tv) :: "memory");
            if (lane < 32) { PW(0, lp); PW(1, lp); PW(2, lp); PW(3, lp); PW(4, lp); PW(5, lp); PW(6, lp); PW(7, lp); }
            *fet = k | FIN;
            STAMP(5);
        } else {
            {
                u32 d[16], x[16];
                const u32 pdst = (P.srcmap >> (4 * xg)) & 7u;
                const bool push = NPUSH && pdst != (u32)rank && !(P.pad & 1);
                if (push) { if (idx < 16) load_unit<false>(d, P, idx, 0, lane, false); else load_unit<true>(d, P, idx, 0, lane, false); }
                const u32 up = 32 + ((rank - pdst - 1) & 7);
                const char* pw = P.ws[pdst];
                const char* q0 = pw + WS_IMG + unit_off(idx, up);
                asm volatile("" : "+s"(q0));
                BAR();
                WAIT_VM0();
                call = UNI(call) + 1;
                if (push) {
                    if (idx < 16) order_unit<false>(x, d); else order_unit<true>(x, d);
                    q0 += (call & 1) * IMG_BYTES;
                    STAMP2(20);
                    st_sys_s(vo, q0, piece(x, 0)); st_sys_s(vo, q0 + 1024, piece(x, 1));
                    arrive(L, lane);
                    WAIT_VM(1); st_sys_s(vo, q0 + 2048, piece(x, 2));
                    WAIT_VM(1); st_sys_s(vo, q0 + 3072, piece(x, 3));
                    WAIT_VM(0); st_flag(pw + WS_UF + idx * 2048 + up * 4, call, lane);
                    STAMP(22);
                    STAMP3(20);
                } else {
                    arrive(L, lane);
                }
            }
            if (P.pad & 1) return;
            u32 job = 0;
            for (u32 spin = 0; spin < SPINS && job == 0; ++spin) { __builtin_amdgcn_s_sleep(1); job = UNI(*(volatile LAS u32*)(L + LDS_JOB)); }
            const u32 src = (P.srcmap >> (4 * (((job - 1) >> 5) & 7))) & 7u;
            if (src == (u32)rank) return;
            const char* dp = myws + WS_IMG + (call & 1) * IMG_BYTES;
            const char* uf = myws + WS_UF;
            const u32 u0 = 32 + ((src - rank - 1) & 7);
            const char* pend = nullptr;
            u32 pn = 1;
            auto publish = [&]() {
                WAIT_VM(0);
                if ((u32)lane < pn) asm volatile("global_store_dword %0, %1, %2 sc0 sc1" :: "v"(lane * 28), "v"(call), "s"(pend) : "memory");
                pend = nullptr;
            };
            for (u32 k = 0;; ++k) {
                u32 f = UNI(*fet);
                if ((f & ~FIN) <= k) {
                    if (pend) publish();
                    for (u32 spin = 0; spin < 2 * SPINS && f <= k; ++spin) { __builtin_amdgcn_s_sleep(1); f = UNI(*fet); }
                    if ((f & ~FIN) <= k) break;
                }
                FENCE();
                LAS unsigned char* sl = L + LDS_RING + (k & 3) * UNIT;
                const u32 mv = meta[k & 3], hx = *(LAS u32*)(sl + 3072), hw_ = *(LAS u32*)(sl + 2560);
                const u32x4 w = *(LAS u32x4*)(sl + lane * 16);
                const u32 cx = *(LAS u32*)(sl + 3076 + lane * 4), cw = *(LAS u32*)(sl + 2564 + lane * 4);
                FENCE(); SB();
                const u32 mt = UNI(mv), panel = (mt >> 5) & 31, r = mt & 31;
                u32x4 e[4];
                if (mt >> 10) {
                    e[0] = w;
#pragma unroll
                    for (int i = 1; i < 4; ++i) e[i] = *(LAS u32x4*)(sl + i * 1024 + lane * 16);
                } else if (panel < 16) {
                    decode_unit<3>(e, UNI(hx), w, cx, sl, L, lane);
                } else {
                    decode_unit<2>(e, UNI(hw_), w, cw, sl, L, lane);
                }
                asm volatile("" : "+v"(e[0]), "+v"(e[1]), "+v"(e[2]), "+v"(e[3]) :: "memory");
                *con = k + 1;
                if (pend) publish();
                const bool half = panel < 16 && r >= R0;
                const u32 u = u0 + 7 * (half ? r & ~1u : r);
                const char* qa = dp + unit_off(panel, u) + (half ? (r & 1) * 2048 : 0);
                const char* qb = half ? dp + unit_off(panel, u + 7) + (r & 1) * 2048 : qa + 2048;
                st_img(vo, qa, e[0]); st_img(vo, qa + 1024, e[1]);
                st_img(vo, qb, e[2]); st_img(vo, qb + 1024, e[3]);
                pend = uf + panel * 2048 + (u + (half && (r & 1) ? 4 * NSPLIT : 0)) * 4;
                pn = half ? 2 : 1;
            }
            STAMP(17);
        }
    }
}

extern "C" void kernel_launch(int rank, int nranks, ncclComm_t comm,
                              hipStream_t stream,
                              void* const* d_in, const int* in_sizes, int n_in,
                              void* d_out, int out_size,
                              const int* in_dtypes, int out_dtype,
                              void* const* d_out_all,
                              void* const* d_ws_all, size_t ws_bytes) {
    (void)comm; (void)in_sizes; (void)n_in; (void)out_size; (void)in_dtypes; (void)out_dtype; (void)d_out_all;
    if (nranks != NR || (unsigned)rank >= (unsigned)NR || ws_bytes < WS_NEED) {
        fprintf(stderr, "kernel_launch: unexpected configuration\n");
        return;
    }
    Params p;
    p.x = (const unsigned char*)d_in[0];
    p.w = (const unsigned char*)d_in[1];
    p.sx = (const float*)d_in[2];
    p.sw = (const float*)d_in[3];
    p.out = (float*)d_out;
    for (int i = 0; i < NR; ++i) p.ws[i] = (char*)d_ws_all[i];
    p.myws = p.ws[rank];
    p.rank = rank;
    p.pad = 0;
    p.pad2 = 0;
    static const u32 near_link[NR] = {0x74510263u, 0x26301475u, 0x17542036u, 0x74153620u,
                                      0x63024157u, 0x60235741u, 0x51476302u, 0x02367514u};
    p.srcmap = near_link[rank];
    hipLaunchKernelGGL(ag_gemm, dim3(NWG), dim3(NTHREADS), 0, stream, p);
    HIP_CHECK(hipGetLastError());
}
```

```cpp
#include <hip/hip_runtime.h>
#include <rccl/rccl.h>
#include <cstdio>
#include <cstddef>

#define HIP_CHECK(expr) do { hipError_t _e = (expr); if (_e != hipSuccess) { \
    fprintf(stderr, "HIP error %s at %s:%d\n", hipGetErrorString(_e), __FILE__, __LINE__); \
    return; } } while (0)

#define MSLEEP 1
#define GAS __attribute__((address_space(1)))
#define LAS __attribute__((address_space(3)))
typedef unsigned u32;
typedef unsigned short u16;
typedef unsigned long long u64;
typedef unsigned u32x2 __attribute__((ext_vector_type(2)));
typedef unsigned u32x4 __attribute__((ext_vector_type(4)));
typedef int i32x4 __attribute__((ext_vector_type(4)));
typedef int i32x8 __attribute__((ext_vector_type(8)));
typedef float f32x4 __attribute__((ext_vector_type(4)));
typedef float f32x16 __attribute__((ext_vector_type(16)));

constexpr int NR = 8;
constexpr int MN = 4096;
constexpr int KS = 512;
constexpr int NWG = 256;
constexpr int NTHREADS = 768;
constexpr u32 ROUNDS = 32;
constexpr u32 NSTAGE = 64;
#define RATE 110
#define LATHI 120
#define PEN 8
#define NOWAIT 12
#define NPW 2
#define PWIN 0
#define QEASY 32
#define NEASY 4
#define ARDY 1
#define RDYLD "global_atomic_or v162, %0, %0, %3 sc0 sc1"
#define TOUCH 1
#define LDELAY 500
#define RAWTAIL 0
#define NSPLIT 8
constexpr u32 S0 = NSTAGE - NSPLIT;
constexpr u32 NQ = NSTAGE + NSPLIT;
constexpr u32 R0 = NSPLIT ? ((S0 * 4 - 32 + 13) / 14) * 2 : ROUNDS;
constexpr u32 US = 32 + 7 * R0;
constexpr u32 NPAIR = (ROUNDS - R0) / 2;
constexpr u32 T0 = 128 - 16 * NPW, T1 = T0 + (R0 - 4) * 32, T2 = T1 + NPAIR * 48, NT = T2 + NPAIR * 16;
constexpr u32 UNIT = 4096;
constexpr size_t IMG_BYTES = 32u << 20;
constexpr size_t CB_BYTES = 4u << 20;
__host__ __device__ constexpr u32 unit_off(u32 panel, u32 u) { return (u >> 2) * 524288u + panel * 16384u + (u & 3) * UNIT; }
constexpr size_t WS_CNT = 0;
constexpr size_t WS_TMO = 48 << 10;
constexpr size_t WS_TIK = 52 << 10;
constexpr size_t WS_CLK = 56 << 10;
constexpr size_t WS_UF = 64 << 10;
constexpr size_t WS_RDY = 128 << 10;
constexpr size_t WS_JOB = 192 << 10;
constexpr size_t WS_DBG = 256 << 10;
constexpr size_t WS_IMG = 1 << 20;
constexpr size_t WS_CB = WS_IMG + 2 * IMG_BYTES;
constexpr size_t WS_NEED = WS_CB + 2 * CB_BYTES;
constexpr u64 DEADLINE = 50000000ull;
constexpr u32 SPINS = 1u << 25;
constexpr u32 LDS_CTL = 0;
constexpr u32 LDS_XCH = LDS_CTL + 136, LDS_PRO = LDS_CTL + 140;
constexpr u32 LDS_JOB = LDS_CTL + 160, LDS_LEN = LDS_CTL + 176;
constexpr u32 LDS_DL = 512, LDS_CL = LDS_DL + 1024;
constexpr u32 LDS_WRK = LDS_CL + 1024, WRK_BYTES = 7168;
constexpr u32 LDS_A = LDS_WRK + 2 * WRK_BYTES, LDS_B = LDS_A + 65536;
constexpr u32 LDS_BYTES = LDS_B + 65536;

struct Params {
    const unsigned char* x;
    const unsigned char* w;
    char* myws;
    int rank;
    u32 srcmap;
    const float* sx;
    const float* sw;
    float* out;
    int pad;
    u32 pad2;
    char* ws[NR];
};

__device__ __forceinline__ u64 rt() { return __builtin_amdgcn_s_memrealtime(); }
#define WAIT_VM(n) asm volatile("s_waitcnt vmcnt(" #n ")" ::: "memory")
#define WAIT_VM0() do { __builtin_amdgcn_s_waitcnt(0x0F70); WAIT_VM(0); } while (0)
#define FENCE() asm volatile("" ::: "memory")
#define SB() __builtin_amdgcn_sched_barrier(0)
#define XCD_ID() (__builtin_amdgcn_s_getreg((3 << 11) | 20) & 7u)
#define UNI(x) __builtin_amdgcn_readfirstlane(x)

#define STAMP(i) do {} while (0)
#define STAMP2(i) do {} while (0)
#define STAMP3(i) do {} while (0)

#define OB0 ""
#define OB1 ""
__device__ __forceinline__ void st_sys(char* p, u32x4 v) {
    asm volatile("global_store_dwordx4 %0, %1, off sc0 sc1\n\ts_nop 1" :: "v"(p), "v"(v) : "memory");
}
__device__ __forceinline__ void st_sys_s(u32 voff, const void* base, u32x4 v) {
    asm volatile("global_store_dwordx4 %0, %1, %2 sc0 sc1\n\ts_nop 1" :: "v"(voff), "v"(v), "s"(base) : "memory");
}
__device__ __forceinline__ void st_flag(const void* base, u32 v, int lane) {
    if (lane == 0) asm volatile("global_store_dword %0, %1, %2 sc0 sc1" :: "v"(0u), "v"(v), "s"(base) : "memory");
}

#define DBITS " sc0 sc1"
#define UBITS " sc0 sc1"
#define FBITS " sc0 sc1"
#define STR2(x) #x
#define STR(x) STR2(x)
#define ISLP 1
#define LS(i) do {} while (0)
struct Ld {
    u32 nops, q, rmask, nf;
    u32 pidx, pb, slot;
    u64 fifo, g, gw;
};
#define G2(v, s_, o) "global_load_lds_dwordx4 " v ", " s_ " offset:" #o DBITS "\n\tglobal_load_lds_dwordx4 " v ", " s_ " offset:" #o "+1024" DBITS "\n\t"
#define EASE ".if %c[ease]\n\ts_sleep %c[ease]\n\t.endif\n\t"
#define M0A(x) "s_add_u32 m0, m0, " x "\n\ts_nop 0\n\t"
#define GW(v) G2(v, "%[g]", 0) M0A("%[land]") G2(v, "%[g]", 2048)
template <int MODE, int EASY = 0>
__device__ __forceinline__ u32 load_run(Ld& S, u32 end, u32 lds, u32 err, u32 call, const void* uf,
                                        u32 vo, u32 vf, u32 vctl, u32 vfull) {
    u32 a, b, land, rc, keep;
    asm volatile(
        "s_mov_b32 %[keep], m0\n\t"
        "s_mov_b32 %[rc], 0\n\t"
        "s_branch .LL%=\n\t"
        ".LZ%=:\n\t"
        ".if %c[md] == 0\n\t"
        "s_sleep " STR(ISLP) "\n\t"
        ".endif\n\t"
        ".LL%=:\n\t"
        "ds_read_b32 v42, %[vctl] offset:8\n\t"
        "s_getreg_b32 %[a], hwreg(HW_REG_IB_STS)\n\t"
        "s_bfe_u32 %[b], %[a], 0x20016\n\t"
        "s_and_b32 %[a], %[a], 15\n\t"
        "s_lshl4_add_u32 %[a], %[b], %[a]\n\t"
        "s_sub_u32 %[land], %[nops], %[a]\n\t"
        "s_mov_b64 vcc, %[fifo]\n\t"
        "s_sub_u32 %[a], %[land], vcc_lo\n\t"
        "s_bitcmp0_b32 %[a], 11\n\t"
        "s_cselect_b32 %[a], %[nf], 0\n\t"
        "s_cmp_eq_u32 %[a], 0\n\t"
        "s_cbranch_scc1 .LP%=\n\t"
        "v_add_u32 v41, 1, v41\n\t"
        "s_lshr_b64 %[fifo], %[fifo], 12\n\t"
        "ds_write_b32 %[vfull], v41\n\t"
        "s_sub_u32 %[nf], %[nf], 1\n\t"
        ".LP%=:\n\t"
        "s_add_u32 %[rc], %[rc], 0x100000\n\t"
        "s_cbranch_scc1 .LT%=\n\t"
        ".if %c[md] == 5\n\t"
        "s_waitcnt lgkmcnt(0)\n\t"
        "s_cmp_lg_u32 %[nf], 0\n\t"
        "s_cbranch_scc1 .LL%=\n\t"
        ".else\n\t"
        ".if %c[md] != 4\n\t"
        "s_or_b32 %[rmask], %[rmask], %[err]\n\t"
        "s_sub_u32 %[a], %[land], %[pidx]\n\t"
        "s_cmp_lt_i32 %[a], 0\n\t"
        "s_cbranch_scc1 .LO%=\n\t"
        "s_cmp_eq_u32 %[pb], -1\n\t"
        "s_cbranch_scc1 .LS%=\n\t"
        "v_cmp_le_u32 vcc, %[call], v40\n\t"
        "s_sub_u32 %[a], %[q], %[pb]\n\t"
        "s_lshl_b32 %[a], %[a], 1\n\t"
        "s_min_u32 %[a], %[a], 16\n\t"
        "s_lshr_b32 %[b], vcc_lo, %[a]\n\t"
        "s_lshr_b32 %[b], %[b], %[a]\n\t"
        "s_or_b32 %[rmask], %[rmask], %[b]\n\t"
        "s_mov_b32 %[pb], -1\n\t"
        "s_andn2_b32 %[b], 15, %[rmask]\n\t"
        "s_cbranch_scc0 .LO%=\n\t"
        ".LS%=:\n\t"
        "s_andn2_b32 %[b], 0xffff, %[rmask]\n\t"
        "s_cbranch_scc0 .LO%=\n\t"
        "s_lshl_b32 %[b], %[q], 4\n\t"
        "v_add_u32 v43, %[b], %[vf]\n\t"
        ".if %c[md] & 1\n\t"
        "v_subrev_u32 v40, 0x400, v43\n\t"
        "v_cmp_gt_u32 vcc, %[whole], v40\n\t"
        "v_subrev_u32 v40, 16*" STR(NSPLIT) ", v43\n\t"
        "s_nop 0\n\t"
        "v_cndmask_b32 v43, v43, v40, vcc\n\t"
        ".endif\n\t"
        "s_add_u32 %[nops], %[nops], 1\n\t"
        "s_mov_b32 exec_hi, 0\n\t"
        "global_atomic_or v40, v43, v47, %[uf] sc0 sc1\n\t"
        "s_mov_b32 exec_hi, -1\n\t"
        "s_mov_b32 %[pidx], %[nops]\n\t"
        "s_mov_b32 %[pb], %[q]\n\t"
        ".LO%=:\n\t"
        ".endif\n\t"
        ".if %c[md] == 0\n\t"
        "s_sub_u32 %[a], %[q], 3\n\t"
        ".else\n\t"
        "s_cmp_ge_u32 %[q], 68-" STR(NSPLIT) "\n\t"
        "s_subb_u32 %[a], %[q], 3\n\t"
        ".endif\n\t"
        "s_waitcnt lgkmcnt(0)\n\t"
        "v_cmp_gt_i32 vcc, %[a], v42\n\t"
        "s_lshl_b32 %[a], %[slot], 14\n\t"
        ".if %c[md]\n\t"
        "s_add_u32 %[b], %[a], 0x10000\n\t"
        "s_cmp_eq_u32 %[slot], 4\n\t"
        "s_cselect_b32 %[a], 2*16384+2048, %[a]\n\t"
        "s_cselect_b32 %[b], 2048, %[b]\n\t"
        "s_cselect_b32 %[land], 16384-2048, 0\n\t"
        ".endif\n\t"
        ".if %c[md] != 4\n\t"
        "s_andn2_b32 m0, 15, %[rmask]\n\t"
        "s_or_b32 m0, m0, vcc_lo\n\t"
        ".else\n\t"
        "s_cmp_lg_u32 vcc_lo, 0\n\t"
        ".endif\n\t"
        "s_cbranch_scc1 .LZ%=\n\t"
        ".if %c[md] == 0\n\t"
        "s_add_u32 m0, %[a], %[lds]\n\t"
        "s_add_u32 %[nops], %[nops], 16\n\t"
        G2("%[vo]", "%[g]", 0) G2("%[vo]", "%[g]", 2048) EASE M0A("0x1000") G2("v44", "%[g]", 0) G2("v44", "%[g]", 2048) EASE M0A("0x1000")
        G2("v45", "%[g]", 0) G2("v45", "%[g]", 2048) EASE M0A("0x1000") G2("v46", "%[g]", 0) G2("v46", "%[g]", 2048)
        ".elseif %c[md] & 1\n\t"
        "s_add_u32 m0, %[a], %[lds]\n\t"
        "s_add_u32 %[nops], %[nops], 6+2*%c[md]\n\t"
        G2("%[vo]", "%[g]", 0) M0A("0x1000") G2("v44", "%[g]", 0) M0A("0x1000") G2("v45", "%[g]", 0) M0A("0x1000") G2("v46", "%[g]", 0)
        ".if %c[md] == 3\n\t"
        "s_add_u32 %[b], %[b], %[lds]\n\t"
        "s_add_u32 m0, %[b], 0x3000\n\t"
        "s_mov_b64 vcc, %[gw]\n\t"
        G2("v46", "vcc", 0) M0A("%[land]") G2("v46", "vcc", 2048)
        "s_add_u32 vcc_lo, vcc_lo, 0x80000\n\t"
        "s_addc_u32 vcc_hi, vcc_hi, 0\n\t"
        "s_mov_b64 %[gw], vcc\n\t"
        ".endif\n\t"
        ".else\n\t"
        "s_add_u32 m0, %[b], %[lds]\n\t"
        "s_sub_u32 %[a], 0x1000, %[land]\n\t"
        "s_add_u32 %[nops], %[nops], 20-2*%c[md]\n\t"
        GW("%[vo]") M0A("%[a]") GW("v44") M0A("%[a]") GW("v45")
        ".if %c[md] == 2\n\t"
        M0A("%[a]") GW("v46")
        ".endif\n\t"
        ".endif\n\t"
        "s_mov_b64 vcc, %[g]\n\t"
        "s_add_u32 vcc_lo, vcc_lo, 0x80000\n\t"
        "s_addc_u32 vcc_hi, vcc_hi, 0\n\t"
        "s_mov_b64 %[g], vcc\n\t"
        "s_and_b32 vcc_lo, %[nops], 0xfff\n\t"
        "s_mov_b32 vcc_hi, 0\n\t"
        "s_mul_i32 %[a], %[nf], 12\n\t"
        "s_lshl_b64 vcc, vcc, %[a]\n\t"
        "s_or_b64 %[fifo], %[fifo], vcc\n\t"
        "s_add_u32 %[nf], %[nf], 1\n\t"
        "s_lshr_b32 %[rmask], %[rmask], 4\n\t"
        "s_add_u32 %[slot], %[slot], 1\n\t"
        "s_cmp_eq_u32 %[slot], 4+(%c[md]+7)/8\n\t"
        "s_cselect_b32 %[slot], 0, %[slot]\n\t"
        "s_add_u32 %[q], %[q], 1\n\t"
        "s_cmp_lt_u32 %[q], %[end]\n\t"
        "s_cbranch_scc1 .LL%=\n\t"
        ".endif\n\t"
        "s_mov_b32 %[rc], 0\n\t"
        "s_branch .LE%=\n\t"
        ".LT%=:\n\t"
        "s_mov_b32 %[rc], 1\n\t"
        "s_waitcnt lgkmcnt(0)\n\t"
        ".LE%=:\n\t"
        "s_mov_b32 m0, %[keep]"
        : [nops] "+s"(S.nops), [q] "+s"(S.q), [rmask] "+s"(S.rmask), [nf] "+s"(S.nf), [pidx] "+s"(S.pidx), [pb] "+s"(S.pb),
          [slot] "+s"(S.slot), [fifo] "+s"(S.fifo), [g] "+s"(S.g), [gw] "+s"(S.gw),
          [a] "=&s"(a), [b] "=&s"(b), [land] "=&s"(land), [rc] "=&s"(rc), [keep] "=&s"(keep)
        : [end] "s"(end), [lds] "s"(lds), [err] "s"(err), [call] "s"(call), [uf] "s"(uf),
          [vo] "v"(vo), [vf] "v"(vf), [vctl] "v"(vctl), [vfull] "v"(vfull), [md] "i"(MODE), [ease] "i"(EASY), [whole] "i"(4 * (US - 4 * S0))
        : "memory", "scc", "vcc", "v40", "v41", "v42", "v43", "v44", "v45", "v46", "v47");
    return rc;
}
#define PIECE(i, g) do { u32 k_; asm volatile("s_mov_b32 %0, m0\n\ts_mov_b32 m0, %3\n\ts_nop 0\n\t" \
    "global_load_lds_dwordx4 %1, %2 offset:1024*" #i FBITS "\n\ts_mov_b32 m0, %0" : "=&s"(k_) : "v"(vo), "s"(g), "s"((u32)(size_t)sl) : "memory"); } while (0)
template <int N> __device__ __forceinline__ void wait_vm() { asm volatile("s_waitcnt vmcnt(%0)" :: "i"(N) : "memory"); }
__device__ __forceinline__ bool vm_idle() {
    u32 r;
    asm volatile("s_getreg_b32 %0, hwreg(HW_REG_IB_STS)" : "=s"(r) :: "memory");
    return (r & 0xC0000Fu) == 0;
}
__device__ __forceinline__ void st_img(u32 voff, const void* base, u32x4 v) {
    asm volatile("global_store_dwordx4 %0, %1, %2" UBITS "\n\ts_nop 1" :: "v"(voff), "v"(v), "s"(base) : "memory");
}
__device__ __forceinline__ u32 perm(u32 hi, u32 lo, u32 sel) { return __builtin_amdgcn_perm(hi, lo, sel); }
__device__ __forceinline__ u32 bfi(u32 m, u32 a, u32 b) { return __builtin_amdgcn_bitop3_b32(m, a, b, 0xCA); }
__device__ __forceinline__ u32 and_or(u32 a, u32 m, u32 c) { return __builtin_amdgcn_bitop3_b32(a, m, c, 0xEA); }
__device__ __forceinline__ u32 xor_and(u32 a, u32 b, u32 m) { return __builtin_amdgcn_bitop3_b32(a, b, m, 0x78); }
#define BAR() asm volatile("s_waitcnt lgkmcnt(0)\n\ts_barrier" ::: "memory")
template <int S> __device__ __forceinline__ u32 shl(u32 v) { return S >= 0 ? v << (S >= 0 ? S : 0) : v >> (S < 0 ? -S : 0); }
__device__ __forceinline__ u32 scan64(u32 v) {
    v += __builtin_amdgcn_update_dpp(0u, v, 0x111, 0xf, 0xf, true);
    v += __builtin_amdgcn_update_dpp(0u, v, 0x112, 0xf, 0xf, true);
    v += __builtin_amdgcn_update_dpp(0u, v, 0x114, 0xf, 0xf, true);
    v += __builtin_amdgcn_update_dpp(0u, v, 0x118, 0xf, 0xf, true);
    v += __builtin_amdgcn_update_dpp(0u, v, 0x142, 0xa, 0xf, false);
    v += __builtin_amdgcn_update_dpp(0u, v, 0x143, 0xc, 0xf, false);
    return v;
}
__device__ __forceinline__ void lds_or(LAS unsigned char* p, u32 v) {
    __hip_atomic_fetch_or((LAS u32*)p, v, __ATOMIC_RELAXED, __HIP_MEMORY_SCOPE_WORKGROUP);
}

__device__ __forceinline__ void load_w(u32 (&d)[16], const unsigned char* w, int panel, int r, int lane) {
    const auto rs = __builtin_amdgcn_make_buffer_rsrc((void*)w, 0, MN * KS, 0x00020000);
    const u32 so = r * 16 * MN + (panel & 15) * 256;
#pragma unroll
    for (int k = 0; k < 16; ++k) d[k] = __builtin_amdgcn_raw_buffer_load_b32(rs, lane * 4, so + k * MN, 0);
}
__device__ __forceinline__ void order_w(u32 (&x)[16], const u32 (&d)[16]) {
#pragma unroll
    for (int q = 0; q < 4; ++q) {
        u32 t0 = perm(d[4 * q + 1], d[4 * q], 0x05010400), t1 = perm(d[4 * q + 1], d[4 * q], 0x07030602);
        u32 t2 = perm(d[4 * q + 3], d[4 * q + 2], 0x05010400), t3 = perm(d[4 * q + 3], d[4 * q + 2], 0x07030602);
        x[q] = perm(t2, t0, 0x05040100); x[4 + q] = perm(t2, t0, 0x07060302);
        x[8 + q] = perm(t3, t1, 0x05040100); x[12 + q] = perm(t3, t1, 0x07060302);
    }
}
__device__ __forceinline__ u32x4 piece(const u32 (&x)[16], int i) { return u32x4{x[4 * i], x[4 * i + 1], x[4 * i + 2], x[4 * i + 3]}; }

#define ES(i) do {} while (0)
template <int MB> struct Fmt {
    static constexpr u32 EM = (MB == 3 ? 0x0Fu : 0x1Fu) * 0x01010101u;
    static constexpr u32 HEAD = MB == 3 ? 3072 : 2048;
    static constexpr u32 OFF_H = MB == 3 ? 0 : 512, OFF_2 = OFF_H + 4;
};
__device__ __forceinline__ u32 zero2(u32 w) { return __builtin_amdgcn_bitop3_b32(w, w >> 1, 0x55555555u, 0x02); }
__device__ __forceinline__ u32 code_of(u32 hi, u32 lo, u32 v) {
    const u32 r = perm(hi, lo, v);
    return __builtin_amdgcn_bitop3_b32(r, r >> 2, 0x03030303u, 0x28);
}
__device__ __forceinline__ void lookup4(u32* sel, u32 w, LAS unsigned char* tab) {
#pragma unroll
    for (int j = 0; j < 4; ++j) sel[j] = *(LAS u32*)(tab + ((w >> (8 * j)) & 255u) * 4);
}
__device__ __forceinline__ void gather4(const u32* v, const u32* sel, u32 z, u32 a) {
#pragma unroll
    for (int j = 0; j < 4; ++j) {
        const u32 o = j ? a + __builtin_popcount(z & ((1u << (8 * j)) - 1)) : a;
        const u32 c = perm(0u, v[j], sel[j]);
        LAS unsigned char* p = (LAS unsigned char*)(size_t)(o & ~3u);
        lds_or(p, __builtin_amdgcn_alignbyte(c, 0u, ~o));
        lds_or(p + 4, __builtin_amdgcn_alignbyte(0u, c, ~o));
    }
}
__device__ __forceinline__ void window4(u32* o, u32x2* q, u32 z, u32 a) {
#pragma unroll
    for (int j = 0; j < 4; ++j) {
        o[j] = j ? a + __builtin_popcount(z & ((1u << (8 * j)) - 1)) : a;
        LAS u32* p = (LAS u32*)(size_t)(o[j] & ~3u);
        q[j] = u32x2{p[0], p[1]};
    }
}
__device__ __forceinline__ u32 spread(u32 tab, u32 o, u32x2 q, u32 sel) {
    return perm(tab, __builtin_amdgcn_alignbyte(q[1], q[0], o), sel);
}
__device__ __forceinline__ u32 pack4(const u32* c) {
    u32 w = __builtin_amdgcn_udot4(c[0], 0x40100401u, 0u, false);
#pragma unroll
    for (int j = 1; j < 4; ++j) w |= __builtin_amdgcn_udot4(c[j], 0x40100401u, 0u, false) << (8 * j);
    return w;
}
__device__ __forceinline__ void encode_clear(LAS unsigned char* T, int lane) {
#pragma unroll
    for (int i = 2; i < 5; ++i) *(LAS u32x4*)(T + i * 1024 + lane * 16) = u32x4{0, 0, 0, 0};
}

template <int MB> __device__ __forceinline__ u32 encode_unit(const u32 (&x)[16], LAS unsigned char* T, LAS unsigned char* L,
                                                             char* dst, int lane, bool raw) {
    typedef Fmt<MB> F;
    u32 sum = 0;
#pragma unroll
    for (int j = 0; j < 16; j += 5) sum = __builtin_amdgcn_udot4(x[j] & 0x7F7F7F7Fu, 0x01010101u, sum, false);
    const int tot = __builtin_amdgcn_readlane(scan64(sum), 63);
    const u32 eb = (u32)((tot - (1516 << MB)) >> (10 + MB)) & (F::EM & 31u);
    const u32 B = (eb << MB) * 0x01010101u;
    u32 v[16], c[16];
    u32x4 w, z;
#pragma unroll
    for (int j = 0; j < 16; ++j) {
        v[j] = (((x[j] | 0x80808080u) - B) >> MB) & F::EM;
        c[j] = code_of(0u, 0x03020100u, v[j]);
    }
    u32 cnt[5] = {0};
#pragma unroll
    for (int q = 0; q < 4; ++q) {
        w[q] = pack4(c + 4 * q); z[q] = zero2(w[q]); cnt[q + 1] = cnt[q] + __builtin_popcount(z[q]);
        lookup4(c + 4 * q, w[q], L + LDS_CL);
    }
    FENCE();
    const u32 incl = scan64(cnt[4]), n1 = __builtin_amdgcn_readlane(incl, 63);
    ES(0);
    u32 n2 = 0, n3 = 0, len64 = 64;
    raw = raw || n1 > 2040;
    const u32 off3 = F::OFF_2 + ((n1 + 15) >> 4) * 4;
    if (!raw) {
        const u32 a1 = incl - cnt[4] + ((u32)(size_t)T + 2047);
#pragma unroll
        for (int q = 0; q < 4; ++q) gather4(v + 4 * q, c + 4 * q, z[q], a1 + cnt[q]);
        ES(1);
        st_sys_s(lane * 16, dst, w);
        if (MB == 3) {
            u32x4 o[2];
#pragma unroll
            for (int p = 0; p < 8; ++p) o[p >> 2][p & 3] = bfi(0x87878787u, x[2 * p], bfi(0xF0F0F0F0u, x[2 * p + 1] << 4, x[2 * p + 1] >> 4));
            st_sys_s(lane * 16, dst + 1024, o[0]);
            st_sys_s(lane * 16, dst + 2048, o[1]);
        } else {
            u32x4 o;
            u32x2 sg;
#pragma unroll
            for (int q = 0; q < 4; ++q)
                o[q] = bfi(0x0F0F0F0Fu, bfi(0x03030303u, x[4 * q], x[4 * q + 1] << 2), bfi(0x03030303u, x[4 * q + 2], x[4 * q + 3] << 2) << 4);
#pragma unroll
            for (int k = 0; k < 2; ++k) {
                u32 t = x[8 * k];
#pragma unroll
                for (int j = 1; j < 8; ++j) t = bfi(0x80808080u >> j, x[8 * k + j] >> j, t);
                sg[k] = t;
            }
            *(LAS u32x2*)(T + lane * 8) = sg;
            st_sys_s(lane * 16, dst + 1024, o);
        }
        FENCE();
        ES(2);
        for (u32 b = 0; b < n1; b += 1024) {
            const u32x4 y = *(LAS u32x4*)(T + 2048 + b + lane * 16);
            u32 u[4], c2[4];
#pragma unroll
            for (int j = 0; j < 4; ++j) {
                c2[j] = code_of(0x00000300u, 0x00000102u, (y[j] + 0x01010101u) & F::EM);
                u[j] = (y[j] + 0x05050505u) & F::EM;
            }
            const u32 w2 = pack4(c2);
            lookup4(c2, w2, L + LDS_CL);
            FENCE();
            const u32 z2 = zero2(w2), k2 = __builtin_popcount(z2), i2 = scan64(k2);
            if (b + lane * 16 < n1) *(LAS u32*)(T + F::OFF_2 + (b >> 2) + lane * 4) = w2;
            const u32 all = n2 + __builtin_amdgcn_readlane(i2, 63);
            if (all <= 1016 && k2) gather4(u, c2, z2, n2 + i2 - k2 + ((u32)(size_t)T + 4095));
            n2 = all;
        }
        raw = n2 > 1016;
        ES(3);
    }
    if (!raw) {
        FENCE();
        const u32x4 y = *(LAS u32x4*)(T + 4096 + lane * 16);
        u32 k3 = 0, m[4];
#pragma unroll
        for (int j = 0; j < 4; ++j) { m[j] = ((y[j] & 0x1C1C1C1Cu) + 0x1C1C1C1Cu) & 0x20202020u; k3 += __builtin_popcount(m[j]); }
        const u32 i3 = scan64(k3);
        n3 = __builtin_amdgcn_readlane(i3, 63);
        const u32 offp = off3 + ((n2 + 15) >> 4) * 4;
        len64 = (F::HEAD + offp + 2 * n3 + 63) >> 6;
        raw = len64 >= 64 || n3 > 63;
        if (!raw) {
            if (lane * 16 < n2)
                *(LAS u32*)(T + off3 + lane * 4) = bfi(0x0F0F0F0Fu, bfi(0x03030303u, y[0], y[1] << 2), bfi(0x03030303u, y[2], y[3] << 2) << 4);
            if (lane == 0) *(LAS u32*)(T + F::OFF_H) = eb | (n1 << 5) | (n2 << 16) | (n3 << 26);
            LAS unsigned char* p = T + offp + 2 * (i3 - k3);
#pragma unroll
            for (int j = 0; j < 4; ++j) {
                u32 mm = m[j];
                while (mm) {
                    const u32 b = __builtin_ctz(mm) & 24u;
                    mm &= mm - 1;
                    *(LAS u16*)p = (u16)(((lane * 16 + j * 4 + (b >> 3)) << 5) | ((y[j] >> b) & 31u));
                    p += 2;
                }
            }
        }
    }
    if (raw) {
        WAIT_VM(0);
#pragma unroll
        for (int i = 0; i < 4; ++i) st_sys_s(lane * 16, dst + i * 1024, piece(x, i));
        return 64;
    }
    FENCE();
    const u32 tl = len64 * 64 - F::HEAD;
    if (lane * 16 < tl) st_sys_s(lane * 16, dst + F::HEAD, *(LAS u32x4*)(T + lane * 16));
    if (MB == 2 && 1024 + lane * 16 < tl) st_sys_s(lane * 16, dst + F::HEAD + 1024, *(LAS u32x4*)(T + 1024 + lane * 16));
    return len64;
}

template <int MB> __device__ __forceinline__ void decode_unit(const char* qa, const char* qb, u32 hdr, u32x4 w, u32 w2,
                                                              LAS unsigned char* S, LAS unsigned char* L, int lane) {
    typedef Fmt<MB> F;
    LAS unsigned char* T = S + F::HEAD;
    const u32 eb = hdr & 31u, n1 = (hdr >> 5) & 2047u, n2 = (hdr >> 16) & 1023u, n3 = hdr >> 26;
    const u32 off3 = F::OFF_2 + ((n1 + 15) >> 4) * 4, offp = off3 + ((n2 + 15) >> 4) * 4;
    const u32 w3 = *(LAS u32*)(T + off3 + lane * 4);
    const u32 en = *(LAS u16*)(T + offp + lane * 2);
    u32 s1[16], s2[4];
    lookup4(s2, w2, L + LDS_DL);
    const u32x4 o0 = *(LAS u32x4*)(S + 1024 + lane * 16);
    u32x4 o1;
    if (MB == 3) o1 = *(LAS u32x4*)(S + 2048 + lane * 16);
    else { const u32x2 sg = *(LAS u32x2*)(T + lane * 8); o1 = u32x4{sg[0], sg[1], 0, 0}; }
    FENCE(); SB();
    const u32 eb4 = eb * 0x01010101u;
    const u32 t1 = ((eb4 + 0x03020100u) & F::EM) << MB, t2 = ((eb4 + 0x041F0000u) & F::EM) << MB, t3 = ((eb4 + 0x1E1D1C1Bu) & F::EM) << MB;
    u32x4 z;
    u32 cnt[5] = {0};
#pragma unroll
    for (int q = 0; q < 4; ++q) { z[q] = zero2(w[q]); cnt[q + 1] = cnt[q] + __builtin_popcount(z[q]); }
    u32 z2 = zero2(w2), k2 = __builtin_popcount(z2);
    const u32 both = cnt[4] | (k2 << 16), incl = scan64(both);
    const u32 excl = incl - both + ((u32)(size_t)S - 2048) + (((u32)(size_t)S - 3072) << 16);
    SB();
    {
        u32x4 y;
#pragma unroll
        for (int j = 0; j < 4; ++j) y[j] = perm(0u, t3, (w3 >> (2 * j)) & 0x03030303u);
        *(LAS u32x4*)(S - 3072 + lane * 16) = y;
        FENCE();
        if ((u32)lane < n3) *(S - 3072 + ((en >> 5) & 1023u)) = (unsigned char)(((en + eb + 27u) & (F::EM & 31u)) << MB);
        FENCE();
    }
    u32 o[16];
    u32x2 q[16];
    window4(o, q, z2, excl >> 16);
    SB();
#pragma unroll
    for (int i = 0; i < 4; ++i) lookup4(s1 + 4 * i, w[i], L + LDS_DL);
    FENCE(); SB();
    *(LAS u32x4*)(S - 2048 + lane * 16) =
        u32x4{spread(t2, o[0], q[0], s2[0]), spread(t2, o[1], q[1], s2[1]), spread(t2, o[2], q[2], s2[2]), spread(t2, o[3], q[3], s2[3])};
    if (n1 > 1024) {
        w2 = *(LAS u32*)(T + F::OFF_2 + 256 + lane * 4);
        lookup4(s2, w2, L + LDS_DL);
        z2 = zero2(w2); k2 = __builtin_popcount(z2);
        window4(o, q, z2, (__builtin_amdgcn_readlane(incl, 63) >> 16) + (u32)(size_t)S - 3072 + scan64(k2) - k2);
        *(LAS u32x4*)(S - 2048 + 1024 + lane * 16) =
            u32x4{spread(t2, o[0], q[0], s2[0]), spread(t2, o[1], q[1], s2[1]), spread(t2, o[2], q[2], s2[2]), spread(t2, o[3], q[3], s2[3])};
    }
    FENCE(); SB();
#pragma unroll
    for (int i = 0; i < 4; ++i) window4(o + 4 * i, q + 4 * i, z[i], (excl & 0xFFFFu) + cnt[i]);
    SB();
    u32x4 e[4];
#pragma unroll
    for (int j = 0; j < 16; ++j) {
        const u32 ex = spread(t1, o[j], q[j], s1[j]);
        if (MB == 3) {
            const u32 v = j < 8 ? o0[(j >> 1) & 3] : o1[(j >> 1) & 3];
            e[j >> 2][j & 3] = (j & 1) ? and_or(v >> 4, 0x07070707u, and_or(v << 4, 0x80808080u, ex)) : and_or(v, 0x87878787u, ex);
        } else {
            e[j >> 2][j & 3] = and_or(o0[j >> 2] >> (2 * (j & 3)), 0x03030303u, and_or(o1[j >> 3] << (j & 7), 0x80808080u, ex));
        }
        if ((j & 3) == 3) st_img(lane * 16, (j < 8 ? qa : qb) + (j & 4) * 256, e[j >> 2]);
    }
}

#define TS(t)
#define TC(t)
#define V10(a) "v" #a "0", "v" #a "1", "v" #a "2", "v" #a "3", "v" #a "4", "v" #a "5", "v" #a "6", "v" #a "7", "v" #a "8", "v" #a "9"
__device__ __forceinline__ void mfma_role(u32 sa, u32 sb, u32 sc, u32 olo, u32 ohi, u32 scale,
                                          u64& ta, u64& tb, u64& tc, u64& td, u64& ca, u64& cb, u64& cc) {
    u32 need, m, spin, scl;
    asm volatile(
        ".macro AG_FRAG d, a, o\n\t"
        "ds_read_b128 v[\\d:\\d+3], \\a offset:\\o\n\t"
        "ds_read_b128 v[\\d+4:\\d+7], \\a offset:\\o+4096\n\t"
        ".endm\n\t"
        ".macro AG_MM c, a, b\n\t"
        "v_mfma_f32_32x32x64_f8f6f4 v[\\c:\\c+15], v[\\a:\\a+7], v[\\b:\\b+7], v[\\c:\\c+15] cbsz:0 blgp:1\n\t"
        ".endm\n\t"
        ".macro AG_MIN\n\t"
        "v_min_u32 v164, v164, v165\n\t"
        "s_nop 0\n\t"
        "v_readfirstlane_b32 %[m], v164\n\t"
        ".endm\n\t"
        ".macro AG_SPIN\n\t"
        "s_mov_b32 %[spin], 0\n\t"
        ".LAG_S\\@:\n\t"
        "ds_read_b64 v[164:165], v162\n\t"
        "s_add_u32 %[spin], %[spin], 1\n\t"
        "s_waitcnt lgkmcnt(0)\n\t"
        "AG_MIN\n\t"
        "s_cmp_ge_u32 %[m], %[need]\n\t"
        "s_cbranch_scc1 .LAG_E\\@\n\t"
        "s_sleep " STR(MSLEEP) "\n\t"
        "s_cmp_lt_u32 %[spin], 0x2000000\n\t"
        "s_cbranch_scc1 .LAG_S\\@\n\t"
        ".LAG_E\\@:\n\t"
        ".endm\n\t"
        ".macro AG_DONE\n\t"
        "v_add_u32 v163, 1, v163\n\t"
        "ds_write_b32 v162, v163 offset:8\n\t"
        ".endm\n\t"
        ".macro AG_FULL so, sn\n\t"
        "s_waitcnt lgkmcnt(4)\n\t"
        "AG_MM 0, 128, 144\n\t"
        "s_waitcnt lgkmcnt(2)\n\t"
        "AG_MM 64, 136, 144\n\t"
        "ds_read_b64 v[164:165], v162\n\t"
        "AG_FRAG 144, v161, \\so+2048\n\t"
        "s_add_u32 %[need], %[need], 1\n\t"
        "s_waitcnt lgkmcnt(3)\n\t"
        "AG_MM 16, 128, 152\n\t"
        "AG_MM 80, 136, 152\n\t"
        "AG_FRAG 152, v161, \\so+3072\n\t"
        "s_waitcnt lgkmcnt(2)\n\t"
        "AG_MM 32, 128, 144\n\t"
        "AG_MIN\n\t"
        "AG_MM 96, 136, 144\n\t"
        "s_cmp_ge_u32 %[m], %[need]\n\t"
        "s_cbranch_scc1 .LAG_F\\@\n\t"
        "s_waitcnt lgkmcnt(0)\n\t"
        "AG_MM 48, 128, 152\n\t"
        "AG_DONE\n\t"
        "AG_MM 112, 136, 152\n\t"
        "AG_SPIN\n\t"
        "AG_FRAG 144, v161, \\sn\n\t"
        "AG_FRAG 128, v160, \\sn\n\t"
        "s_branch .LAG_J\\@\n\t"
        ".LAG_F\\@:\n\t"
        "AG_FRAG 144, v161, \\sn\n\t"
        "s_waitcnt lgkmcnt(2)\n\t"
        "AG_MM 48, 128, 152\n\t"
        "AG_DONE\n\t"
        "AG_FRAG 128, v160, \\sn\n\t"
        "AG_MM 112, 136, 152\n\t"
        ".LAG_J\\@:\n\t"
        "AG_FRAG 136, v160, \\sn+2048\n\t"
        "AG_FRAG 152, v161, \\sn+1024\n\t"
        ".endm\n\t"
        ".macro AG_PREP c, b\n\t"
        ".irp j, 0, 16, 32, 48\n\t"
        ".irp i, 0, 1, 2, 3\n\t"
        "v_mul_f32 v[\\c+\\j+4*\\b+\\i], %[scl], v[\\c+\\j+4*\\b+\\i]\n\t"
        ".endr\n\t"
        ".endr\n\t"
        "v_swap_b32 v[\\c+4*\\b+1], v[\\c+16+4*\\b]\n\t"
        "v_swap_b32 v[\\c+4*\\b+2], v[\\c+32+4*\\b]\n\t"
        "v_swap_b32 v[\\c+4*\\b+3], v[\\c+48+4*\\b]\n\t"
        "v_swap_b32 v[\\c+16+4*\\b+2], v[\\c+32+4*\\b+1]\n\t"
        "v_swap_b32 v[\\c+16+4*\\b+3], v[\\c+48+4*\\b+1]\n\t"
        "v_swap_b32 v[\\c+32+4*\\b+3], v[\\c+48+4*\\b+2]\n\t"
        ".endm\n\t"
        ".macro AG_ST c, r\n\t"
        ".if (\\r) >= 0\n\t"
        ".if ((\\r) & 3) == 0\n\t"
        "AG_PREP \\c, (\\r)/4\n\t"
        ".endif\n\t"
        ".if \\c\n\t"
        "global_store_dwordx4 v166, v[\\c+16*((\\r)&3)+((\\r)&12):\\c+16*((\\r)&3)+((\\r)&12)+3], vcc" OB1 "\n\t"
        ".else\n\t"
        "global_store_dwordx4 v166, v[\\c+16*((\\r)&3)+((\\r)&12):\\c+16*((\\r)&3)+((\\r)&12)+3], vcc" OB0 "\n\t"
        ".endif\n\t"
        ".if ((\\r) & 3) == 3\n\t"
        "s_add_u32 vcc_lo, vcc_lo, 5*16384\n\t"
        ".else\n\t"
        "s_add_u32 vcc_lo, vcc_lo, 16384\n\t"
        ".endif\n\t"
        "s_addc_u32 vcc_hi, vcc_hi, 0\n\t"
        ".endif\n\t"
        ".endm\n\t"
        ".macro AG_XF d, q\n\t"
        ".if ((\\q) %% 5) == 4\n\t"
        "AG_FRAG \\d, v160, 2*16384+2048\n\t"
        ".else\n\t"
        "AG_FRAG \\d, v160, ((\\q) %% 5)*16384\n\t"
        ".endif\n\t"
        ".endm\n\t"
        ".macro AG_WF d, q, i\n\t"
        ".if ((\\q) %% 5) == 4\n\t"
        "AG_FRAG \\d, v167, (\\i/2)*16384+(\\i&1)*1024\n\t"
        ".else\n\t"
        "AG_FRAG \\d, v161, ((\\q) %% 5)*16384+\\i*1024\n\t"
        ".endif\n\t"
        ".endm\n\t"
        ".macro AG_HALF c, q, last, r0, r1\n\t"
        ".set AG_B0, 136+8*(((\\q)+1) %% 3)\n\t"
        ".set AG_B1, 136+8*(((\\q)+2) %% 3)\n\t"
        ".set AG_B2, 136+8*((\\q) %% 3)\n\t"
        "s_waitcnt lgkmcnt(2)\n\t"
        "AG_MM \\c, 128, AG_B0\n\t"
        "AG_WF AG_B0, \\q, 3\n\t"
        "s_add_u32 %[need], %[need], 1\n\t"
        "AG_MIN\n\t"
        "AG_MM \\c+16, 128, AG_B1\n\t"
        ".if \\last == 0\n\t"
        "s_cmp_ge_u32 %[m], %[need]\n\t"
        "s_cbranch_scc1 .LAG_F\\@\n\t"
        ".endif\n\t"
        "AG_ST 0, \\r0\n\t"
        "s_waitcnt lgkmcnt(2)\n\t"
        "AG_MM \\c+32, 128, AG_B2\n\t"
        "s_waitcnt lgkmcnt(0)\n\t"
        "AG_MM \\c+48, 128, AG_B0\n\t"
        ".if \\last == 0\n\t"
        "AG_DONE\n\t"
        "AG_SPIN\n\t"
        "AG_WF AG_B1, (\\q)+1, 0\n\t"
        "AG_WF AG_B2, (\\q)+1, 1\n\t"
        "s_branch .LAG_J\\@\n\t"
        ".LAG_F\\@:\n\t"
        "AG_WF AG_B1, (\\q)+1, 0\n\t"
        "AG_ST 0, \\r0\n\t"
        "s_waitcnt lgkmcnt(4)\n\t"
        "AG_MM \\c+32, 128, AG_B2\n\t"
        "AG_WF AG_B2, (\\q)+1, 1\n\t"
        "s_waitcnt lgkmcnt(4)\n\t"
        "AG_MM \\c+48, 128, AG_B0\n\t"
        "AG_DONE\n\t"
        ".LAG_J\\@:\n\t"
        "AG_XF 128, (\\q)+1\n\t"
        "ds_read_b64 v[164:165], v162\n\t"
        "AG_WF AG_B0, (\\q)+1, 2\n\t"
        ".endif\n\t"
        "AG_ST 0, \\r1\n\t"
        ".endm\n\t"
        ".macro AG_SCALE\n\t"
        "ds_read_b32 v164, v162 offset:12\n\t"
        "s_nop 15\n\t"
        "s_nop 7\n\t"
        "s_waitcnt lgkmcnt(0)\n\t"
        "v_readfirstlane_b32 %[m], v164\n\t"
        "s_cmp_eq_u32 %[m], 0\n\t"
        "s_cselect_b32 %[scl], %[scale], 0x7fc00000\n\t"
        "ds_read_b64 v[164:165], v162\n\t"
        "s_waitcnt lgkmcnt(0)\n\t"
        ".endm\n\t"

        "v_mbcnt_lo_u32_b32 v167, -1, 0\n\t"
        "v_mbcnt_hi_u32_b32 v167, -1, v167\n\t"
        "v_lshrrev_b32 v165, 5, v167\n\t"
        "v_and_b32 v167, 31, v167\n\t"
        "v_lshlrev_b32 v167, 4, v167\n\t"
        "v_lshl_add_u32 v166, v165, 16, v167\n\t"
        "v_lshl_add_u32 v167, v165, 13, v167\n\t"
        "v_add_u32 v160, %[sa], v167\n\t"
        "v_add_u32 v161, %[sb], v167\n\t"
        "v_subrev_u32 v167, 65536-2048, v161\n\t"
        "v_mov_b32 v162, %[sc]\n\t"
        "v_mov_b32 v163, 0\n\t"
        ".set AG_I, 0\n\t"
        ".rept 128\n\t"
        "v_mov_b32 v[AG_I], 0\n\t"
        ".set AG_I, AG_I+1\n\t"
        ".endr\n\t"
        "s_mov_b32 vcc_lo, %[olo]\n\t"
        "s_mov_b32 vcc_hi, %[ohi]\n\t"
        "s_mov_b32 %[need], 1\n\t"
        "AG_SPIN\n\t"
        TS(ta) TC(ca)
        "AG_FRAG 144, v161, 0\n\t"
        "AG_FRAG 128, v160, 0\n\t"
        "AG_FRAG 136, v160, 2048\n\t"
        "AG_FRAG 152, v161, 1024\n\t"
        ".LAG_LOOP:\n\t"
        "AG_FULL 0, 16384\n\t"
        "AG_FULL 16384, 32768\n\t"
        "AG_FULL 32768, 49152\n\t"
        "AG_FULL 49152, 0\n\t"
        "s_cmp_le_u32 %[need], 64-" STR(NSPLIT) "\n\t"
        "s_cbranch_scc1 .LAG_LOOP\n\t"
        TS(td)
        ".if " STR(NSPLIT) "\n\t"
        "ds_read_b64 v[164:165], v162\n\t"
        "AG_WF 136, 0, 2\n\t"
        ".set AG_I, 0\n\t"
        ".rept " STR(NSPLIT) "\n\t"
        "AG_HALF 0, AG_I, 0, -1, -1\n\t"
        ".set AG_I, AG_I+1\n\t"
        ".endr\n\t"
        "AG_SCALE\n\t"
        TS(tb) TC(cc)
        ".set AG_I, 0\n\t"
        ".rept " STR(NSPLIT) "\n\t"
        ".set AG_R0, (AG_I*16+" STR(NSPLIT) "-1)/" STR(NSPLIT) "\n\t"
        ".set AG_R1, (AG_I*16+16+" STR(NSPLIT) "-1)/" STR(NSPLIT) "-1\n\t"
        ".if AG_R1 == AG_R0\n\t"
        ".set AG_R1, -1\n\t"
        ".endif\n\t"
        "AG_HALF 64, AG_I+" STR(NSPLIT) ", (AG_I==" STR(NSPLIT) "-1), AG_R0, AG_R1\n\t"
        ".set AG_I, AG_I+1\n\t"
        ".endr\n\t"
        "AG_SCALE\n\t"
        TS(tc) TC(cb)
        ".else\n\t"
        "AG_SCALE\n\t"
        TS(tc) TC(cb)
        ".irp r, 0, 1, 2, 3, 4, 5, 6, 7, 8, 9, 10, 11, 12, 13, 14, 15\n\t"
        "AG_ST 0, \\r\n\t"
        ".endr\n\t"
        ".endif\n\t"
        "s_add_u32 vcc_lo, vcc_lo, 96*16384\n\t"
        "s_addc_u32 vcc_hi, vcc_hi, 0\n\t"
        ".irp r, 0, 1, 2, 3, 4, 5, 6, 7, 8, 9, 10, 11, 12, 13, 14, 15\n\t"
        "AG_ST 64, \\r\n\t"
        ".endr\n\t"
        "s_nop 1\n\t"
        : [need] "=&s"(need), [m] "=&s"(m), [spin] "=&s"(spin), [scl] "=&s"(scl)
        : [sa] "s"(sa), [sb] "s"(sb), [sc] "s"(sc), [olo] "s"(olo), [ohi] "s"(ohi), [scale] "s"(scale)
        : "memory", "scc", "vcc", V10(), V10(1), V10(2), V10(3), V10(4), V10(5), V10(6), V10(7), V10(8), V10(9),
          V10(10), V10(11), V10(12), V10(13), V10(14), V10(15),
          "v160", "v161", "v162", "v163", "v164", "v165", "v166", "v167");
}

__device__ __forceinline__ char* peer_of_lane(const Params& P) {
    u32 lo = 0, hi = 0;
#pragma unroll
    for (int p = 0; p < NR; ++p) {
        asm volatile("v_writelane_b32 %0, %2, %4\n\tv_writelane_b32 %1, %3, %4"
                     : "+v"(lo), "+v"(hi) : "s"((u32)(u64)P.ws[p]), "s"((u32)((u64)P.ws[p] >> 32)), "i"(p));
    }
    return (char*)(((u64)hi << 32) | lo);
}
__device__ __forceinline__ void arrive(LAS unsigned char* L, int lane) {
    if (lane == 0) __hip_atomic_fetch_add((LAS u32*)(L + LDS_PRO), 1u, __ATOMIC_RELAXED, __HIP_MEMORY_SCOPE_WORKGROUP);
}

template <bool W> __device__ __forceinline__ void prologue_unit(const Params& P, LAS unsigned char* L, char* myws, u32 callv,
                                                                 const u32 (&d)[16], int idx, int xg, int wave, int lane) {
    u32 x[16];
    __builtin_amdgcn_s_setprio(3);
    const int u = xg * 4 + wave;
    const bool half = !W && u >= (int)R0;
    char* peer = peer_of_lane(P);
    WAIT_VM0();
    if (W) {
        order_w(x, d);
    } else {
        LAS u32* xc = (LAS u32*)(L + LDS_XCH);
        if (lane == 0) __hip_atomic_fetch_add(xc, 1u, __ATOMIC_RELAXED, __HIP_MEMORY_SCOPE_WORKGROUP);
        for (u32 spin = 0; spin < SPINS && UNI(*(volatile LAS u32*)xc) < 4; ++spin) {}
        FENCE();
        LAS unsigned char* b = L + LDS_A + 32768 + lane * 64 + (half ? (wave & 1) * 8192 + (wave & 2) * 16 : wave * 16);
        const u32 hi = half ? 16 : 8192;
#pragma unroll
        for (int i = 0; i < 4; ++i) {
            const u32x4 v = *(LAS u32x4*)(b + (i & 1) * 4096 + (i >> 1) * hi);
            x[4 * i] = v[0]; x[4 * i + 1] = v[1]; x[4 * i + 2] = v[2]; x[4 * i + 3] = v[3];
        }
    }
    const u32 par = (UNI(callv) + 1) & 1;
    bool raw = RAWTAIL == 1 ? u == (int)ROUNDS - 1 && (!W || !NSPLIT)
             : RAWTAIL >= 2 && (W ? u == (int)ROUNDS - 1 : (RAWTAIL == 3 && u == (int)ROUNDS - 2) || (RAWTAIL == 4 && u >= (int)ROUNDS - 2) || (RAWTAIL == 5 && u >= (int)ROUNDS - 3));
    const char* qa = myws + WS_IMG + par * IMG_BYTES + unit_off(idx, half ? u & ~1 : u) + (half ? (u & 1) * 2048 : 0);
    const char* qb = qa + (half ? 4096 : 2048);
    st_sys_s(lane * 16, qa, piece(x, 0)); st_sys_s(lane * 16, qa + 1024, piece(x, 1));
    st_sys_s(lane * 16, qb, piece(x, 2)); st_sys_s(lane * 16, qb + 1024, piece(x, 3));
    u32 len64 = 64;
    if (!W || u >= NPW) len64 = encode_unit<W ? 2 : 3>(x, L + LDS_A + wave * 8192, L, myws + WS_CB + par * CB_BYTES + unit_off(idx, u), lane, raw);
    if (lane == 0) *(volatile LAS u32*)(L + LDS_LEN + wave * 4) = len64;
    WAIT_VM0();
    arrive(L, lane);
    if (xg == 0 && (!W || u >= NPW) && lane < NR && lane != P.rank)
        asm volatile("global_store_dword %0, %1, off sc0 sc1" :: "v"(peer + WS_RDY + par * 32768 + (P.rank * 32 + idx) * 128 + u * 4),
                     "v"(((UNI(callv) + 1) << 7) | len64) : "memory");
    __builtin_amdgcn_s_setprio(0);
}

__global__ void __launch_bounds__(NTHREADS, 3) ag_gemm(Params P) {
    __shared__ __attribute__((aligned(16))) unsigned char lds_raw[LDS_BYTES];
    LAS unsigned char* L = (LAS unsigned char*)lds_raw;
    const int hw = UNI(threadIdx.x >> 6);
    const int wave = hw < 4 ? 11 - hw : hw - 4;
    const int lane = threadIdx.x & 63;
    const int bid = blockIdx.x;
    const unsigned char* px = P.x;
    const unsigned char* pw_ = P.w;
    char* myws = P.myws;
    int rank = P.rank;
    u32 srcmap = P.srcmap;
    asm volatile("" : "+s"(px), "+s"(pw_), "+s"(myws), "+s"(rank), "+s"(srcmap) : "s"(P.ws[0]));
    const int xg = bid & 7, idx = bid >> 3;
    const int rg = (xg + 5) & 7;
    const int tm = (xg >> 1) * 4 + (idx >> 3), tn = (xg & 1) * 8 + (idx & 7);
    GAS u32* tmo = (GAS u32*)(myws + WS_TMO);

    u32 xo = (lane >> 2) * KS + (lane & 3) * 16;
    u32 d[16];
    if (wave < 4 && idx < 16) {
        const unsigned char* g = px + (size_t)(idx * 256 + wave * 64) * KS + rg * 64;
        asm volatile("s_mov_b32 m0, %2\n\ts_nop 0\n\tglobal_load_lds_dwordx4 %0, %1\n\t"
                     ".rept 3\n\ts_add_u32 m0, m0, 0x400\n\tv_add_u32 %0, 16*512, %0\n\tglobal_load_lds_dwordx4 %0, %1\n\t.endr"
                     : "+v"(xo) : "s"(g), "s"((u32)(size_t)L + LDS_A + 32768 + wave * 4096) : "memory", "scc");
    }
    FENCE();
    const int pr = (idx >> 4) + 2 * (11 - wave);
    if (wave < 4 ? idx >= 16 : wave >= 10 && pr < NPW) load_w(d, pw_, idx, wave < 4 ? rg * 4 + wave : pr, lane);
    u32 call = __hip_atomic_load((GAS u32*)(myws + WS_CNT + bid * 128), __ATOMIC_RELAXED, __HIP_MEMORY_SCOPE_AGENT);
    if (wave < 4) {
        encode_clear(L + LDS_A + wave * 8192, lane);
    } else if (wave < 8) {
        const u32 f = (wave - 4) * 64 + lane;
        u32 dl = 0, cl = 0x0C0C0C0Cu, k = 0;
#pragma unroll
        for (u32 i = 0; i < 4; ++i) {
            const u32 c = (f >> (2 * i)) & 3u;
            dl |= (c ? 4 + c : k) << (8 * i);
            cl ^= c ? 0u : (12u ^ i) << (8 * k);
            k += c == 0;
        }
        *(LAS u32*)(L + LDS_DL + f * 4) = dl;
        *(LAS u32*)(L + LDS_CL + f * 4) = cl;
    } else if (wave == 8) {
        ((LAS u32*)(L + LDS_CTL))[lane] = 0;
    }
    BAR();

    if (wave < 8) {
        if (wave < 4) {
            if (idx < 16) prologue_unit<false>(P, L, myws, call, d, idx, rg, wave, lane);
            else prologue_unit<true>(P, L, myws, call, d, idx, rg, wave, lane);
        }
        const u32 scb = UNI(__builtin_bit_cast(u32, P.sx[0] * P.sw[0]));
        const int g = wave >> 1, nh = wave & 1;
        const u64 ob = (u64)P.out + ((size_t)(tm * 256 + g * 32) * MN + tn * 256 + nh * 128) * 4;
        u32 olo = (u32)ob, ohi = (u32)(ob >> 32);
        u64 ta = 0, tb = 0, tc = 0, td = 0, ca = 0, cb = 0, cc = 0;
        mfma_role((u32)(size_t)L + LDS_A + g * 512, (u32)(size_t)L + LDS_B + nh * 512, (u32)(size_t)L + LDS_CTL + wave * 16,
                  olo, ohi, scb, ta, tb, tc, td, ca, cb, cc);
    } else if (wave < 10) {
        const u64 tk = rt();
        __builtin_amdgcn_s_setprio(2);
        if (wave == 8) {
            char* peer = peer_of_lane(P);
            call = UNI(call) + 1;
            STAMP(0);
            WAIT_VM0();
            for (u32 spin = 0; spin < SPINS && UNI(*(volatile LAS u32*)(L + LDS_PRO)) < 7; ++spin) __builtin_amdgcn_s_sleep(1);
            FENCE();
            const u32x4 ln = *(LAS u32x4*)(L + LDS_LEN);
            if (lane < NR) {
                if (lane != rank) { if (rg) st_sys(peer + WS_RDY + (call & 1) * 32768 + (rank * 32 + idx) * 128 + rg * 16, ln | (call << 7)); }
                else st_sys(peer + WS_UF + idx * 2048 + rg * 16, u32x4{call, call, call, call});
            }
            if (lane == 0) __hip_atomic_store((GAS u32*)(myws + WS_CNT + bid * 128), call, __ATOMIC_RELAXED, __HIP_MEMORY_SCOPE_AGENT);
            STAMP(7);
        } else {
            call = UNI(call) + 1;
            arrive(L, lane);
            u32 job = XCD_ID() * 32 + idx;
#pragma unroll 1
            for (int k = 0; k < NWG; ++k, job = (job + 1) & (NWG - 1)) {
                u32 old = 0;
                if (lane == 0) {
                    __hip_atomic_fetch_max((GAS u32*)(myws + WS_TIK + (job >> 5) * 128), call * 2048, __ATOMIC_RELAXED, __HIP_MEMORY_SCOPE_AGENT);
                    FENCE();
                    old = __hip_atomic_fetch_max((GAS u32*)(myws + WS_JOB + job * 128), call, __ATOMIC_RELAXED, __HIP_MEMORY_SCOPE_AGENT);
                }
                WAIT_VM0();
                if (UNI(old) < call) break;
            }
            *(volatile LAS u32*)(L + LDS_JOB) = job + 1;
        }
        const int isB = wave - 8;
        const int panel = isB ? 16 + tn : tm;
        const char* img = myws + WS_IMG + (call & 1) * IMG_BYTES + unit_off(panel, 0);
        const u32 ldsb = (u32)(size_t)L + (isB ? LDS_B : LDS_A);
        volatile LAS u32* ctl = (volatile LAS u32*)(L + LDS_CTL + (lane & 7) * 16);
        const char* uf = myws + WS_UF + panel * 2048;
        const u32 voff = lane * 16;
        Ld S = {0, 0, 0, 0, 0, ~0u, 0, 0, (u64)img, (u64)(img + (16 + tn - panel) * 16384 + (size_t)S0 * 524288)};
        u32 err = 0;
        const u32 vctl = (u32)(size_t)ctl, vfull = vctl + isB * 4, vf = (lane & 31) * 4;
        const u64 t0 = rt();
        WAIT_VM0();
        asm volatile("v_mov_b32 v40, 0\n\tv_mov_b32 v41, 0\n\tv_mov_b32 v47, 0\n\tv_add_u32 v44, 0x1000, %0\n\t"
                     "v_add_u32 v45, 0x2000, %0\n\tv_add_u32 v46, 0x3000, %0" :: "v"(voff) : "v40", "v41", "v44", "v45", "v46", "v47");
        auto late = [&]() {
            if (UNI(__hip_atomic_load(tmo, __ATOMIC_RELAXED, __HIP_MEMORY_SCOPE_AGENT)) == call || rt() - t0 > DEADLINE) {
                if (lane == 0) __hip_atomic_store(tmo, call, __ATOMIC_RELAXED, __HIP_MEMORY_SCOPE_AGENT);
                ctl[3] = 1;
                err = ~0u;
            }
            asm volatile("s_waitcnt vmcnt(0) lgkmcnt(0)" ::: "memory");
            return rt() - t0 < 3 * DEADLINE;
        };
#define RUN(M, end, l) while (load_run<M>(S, end, l, err, call, uf, voff, vf, vctl, vfull) && late()) {}
        if (LDELAY) while (rt() - tk < LDELAY) __builtin_amdgcn_s_sleep(4);
        if (QEASY) while (load_run<0, NEASY>(S, QEASY, ldsb, err, call, uf, voff, vf, vctl, vfull) && late()) {}
        RUN(0, S0, ldsb);
        LS(9 + isB);
        if (NSPLIT) {
            const u32 l0 = (u32)(size_t)L + LDS_A;
            if (isB) { RUN(2, NSTAGE, l0); LS(16); S.g -= (size_t)NSPLIT * 524288; RUN(4, NQ, l0); }
            else { RUN(1, NSTAGE, l0); S.g -= (size_t)NSPLIT * 524288 - 2048; RUN(3, NQ, l0); }
        }
        LS(14 + isB);
        RUN(5, 0, 0);
        ctl[isB] = 0xFFFFu;
        WAIT_VM(0);
        STAMP(3 + isB);
    } else {
        __builtin_amdgcn_s_setprio(2);
        const u32 vo = lane * 16, wk = wave - 10;
#define FS(i) do {} while (0)
        {
            u32 x[16];
            const u32 pdst = (srcmap >> (4 * xg)) & 7u;
            const bool push = pr < NPW && pdst != (u32)rank && !(P.pad & 1);
            const u32 up = 32 + ((rank - pdst - 1) & 7) + 7 * pr, pp = idx | 16;
            const char* pw = P.ws[pdst];
            const char* q0 = pw + WS_IMG + unit_off(pp, up);
            asm volatile("" : "+s"(q0));
            WAIT_VM0();
            call = UNI(call) + 1;
            if (push) {
                order_w(x, d);
                q0 += (call & 1) * IMG_BYTES;
                FS(0);
                st_sys_s(vo, q0, piece(x, 0));
                arrive(L, lane);
                wait_vm<PWIN>(); st_sys_s(vo, q0 + 1024, piece(x, 1));
                wait_vm<PWIN>(); st_sys_s(vo, q0 + 2048, piece(x, 2));
                wait_vm<PWIN>(); st_sys_s(vo, q0 + 3072, piece(x, 3));
                WAIT_VM(0); st_flag(pw + WS_UF + pp * 2048 + up * 4, call, lane);
                FS(2);
            } else {
                arrive(L, lane);
            }
        }
        call = UNI(call);
        u32 job = 0;
        for (u32 spin = 0; spin < SPINS && job == 0; ++spin) { __builtin_amdgcn_s_sleep(1); job = UNI(*(volatile LAS u32*)(L + LDS_JOB)); }
        job = (job - 1) & (NWG - 1);
        const u32 slot = job >> 5, src = (srcmap >> (4 * slot)) & 7u;
        if (src == (u32)rank || (P.pad & 1)) return;
        const char* tik = myws + WS_TIK + slot * 128;
        const char* scb = P.ws[src] + WS_CB + (call & 1) * CB_BYTES;
        const char* rdy = myws + WS_RDY + (call & 1) * 32768 + src * 4096;
        const char* dp = myws + WS_IMG + (call & 1) * IMG_BYTES;
        const u32 u0 = 32 + ((src - rank - 1) & 7);
        LAS unsigned char* sl = L + LDS_WRK + wk * WRK_BYTES + 3072;
        auto unit_of = [](u32 t) -> u32 {
            auto px = [](u32 k) { return 4 * (k & 3) + (k >> 2); };
            auto pw = [](u32 k) { return 16 + 8 * (k & 1) + (k >> 1); };
            if (t < T0) return t < 64 ? px(t & 15) * 32 + (t >> 4) : pw(t & 15) * 32 + NPW + (t >> 4) - 4;
            if (t < T1) { t -= T0; return ((t & 1) ? pw((t >> 1) & 15) : px((t >> 1) & 15)) * 32 + 4 + (t >> 5); }
            if (t < T2) {
                const u32 pr = (t - T1) / 48, i = (t - T1) % 48;
                return ((i & 16) ? px(i & 15) : pw(i & 15)) * 32 + R0 + 2 * pr + (i >> 5);
            }
            return t < NT ? px((t - T2) & 15) * 32 + R0 + 2 * ((t - T2) >> 4) + 1 : 0u;
        };
        const u64 t0 = rt();
        const char* clk = myws + WS_CLK + slot * 128;
        auto await = [&](u32 un) -> u32 {
            for (u32 n = 1;; ++n) {
                u32 rv;
                if (lane == 0) asm volatile("global_atomic_or v162, %0, %0, %1 sc0 sc1\n\ts_waitcnt vmcnt(0)" :: "v"(0u), "s"(rdy + un * 4) : "memory", "v162");
                asm volatile("v_readfirstlane_b32 %0, v162" : "=s"(rv) :: "memory", "v162");
                if ((rv >> 7) >= call) return rv;
                if ((n & 31) == 0 && (UNI(__hip_atomic_load(tmo, __ATOMIC_RELAXED, __HIP_MEMORY_SCOPE_AGENT)) == call || rt() - t0 > DEADLINE)) {
                    if (lane == 0) __hip_atomic_store(tmo, call, __ATOMIC_RELAXED, __HIP_MEMORY_SCOPE_AGENT);
                    return 0;
                }
                __builtin_amdgcn_s_sleep(2);
            }
        };
        auto book = [&](u32 cost) {
            const u64 now = rt() << 10, c = cost;
            if (lane == 0)
                asm volatile("global_atomic_umax_x2 %0, %1, %3\n\tglobal_atomic_add_x2 v[160:161], %0, %2, %3 sc0"
                             :: "v"(0u), "v"(now), "v"(c), "s"(clk) : "memory", "v160", "v161");
        };
        const char* pend = nullptr;
        u32 pn = 0;
        auto flag = [&]() {
            if ((u32)lane <= pn) asm volatile("global_store_dword %0, %1, %2 sc0 sc1" :: "v"(lane * 28), "v"(call), "s"(pend) : "memory");
            pend = nullptr;
        };
        u32 t = (job & 31) + 32 * wk, un = unit_of(t), k = 0, lat = 0, plat = 0;
        if (TOUCH && lane < 2)
            asm volatile("global_load_dword v163, %0, %1 sc0 sc1" :: "v"(lane * (2u << 20) + 4092 + (job & 31) * 4096), "s"(scb) : "memory", "v163");
        u32 rv = await(un);
        if (!rv) return;
        WAIT_VM0();
        book((rv & 127u) * RATE);
        bool now_ = t < NOWAIT;
        if (!now_) WAIT_VM(0);
        t += 64;
        for (;; ++k) {
            u32 len64 = rv & 127u;
            len64 = len64 < 41 ? 41 : (len64 > 64 ? 64 : len64);
            const u32 len = len64 * 64, panel = un >> 5, r = un & 31;
            const char* q = scb + unit_off(panel, r);
            const bool more = t < NT;
            un = unit_of(t);
            const char* rp = rdy + un * 4;
            asm volatile("" : "+s"(q), "+s"(rp));
            if (!now_) {
                u32 lo, hi;
                asm volatile("v_readfirstlane_b32 %0, v160\n\tv_readfirstlane_b32 %1, v161" : "=s"(lo), "=s"(hi) :: "memory", "v160", "v161");
                const u32 start = (u32)((((u64)hi << 32) | lo) >> 10);
                for (u32 d = start - (u32)rt(); d - 1 < 1000u; d = start - (u32)rt()) {
                    if (pend) { if (vm_idle()) { WAIT_VM(0); flag(); } }
                    else if (d > 12) __builtin_amdgcn_s_sleep(2);
                }
            }
            now_ = false;
            const u32 i0 = (u32)rt();
            if (lane == 0) asm volatile(RDYLD "\n\tglobal_atomic_add v163, %0, %1, %2 sc0"
                                        :: "v"(0u), "v"(1u), "s"(tik), "s"(rp) : "memory", "v162", "v163");
            PIECE(0, q); PIECE(1, q);
            if (2048 + vo < len) PIECE(2, q);
            if (3072 + vo < len) PIECE(3, q);
            const bool half = panel < 16 && r >= R0;
            const u32 u = u0 + 7 * (half ? r & ~1u : r);
            const char* qa = dp + unit_off(panel, u) + (half ? (r & 1) * 2048 : 0);
            const char* qb = half ? dp + unit_off(panel, u + 7) + (r & 1) * 2048 : qa + 2048;
            const u32 rate = (lat < plat ? lat : plat) > LATHI ? RATE + RATE / PEN : RATE;
            if (pend) WAIT_VM(5);
            asm volatile("" : "+s"(qa), "+s"(qb) :: "memory");
            if (pend) { flag(); WAIT_VM(4); } else { WAIT_VM(3); }
            asm volatile("v_readfirstlane_b32 %0, v163\n\tv_readfirstlane_b32 %1, v162" : "=s"(t), "=s"(rv) :: "memory", "v162", "v163");
            t += 128 - call * 2048;
            const bool known = (rv >> 7) >= call;
            if (more && known) { book((rv & 127u) * rate); WAIT_VM(2); } else { WAIT_VM(0); }
            const u32x4 w = *(LAS u32x4*)(sl + lane * 16);
            const u32 ho = panel < 16 ? 3072 : 2560;
            const u32 hd = *(LAS u32*)(sl + ho), w2 = *(LAS u32*)(sl + ho + 4 + lane * 4);
            plat = lat;
            lat = (u32)rt() - i0;
            FENCE(); SB();
            if (len64 == 64) {
                const u32x4 e1 = *(LAS u32x4*)(sl + 1024 + lane * 16), e2 = *(LAS u32x4*)(sl + 2048 + lane * 16), e3 = *(LAS u32x4*)(sl + 3072 + lane * 16);
                st_img(vo, qa, w); st_img(vo, qa + 1024, e1); st_img(vo, qb, e2); st_img(vo, qb + 1024, e3);
            } else if (panel < 16) {
                decode_unit<3>(qa, qb, UNI(hd), w, w2, sl, L, lane);
            } else {
                decode_unit<2>(qa, qb, UNI(hd), w, w2, sl, L, lane);
            }
            pend = myws + WS_UF + panel * 2048 + (u + (half && (r & 1) ? 4 * NSPLIT : 0)) * 4;
            pn = half;
            if (!more) break;
            if (!known) {
                WAIT_VM(0); flag();
                rv = await(un);
                if (!rv) return;
                WAIT_VM0();
                book((rv & 127u) * rate);
                WAIT_VM(0);
            }
            WAIT_VM(4);
        }
        WAIT_VM(0);
        flag();
    }
}

extern "C" void kernel_launch(int rank, int nranks, ncclComm_t comm,
                              hipStream_t stream,
                              void* const* d_in, const int* in_sizes, int n_in,
                              void* d_out, int out_size,
                              const int* in_dtypes, int out_dtype,
                              void* const* d_out_all,
                              void* const* d_ws_all, size_t ws_bytes) {
    (void)comm; (void)in_sizes; (void)n_in; (void)out_size; (void)in_dtypes; (void)out_dtype; (void)d_out_all;
    if (nranks != NR || (unsigned)rank >= (unsigned)NR || ws_bytes < WS_NEED) {
        fprintf(stderr, "kernel_launch: unexpected configuration\n");
        return;
    }
    Params p;
    p.x = (const unsigned char*)d_in[0];
    p.w = (const unsigned char*)d_in[1];
    p.sx = (const float*)d_in[2];
    p.sw = (const float*)d_in[3];
    p.out = (float*)d_out;
    for (int i = 0; i < NR; ++i) p.ws[i] = (char*)d_ws_all[i];
    p.myws = p.ws[rank];
    p.rank = rank;
    p.pad = 0;
    p.pad2 = 0;
    static const u32 near_link[NR] = {0x74510263u, 0x26301475u, 0x17542036u, 0x74153620u,
                                      0x63024157u, 0x60235741u, 0x51476302u, 0x02367514u};
    p.srcmap = near_link[rank];
    hipLaunchKernelGGL(ag_gemm, dim3(NWG), dim3(NTHREADS), 0, stream, p);
    HIP_CHECK(hipGetLastError());
}
```

```cpp
#include <hip/hip_runtime.h>
#include <rccl/rccl.h>
#include <cstdio>
#include <cstddef>

#define HIP_CHECK(expr) do { hipError_t _e = (expr); if (_e != hipSuccess) { \
    fprintf(stderr, "HIP error %s at %s:%d\n", hipGetErrorString(_e), __FILE__, __LINE__); \
    return; } } while (0)

#define MSLEEP 1
#define GAS __attribute__((address_space(1)))
#define LAS __attribute__((address_space(3)))
typedef unsigned u32;
typedef unsigned short u16;
typedef unsigned long long u64;
typedef unsigned u32x2 __attribute__((ext_vector_type(2)));
typedef unsigned u32x4 __attribute__((ext_vector_type(4)));
typedef int i32x4 __attribute__((ext_vector_type(4)));
typedef int i32x8 __attribute__((ext_vector_type(8)));
typedef float f32x4 __attribute__((ext_vector_type(4)));
typedef float f32x16 __attribute__((ext_vector_type(16)));

constexpr int NR = 8;
constexpr int MN = 4096;
constexpr int KS = 512;
constexpr int NWG = 256;
constexpr int NTHREADS = 768;
constexpr u32 ROUNDS = 32;
constexpr u32 NSTAGE = 64;
#define RATE 110
#define LATHI 105
#define PEN 8
#define NOWAIT 12
#define NPW 2
#define PWIN 3
#define QEASY 32
#define NEASY 4
#define ARDY 1
#define RDYLD "global_atomic_or v162, %0, %0, %3 sc0 sc1"
#define TOUCH 1
#define LDELAY 500
#define OWNFIRST 1
#define ROT 2
#define KK 0
#define NE 1
#define RAWTAIL 2
#define NSPLIT 8
constexpr u32 S0 = NSTAGE - NSPLIT;
constexpr u32 NQ = NSTAGE + NSPLIT;
constexpr u32 R0 = NSPLIT ? ((S0 * 4 - 32 + 13) / 14) * 2 : ROUNDS;
constexpr u32 US = 32 + 7 * R0;
constexpr u32 NPAIR = (ROUNDS - R0) / 2;
constexpr u32 T0 = 128 - 16 * NPW, T1 = T0 + (R0 - 4) * 32, T2 = T1 + NPAIR * 48, NT = T2 + NPAIR * 16;
constexpr u32 UNIT = 4096;
constexpr size_t IMG_BYTES = 32u << 20;
constexpr size_t CB_BYTES = 5u << 20;
__host__ __device__ constexpr u32 unit_off(u32 panel, u32 u) { return (u >> 2) * 524288u + panel * 16384u + (u & 3) * UNIT; }
#define CBL 2
__host__ __device__ constexpr u32 cb_off(u32 panel, u32 r) { return CBL == 1 ? (r * 32 + panel) * UNIT : CBL == 2 ? (r * 32 + panel) * (UNIT + 256) : unit_off(panel, r); }
constexpr size_t WS_CNT = 0;
constexpr size_t WS_TMO = 48 << 10;
constexpr size_t WS_TIK = 52 << 10;
constexpr size_t WS_CLK = WS_TIK + 2048;
constexpr size_t WS_UF = 64 << 10;
constexpr size_t WS_RDY = 128 << 10;
constexpr size_t WS_JOB = 192 << 10;
constexpr size_t WS_DBG = 256 << 10;
constexpr size_t WS_IMG = 1 << 20;
constexpr size_t WS_CB = WS_IMG + 2 * IMG_BYTES;
constexpr size_t WS_NEED = WS_CB + 2 * CB_BYTES;
constexpr u64 DEADLINE = 50000000ull;
constexpr u32 SPINS = 1u << 25;
constexpr u32 LDS_CTL = 0;
constexpr u32 LDS_PSH = LDS_CTL + 132, LDS_XCH = LDS_CTL + 136, LDS_PRO = LDS_CTL + 140;
constexpr u32 LDS_JOB = LDS_CTL + 160, LDS_LEN = LDS_CTL + 176;
constexpr u32 LDS_DL = 512, LDS_CL = LDS_DL + 1024;
constexpr u32 LDS_WRK = LDS_CL + 1024, WRK_BYTES = 7168;
constexpr u32 LDS_A = LDS_WRK + 2 * WRK_BYTES, LDS_B = LDS_A + 65536;
constexpr u32 LDS_BYTES = LDS_B + 65536;

struct Params {
    const unsigned char* x;
    const unsigned char* w;
    char* myws;
    int rank;
    u32 srcmap;
    const float* sx;
    const float* sw;
    float* out;
    int pad;
    u32 pad2;
    char* ws[NR];
};

__device__ __forceinline__ u64 rt() { return __builtin_amdgcn_s_memrealtime(); }
#define WAIT_VM(n) asm volatile("s_waitcnt vmcnt(" #n ")" ::: "memory")
#define WAIT_VM0() do { __builtin_amdgcn_s_waitcnt(0x0F70); WAIT_VM(0); } while (0)
#define FENCE() asm volatile("" ::: "memory")
#define SB() __builtin_amdgcn_sched_barrier(0)
#define XCD_ID() (__builtin_amdgcn_s_getreg((3 << 11) | 20) & 7u)
#define UNI(x) __builtin_amdgcn_readfirstlane(x)

#define STAMP(i) do {} while (0)
#define STAMP2(i) do {} while (0)
#define STAMP3(i) do {} while (0)

#define OB0 ""
#define OB1 ""
__device__ __forceinline__ void st_sys(char* p, u32x4 v) {
    asm volatile("global_store_dwordx4 %0, %1, off sc0 sc1\n\ts_nop 1" :: "v"(p), "v"(v) : "memory");
}
__device__ __forceinline__ void st_sys_s(u32 voff, const void* base, u32x4 v) {
    asm volatile("global_store_dwordx4 %0, %1, %2 sc0 sc1\n\ts_nop 1" :: "v"(voff), "v"(v), "s"(base) : "memory");
}
__device__ __forceinline__ void st_flag(const void* base, u32 v, int lane) {
    if (lane == 0) asm volatile("global_store_dword %0, %1, %2 sc0 sc1" :: "v"(0u), "v"(v), "s"(base) : "memory");
}

#define DBITS " sc0 sc1"
#define UBITS " sc0 sc1"
#define FBITS " sc0 sc1"
#define STR2(x) #x
#define STR(x) STR2(x)
#define ISLP 1
#define LS(i) do {} while (0)
struct Ld {
    u32 nops, q, rmask, nf;
    u32 pidx, pb, slot;
    u64 fifo, g, gw;
};
#define G2(v, s_, o) "global_load_lds_dwordx4 " v ", " s_ " offset:" #o DBITS "\n\tglobal_load_lds_dwordx4 " v ", " s_ " offset:" #o "+1024" DBITS "\n\t"
#define EASE ".if %c[ease]\n\ts_sleep %c[ease]\n\t.endif\n\t"
#define M0A(x) "s_add_u32 m0, m0, " x "\n\ts_nop 0\n\t"
#define GW(v) G2(v, "%[g]", 0) M0A("%[land]") G2(v, "%[g]", 2048)
template <int MODE, int EASY = 0>
__device__ __forceinline__ u32 load_run(Ld& S, u32 end, u32 lds, u32 err, u32 call, const void* uf,
                                        u32 vo, u32 vf, u32 vctl, u32 vfull) {
    u32 a, b, land, rc, keep;
    asm volatile(
        "s_mov_b32 %[keep], m0\n\t"
        "s_mov_b32 %[rc], 0\n\t"
        "s_branch .LL%=\n\t"
        ".LZ%=:\n\t"
        ".if %c[md] == 0\n\t"
        "s_sleep " STR(ISLP) "\n\t"
        ".endif\n\t"
        ".LL%=:\n\t"
        "ds_read_b32 v42, %[vctl] offset:8\n\t"
        "s_getreg_b32 %[a], hwreg(HW_REG_IB_STS)\n\t"
        "s_bfe_u32 %[b], %[a], 0x20016\n\t"
        "s_and_b32 %[a], %[a], 15\n\t"
        "s_lshl4_add_u32 %[a], %[b], %[a]\n\t"
        "s_sub_u32 %[land], %[nops], %[a]\n\t"
        "s_mov_b64 vcc, %[fifo]\n\t"
        "s_sub_u32 %[a], %[land], vcc_lo\n\t"
        "s_bitcmp0_b32 %[a], 11\n\t"
        "s_cselect_b32 %[a], %[nf], 0\n\t"
        "s_cmp_eq_u32 %[a], 0\n\t"
        "s_cbranch_scc1 .LP%=\n\t"
        "v_add_u32 v41, 1, v41\n\t"
        "s_lshr_b64 %[fifo], %[fifo], 12\n\t"
        "ds_write_b32 %[vfull], v41\n\t"
        "s_sub_u32 %[nf], %[nf], 1\n\t"
        ".LP%=:\n\t"
        "s_add_u32 %[rc], %[rc], 0x100000\n\t"
        "s_cbranch_scc1 .LT%=\n\t"
        ".if %c[md] == 5\n\t"
        "s_waitcnt lgkmcnt(0)\n\t"
        "s_cmp_lg_u32 %[nf], 0\n\t"
        "s_cbranch_scc1 .LL%=\n\t"
        ".else\n\t"
        ".if %c[md] != 4\n\t"
        "s_or_b32 %[rmask], %[rmask], %[err]\n\t"
        "s_sub_u32 %[a], %[land], %[pidx]\n\t"
        "s_cmp_lt_i32 %[a], 0\n\t"
        "s_cbranch_scc1 .LO%=\n\t"
        "s_cmp_eq_u32 %[pb], -1\n\t"
        "s_cbranch_scc1 .LS%=\n\t"
        "v_cmp_le_u32 vcc, %[call], v40\n\t"
        "s_sub_u32 %[a], %[q], %[pb]\n\t"
        "s_lshl_b32 %[a], %[a], 1\n\t"
        "s_min_u32 %[a], %[a], 16\n\t"
        "s_lshr_b32 %[b], vcc_lo, %[a]\n\t"
        "s_lshr_b32 %[b], %[b], %[a]\n\t"
        "s_or_b32 %[rmask], %[rmask], %[b]\n\t"
        "s_mov_b32 %[pb], -1\n\t"
        "s_andn2_b32 %[b], 15, %[rmask]\n\t"
        "s_cbranch_scc0 .LO%=\n\t"
        ".LS%=:\n\t"
        "s_andn2_b32 %[b], 0xffff, %[rmask]\n\t"
        "s_cbranch_scc0 .LO%=\n\t"
        "s_lshl_b32 %[b], %[q], 4\n\t"
        "v_add_u32 v43, %[b], %[vf]\n\t"
        ".if %c[md] & 1\n\t"
        "v_subrev_u32 v40, 0x400, v43\n\t"
        "v_cmp_gt_u32 vcc, %[whole], v40\n\t"
        "v_subrev_u32 v40, 16*" STR(NSPLIT) ", v43\n\t"
        "s_nop 0\n\t"
        "v_cndmask_b32 v43, v43, v40, vcc\n\t"
        ".endif\n\t"
        "s_add_u32 %[nops], %[nops], 1\n\t"
        "s_mov_b32 exec_hi, 0\n\t"
        "global_atomic_or v40, v43, v47, %[uf] sc0 sc1\n\t"
        "s_mov_b32 exec_hi, -1\n\t"
        "s_mov_b32 %[pidx], %[nops]\n\t"
        "s_mov_b32 %[pb], %[q]\n\t"
        ".LO%=:\n\t"
        ".endif\n\t"
        ".if %c[md] == 0\n\t"
        "s_sub_u32 %[a], %[q], 3\n\t"
        ".else\n\t"
        "s_cmp_ge_u32 %[q], 68-" STR(NSPLIT) "\n\t"
        "s_subb_u32 %[a], %[q], 3\n\t"
        ".endif\n\t"
        "s_waitcnt lgkmcnt(0)\n\t"
        "v_cmp_gt_i32 vcc, %[a], v42\n\t"
        "s_lshl_b32 %[a], %[slot], 14\n\t"
        ".if %c[md]\n\t"
        "s_add_u32 %[b], %[a], 0x10000\n\t"
        "s_cmp_eq_u32 %[slot], 4\n\t"
        "s_cselect_b32 %[a], 2*16384+2048, %[a]\n\t"
        "s_cselect_b32 %[b], 2048, %[b]\n\t"
        "s_cselect_b32 %[land], 16384-2048, 0\n\t"
        ".endif\n\t"
        ".if %c[md] != 4\n\t"
        "s_andn2_b32 m0, 15, %[rmask]\n\t"
        "s_or_b32 m0, m0, vcc_lo\n\t"
        ".else\n\t"
        "s_cmp_lg_u32 vcc_lo, 0\n\t"
        ".endif\n\t"
        "s_cbranch_scc1 .LZ%=\n\t"
        ".if %c[md] == 0\n\t"
        "s_add_u32 m0, %[a], %[lds]\n\t"
        "s_add_u32 %[nops], %[nops], 16\n\t"
        G2("%[vo]", "%[g]", 0) G2("%[vo]", "%[g]", 2048) EASE M0A("0x1000") G2("v44", "%[g]", 0) G2("v44", "%[g]", 2048) EASE M0A("0x1000")
        G2("v45", "%[g]", 0) G2("v45", "%[g]", 2048) EASE M0A("0x1000") G2("v46", "%[g]", 0) G2("v46", "%[g]", 2048)
        ".elseif %c[md] & 1\n\t"
        "s_add_u32 m0, %[a], %[lds]\n\t"
        "s_add_u32 %[nops], %[nops], 6+2*%c[md]\n\t"
        G2("%[vo]", "%[g]", 0) M0A("0x1000") G2("v44", "%[g]", 0) M0A("0x1000") G2("v45", "%[g]", 0) M0A("0x1000") G2("v46", "%[g]", 0)
        ".if %c[md] == 3\n\t"
        "s_add_u32 %[b], %[b], %[lds]\n\t"
        "s_add_u32 m0, %[b], 0x3000\n\t"
        "s_mov_b64 vcc, %[gw]\n\t"
        G2("v46", "vcc", 0) M0A("%[land]") G2("v46", "vcc", 2048)
        "s_add_u32 vcc_lo, vcc_lo, 0x80000\n\t"
        "s_addc_u32 vcc_hi, vcc_hi, 0\n\t"
        "s_mov_b64 %[gw], vcc\n\t"
        ".endif\n\t"
        ".else\n\t"
        "s_add_u32 m0, %[b], %[lds]\n\t"
        "s_sub_u32 %[a], 0x1000, %[land]\n\t"
        "s_add_u32 %[nops], %[nops], 20-2*%c[md]\n\t"
        GW("%[vo]") M0A("%[a]") GW("v44") M0A("%[a]") GW("v45")
        ".if %c[md] == 2\n\t"
        M0A("%[a]") GW("v46")
        ".endif\n\t"
        ".endif\n\t"
        "s_mov_b64 vcc, %[g]\n\t"
        "s_add_u32 vcc_lo, vcc_lo, 0x80000\n\t"
        "s_addc_u32 vcc_hi, vcc_hi, 0\n\t"
        "s_mov_b64 %[g], vcc\n\t"
        "s_and_b32 vcc_lo, %[nops], 0xfff\n\t"
        "s_mov_b32 vcc_hi, 0\n\t"
        "s_mul_i32 %[a], %[nf], 12\n\t"
        "s_lshl_b64 vcc, vcc, %[a]\n\t"
        "s_or_b64 %[fifo], %[fifo], vcc\n\t"
        "s_add_u32 %[nf], %[nf], 1\n\t"
        "s_lshr_b32 %[rmask], %[rmask], 4\n\t"
        "s_add_u32 %[slot], %[slot], 1\n\t"
        "s_cmp_eq_u32 %[slot], 4+(%c[md]+7)/8\n\t"
        "s_cselect_b32 %[slot], 0, %[slot]\n\t"
        "s_add_u32 %[q], %[q], 1\n\t"
        "s_cmp_lt_u32 %[q], %[end]\n\t"
        "s_cbranch_scc1 .LL%=\n\t"
        ".endif\n\t"
        "s_mov_b32 %[rc], 0\n\t"
        "s_branch .LE%=\n\t"
        ".LT%=:\n\t"
        "s_mov_b32 %[rc], 1\n\t"
        "s_waitcnt lgkmcnt(0)\n\t"
        ".LE%=:\n\t"
        "s_mov_b32 m0, %[keep]"
        : [nops] "+s"(S.nops), [q] "+s"(S.q), [rmask] "+s"(S.rmask), [nf] "+s"(S.nf), [pidx] "+s"(S.pidx), [pb] "+s"(S.pb),
          [slot] "+s"(S.slot), [fifo] "+s"(S.fifo), [g] "+s"(S.g), [gw] "+s"(S.gw),
          [a] "=&s"(a), [b] "=&s"(b), [land] "=&s"(land), [rc] "=&s"(rc), [keep] "=&s"(keep)
        : [end] "s"(end), [lds] "s"(lds), [err] "s"(err), [call] "s"(call), [uf] "s"(uf),
          [vo] "v"(vo), [vf] "v"(vf), [vctl] "v"(vctl), [vfull] "v"(vfull), [md] "i"(MODE), [ease] "i"(EASY), [whole] "i"(4 * (US - 4 * S0))
        : "memory", "scc", "vcc", "v40", "v41", "v42", "v43", "v44", "v45", "v46", "v47");
    return rc;
}
#define PIECE(i, g) do { u32 k_; asm volatile("s_mov_b32 %0, m0\n\ts_mov_b32 m0, %3\n\ts_nop 0\n\t" \
    "global_load_lds_dwordx4 %1, %2 offset:1024*" #i FBITS "\n\ts_mov_b32 m0, %0" : "=&s"(k_) : "v"(vo), "s"(g), "s"((u32)(size_t)sl) : "memory"); } while (0)
template <int N> __device__ __forceinline__ void wait_vm() { asm volatile("s_waitcnt vmcnt(%0)" :: "i"(N) : "memory"); }
__device__ __forceinline__ bool vm_idle() {
    u32 r;
    asm volatile("s_getreg_b32 %0, hwreg(HW_REG_IB_STS)" : "=s"(r) :: "memory");
    return (r & 0xC0000Fu) == 0;
}
__device__ __forceinline__ void st_img(u32 voff, const void* base, u32x4 v) {
    asm volatile("global_store_dwordx4 %0, %1, %2" UBITS "\n\ts_nop 1" :: "v"(voff), "v"(v), "s"(base) : "memory");
}
__device__ __forceinline__ u32 perm(u32 hi, u32 lo, u32 sel) { return __builtin_amdgcn_perm(hi, lo, sel); }
__device__ __forceinline__ u32 bfi(u32 m, u32 a, u32 b) { return __builtin_amdgcn_bitop3_b32(m, a, b, 0xCA); }
__device__ __forceinline__ u32 and_or(u32 a, u32 m, u32 c) { return __builtin_amdgcn_bitop3_b32(a, m, c, 0xEA); }
__device__ __forceinline__ u32 xor_and(u32 a, u32 b, u32 m) { return __builtin_amdgcn_bitop3_b32(a, b, m, 0x78); }
#define BAR() asm volatile("s_waitcnt lgkmcnt(0)\n\ts_barrier" ::: "memory")
template <int S> __device__ __forceinline__ u32 shl(u32 v) { return S >= 0 ? v << (S >= 0 ? S : 0) : v >> (S < 0 ? -S : 0); }
__device__ __forceinline__ u32 scan64(u32 v) {
    v += __builtin_amdgcn_update_dpp(0u, v, 0x111, 0xf, 0xf, true);
    v += __builtin_amdgcn_update_dpp(0u, v, 0x112, 0xf, 0xf, true);
    v += __builtin_amdgcn_update_dpp(0u, v, 0x114, 0xf, 0xf, true);
    v += __builtin_amdgcn_update_dpp(0u, v, 0x118, 0xf, 0xf, true);
    v += __builtin_amdgcn_update_dpp(0u, v, 0x142, 0xa, 0xf, false);
    v += __builtin_amdgcn_update_dpp(0u, v, 0x143, 0xc, 0xf, false);
    return v;
}
__device__ __forceinline__ void lds_or(LAS unsigned char* p, u32 v) {
    __hip_atomic_fetch_or((LAS u32*)p, v, __ATOMIC_RELAXED, __HIP_MEMORY_SCOPE_WORKGROUP);
}

__device__ __forceinline__ void load_w(u32 (&d)[16], const unsigned char* w, int panel, int r, int lane) {
    const auto rs = __builtin_amdgcn_make_buffer_rsrc((void*)w, 0, MN * KS, 0x00020000);
    const u32 so = r * 16 * MN + (panel & 15) * 256;
#pragma unroll
    for (int k = 0; k < 16; ++k) d[k] = __builtin_amdgcn_raw_buffer_load_b32(rs, lane * 4, so + k * MN, 0);
}
__device__ __forceinline__ void order_w(u32 (&x)[16], const u32 (&d)[16]) {
#pragma unroll
    for (int q = 0; q < 4; ++q) {
        u32 t0 = perm(d[4 * q + 1], d[4 * q], 0x05010400), t1 = perm(d[4 * q + 1], d[4 * q], 0x07030602);
        u32 t2 = perm(d[4 * q + 3], d[4 * q + 2], 0x05010400), t3 = perm(d[4 * q + 3], d[4 * q + 2], 0x07030602);
        x[q] = perm(t2, t0, 0x05040100); x[4 + q] = perm(t2, t0, 0x07060302);
        x[8 + q] = perm(t3, t1, 0x05040100); x[12 + q] = perm(t3, t1, 0x07060302);
    }
}
__device__ __forceinline__ u32x4 piece(const u32 (&x)[16], int i) { return u32x4{x[4 * i], x[4 * i + 1], x[4 * i + 2], x[4 * i + 3]}; }

#define ES(i) do {} while (0)
template <int MB> struct Fmt {
    static constexpr u32 EM = (MB == 3 ? 0x0Fu : 0x1Fu) * 0x01010101u;
    static constexpr u32 HEAD = MB == 3 ? 3072 : 2048;
    static constexpr u32 OFF_H = MB == 3 ? 0 : 512, OFF_2 = OFF_H + 4;
};
__device__ __forceinline__ u32 zero2(u32 w) { return __builtin_amdgcn_bitop3_b32(w, w >> 1, 0x55555555u, 0x02); }
__device__ __forceinline__ u32 code_of(u32 hi, u32 lo, u32 v) {
    const u32 r = perm(hi, lo, v);
    return __builtin_amdgcn_bitop3_b32(r, r >> 2, 0x03030303u, 0x28);
}
__device__ __forceinline__ void lookup4(u32* sel, u32 w, LAS unsigned char* tab) {
#pragma unroll
    for (int j = 0; j < 4; ++j) sel[j] = *(LAS u32*)(tab + ((w >> (8 * j)) & 255u) * 4);
}
__device__ __forceinline__ void gather4(const u32* v, const u32* sel, u32 z, u32 a) {
#pragma unroll
    for (int j = 0; j < 4; ++j) {
        const u32 o = j ? a + __builtin_popcount(z & ((1u << (8 * j)) - 1)) : a;
        const u32 c = perm(0u, v[j], sel[j]);
        LAS unsigned char* p = (LAS unsigned char*)(size_t)(o & ~3u);
        lds_or(p, __builtin_amdgcn_alignbyte(c, 0u, ~o));
        lds_or(p + 4, __builtin_amdgcn_alignbyte(0u, c, ~o));
    }
}
__device__ __forceinline__ void window4(u32* o, u32x2* q, u32 z, u32 a) {
#pragma unroll
    for (int j = 0; j < 4; ++j) {
        o[j] = j ? a + __builtin_popcount(z & ((1u << (8 * j)) - 1)) : a;
        LAS u32* p = (LAS u32*)(size_t)(o[j] & ~3u);
        q[j] = u32x2{p[0], p[1]};
    }
}
__device__ __forceinline__ u32 spread(u32 tab, u32 o, u32x2 q, u32 sel) {
    return perm(tab, __builtin_amdgcn_alignbyte(q[1], q[0], o), sel);
}
__device__ __forceinline__ u32 pack4(const u32* c) {
    u32 w = __builtin_amdgcn_udot4(c[0], 0x40100401u, 0u, false);
#pragma unroll
    for (int j = 1; j < 4; ++j) w |= __builtin_amdgcn_udot4(c[j], 0x40100401u, 0u, false) << (8 * j);
    return w;
}
__device__ __forceinline__ void encode_clear(LAS unsigned char* T, int lane) {
#pragma unroll
    for (int i = 2; i < 5; ++i) *(LAS u32x4*)(T + i * 1024 + lane * 16) = u32x4{0, 0, 0, 0};
}

template <int MB> __device__ __forceinline__ u32 encode_unit(const u32 (&x)[16], LAS unsigned char* T, LAS unsigned char* L,
                                                             char* dst, int lane, bool raw) {
    typedef Fmt<MB> F;
    u32 sum = 0;
#pragma unroll
    for (int j = 0; j < 16; j += 5) sum = __builtin_amdgcn_udot4(x[j] & 0x7F7F7F7Fu, 0x01010101u, sum, false);
    const int tot = __builtin_amdgcn_readlane(scan64(sum), 63);
    const u32 eb = (u32)((tot - (1516 << MB)) >> (10 + MB)) & (F::EM & 31u);
    const u32 B = (eb << MB) * 0x01010101u;
    u32 v[16], c[16];
    u32x4 w, z;
#pragma unroll
    for (int j = 0; j < 16; ++j) {
        v[j] = (((x[j] | 0x80808080u) - B) >> MB) & F::EM;
        c[j] = code_of(0u, 0x03020100u, v[j]);
    }
    u32 cnt[5] = {0};
#pragma unroll
    for (int q = 0; q < 4; ++q) {
        w[q] = pack4(c + 4 * q); z[q] = zero2(w[q]); cnt[q + 1] = cnt[q] + __builtin_popcount(z[q]);
        lookup4(c + 4 * q, w[q], L + LDS_CL);
    }
    FENCE();
    const u32 incl = scan64(cnt[4]), n1 = __builtin_amdgcn_readlane(incl, 63);
    ES(0);
    u32 n2 = 0, n3 = 0, len64 = 64;
    raw = raw || n1 > 2040;
    const u32 off3 = F::OFF_2 + ((n1 + 15) >> 4) * 4;
    if (!raw) {
        const u32 a1 = incl - cnt[4] + ((u32)(size_t)T + 2047);
#pragma unroll
        for (int q = 0; q < 4; ++q) gather4(v + 4 * q, c + 4 * q, z[q], a1 + cnt[q]);
        ES(1);
        st_sys_s(lane * 16, dst, w);
        if (MB == 3) {
            u32x4 o[2];
#pragma unroll
            for (int p = 0; p < 8; ++p) o[p >> 2][p & 3] = bfi(0x87878787u, x[2 * p], bfi(0xF0F0F0F0u, x[2 * p + 1] << 4, x[2 * p + 1] >> 4));
            st_sys_s(lane * 16, dst + 1024, o[0]);
            st_sys_s(lane * 16, dst + 2048, o[1]);
        } else {
            u32x4 o;
            u32x2 sg;
#pragma unroll
            for (int q = 0; q < 4; ++q)
                o[q] = bfi(0x0F0F0F0Fu, bfi(0x03030303u, x[4 * q], x[4 * q + 1] << 2), bfi(0x03030303u, x[4 * q + 2], x[4 * q + 3] << 2) << 4);
#pragma unroll
            for (int k = 0; k < 2; ++k) {
                u32 t = x[8 * k];
#pragma unroll
                for (int j = 1; j < 8; ++j) t = bfi(0x80808080u >> j, x[8 * k + j] >> j, t);
                sg[k] = t;
            }
            *(LAS u32x2*)(T + lane * 8) = sg;
            st_sys_s(lane * 16, dst + 1024, o);
        }
        FENCE();
        ES(2);
        for (u32 b = 0; b < n1; b += 1024) {
            const u32x4 y = *(LAS u32x4*)(T + 2048 + b + lane * 16);
            u32 u[4], c2[4];
#pragma unroll
            for (int j = 0; j < 4; ++j) {
                c2[j] = code_of(0x00000300u, 0x00000102u, (y[j] + 0x01010101u) & F::EM);
                u[j] = (y[j] + 0x05050505u) & F::EM;
            }
            const u32 w2 = pack4(c2);
            lookup4(c2, w2, L + LDS_CL);
            FENCE();
            const u32 z2 = zero2(w2), k2 = __builtin_popcount(z2), i2 = scan64(k2);
            if (b + lane * 16 < n1) *(LAS u32*)(T + F::OFF_2 + (b >> 2) + lane * 4) = w2;
            const u32 all = n2 + __builtin_amdgcn_readlane(i2, 63);
            if (all <= 1016 && k2) gather4(u, c2, z2, n2 + i2 - k2 + ((u32)(size_t)T + 4095));
            n2 = all;
        }
        raw = n2 > 1016;
        ES(3);
    }
    if (!raw) {
        FENCE();
        const u32x4 y = *(LAS u32x4*)(T + 4096 + lane * 16);
        u32 k3 = 0, m[4];
#pragma unroll
        for (int j = 0; j < 4; ++j) { m[j] = ((y[j] & 0x1C1C1C1Cu) + 0x1C1C1C1Cu) & 0x20202020u; k3 += __builtin_popcount(m[j]); }
        const u32 i3 = scan64(k3);
        n3 = __builtin_amdgcn_readlane(i3, 63);
        const u32 offp = off3 + ((n2 + 15) >> 4) * 4;
        len64 = (F::HEAD + offp + 2 * n3 + 63) >> 6;
        raw = len64 >= 64 || n3 > 63;
        if (!raw) {
            if (lane * 16 < n2)
                *(LAS u32*)(T + off3 + lane * 4) = bfi(0x0F0F0F0Fu, bfi(0x03030303u, y[0], y[1] << 2), bfi(0x03030303u, y[2], y[3] << 2) << 4);
            if (lane == 0) *(LAS u32*)(T + F::OFF_H) = eb | (n1 << 5) | (n2 << 16) | (n3 << 26);
            LAS unsigned char* p = T + offp + 2 * (i3 - k3);
#pragma unroll
            for (int j = 0; j < 4; ++j) {
                u32 mm = m[j];
                while (mm) {
                    const u32 b = __builtin_ctz(mm) & 24u;
                    mm &= mm - 1;
                    *(LAS u16*)p = (u16)(((lane * 16 + j * 4 + (b >> 3)) << 5) | ((y[j] >> b) & 31u));
                    p += 2;
                }
            }
        }
    }
    if (raw) {
        WAIT_VM(0);
#pragma unroll
        for (int i = 0; i < 4; ++i) st_sys_s(lane * 16, dst + i * 1024, piece(x, i));
        return 64;
    }
    FENCE();
    const u32 tl = len64 * 64 - F::HEAD;
    if (lane * 16 < tl) st_sys_s(lane * 16, dst + F::HEAD, *(LAS u32x4*)(T + lane * 16));
    if (MB == 2 && 1024 + lane * 16 < tl) st_sys_s(lane * 16, dst + F::HEAD + 1024, *(LAS u32x4*)(T + 1024 + lane * 16));
    return len64;
}

template <int MB> __device__ __forceinline__ void decode_unit(const char* qa, const char* qb, u32 hdr, u32x4 w, u32 w2,
                                                              LAS unsigned char* S, LAS unsigned char* L, int lane) {
    typedef Fmt<MB> F;
    LAS unsigned char* T = S + F::HEAD;
    const u32 eb = hdr & 31u, n1 = (hdr >> 5) & 2047u, n2 = (hdr >> 16) & 1023u, n3 = hdr >> 26;
    const u32 off3 = F::OFF_2 + ((n1 + 15) >> 4) * 4, offp = off3 + ((n2 + 15) >> 4) * 4;
    const u32 w3 = *(LAS u32*)(T + off3 + lane * 4);
    const u32 en = *(LAS u16*)(T + offp + lane * 2);
    u32 s1[16], s2[4];
    lookup4(s2, w2, L + LDS_DL);
    const u32x4 o0 = *(LAS u32x4*)(S + 1024 + lane * 16);
    u32x4 o1;
    if (MB == 3) o1 = *(LAS u32x4*)(S + 2048 + lane * 16);
    else { const u32x2 sg = *(LAS u32x2*)(T + lane * 8); o1 = u32x4{sg[0], sg[1], 0, 0}; }
    FENCE(); SB();
    const u32 eb4 = eb * 0x01010101u;
    const u32 t1 = ((eb4 + 0x03020100u) & F::EM) << MB, t2 = ((eb4 + 0x041F0000u) & F::EM) << MB, t3 = ((eb4 + 0x1E1D1C1Bu) & F::EM) << MB;
    u32x4 z;
    u32 cnt[5] = {0};
#pragma unroll
    for (int q = 0; q < 4; ++q) { z[q] = zero2(w[q]); cnt[q + 1] = cnt[q] + __builtin_popcount(z[q]); }
    u32 z2 = zero2(w2), k2 = __builtin_popcount(z2);
    const u32 both = cnt[4] | (k2 << 16), incl = scan64(both);
    const u32 excl = incl - both + ((u32)(size_t)S - 2048) + (((u32)(size_t)S - 3072) << 16);
    SB();
    {
        u32x4 y;
#pragma unroll
        for (int j = 0; j < 4; ++j) y[j] = perm(0u, t3, (w3 >> (2 * j)) & 0x03030303u);
        *(LAS u32x4*)(S - 3072 + lane * 16) = y;
        FENCE();
        if ((u32)lane < n3) *(S - 3072 + ((en >> 5) & 1023u)) = (unsigned char)(((en + eb + 27u) & (F::EM & 31u)) << MB);
        FENCE();
    }
    u32 o[16];
    u32x2 q[16];
    window4(o, q, z2, excl >> 16);
    SB();
#pragma unroll
    for (int i = 0; i < 4; ++i) lookup4(s1 + 4 * i, w[i], L + LDS_DL);
    FENCE(); SB();
    *(LAS u32x4*)(S - 2048 + lane * 16) =
        u32x4{spread(t2, o[0], q[0], s2[0]), spread(t2, o[1], q[1], s2[1]), spread(t2, o[2], q[2], s2[2]), spread(t2, o[3], q[3], s2[3])};
    if (n1 > 1024) {
        w2 = *(LAS u32*)(T + F::OFF_2 + 256 + lane * 4);
        lookup4(s2, w2, L + LDS_DL);
        z2 = zero2(w2); k2 = __builtin_popcount(z2);
        window4(o, q, z2, (__builtin_amdgcn_readlane(incl, 63) >> 16) + (u32)(size_t)S - 3072 + scan64(k2) - k2);
        *(LAS u32x4*)(S - 2048 + 1024 + lane * 16) =
            u32x4{spread(t2, o[0], q[0], s2[0]), spread(t2, o[1], q[1], s2[1]), spread(t2, o[2], q[2], s2[2]), spread(t2, o[3], q[3], s2[3])};
    }
    FENCE(); SB();
#pragma unroll
    for (int i = 0; i < 4; ++i) window4(o + 4 * i, q + 4 * i, z[i], (excl & 0xFFFFu) + cnt[i]);
    SB();
    u32x4 e[4];
#pragma unroll
    for (int j = 0; j < 16; ++j) {
        const u32 ex = spread(t1, o[j], q[j], s1[j]);
        if (MB == 3) {
            const u32 v = j < 8 ? o0[(j >> 1) & 3] : o1[(j >> 1) & 3];
            e[j >> 2][j & 3] = (j & 1) ? and_or(v >> 4, 0x07070707u, and_or(v << 4, 0x80808080u, ex)) : and_or(v, 0x87878787u, ex);
        } else {
            e[j >> 2][j & 3] = and_or(o0[j >> 2] >> (2 * (j & 3)), 0x03030303u, and_or(o1[j >> 3] << (j & 7), 0x80808080u, ex));
        }
        if ((j & 3) == 3) st_img(lane * 16, (j < 8 ? qa : qb) + (j & 4) * 256, e[j >> 2]);
    }
}

#define TS(t)
#define TC(t)
#define V10(a) "v" #a "0", "v" #a "1", "v" #a "2", "v" #a "3", "v" #a "4", "v" #a "5", "v" #a "6", "v" #a "7", "v" #a "8", "v" #a "9"
__device__ __forceinline__ void mfma_role(u32 sa, u32 sb, u32 sc, u32 olo, u32 ohi, u32 scale,
                                          u64& ta, u64& tb, u64& tc, u64& td, u64& ca, u64& cb, u64& cc) {
    u32 need, m, spin, scl;
    asm volatile(
        ".macro AG_FRAG d, a, o\n\t"
        "ds_read_b128 v[\\d:\\d+3], \\a offset:\\o\n\t"
        "ds_read_b128 v[\\d+4:\\d+7], \\a offset:\\o+4096\n\t"
        ".endm\n\t"
        ".macro AG_MM c, a, b\n\t"
        "v_mfma_f32_32x32x64_f8f6f4 v[\\c:\\c+15], v[\\a:\\a+7], v[\\b:\\b+7], v[\\c:\\c+15] cbsz:0 blgp:1\n\t"
        ".endm\n\t"
        ".macro AG_MIN\n\t"
        "v_min_u32 v164, v164, v165\n\t"
        "s_nop 0\n\t"
        "v_readfirstlane_b32 %[m], v164\n\t"
        ".endm\n\t"
        ".macro AG_SPIN\n\t"
        "s_mov_b32 %[spin], 0\n\t"
        ".LAG_S\\@:\n\t"
        "ds_read_b64 v[164:165], v162\n\t"
        "s_add_u32 %[spin], %[spin], 1\n\t"
        "s_waitcnt lgkmcnt(0)\n\t"
        "AG_MIN\n\t"
        "s_cmp_ge_u32 %[m], %[need]\n\t"
        "s_cbranch_scc1 .LAG_E\\@\n\t"
        "s_sleep " STR(MSLEEP) "\n\t"
        "s_cmp_lt_u32 %[spin], 0x2000000\n\t"
        "s_cbranch_scc1 .LAG_S\\@\n\t"
        ".LAG_E\\@:\n\t"
        ".endm\n\t"
        ".macro AG_DONE\n\t"
        "v_add_u32 v163, 1, v163\n\t"
        "ds_write_b32 v162, v163 offset:8\n\t"
        ".endm\n\t"
        ".macro AG_FULL so, sn\n\t"
        "s_waitcnt lgkmcnt(4)\n\t"
        "AG_MM 0, 128, 144\n\t"
        "s_waitcnt lgkmcnt(2)\n\t"
        "AG_MM 64, 136, 144\n\t"
        "ds_read_b64 v[164:165], v162\n\t"
        "AG_FRAG 144, v161, \\so+2048\n\t"
        "s_add_u32 %[need], %[need], 1\n\t"
        "s_waitcnt lgkmcnt(3)\n\t"
        "AG_MM 16, 128, 152\n\t"
        "AG_MM 80, 136, 152\n\t"
        "AG_FRAG 152, v161, \\so+3072\n\t"
        "s_waitcnt lgkmcnt(2)\n\t"
        "AG_MM 32, 128, 144\n\t"
        "AG_MIN\n\t"
        "AG_MM 96, 136, 144\n\t"
        "s_cmp_ge_u32 %[m], %[need]\n\t"
        "s_cbranch_scc1 .LAG_F\\@\n\t"
        "s_waitcnt lgkmcnt(0)\n\t"
        "AG_MM 48, 128, 152\n\t"
        "AG_DONE\n\t"
        "AG_MM 112, 136, 152\n\t"
        "AG_SPIN\n\t"
        "AG_FRAG 144, v161, \\sn\n\t"
        "AG_FRAG 128, v160, \\sn\n\t"
        "s_branch .LAG_J\\@\n\t"
        ".LAG_F\\@:\n\t"
        "AG_FRAG 144, v161, \\sn\n\t"
        "s_waitcnt lgkmcnt(2)\n\t"
        "AG_MM 48, 128, 152\n\t"
        "AG_DONE\n\t"
        "AG_FRAG 128, v160, \\sn\n\t"
        "AG_MM 112, 136, 152\n\t"
        ".LAG_J\\@:\n\t"
        "AG_FRAG 136, v160, \\sn+2048\n\t"
        "AG_FRAG 152, v161, \\sn+1024\n\t"
        ".endm\n\t"
        ".macro AG_PREP c, b\n\t"
        ".irp j, 0, 16, 32, 48\n\t"
        ".irp i, 0, 1, 2, 3\n\t"
        "v_mul_f32 v[\\c+\\j+4*\\b+\\i], %[scl], v[\\c+\\j+4*\\b+\\i]\n\t"
        ".endr\n\t"
        ".endr\n\t"
        "v_swap_b32 v[\\c+4*\\b+1], v[\\c+16+4*\\b]\n\t"
        "v_swap_b32 v[\\c+4*\\b+2], v[\\c+32+4*\\b]\n\t"
        "v_swap_b32 v[\\c+4*\\b+3], v[\\c+48+4*\\b]\n\t"
        "v_swap_b32 v[\\c+16+4*\\b+2], v[\\c+32+4*\\b+1]\n\t"
        "v_swap_b32 v[\\c+16+4*\\b+3], v[\\c+48+4*\\b+1]\n\t"
        "v_swap_b32 v[\\c+32+4*\\b+3], v[\\c+48+4*\\b+2]\n\t"
        ".endm\n\t"
        ".macro AG_ST c, r\n\t"
        ".if (\\r) >= 0\n\t"
        ".if ((\\r) & 3) == 0\n\t"
        "AG_PREP \\c, (\\r)/4\n\t"
        ".endif\n\t"
        ".if \\c\n\t"
        "global_store_dwordx4 v166, v[\\c+16*((\\r)&3)+((\\r)&12):\\c+16*((\\r)&3)+((\\r)&12)+3], vcc" OB1 "\n\t"
        ".else\n\t"
        "global_store_dwordx4 v166, v[\\c+16*((\\r)&3)+((\\r)&12):\\c+16*((\\r)&3)+((\\r)&12)+3], vcc" OB0 "\n\t"
        ".endif\n\t"
        ".if ((\\r) & 3) == 3\n\t"
        "s_add_u32 vcc_lo, vcc_lo, 5*16384\n\t"
        ".else\n\t"
        "s_add_u32 vcc_lo, vcc_lo, 16384\n\t"
        ".endif\n\t"
        "s_addc_u32 vcc_hi, vcc_hi, 0\n\t"
        ".endif\n\t"
        ".endm\n\t"
        ".macro AG_XF d, q\n\t"
        ".if ((\\q) %% 5) == 4\n\t"
        "AG_FRAG \\d, v160, 2*16384+2048\n\t"
        ".else\n\t"
        "AG_FRAG \\d, v160, ((\\q) %% 5)*16384\n\t"
        ".endif\n\t"
        ".endm\n\t"
        ".macro AG_WF d, q, i\n\t"
        ".if ((\\q) %% 5) == 4\n\t"
        "AG_FRAG \\d, v167, (\\i/2)*16384+(\\i&1)*1024\n\t"
        ".else\n\t"
        "AG_FRAG \\d, v161, ((\\q) %% 5)*16384+\\i*1024\n\t"
        ".endif\n\t"
        ".endm\n\t"
        ".macro AG_HALF c, q, last, r0, r1\n\t"
        ".set AG_B0, 136+8*(((\\q)+1) %% 3)\n\t"
        ".set AG_B1, 136+8*(((\\q)+2) %% 3)\n\t"
        ".set AG_B2, 136+8*((\\q) %% 3)\n\t"
        "s_waitcnt lgkmcnt(2)\n\t"
        "AG_MM \\c, 128, AG_B0\n\t"
        "AG_WF AG_B0, \\q, 3\n\t"
        "s_add_u32 %[need], %[need], 1\n\t"
        "AG_MIN\n\t"
        "AG_MM \\c+16, 128, AG_B1\n\t"
        ".if \\last == 0\n\t"
        "s_cmp_ge_u32 %[m], %[need]\n\t"
        "s_cbranch_scc1 .LAG_F\\@\n\t"
        ".endif\n\t"
        "AG_ST 0, \\r0\n\t"
        "s_waitcnt lgkmcnt(2)\n\t"
        "AG_MM \\c+32, 128, AG_B2\n\t"
        "s_waitcnt lgkmcnt(0)\n\t"
        "AG_MM \\c+48, 128, AG_B0\n\t"
        ".if \\last == 0\n\t"
        "AG_DONE\n\t"
        "AG_SPIN\n\t"
        "AG_WF AG_B1, (\\q)+1, 0\n\t"
        "AG_WF AG_B2, (\\q)+1, 1\n\t"
        "s_branch .LAG_J\\@\n\t"
        ".LAG_F\\@:\n\t"
        "AG_WF AG_B1, (\\q)+1, 0\n\t"
        "AG_ST 0, \\r0\n\t"
        "s_waitcnt lgkmcnt(4)\n\t"
        "AG_MM \\c+32, 128, AG_B2\n\t"
        "AG_WF AG_B2, (\\q)+1, 1\n\t"
        "s_waitcnt lgkmcnt(4)\n\t"
        "AG_MM \\c+48, 128, AG_B0\n\t"
        "AG_DONE\n\t"
        ".LAG_J\\@:\n\t"
        "AG_XF 128, (\\q)+1\n\t"
        "ds_read_b64 v[164:165], v162\n\t"
        "AG_WF AG_B0, (\\q)+1, 2\n\t"
        ".endif\n\t"
        "AG_ST 0, \\r1\n\t"
        ".endm\n\t"
        ".macro AG_SCALE\n\t"
        "ds_read_b32 v164, v162 offset:12\n\t"
        "s_nop 15\n\t"
        "s_nop 7\n\t"
        "s_waitcnt lgkmcnt(0)\n\t"
        "v_readfirstlane_b32 %[m], v164\n\t"
        "s_cmp_eq_u32 %[m], 0\n\t"
        "s_cselect_b32 %[scl], %[scale], 0x7fc00000\n\t"
        "ds_read_b64 v[164:165], v162\n\t"
        "s_waitcnt lgkmcnt(0)\n\t"
        ".endm\n\t"

        "v_mbcnt_lo_u32_b32 v167, -1, 0\n\t"
        "v_mbcnt_hi_u32_b32 v167, -1, v167\n\t"
        "v_lshrrev_b32 v165, 5, v167\n\t"
        "v_and_b32 v167, 31, v167\n\t"
        "v_lshlrev_b32 v167, 4, v167\n\t"
        "v_lshl_add_u32 v166, v165, 16, v167\n\t"
        "v_lshl_add_u32 v167, v165, 13, v167\n\t"
        "v_add_u32 v160, %[sa], v167\n\t"
        "v_add_u32 v161, %[sb], v167\n\t"
        "v_subrev_u32 v167, 65536-2048, v161\n\t"
        "v_mov_b32 v162, %[sc]\n\t"
        "v_mov_b32 v163, 0\n\t"
        ".set AG_I, 0\n\t"
        ".rept 128\n\t"
        "v_mov_b32 v[AG_I], 0\n\t"
        ".set AG_I, AG_I+1\n\t"
        ".endr\n\t"
        "s_mov_b32 vcc_lo, %[olo]\n\t"
        "s_mov_b32 vcc_hi, %[ohi]\n\t"
        "s_mov_b32 %[need], 1\n\t"
        "AG_SPIN\n\t"
        TS(ta) TC(ca)
        "AG_FRAG 144, v161, 0\n\t"
        "AG_FRAG 128, v160, 0\n\t"
        "AG_FRAG 136, v160, 2048\n\t"
        "AG_FRAG 152, v161, 1024\n\t"
        ".LAG_LOOP:\n\t"
        "AG_FULL 0, 16384\n\t"
        "AG_FULL 16384, 32768\n\t"
        "AG_FULL 32768, 49152\n\t"
        "AG_FULL 49152, 0\n\t"
        "s_cmp_le_u32 %[need], 64-" STR(NSPLIT) "\n\t"
        "s_cbranch_scc1 .LAG_LOOP\n\t"
        TS(td)
        ".if " STR(NSPLIT) "\n\t"
        "ds_read_b64 v[164:165], v162\n\t"
        "AG_WF 136, 0, 2\n\t"
        ".set AG_I, 0\n\t"
        ".rept " STR(NSPLIT) "\n\t"
        "AG_HALF 0, AG_I, 0, -1, -1\n\t"
        ".set AG_I, AG_I+1\n\t"
        ".endr\n\t"
        "AG_SCALE\n\t"
        TS(tb) TC(cc)
        ".set AG_I, 0\n\t"
        ".rept " STR(NSPLIT) "\n\t"
        ".set AG_R0, (AG_I*16+" STR(NSPLIT) "-1)/" STR(NSPLIT) "\n\t"
        ".set AG_R1, (AG_I*16+16+" STR(NSPLIT) "-1)/" STR(NSPLIT) "-1\n\t"
        ".if AG_R1 == AG_R0\n\t"
        ".set AG_R1, -1\n\t"
        ".endif\n\t"
        "AG_HALF 64, AG_I+" STR(NSPLIT) ", (AG_I==" STR(NSPLIT) "-1), AG_R0, AG_R1\n\t"
        ".set AG_I, AG_I+1\n\t"
        ".endr\n\t"
        "AG_SCALE\n\t"
        TS(tc) TC(cb)
        ".else\n\t"
        "AG_SCALE\n\t"
        TS(tc) TC(cb)
        ".irp r, 0, 1, 2, 3, 4, 5, 6, 7, 8, 9, 10, 11, 12, 13, 14, 15\n\t"
        "AG_ST 0, \\r\n\t"
        ".endr\n\t"
        ".endif\n\t"
        "s_add_u32 vcc_lo, vcc_lo, 96*16384\n\t"
        "s_addc_u32 vcc_hi, vcc_hi, 0\n\t"
        ".irp r, 0, 1, 2, 3, 4, 5, 6, 7, 8, 9, 10, 11, 12, 13, 14, 15\n\t"
        "AG_ST 64, \\r\n\t"
        ".endr\n\t"
        "s_nop 1\n\t"
        : [need] "=&s"(need), [m] "=&s"(m), [spin] "=&s"(spin), [scl] "=&s"(scl)
        : [sa] "s"(sa), [sb] "s"(sb), [sc] "s"(sc), [olo] "s"(olo), [ohi] "s"(ohi), [scale] "s"(scale)
        : "memory", "scc", "vcc", V10(), V10(1), V10(2), V10(3), V10(4), V10(5), V10(6), V10(7), V10(8), V10(9),
          V10(10), V10(11), V10(12), V10(13), V10(14), V10(15),
          "v160", "v161", "v162", "v163", "v164", "v165", "v166", "v167");
}

__device__ __forceinline__ char* peer_of_lane(const Params& P) {
    u32 lo = 0, hi = 0;
#pragma unroll
    for (int p = 0; p < NR; ++p) {
        asm volatile("v_writelane_b32 %0, %2, %4\n\tv_writelane_b32 %1, %3, %4"
                     : "+v"(lo), "+v"(hi) : "s"((u32)(u64)P.ws[p]), "s"((u32)((u64)P.ws[p] >> 32)), "i"(p));
    }
    return (char*)(((u64)hi << 32) | lo);
}
__device__ __forceinline__ void arrive(LAS unsigned char* L, int lane) {
    if (lane == 0) __hip_atomic_fetch_add((LAS u32*)(L + LDS_PRO), 1u, __ATOMIC_RELAXED, __HIP_MEMORY_SCOPE_WORKGROUP);
}

template <bool W> __device__ __forceinline__ void prologue_unit(const Params& P, LAS unsigned char* L, char* myws, u32 callv,
                                                                 const u32 (&d)[16], int idx, int xg, int wave, int lane) {
    u32 x[16];
    __builtin_amdgcn_s_setprio(3);
    const int u = xg * 4 + wave;
    const bool half = !W && u >= (int)R0;
    char* peer = peer_of_lane(P);
    WAIT_VM0();
    if (W) {
        order_w(x, d);
    } else {
        LAS u32* xc = (LAS u32*)(L + LDS_XCH);
        if (lane == 0) __hip_atomic_fetch_add(xc, 1u, __ATOMIC_RELAXED, __HIP_MEMORY_SCOPE_WORKGROUP);
        for (u32 spin = 0; spin < SPINS && UNI(*(volatile LAS u32*)xc) < 4; ++spin) {}
        FENCE();
        LAS unsigned char* b = L + LDS_A + 32768 + lane * 64 + (half ? (wave & 1) * 8192 + (wave & 2) * 16 : wave * 16);
        const u32 hi = half ? 16 : 8192;
#pragma unroll
        for (int i = 0; i < 4; ++i) {
            const u32x4 v = *(LAS u32x4*)(b + (i & 1) * 4096 + (i >> 1) * hi);
            x[4 * i] = v[0]; x[4 * i + 1] = v[1]; x[4 * i + 2] = v[2]; x[4 * i + 3] = v[3];
        }
    }
    const u32 par = (UNI(callv) + 1) & 1;
    bool raw = RAWTAIL == 1 ? u == (int)ROUNDS - 1 && (!W || !NSPLIT)
             : RAWTAIL >= 2 && (W ? u == (int)ROUNDS - 1 : (RAWTAIL == 3 && u == (int)ROUNDS - 2) || (RAWTAIL == 4 && u >= (int)ROUNDS - 2) || (RAWTAIL == 5 && u >= (int)ROUNDS - 3));
    const char* qa = myws + WS_IMG + par * IMG_BYTES + unit_off(idx, half ? u & ~1 : u) + (half ? (u & 1) * 2048 : 0);
    const char* qb = qa + (half ? 4096 : 2048);
    st_sys_s(lane * 16, qa, piece(x, 0)); st_sys_s(lane * 16, qa + 1024, piece(x, 1));
    st_sys_s(lane * 16, qb, piece(x, 2)); st_sys_s(lane * 16, qb + 1024, piece(x, 3));
    u32 len64 = 64;
    if (!W && u < NE) {
        char* dst = myws + WS_CB + par * CB_BYTES + cb_off(idx, u);
#pragma unroll
        for (int i = 0; i < 4; ++i) st_sys_s(lane * 16, dst + i * 1024, piece(x, i));
    } else if (!W || u >= NPW) len64 = encode_unit<W ? 2 : 3>(x, L + LDS_A + wave * 8192, L, myws + WS_CB + par * CB_BYTES + cb_off(idx, u), lane, raw);
    if (lane == 0) *(volatile LAS u32*)(L + LDS_LEN + wave * 4) = len64;
    WAIT_VM0();
    arrive(L, lane);
    if (xg == 0 && (!W || u >= NPW) && lane < NR && lane != P.rank)
        asm volatile("global_store_dword %0, %1, off sc0 sc1" :: "v"(peer + WS_RDY + par * 32768 + (P.rank * 32 + idx) * 128 + u * 4),
                     "v"(((UNI(callv) + 1) << 7) | len64) : "memory");
    __builtin_amdgcn_s_setprio(0);
}

__global__ void __launch_bounds__(NTHREADS, 3) ag_gemm(Params P) {
    __shared__ __attribute__((aligned(16))) unsigned char lds_raw[LDS_BYTES];
    LAS unsigned char* L = (LAS unsigned char*)lds_raw;
    const int hw = UNI(threadIdx.x >> 6);
    const int wave = hw < 4 ? 11 - hw : hw - 4;
    const int lane = threadIdx.x & 63;
    const int bid = blockIdx.x;
    const unsigned char* px = P.x;
    const unsigned char* pw_ = P.w;
    char* myws = P.myws;
    int rank = P.rank;
    u32 srcmap = P.srcmap;
    asm volatile("" : "+s"(px), "+s"(pw_), "+s"(myws), "+s"(rank), "+s"(srcmap) : "s"(P.ws[0]));
    const int xg = bid & 7, idx = bid >> 3;
    const int rg = (xg + 5) & 7;
    const int tm = (xg >> 1) * 4 + (idx >> 3), tn = (xg & 1) * 8 + (idx & 7);
    GAS u32* tmo = (GAS u32*)(myws + WS_TMO);

    u32 xo = (lane >> 2) * KS + (lane & 3) * 16;
    u32 d[16];
    if (wave < 4 && idx < 16) {
        const unsigned char* g = px + (size_t)(idx * 256 + wave * 64) * KS + rg * 64;
        asm volatile("s_mov_b32 m0, %2\n\ts_nop 0\n\tglobal_load_lds_dwordx4 %0, %1\n\t"
                     ".rept 3\n\ts_add_u32 m0, m0, 0x400\n\tv_add_u32 %0, 16*512, %0\n\tglobal_load_lds_dwordx4 %0, %1\n\t.endr"
                     : "+v"(xo) : "s"(g), "s"((u32)(size_t)L + LDS_A + 32768 + wave * 4096) : "memory", "scc");
    }
    FENCE();
    const int pr = (idx >> 4) + 2 * (11 - wave);
    if (wave < 4 ? idx >= 16 : wave >= 10 && pr < NPW) load_w(d, pw_, idx, wave < 4 ? rg * 4 + wave : pr, lane);
    u32 call = __hip_atomic_load((GAS u32*)(myws + WS_CNT + bid * 128), __ATOMIC_RELAXED, __HIP_MEMORY_SCOPE_AGENT);
    if (wave < 4) {
        encode_clear(L + LDS_A + wave * 8192, lane);
    } else if (wave < 8) {
        const u32 f = (wave - 4) * 64 + lane;
        u32 dl = 0, cl = 0x0C0C0C0Cu, k = 0;
#pragma unroll
        for (u32 i = 0; i < 4; ++i) {
            const u32 c = (f >> (2 * i)) & 3u;
            dl |= (c ? 4 + c : k) << (8 * i);
            cl ^= c ? 0u : (12u ^ i) << (8 * k);
            k += c == 0;
        }
        *(LAS u32*)(L + LDS_DL + f * 4) = dl;
        *(LAS u32*)(L + LDS_CL + f * 4) = cl;
    } else if (wave == 8) {
        ((LAS u32*)(L + LDS_CTL))[lane] = 0;
    }
    BAR();

    if (wave < 8) {
        if (wave < 4) {
            if (idx < 16) prologue_unit<false>(P, L, myws, call, d, idx, rg, wave, lane);
            else prologue_unit<true>(P, L, myws, call, d, idx, rg, wave, lane);
        }
        const u32 scb = UNI(__builtin_bit_cast(u32, P.sx[0] * P.sw[0]));
        const int g = wave >> 1, nh = wave & 1;
        const u64 ob = (u64)P.out + ((size_t)(tm * 256 + g * 32) * MN + tn * 256 + nh * 128) * 4;
        u32 olo = (u32)ob, ohi = (u32)(ob >> 32);
        u64 ta = 0, tb = 0, tc = 0, td = 0, ca = 0, cb = 0, cc = 0;
        mfma_role((u32)(size_t)L + LDS_A + g * 512, (u32)(size_t)L + LDS_B + nh * 512, (u32)(size_t)L + LDS_CTL + wave * 16,
                  olo, ohi, scb, ta, tb, tc, td, ca, cb, cc);
    } else if (wave < 10) {
        const u64 tk = rt();
        __builtin_amdgcn_s_setprio(2);
        if (wave == 8) {
            char* peer = peer_of_lane(P);
            call = UNI(call) + 1;
            STAMP(0);
            WAIT_VM0();
            for (u32 spin = 0; spin < SPINS && UNI(*(volatile LAS u32*)(L + LDS_PRO)) < 7; ++spin) __builtin_amdgcn_s_sleep(1);
            FENCE();
            const u32x4 ln = *(LAS u32x4*)(L + LDS_LEN);
            if (lane < NR) {
                if (lane != rank) { if (rg) st_sys(peer + WS_RDY + (call & 1) * 32768 + (rank * 32 + idx) * 128 + rg * 16, ln | (call << 7)); }
                else st_sys(peer + WS_UF + idx * 2048 + rg * 16, u32x4{call, call, call, call});
            }
            if (lane == 0) __hip_atomic_store((GAS u32*)(myws + WS_CNT + bid * 128), call, __ATOMIC_RELAXED, __HIP_MEMORY_SCOPE_AGENT);
            STAMP(7);
        } else {
            call = UNI(call) + 1;
            arrive(L, lane);
            u32 job = XCD_ID() * 32 + idx;
#pragma unroll 1
            for (int k = 0; k < NWG; ++k, job = (job + 1) & (NWG - 1)) {
                u32 old = 0;
                if (lane == 0) {
                    __hip_atomic_fetch_max((GAS u32*)(myws + WS_TIK + (job >> 5) * 128), call * 2048, __ATOMIC_RELAXED, __HIP_MEMORY_SCOPE_AGENT);
                    FENCE();
                    old = __hip_atomic_fetch_max((GAS u32*)(myws + WS_JOB + job * 128), call, __ATOMIC_RELAXED, __HIP_MEMORY_SCOPE_AGENT);
                }
                WAIT_VM0();
                if (UNI(old) < call) break;
            }
            *(volatile LAS u32*)(L + LDS_JOB) = job + 1;
        }
        const int isB = wave - 8;
        const int panel = isB ? 16 + tn : tm;
        const char* img = myws + WS_IMG + (call & 1) * IMG_BYTES + unit_off(panel, 0);
        const u32 ldsb = (u32)(size_t)L + (isB ? LDS_B : LDS_A);
        volatile LAS u32* ctl = (volatile LAS u32*)(L + LDS_CTL + (lane & 7) * 16);
        const char* uf = myws + WS_UF + panel * 2048;
        const u32 voff = lane * 16;
        Ld S = {0, 0, 0, 0, 0, ~0u, 0, 0, (u64)img, (u64)(img + (16 + tn - panel) * 16384 + (size_t)S0 * 524288)};
        u32 err = 0;
        const u32 vctl = (u32)(size_t)ctl, vfull = vctl + isB * 4, vf = (lane & 31) * 4;
        const u64 t0 = rt();
        WAIT_VM0();
        asm volatile("v_mov_b32 v40, 0\n\tv_mov_b32 v41, 0\n\tv_mov_b32 v47, 0\n\tv_add_u32 v44, 0x1000, %0\n\t"
                     "v_add_u32 v45, 0x2000, %0\n\tv_add_u32 v46, 0x3000, %0" :: "v"(voff) : "v40", "v41", "v44", "v45", "v46", "v47");
        auto late = [&]() {
            if (UNI(__hip_atomic_load(tmo, __ATOMIC_RELAXED, __HIP_MEMORY_SCOPE_AGENT)) == call || rt() - t0 > DEADLINE) {
                if (lane == 0) __hip_atomic_store(tmo, call, __ATOMIC_RELAXED, __HIP_MEMORY_SCOPE_AGENT);
                ctl[3] = 1;
                err = ~0u;
            }
            asm volatile("s_waitcnt vmcnt(0) lgkmcnt(0)" ::: "memory");
            return rt() - t0 < 3 * DEADLINE;
        };
#define RUN(M, end, l) while (load_run<M>(S, end, l, err, call, uf, voff, vf, vctl, vfull) && late()) {}
        if (LDELAY) while (rt() - tk < LDELAY) __builtin_amdgcn_s_sleep(4);
        if (QEASY) while (load_run<0, NEASY>(S, QEASY, ldsb, err, call, uf, voff, vf, vctl, vfull) && late()) {}
        RUN(0, S0, ldsb);
        LS(9 + isB);
        if (NSPLIT) {
            const u32 l0 = (u32)(size_t)L + LDS_A;
            if (isB) { RUN(2, NSTAGE, l0); LS(16); S.g -= (size_t)NSPLIT * 524288; RUN(4, NQ, l0); }
            else { RUN(1, NSTAGE, l0); S.g -= (size_t)NSPLIT * 524288 - 2048; RUN(3, NQ, l0); }
        }
        LS(14 + isB);
        RUN(5, 0, 0);
        ctl[isB] = 0xFFFFu;
        WAIT_VM(0);
        STAMP(3 + isB);
    } else {
        __builtin_amdgcn_s_setprio(2);
        const u32 vo = lane * 16, wk = wave - 10;
#define KS_(i) do {} while (0)
#define FS(i) do {} while (0)
        {
            u32 x[16];
            const u32 pdst = (srcmap >> (4 * xg)) & 7u;
            const bool push = pr < NPW && pdst != (u32)rank && !(P.pad & 1);
            const u32 up = 32 + ((rank - pdst - 1) & 7) + 7 * pr, pp = idx | 16;
            const char* pw = P.ws[pdst];
            const char* q0 = pw + WS_IMG + unit_off(pp, up);
            asm volatile("" : "+s"(q0));
            WAIT_VM0();
            call = UNI(call) + 1;
            if (push) {
                order_w(x, d);
                q0 += (call & 1) * IMG_BYTES;
                FS(0);
                st_sys_s(vo, q0, piece(x, 0));
                arrive(L, lane);
                wait_vm<PWIN>(); st_sys_s(vo, q0 + 1024, piece(x, 1));
                wait_vm<PWIN>(); st_sys_s(vo, q0 + 2048, piece(x, 2));
                wait_vm<PWIN>(); st_sys_s(vo, q0 + 3072, piece(x, 3));
                WAIT_VM(0); st_flag(pw + WS_UF + pp * 2048 + up * 4, call, lane);
                FS(2);
            } else {
                arrive(L, lane);
            }
            if (wk) *(volatile LAS u32*)(L + LDS_PSH) = 1;
        }
        call = UNI(call);
        u32 job = 0;
        for (u32 spin = 0; spin < SPINS && job == 0; ++spin) { __builtin_amdgcn_s_sleep(1); job = UNI(*(volatile LAS u32*)(L + LDS_JOB)); }
        job = (job - 1) & (NWG - 1);
        const u32 slot = job >> 5, src = (srcmap >> (4 * slot)) & 7u;
        if (src == (u32)rank || (P.pad & 1)) return;
        const char* tik = myws + WS_TIK + slot * 128;
        const char* scb = P.ws[src] + WS_CB + (call & 1) * CB_BYTES;
        const char* rdy = myws + WS_RDY + (call & 1) * 32768 + src * 4096;
        const char* dp = myws + WS_IMG + (call & 1) * IMG_BYTES;
        const u32 u0 = 32 + ((src - rank - 1) & 7);
        LAS unsigned char* sl = L + LDS_WRK + wk * WRK_BYTES + 3072;
        const u32 rot = ROT * rank;
        auto unit_of = [rot](u32 t) -> u32 {
            auto px = [rot](u32 k) { k = (k + rot) & 15; return 4 * (k & 3) + (k >> 2); };
            auto pw = [rot](u32 k) { k = (k + rot) & 15; return 16 + 8 * (k & 1) + (k >> 1); };
            if (t < T0) return t < 64 ? px(t & 15) * 32 + (t >> 4) : pw(t & 15) * 32 + NPW + (t >> 4) - 4;
            if (t < T1) { t -= T0; return ((t & 1) ? pw((t >> 1) & 15) : px((t >> 1) & 15)) * 32 + 4 + (t >> 5); }
            if (t < T2) {
                const u32 pr = (t - T1) / 48, i = (t - T1) % 48;
                return ((i & 16) ? px(i & 15) : pw(i & 15)) * 32 + R0 + 2 * pr + (i >> 5);
            }
            return t < NT ? px((t - T2) & 15) * 32 + R0 + 2 * ((t - T2) >> 4) + 1 : 0u;
        };
        const u64 t0 = rt();
        auto await = [&](u32 un) -> u32 {
            for (u32 n = 1;; ++n) {
                u32 rv;
                if (lane == 0) asm volatile("global_atomic_or v162, %0, %0, %1 sc0 sc1\n\ts_waitcnt vmcnt(0)" :: "v"(0u), "s"(rdy + un * 4) : "memory", "v162");
                asm volatile("v_readfirstlane_b32 %0, v162" : "=s"(rv) :: "memory", "v162");
                if ((rv >> 7) >= call) return rv;
                if ((n & 31) == 0 && (UNI(__hip_atomic_load(tmo, __ATOMIC_RELAXED, __HIP_MEMORY_SCOPE_AGENT)) == call || rt() - t0 > DEADLINE)) {
                    if (lane == 0) __hip_atomic_store(tmo, call, __ATOMIC_RELAXED, __HIP_MEMORY_SCOPE_AGENT);
                    return 0;
                }
                __builtin_amdgcn_s_sleep(2);
            }
        };
        auto book = [&](u64 at, u32 cost) {
            const u64 now = at << 10, c = cost;
            if (lane == 0)
                asm volatile("global_atomic_umax_x2 %0, %1, %3 offset:2048\n\tglobal_atomic_add_x2 v[160:161], %0, %2, %3 offset:2048 sc0"
                             :: "v"(0u), "v"(now), "v"(c), "s"(tik) : "memory", "v160", "v161");
        };
        const char* pend = nullptr;
        u32 pn = 0;
        auto flag = [&]() {
            if ((u32)lane <= pn) asm volatile("global_store_dword %0, %1, %2 sc0 sc1" :: "v"(lane * 28), "v"(call), "s"(pend) : "memory");
            pend = nullptr;
        };
        u32 t = (job & 31) + 32 * wk, un = unit_of(t), k = 0, lat = 0, plat = 0;
        if (TOUCH && lane < 2)
            asm volatile("global_load_dword v163, %0, %1 sc0 sc1" :: "v"(lane * (2u << 20) + 4092 + (job & 31) * 4096), "s"(scb) : "memory", "v163");
        if (OWNFIRST)
            for (u32 spin = 0; spin < SPINS && UNI(*(volatile LAS u32*)(L + (t < 16 * NE ? LDS_PSH : LDS_PRO))) < (t < 16 * NE ? 1 : 7); ++spin) __builtin_amdgcn_s_sleep(2);
        u32 rv = await(un);
        if (!rv) return;
        WAIT_VM0();
        book(rt(), (rv & 127u) * RATE);
        bool now_ = t < NOWAIT;
        if (!now_) WAIT_VM(0);
        t += 64;
        for (;; ++k) {
            u32 len64 = rv & 127u;
            len64 = len64 < 41 ? 41 : (len64 > 64 ? 64 : len64);
            const u32 len = len64 * 64, panel = un >> 5, r = un & 31;
            const char* q = scb + cb_off(panel, r);
            const bool more = t < NT;
            un = unit_of(t);
            const char* rp = rdy + un * 4;
            asm volatile("" : "+s"(q), "+s"(rp));
            if (!now_) {
                u32 lo, hi;
                asm volatile("v_readfirstlane_b32 %0, v160\n\tv_readfirstlane_b32 %1, v161" : "=s"(lo), "=s"(hi) :: "memory", "v160", "v161");
                const u32 start = (u32)((((u64)hi << 32) | lo) >> 10);
                for (u32 d = start - (u32)rt(); d - 1 < 1000u; d = start - (u32)rt()) {
                    if (pend) { if (vm_idle()) { WAIT_VM(0); flag(); } }
                    else if (d > 12) __builtin_amdgcn_s_sleep(2);
                }
            }
            now_ = false;
            const u32 i0 = (u32)rt();
            PIECE(0, q); PIECE(1, q);
            if (2048 + vo < len) PIECE(2, q);
            if (3072 + vo < len) PIECE(3, q);
            if (lane == 0) asm volatile(RDYLD "\n\tglobal_atomic_add v163, %0, %1, %2 sc0"
                                        :: "v"(0u), "v"(1u), "s"(tik), "s"(rp) : "memory", "v162", "v163");
            KS_(0);
            const bool half = panel < 16 && r >= R0;
            const u32 u = u0 + 7 * (half ? r & ~1u : r);
            const char* qa = dp + unit_off(panel, u) + (half ? (r & 1) * 2048 : 0);
            const char* qb = half ? dp + unit_off(panel, u + 7) + (r & 1) * 2048 : qa + 2048;
            const u32 ho = panel < 16 ? 3072 : 2560;
            const bool late = pend != nullptr;
            if (late) WAIT_VM(5);
            asm volatile("" : "+s"(qa), "+s"(qb) :: "memory");
            if (late) { flag(); WAIT_VM(3); } else { WAIT_VM(2); }
            KS_(1);
            const u32x4 w = *(LAS u32x4*)(sl + lane * 16);
            const u32 hd = *(LAS u32*)(sl + ho), w2 = *(LAS u32*)(sl + ho + 4 + lane * 4);
            const u64 now = rt();
            if (late) WAIT_VM(1); else WAIT_VM(0);
            KS_(2);
            asm volatile("v_readfirstlane_b32 %0, v163\n\tv_readfirstlane_b32 %1, v162" : "=s"(t), "=s"(rv) :: "memory", "v162", "v163");
            plat = lat;
            lat = (u32)now - i0;
            const u32 rate = (lat < plat ? lat : plat) > LATHI ? RATE + RATE / PEN : RATE;
            t += 128 - call * 2048;
            const bool known = (rv >> 7) >= call;
            if (more && known) book(now, (rv & 127u) * rate);
            KS_(3);
            FENCE(); SB();
            if (len64 == 64) {
                const u32x4 e1 = *(LAS u32x4*)(sl + 1024 + lane * 16), e2 = *(LAS u32x4*)(sl + 2048 + lane * 16), e3 = *(LAS u32x4*)(sl + 3072 + lane * 16);
                st_img(vo, qa, w); st_img(vo, qa + 1024, e1); st_img(vo, qb, e2); st_img(vo, qb + 1024, e3);
            } else if (panel < 16) {
                decode_unit<3>(qa, qb, UNI(hd), w, w2, sl, L, lane);
            } else {
                decode_unit<2>(qa, qb, UNI(hd), w, w2, sl, L, lane);
            }
            pend = myws + WS_UF + panel * 2048 + (u + (half && (r & 1) ? 4 * NSPLIT : 0)) * 4;
            pn = half;
            if (!more) break;
            if (!known) {
                WAIT_VM(0); flag();
                rv = await(un);
                if (!rv) return;
                WAIT_VM0();
                book(rt(), (rv & 127u) * rate);
                WAIT_VM(0);
            }
            WAIT_VM(4);
        }
        WAIT_VM(0);
        flag();
    }
}

extern "C" void kernel_launch(int rank, int nranks, ncclComm_t comm,
                              hipStream_t stream,
                              void* const* d_in, const int* in_sizes, int n_in,
                              void* d_out, int out_size,
                              const int* in_dtypes, int out_dtype,
                              void* const* d_out_all,
                              void* const* d_ws_all, size_t ws_bytes) {
    (void)comm; (void)in_sizes; (void)n_in; (void)out_size; (void)in_dtypes; (void)out_dtype; (void)d_out_all;
    if (nranks != NR || (unsigned)rank >= (unsigned)NR || ws_bytes < WS_NEED) {
        fprintf(stderr, "kernel_launch: unexpected configuration\n");
        return;
    }
    Params p;
    p.x = (const unsigned char*)d_in[0];
    p.w = (const unsigned char*)d_in[1];
    p.sx = (const float*)d_in[2];
    p.sw = (const float*)d_in[3];
    p.out = (float*)d_out;
    for (int i = 0; i < NR; ++i) p.ws[i] = (char*)d_ws_all[i];
    p.myws = p.ws[rank];
    p.rank = rank;
    p.pad = 0;
    p.pad2 = 0;
    static const u32 near_link[NR] = {0x74510263u, 0x26301475u, 0x17542036u, 0x74153620u,
                                      0x63024157u, 0x60235741u, 0x51476302u, 0x02367514u};
    p.srcmap = near_link[rank];
    hipLaunchKernelGGL(ag_gemm, dim3(NWG), dim3(NTHREADS), 0, stream, p);
    HIP_CHECK(hipGetLastError());
}
```

```cpp
#include <hip/hip_runtime.h>
#include <rccl/rccl.h>
#include <cstdio>

#define HIP_CHECK(expr) do { hipError_t _e = (expr); if (_e != hipSuccess) { \
    fprintf(stderr, "HIP error %s at %s:%d\n", hipGetErrorString(_e), __FILE__, __LINE__); \
    return; } } while (0)

#define GAS __attribute__((address_space(1)))
#define LAS __attribute__((address_space(3)))
typedef unsigned u32;
typedef unsigned short u16;
typedef unsigned long long u64;
typedef unsigned u32x2 __attribute__((ext_vector_type(2)));
typedef unsigned u32x4 __attribute__((ext_vector_type(4)));

constexpr int NR = 8;
constexpr int MN = 4096;
constexpr int KS = 512;
constexpr int NWG = 256;
constexpr int NTHREADS = 768;
constexpr u32 ROUNDS = 32;
constexpr u32 NSTAGE = 64;
#define NSPLIT 8
#define NS0 4
#define NST 4
constexpr int NPW = 2;
constexpr u32 RATE = 217;
constexpr u32 LATHI = 105;
constexpr u32 NOWAIT = 12;
constexpr u32 QEASY = 32, LDELAY = 500;
constexpr u32 S0 = NSTAGE - NSPLIT;
constexpr u32 NQ = NSTAGE + NSPLIT;
constexpr u32 R0 = ((S0 * 4 - 32 + 13) / 14) * 2;
constexpr u32 US = 32 + 7 * R0;
constexpr u32 NPAIR = (ROUNDS - R0) / 2;
constexpr u32 T0 = 128 - 16 * NPW, T1 = T0 + (R0 - 4) * 32, T2 = T1 + NPAIR * 48, NT = T2 + NPAIR * 16;
constexpr u32 UNIT = 4096;
constexpr size_t IMG_BYTES = 32u << 20;
constexpr size_t CB_BYTES = 5u << 20;
__host__ __device__ constexpr u32 unit_off(u32 panel, u32 u) { return (u >> 2) * 524288u + panel * 16384u + ((u + panel) & 3) * UNIT; }
constexpr u32 SLOT = UNIT + 256;
__host__ __device__ constexpr u32 cb_off(u32 panel, u32 r) { return (r * 32 + panel) * SLOT; }
constexpr size_t WS_CNT = 0;
constexpr size_t WS_TMO = 48 << 10;
constexpr size_t WS_TIK = 52 << 10;
constexpr size_t WS_UF = 64 << 10;
constexpr size_t WS_RDY = 128 << 10;
constexpr size_t WS_JOB = 192 << 10;
constexpr size_t WS_PAD = 224 << 10;
constexpr size_t WS_IMG = 1 << 20;
constexpr size_t WS_CB = WS_IMG + 2 * IMG_BYTES;
constexpr size_t WS_NEED = WS_CB + 2 * CB_BYTES;
constexpr u64 DEADLINE = 50000000ull;
constexpr u32 SPINS = 1u << 25;
constexpr u32 LDS_CTL = 0;
constexpr u32 LDS_PSH = LDS_CTL + 132, LDS_XCH = LDS_CTL + 136, LDS_PRO = LDS_CTL + 140;
constexpr u32 LDS_JOB = LDS_CTL + 160, LDS_LEN = LDS_CTL + 176;
constexpr u32 LDS_DL = 512, LDS_CL = LDS_DL + 1024;
constexpr u32 LDS_WRK = LDS_CL + 1024, WRK_BYTES = 7168;
constexpr u32 LDS_A = LDS_WRK + 2 * WRK_BYTES, LDS_B = LDS_A + 65536;
constexpr u32 LDS_BYTES = LDS_B + 65536;

struct Params {
    const unsigned char* x;
    const unsigned char* w;
    char* myws;
    int rank;
    u32 srcmap;
    const float* sx;
    const float* sw;
    float* out;
    u64 pad;
    char* ws[NR];
};

__device__ __forceinline__ u64 rt() { return __builtin_amdgcn_s_memrealtime(); }
#define WAIT_VM(n) asm volatile("s_waitcnt vmcnt(" #n ")" ::: "memory")
#define WAIT_VM0() do { __builtin_amdgcn_s_waitcnt(0x0F70); WAIT_VM(0); } while (0)
#define FENCE() asm volatile("" ::: "memory")
#define SB() __builtin_amdgcn_sched_barrier(0)
#define XCD_ID() (__builtin_amdgcn_s_getreg((3 << 11) | 20) & 7u)
#define UNI(x) __builtin_amdgcn_readfirstlane(x)

__device__ __forceinline__ void st_sys(char* p, u32x4 v) {
    asm volatile("global_store_dwordx4 %0, %1, off sc0 sc1\n\ts_nop 1" :: "v"(p), "v"(v) : "memory");
}
__device__ __forceinline__ void st_sys_s(u32 voff, const void* base, u32x4 v) {
    asm volatile("global_store_dwordx4 %0, %1, %2 sc0 sc1\n\ts_nop 1" :: "v"(voff), "v"(v), "s"(base) : "memory");
}
__device__ __forceinline__ void st_flag(const void* base, u32 v, int lane) {
    if (lane == 0) asm volatile("global_store_dword %0, %1, %2 sc0 sc1" :: "v"(0u), "v"(v), "s"(base) : "memory");
}

#define STR2(x) #x
#define STR(x) STR2(x)
struct Ld {
    u32 nops, q, rmask, nf;
    u32 pidx, pb, slot;
    u64 fifo, g, gw;
};
#define G2(v, s_, o) "global_load_lds_dwordx4 " v ", " s_ " offset:" #o " sc0 sc1\n\tglobal_load_lds_dwordx4 " v ", " s_ " offset:" #o "+1024" " sc0 sc1\n\t"
#define EASE ".if %c[ease]\n\ts_sleep %c[ease]\n\t.endif\n\t"
#define M0A(x) "s_add_u32 m0, m0, " x "\n\ts_nop 0\n\t"
#define GW(v) G2(v, "%[g]", 0) M0A("%[land]") G2(v, "%[g]", 2048)
template <int MODE, int EASY = 0>
__device__ __forceinline__ u32 load_run(Ld& S, u32 end, u32 lds, u32 err, u32 call, const void* uf,
                                        u32 vo, u32 vf, u32 vctl, u32 vfull) {
    u32 a, b, land, rc, keep;
    asm volatile(
        "s_mov_b32 %[keep], m0\n\t"
        "s_mov_b32 %[rc], 0\n\t"
        "s_branch .LL%=\n\t"
        ".LZ%=:\n\t"
        ".if %c[md] == 0\n\t"
        "s_sleep 1\n\t"
        ".endif\n\t"
        ".LL%=:\n\t"
        "ds_read_b32 v42, %[vctl] offset:8\n\t"
        "s_getreg_b32 %[a], hwreg(HW_REG_IB_STS)\n\t"
        "s_bfe_u32 %[b], %[a], 0x20016\n\t"
        "s_and_b32 %[a], %[a], 15\n\t"
        "s_lshl4_add_u32 %[a], %[b], %[a]\n\t"
        "s_sub_u32 %[land], %[nops], %[a]\n\t"
        "s_mov_b64 vcc, %[fifo]\n\t"
        "s_sub_u32 %[a], %[land], vcc_lo\n\t"
        "s_bitcmp0_b32 %[a], 11\n\t"
        "s_cselect_b32 %[a], %[nf], 0\n\t"
        "s_cmp_eq_u32 %[a], 0\n\t"
        "s_cbranch_scc1 .LP%=\n\t"
        "v_add_u32 v41, 1, v41\n\t"
        "s_lshr_b64 %[fifo], %[fifo], 12\n\t"
        "ds_write_b32 %[vfull], v41\n\t"
        "s_sub_u32 %[nf], %[nf], 1\n\t"
        ".LP%=:\n\t"
        "s_add_u32 %[rc], %[rc], 0x100000\n\t"
        "s_cbranch_scc1 .LT%=\n\t"
        ".if %c[md] == 5\n\t"
        "s_waitcnt lgkmcnt(0)\n\t"
        "s_cmp_lg_u32 %[nf], 0\n\t"
        "s_cbranch_scc1 .LL%=\n\t"
        ".else\n\t"
        ".if %c[md] != 4\n\t"
        "s_or_b32 %[rmask], %[rmask], %[err]\n\t"
        "s_sub_u32 %[a], %[land], %[pidx]\n\t"
        "s_cmp_lt_i32 %[a], 0\n\t"
        "s_cbranch_scc1 .LO%=\n\t"
        "s_cmp_eq_u32 %[pb], -1\n\t"
        "s_cbranch_scc1 .LS%=\n\t"
        "v_cmp_le_u32 vcc, %[call], v40\n\t"
        "s_sub_u32 %[a], %[q], %[pb]\n\t"
        "s_lshl_b32 %[a], %[a], 1\n\t"
        "s_min_u32 %[a], %[a], 16\n\t"
        "s_lshr_b32 %[b], vcc_lo, %[a]\n\t"
        "s_lshr_b32 %[b], %[b], %[a]\n\t"
        "s_or_b32 %[rmask], %[rmask], %[b]\n\t"
        "s_mov_b32 %[pb], -1\n\t"
        "s_andn2_b32 %[b], 15, %[rmask]\n\t"
        "s_cbranch_scc0 .LO%=\n\t"
        ".LS%=:\n\t"
        "s_andn2_b32 %[b], 0xffff, %[rmask]\n\t"
        "s_cbranch_scc0 .LO%=\n\t"
        "s_lshl_b32 %[b], %[q], 4\n\t"
        "v_add_u32 v43, %[b], %[vf]\n\t"
        ".if %c[md] & 1\n\t"
        "v_subrev_u32 v40, 0x400, v43\n\t"
        "v_cmp_gt_u32 vcc, %[whole], v40\n\t"
        "v_subrev_u32 v40, 16*" STR(NSPLIT) ", v43\n\t"
        "s_nop 0\n\t"
        "v_cndmask_b32 v43, v43, v40, vcc\n\t"
        ".endif\n\t"
        "s_add_u32 %[nops], %[nops], 1\n\t"
        "s_mov_b32 exec_hi, 0\n\t"
        "global_atomic_or v40, v43, v47, %[uf] sc0 sc1\n\t"
        "s_mov_b32 exec_hi, -1\n\t"
        "s_mov_b32 %[pidx], %[nops]\n\t"
        "s_mov_b32 %[pb], %[q]\n\t"
        ".LO%=:\n\t"
        ".endif\n\t"
        ".if %c[md] == 0\n\t"
        "s_sub_u32 %[a], %[q], 3\n\t"
        ".else\n\t"
        "s_cmp_ge_u32 %[q], 68-" STR(NSPLIT) "\n\t"
        "s_subb_u32 %[a], %[q], 3\n\t"
        ".endif\n\t"
        "s_waitcnt lgkmcnt(0)\n\t"
        "v_cmp_gt_i32 vcc, %[a], v42\n\t"
        "s_lshl_b32 %[a], %[slot], 14\n\t"
        ".if %c[md]\n\t"
        "s_add_u32 %[b], %[a], 0x10000\n\t"
        "s_cmp_eq_u32 %[slot], 4\n\t"
        "s_cselect_b32 %[a], 2*16384+2048, %[a]\n\t"
        "s_cselect_b32 %[b], 2048, %[b]\n\t"
        "s_cselect_b32 %[land], 16384-2048, 0\n\t"
        ".endif\n\t"
        ".if %c[md] != 4\n\t"
        "s_andn2_b32 m0, 15, %[rmask]\n\t"
        "s_or_b32 m0, m0, vcc_lo\n\t"
        ".else\n\t"
        "s_cmp_lg_u32 vcc_lo, 0\n\t"
        ".endif\n\t"
        "s_cbranch_scc1 .LZ%=\n\t"
        ".if %c[md] == 0\n\t"
        "s_add_u32 m0, %[a], %[lds]\n\t"
        "s_add_u32 %[nops], %[nops], 16\n\t"
        G2("%[vo]", "%[g]", 0) G2("%[vo]", "%[g]", 2048) EASE M0A("0x1000") G2("v44", "%[g]", 0) G2("v44", "%[g]", 2048) EASE M0A("0x1000")
        G2("v45", "%[g]", 0) G2("v45", "%[g]", 2048) EASE M0A("0x1000") G2("v46", "%[g]", 0) G2("v46", "%[g]", 2048)
        ".elseif %c[md] & 1\n\t"
        "s_add_u32 m0, %[a], %[lds]\n\t"
        "s_add_u32 %[nops], %[nops], 6+2*%c[md]\n\t"
        G2("%[vo]", "%[g]", 0) M0A("0x1000") G2("v44", "%[g]", 0) M0A("0x1000") G2("v45", "%[g]", 0) M0A("0x1000") G2("v46", "%[g]", 0)
        ".if %c[md] == 3\n\t"
        "s_add_u32 %[b], %[b], %[lds]\n\t"
        "s_add_u32 m0, %[b], 0x3000\n\t"
        "s_mov_b64 vcc, %[gw]\n\t"
        G2("v48", "vcc", 0) M0A("%[land]") G2("v48", "vcc", 2048)
        "s_add_u32 vcc_lo, vcc_lo, 0x80000\n\t"
        "s_addc_u32 vcc_hi, vcc_hi, 0\n\t"
        "s_mov_b64 %[gw], vcc\n\t"
        ".endif\n\t"
        ".else\n\t"
        "s_add_u32 m0, %[b], %[lds]\n\t"
        "s_sub_u32 %[a], 0x1000, %[land]\n\t"
        "s_add_u32 %[nops], %[nops], 20-2*%c[md]\n\t"
        GW("%[vo]") M0A("%[a]") GW("v44") M0A("%[a]") GW("v45")
        ".if %c[md] == 2\n\t"
        M0A("%[a]") GW("v46")
        ".endif\n\t"
        ".endif\n\t"
        "s_mov_b64 vcc, %[g]\n\t"
        "s_add_u32 vcc_lo, vcc_lo, 0x80000\n\t"
        "s_addc_u32 vcc_hi, vcc_hi, 0\n\t"
        "s_mov_b64 %[g], vcc\n\t"
        "s_and_b32 vcc_lo, %[nops], 0xfff\n\t"
        "s_mov_b32 vcc_hi, 0\n\t"
        "s_mul_i32 %[a], %[nf], 12\n\t"
        "s_lshl_b64 vcc, vcc, %[a]\n\t"
        "s_or_b64 %[fifo], %[fifo], vcc\n\t"
        "s_add_u32 %[nf], %[nf], 1\n\t"
        "s_lshr_b32 %[rmask], %[rmask], 4\n\t"
        "s_add_u32 %[slot], %[slot], 1\n\t"
        "s_cmp_eq_u32 %[slot], 4+(%c[md]+7)/8\n\t"
        "s_cselect_b32 %[slot], 0, %[slot]\n\t"
        "s_add_u32 %[q], %[q], 1\n\t"
        "s_cmp_lt_u32 %[q], %[end]\n\t"
        "s_cbranch_scc1 .LL%=\n\t"
        ".endif\n\t"
        "s_mov_b32 %[rc], 0\n\t"
        "s_branch .LE%=\n\t"
        ".LT%=:\n\t"
        "s_mov_b32 %[rc], 1\n\t"
        "s_waitcnt lgkmcnt(0)\n\t"
        ".LE%=:\n\t"
        "s_mov_b32 m0, %[keep]"
        : [nops] "+s"(S.nops), [q] "+s"(S.q), [rmask] "+s"(S.rmask), [nf] "+s"(S.nf), [pidx] "+s"(S.pidx), [pb] "+s"(S.pb),
          [slot] "+s"(S.slot), [fifo] "+s"(S.fifo), [g] "+s"(S.g), [gw] "+s"(S.gw),
          [a] "=&s"(a), [b] "=&s"(b), [land] "=&s"(land), [rc] "=&s"(rc), [keep] "=&s"(keep)
        : [end] "s"(end), [lds] "s"(lds), [err] "s"(err), [call] "s"(call), [uf] "s"(uf),
          [vo] "v"(vo), [vf] "v"(vf), [vctl] "v"(vctl), [vfull] "v"(vfull), [md] "i"(MODE), [ease] "i"(EASY), [whole] "i"(4 * (US - 4 * S0))
        : "memory", "scc", "vcc", "v40", "v41", "v42", "v43", "v44", "v45", "v46", "v47", "v48");
    return rc;
}
#define PIECE(i, g) do { u32 k_; asm volatile("s_mov_b32 %0, m0\n\ts_mov_b32 m0, %3\n\ts_nop 0\n\t" \
    "global_load_lds_dwordx4 %1, %2 offset:1024*" #i " sc0 sc1\n\ts_mov_b32 m0, %0" : "=&s"(k_) : "v"(vo), "s"(g), "s"((u32)(size_t)sl) : "memory"); } while (0)
__device__ __forceinline__ bool vm_idle() {
    u32 r;
    asm volatile("s_getreg_b32 %0, hwreg(HW_REG_IB_STS)" : "=s"(r) :: "memory");
    return (r & 0xC0000Fu) == 0;
}
__device__ __forceinline__ u32 perm(u32 hi, u32 lo, u32 sel) { return __builtin_amdgcn_perm(hi, lo, sel); }
__device__ __forceinline__ u32 bfi(u32 m, u32 a, u32 b) { return __builtin_amdgcn_bitop3_b32(m, a, b, 0xCA); }
__device__ __forceinline__ u32 and_or(u32 a, u32 m, u32 c) { return __builtin_amdgcn_bitop3_b32(a, m, c, 0xEA); }
#define BAR() asm volatile("s_waitcnt lgkmcnt(0)\n\ts_barrier" ::: "memory")
__device__ __forceinline__ u32 scan64(u32 v) {
    v += __builtin_amdgcn_update_dpp(0u, v, 0x111, 0xf, 0xf, true);
    v += __builtin_amdgcn_update_dpp(0u, v, 0x112, 0xf, 0xf, true);
    v += __builtin_amdgcn_update_dpp(0u, v, 0x114, 0xf, 0xf, true);
    v += __builtin_amdgcn_update_dpp(0u, v, 0x118, 0xf, 0xf, true);
    v += __builtin_amdgcn_update_dpp(0u, v, 0x142, 0xa, 0xf, false);
    v += __builtin_amdgcn_update_dpp(0u, v, 0x143, 0xc, 0xf, false);
    return v;
}
__device__ __forceinline__ void lds_or(LAS unsigned char* p, u32 v) {
    __hip_atomic_fetch_or((LAS u32*)p, v, __ATOMIC_RELAXED, __HIP_MEMORY_SCOPE_WORKGROUP);
}

__device__ __forceinline__ void load_w(u32 (&d)[16], const unsigned char* w, int panel, int r, int lane) {
    const auto rs = __builtin_amdgcn_make_buffer_rsrc((void*)w, 0, MN * KS, 0x00020000);
    const u32 so = r * 16 * MN + (panel & 15) * 256;
#pragma unroll
    for (int k = 0; k < 16; ++k) d[k] = __builtin_amdgcn_raw_buffer_load_b32(rs, lane * 4, so + k * MN, 0);
}
__device__ __forceinline__ void order_w(u32 (&x)[16], const u32 (&d)[16]) {
#pragma unroll
    for (int q = 0; q < 4; ++q) {
        u32 t0 = perm(d[4 * q + 1], d[4 * q], 0x05010400), t1 = perm(d[4 * q + 1], d[4 * q], 0x07030602);
        u32 t2 = perm(d[4 * q + 3], d[4 * q + 2], 0x05010400), t3 = perm(d[4 * q + 3], d[4 * q + 2], 0x07030602);
        x[q] = perm(t2, t0, 0x05040100); x[4 + q] = perm(t2, t0, 0x07060302);
        x[8 + q] = perm(t3, t1, 0x05040100); x[12 + q] = perm(t3, t1, 0x07060302);
    }
}
__device__ __forceinline__ u32x4 piece(const u32 (&x)[16], int i) { return u32x4{x[4 * i], x[4 * i + 1], x[4 * i + 2], x[4 * i + 3]}; }

template <int MB> struct Fmt {
    static constexpr u32 EM = (MB == 3 ? 0x0Fu : 0x1Fu) * 0x01010101u;
    static constexpr u32 HEAD = MB == 3 ? 3072 : 2048;
    static constexpr u32 OFF_H = MB == 3 ? 0 : 512, OFF_2 = OFF_H + 4;
};
__device__ __forceinline__ u32 zero2(u32 w) { return __builtin_amdgcn_bitop3_b32(w, w >> 1, 0x55555555u, 0x02); }
__device__ __forceinline__ u32 code_of(u32 hi, u32 lo, u32 v) {
    const u32 r = perm(hi, lo, v);
    return __builtin_amdgcn_bitop3_b32(r, r >> 2, 0x03030303u, 0x28);
}
__device__ __forceinline__ void lookup4(u32* sel, u32 w, LAS unsigned char* tab) {
#pragma unroll
    for (int j = 0; j < 4; ++j) sel[j] = *(LAS u32*)(tab + ((w >> (8 * j)) & 255u) * 4);
}
__device__ __forceinline__ void gather4(const u32* v, const u32* sel, u32 z, u32 a) {
#pragma unroll
    for (int j = 0; j < 4; ++j) {
        const u32 o = j ? a + __builtin_popcount(z & ((1u << (8 * j)) - 1)) : a;
        const u32 c = perm(0u, v[j], sel[j]);
        LAS unsigned char* p = (LAS unsigned char*)(size_t)(o & ~3u);
        lds_or(p, __builtin_amdgcn_alignbyte(c, 0u, ~o));
        lds_or(p + 4, __builtin_amdgcn_alignbyte(0u, c, ~o));
    }
}
__device__ __forceinline__ void window4(u32* o, u32x2* q, u32 z, u32 a) {
#pragma unroll
    for (int j = 0; j < 4; ++j) {
        o[j] = j ? a + __builtin_popcount(z & ((1u << (8 * j)) - 1)) : a;
        LAS u32* p = (LAS u32*)(size_t)(o[j] & ~3u);
        q[j] = u32x2{p[0], p[1]};
    }
}
__device__ __forceinline__ u32 spread(u32 tab, u32 o, u32x2 q, u32 sel) {
    return perm(tab, __builtin_amdgcn_alignbyte(q[1], q[0], o), sel);
}
__device__ __forceinline__ u32 pack4(const u32* c) {
    u32 w = __builtin_amdgcn_udot4(c[0], 0x40100401u, 0u, false);
#pragma unroll
    for (int j = 1; j < 4; ++j) w |= __builtin_amdgcn_udot4(c[j], 0x40100401u, 0u, false) << (8 * j);
    return w;
}
__device__ __forceinline__ void encode_clear(LAS unsigned char* T, int lane) {
#pragma unroll
    for (int i = 2; i < 5; ++i) *(LAS u32x4*)(T + i * 1024 + lane * 16) = u32x4{0, 0, 0, 0};
}

template <int MB> __device__ __forceinline__ u32 encode_unit(const u32 (&x)[16], LAS unsigned char* T, LAS unsigned char* L,
                                                             char* dst, int lane, bool raw) {
    typedef Fmt<MB> F;
    u32 sum = 0;
#pragma unroll
    for (int j = 0; j < 16; j += 5) sum = __builtin_amdgcn_udot4(x[j] & 0x7F7F7F7Fu, 0x01010101u, sum, false);
    const int tot = __builtin_amdgcn_readlane(scan64(sum), 63);
    const u32 eb = (u32)((tot - (1516 << MB)) >> (10 + MB)) & (F::EM & 31u);
    const u32 B = (eb << MB) * 0x01010101u;
    u32 v[16], c[16];
    u32x4 w, z;
#pragma unroll
    for (int j = 0; j < 16; ++j) {
        v[j] = (((x[j] | 0x80808080u) - B) >> MB) & F::EM;
        c[j] = code_of(0u, 0x03020100u, v[j]);
    }
    u32 cnt[5] = {0};
#pragma unroll
    for (int q = 0; q < 4; ++q) {
        w[q] = pack4(c + 4 * q); z[q] = zero2(w[q]); cnt[q + 1] = cnt[q] + __builtin_popcount(z[q]);
        lookup4(c + 4 * q, w[q], L + LDS_CL);
    }
    FENCE();
    const u32 incl = scan64(cnt[4]), n1 = __builtin_amdgcn_readlane(incl, 63);
    u32 n2 = 0, n3 = 0, len64 = 64;
    raw = raw || n1 > 2040;
    const u32 off3 = F::OFF_2 + ((n1 + 15) >> 4) * 4;
    if (!raw) {
        const u32 a1 = incl - cnt[4] + ((u32)(size_t)T + 2047);
#pragma unroll
        for (int q = 0; q < 4; ++q) gather4(v + 4 * q, c + 4 * q, z[q], a1 + cnt[q]);
        st_sys_s(lane * 16, dst, w);
        if (MB == 3) {
            u32x4 o[2];
#pragma unroll
            for (int p = 0; p < 8; ++p) o[p >> 2][p & 3] = bfi(0x87878787u, x[2 * p], bfi(0xF0F0F0F0u, x[2 * p + 1] << 4, x[2 * p + 1] >> 4));
            st_sys_s(lane * 16, dst + 1024, o[0]);
            st_sys_s(lane * 16, dst + 2048, o[1]);
        } else {
            u32x4 o;
            u32x2 sg;
#pragma unroll
            for (int q = 0; q < 4; ++q)
                o[q] = bfi(0x0F0F0F0Fu, bfi(0x03030303u, x[4 * q], x[4 * q + 1] << 2), bfi(0x03030303u, x[4 * q + 2], x[4 * q + 3] << 2) << 4);
#pragma unroll
            for (int k = 0; k < 2; ++k) {
                u32 t = x[8 * k];
#pragma unroll
                for (int j = 1; j < 8; ++j) t = bfi(0x80808080u >> j, x[8 * k + j] >> j, t);
                sg[k] = t;
            }
            *(LAS u32x2*)(T + lane * 8) = sg;
            st_sys_s(lane * 16, dst + 1024, o);
        }
        FENCE();
        for (u32 b = 0; b < n1; b += 1024) {
            const u32x4 y = *(LAS u32x4*)(T + 2048 + b + lane * 16);
            u32 u[4], c2[4];
#pragma unroll
            for (int j = 0; j < 4; ++j) {
                c2[j] = code_of(0x00000300u, 0x00000102u, (y[j] + 0x01010101u) & F::EM);
                u[j] = (y[j] + 0x05050505u) & F::EM;
            }
            const u32 w2 = pack4(c2);
            lookup4(c2, w2, L + LDS_CL);
            FENCE();
            const u32 z2 = zero2(w2), k2 = __builtin_popcount(z2), i2 = scan64(k2);
            if (b + lane * 16 < n1) *(LAS u32*)(T + F::OFF_2 + (b >> 2) + lane * 4) = w2;
            const u32 all = n2 + __builtin_amdgcn_readlane(i2, 63);
            if (all <= 1016 && k2) gather4(u, c2, z2, n2 + i2 - k2 + ((u32)(size_t)T + 4095));
            n2 = all;
        }
        raw = n2 > 1016;
    }
    if (!raw) {
        FENCE();
        const u32x4 y = *(LAS u32x4*)(T + 4096 + lane * 16);
        u32 k3 = 0, m[4];
#pragma unroll
        for (int j = 0; j < 4; ++j) { m[j] = ((y[j] & 0x1C1C1C1Cu) + 0x1C1C1C1Cu) & 0x20202020u; k3 += __builtin_popcount(m[j]); }
        u32 i3 = 0;
        if (__builtin_amdgcn_ballot_w64(k3 != 0)) { i3 = scan64(k3); n3 = __builtin_amdgcn_readlane(i3, 63); }
        const u32 offp = off3 + ((n2 + 15) >> 4) * 4;
        len64 = (F::HEAD + offp + 2 * n3 + 63) >> 6;
        raw = len64 >= 64 || n3 > 63;
        if (!raw) {
            if (lane * 16 < n2)
                *(LAS u32*)(T + off3 + lane * 4) = bfi(0x0F0F0F0Fu, bfi(0x03030303u, y[0], y[1] << 2), bfi(0x03030303u, y[2], y[3] << 2) << 4);
            if (lane == 0) *(LAS u32*)(T + F::OFF_H) = eb | (n1 << 5) | (n2 << 16) | (n3 << 26);
            if (n3) {
                LAS unsigned char* p = T + offp + 2 * (i3 - k3);
                u32 mm = (m[0] >> 5) | (m[1] >> 4) | (m[2] >> 3) | (m[3] >> 2);
                while (mm) {
                    const u32 b = __builtin_ctz(mm), i = lane * 16 + (b & 3u) * 4 + (b >> 3);
                    mm &= mm - 1;
                    *(LAS u16*)p = (u16)((i << 5) | T[4096 + i]);
                    p += 2;
                }
            }
        }
    }
    if (raw) {
        WAIT_VM(0);
#pragma unroll
        for (int i = 0; i < 4; ++i) st_sys_s(lane * 16, dst + i * 1024, piece(x, i));
        return 64;
    }
    FENCE();
    const u32 tl = len64 * 64 - F::HEAD;
    if (lane * 16 < tl) st_sys_s(lane * 16, dst + F::HEAD, *(LAS u32x4*)(T + lane * 16));
    if (MB == 2 && 1024 + lane * 16 < tl) st_sys_s(lane * 16, dst + F::HEAD + 1024, *(LAS u32x4*)(T + 1024 + lane * 16));
    return len64;
}

template <int MB> __device__ __forceinline__ void decode_unit(const char* qa, const char* qb, u32 hdr, u32x4 w, u32 w2,
                                                              LAS unsigned char* S, LAS unsigned char* L, int lane) {
    typedef Fmt<MB> F;
    LAS unsigned char* T = S + F::HEAD;
    const u32 eb = hdr & 31u, n1 = (hdr >> 5) & 2047u, n2 = (hdr >> 16) & 1023u, n3 = hdr >> 26;
    const u32 off3 = F::OFF_2 + ((n1 + 15) >> 4) * 4, offp = off3 + ((n2 + 15) >> 4) * 4;
    const u32 w3 = *(LAS u32*)(T + off3 + lane * 4);
    const u32 en = *(LAS u16*)(T + offp + lane * 2);
    u32 s1[16], s2[4];
    lookup4(s2, w2, L + LDS_DL);
    const u32x4 o0 = *(LAS u32x4*)(S + 1024 + lane * 16);
    u32x4 o1;
    if (MB == 3) o1 = *(LAS u32x4*)(S + 2048 + lane * 16);
    else { const u32x2 sg = *(LAS u32x2*)(T + lane * 8); o1 = u32x4{sg[0], sg[1], 0, 0}; }
    FENCE(); SB();
    const u32 eb4 = eb * 0x01010101u;
    const u32 t1 = ((eb4 + 0x03020100u) & F::EM) << MB, t2 = ((eb4 + 0x041F0000u) & F::EM) << MB, t3 = ((eb4 + 0x1E1D1C1Bu) & F::EM) << MB;
    u32x4 z;
    u32 cnt[5] = {0};
#pragma unroll
    for (int q = 0; q < 4; ++q) { z[q] = zero2(w[q]); cnt[q + 1] = cnt[q] + __builtin_popcount(z[q]); }
    u32 z2 = zero2(w2), k2 = __builtin_popcount(z2);
    const u32 both = cnt[4] | (k2 << 16), incl = scan64(both);
    const u32 excl = incl - both + ((u32)(size_t)S - 2048) + (((u32)(size_t)S - 3072) << 16);
    SB();
    {
        u32x4 y;
#pragma unroll
        for (int j = 0; j < 4; ++j) y[j] = perm(0u, t3, (w3 >> (2 * j)) & 0x03030303u);
        *(LAS u32x4*)(S - 3072 + lane * 16) = y;
        FENCE();
        if ((u32)lane < n3) *(S - 3072 + ((en >> 5) & 1023u)) = (unsigned char)(((en + eb + 27u) & (F::EM & 31u)) << MB);
        FENCE();
    }
    u32 o[16];
    u32x2 q[16];
    window4(o, q, z2, excl >> 16);
    SB();
#pragma unroll
    for (int i = 0; i < 4; ++i) lookup4(s1 + 4 * i, w[i], L + LDS_DL);
    FENCE(); SB();
    *(LAS u32x4*)(S - 2048 + lane * 16) =
        u32x4{spread(t2, o[0], q[0], s2[0]), spread(t2, o[1], q[1], s2[1]), spread(t2, o[2], q[2], s2[2]), spread(t2, o[3], q[3], s2[3])};
    if (n1 > 1024) {
        w2 = *(LAS u32*)(T + F::OFF_2 + 256 + lane * 4);
        lookup4(s2, w2, L + LDS_DL);
        z2 = zero2(w2); k2 = __builtin_popcount(z2);
        window4(o, q, z2, (__builtin_amdgcn_readlane(incl, 63) >> 16) + (u32)(size_t)S - 3072 + scan64(k2) - k2);
        *(LAS u32x4*)(S - 2048 + 1024 + lane * 16) =
            u32x4{spread(t2, o[0], q[0], s2[0]), spread(t2, o[1], q[1], s2[1]), spread(t2, o[2], q[2], s2[2]), spread(t2, o[3], q[3], s2[3])};
    }
    FENCE(); SB();
#pragma unroll
    for (int i = 0; i < 4; ++i) window4(o + 4 * i, q + 4 * i, z[i], (excl & 0xFFFFu) + cnt[i]);
    SB();
    u32x4 e[4];
#pragma unroll
    for (int j = 0; j < 16; ++j) {
        const u32 ex = spread(t1, o[j], q[j], s1[j]);
        if (MB == 3) {
            const u32 v = j < 8 ? o0[(j >> 1) & 3] : o1[(j >> 1) & 3];
            e[j >> 2][j & 3] = (j & 1) ? and_or(v >> 4, 0x07070707u, and_or(v << 4, 0x80808080u, ex)) : and_or(v, 0x87878787u, ex);
        } else {
            e[j >> 2][j & 3] = and_or(o0[j >> 2] >> (2 * (j & 3)), 0x03030303u, and_or(o1[j >> 3] << (j & 7), 0x80808080u, ex));
        }
        if ((j & 3) == 3) st_sys_s(lane * 16, (j < 8 ? qa : qb) + (j & 4) * 256, e[j >> 2]);
    }
}

#define V10(a) "v" #a "0", "v" #a "1", "v" #a "2", "v" #a "3", "v" #a "4", "v" #a "5", "v" #a "6", "v" #a "7", "v" #a "8", "v" #a "9"
__device__ __forceinline__ void mfma_role(u32 sa, u32 sb, u32 sc, u32 olo, u32 ohi, u32 scale) {
    u32 need, m, spin, scl;
    asm volatile(
        ".macro AG_FRAG d, a, o\n\t"
        "ds_read_b128 v[\\d:\\d+3], \\a offset:\\o\n\t"
        "ds_read_b128 v[\\d+4:\\d+7], \\a offset:\\o+4096\n\t"
        ".endm\n\t"
        ".macro AG_MM c, a, b\n\t"
        "v_mfma_f32_32x32x64_f8f6f4 v[\\c:\\c+15], v[\\a:\\a+7], v[\\b:\\b+7], v[\\c:\\c+15] cbsz:0 blgp:1\n\t"
        ".endm\n\t"
        ".macro AG_MIN\n\t"
        "v_min_u32 v164, v164, v165\n\t"
        "s_nop 0\n\t"
        "v_readfirstlane_b32 %[m], v164\n\t"
        ".endm\n\t"
        ".macro AG_SPIN\n\t"
        "s_mov_b32 %[spin], 0\n\t"
        ".LAG_S\\@:\n\t"
        "ds_read_b64 v[164:165], v162\n\t"
        "s_add_u32 %[spin], %[spin], 1\n\t"
        "s_waitcnt lgkmcnt(0)\n\t"
        "AG_MIN\n\t"
        "s_cmp_ge_u32 %[m], %[need]\n\t"
        "s_cbranch_scc1 .LAG_E\\@\n\t"
        "s_sleep 1\n\t"
        "s_cmp_lt_u32 %[spin], 0x2000000\n\t"
        "s_cbranch_scc1 .LAG_S\\@\n\t"
        "s_mov_b32 %[scale], 0x7fc00000\n\t"
        ".LAG_E\\@:\n\t"
        ".endm\n\t"
        ".macro AG_DONE\n\t"
        "v_add_u32 v163, 1, v163\n\t"
        "ds_write_b32 v162, v163 offset:8\n\t"
        ".endm\n\t"
        ".macro AG_FULL so, sn\n\t"
        "s_waitcnt lgkmcnt(4)\n\t"
        "AG_MM 0, 128, 144\n\t"
        "s_waitcnt lgkmcnt(2)\n\t"
        "AG_MM 64, 136, 144\n\t"
        "ds_read_b64 v[164:165], v162\n\t"
        "AG_FRAG 144, v161, \\so+2048\n\t"
        "s_add_u32 %[need], %[need], 1\n\t"
        "s_waitcnt lgkmcnt(3)\n\t"
        "AG_MM 16, 128, 152\n\t"
        "AG_MM 80, 136, 152\n\t"
        "AG_FRAG 152, v161, \\so+3072\n\t"
        "s_waitcnt lgkmcnt(2)\n\t"
        "AG_MM 32, 128, 144\n\t"
        "AG_MIN\n\t"
        "AG_MM 96, 136, 144\n\t"
        "s_cmp_ge_u32 %[m], %[need]\n\t"
        "s_cbranch_scc1 .LAG_F\\@\n\t"
        "s_waitcnt lgkmcnt(0)\n\t"
        "AG_MM 48, 128, 152\n\t"
        "AG_DONE\n\t"
        "AG_MM 112, 136, 152\n\t"
        "AG_SPIN\n\t"
        "AG_FRAG 144, v161, \\sn\n\t"
        "AG_FRAG 128, v160, \\sn\n\t"
        "s_branch .LAG_J\\@\n\t"
        ".LAG_F\\@:\n\t"
        "AG_FRAG 144, v161, \\sn\n\t"
        "s_waitcnt lgkmcnt(2)\n\t"
        "AG_MM 48, 128, 152\n\t"
        "AG_DONE\n\t"
        "AG_FRAG 128, v160, \\sn\n\t"
        "AG_MM 112, 136, 152\n\t"
        ".LAG_J\\@:\n\t"
        "AG_FRAG 136, v160, \\sn+2048\n\t"
        "AG_FRAG 152, v161, \\sn+1024\n\t"
        ".endm\n\t"
        ".macro AG_PREP c, b\n\t"
        ".irp j, 0, 16, 32, 48\n\t"
        ".irp i, 0, 1, 2, 3\n\t"
        "v_mul_f32 v[\\c+\\j+4*\\b+\\i], %[scl], v[\\c+\\j+4*\\b+\\i]\n\t"
        ".endr\n\t"
        ".endr\n\t"
        "v_swap_b32 v[\\c+4*\\b+1], v[\\c+16+4*\\b]\n\t"
        "v_swap_b32 v[\\c+4*\\b+2], v[\\c+32+4*\\b]\n\t"
        "v_swap_b32 v[\\c+4*\\b+3], v[\\c+48+4*\\b]\n\t"
        "v_swap_b32 v[\\c+16+4*\\b+2], v[\\c+32+4*\\b+1]\n\t"
        "v_swap_b32 v[\\c+16+4*\\b+3], v[\\c+48+4*\\b+1]\n\t"
        "v_swap_b32 v[\\c+32+4*\\b+3], v[\\c+48+4*\\b+2]\n\t"
        ".endm\n\t"
        ".macro AG_ST c, r\n\t"
        ".if ((\\r) & 3) == 0\n\t"
        "AG_PREP \\c, (\\r)/4\n\t"
        ".endif\n\t"
        "global_store_dwordx4 v166, v[\\c+16*((\\r)&3)+((\\r)&12):\\c+16*((\\r)&3)+((\\r)&12)+3], vcc\n\t"
        ".if ((\\r) & 3) == 3\n\t"
        "s_add_u32 vcc_lo, vcc_lo, 5*16384\n\t"
        ".else\n\t"
        "s_add_u32 vcc_lo, vcc_lo, 16384\n\t"
        ".endif\n\t"
        "s_addc_u32 vcc_hi, vcc_hi, 0\n\t"
        ".endm\n\t"
        ".macro AG_STN lo, hi\n\t"
        ".set AG_K, \\lo\n\t"
        ".rept (\\hi)-(\\lo)\n\t"
        "AG_ST 0, AG_K\n\t"
        ".set AG_K, AG_K+1\n\t"
        ".endr\n\t"
        ".endm\n\t"
        ".macro AG_XF d, q\n\t"
        ".if ((\\q) %% 5) == 4\n\t"
        "AG_FRAG \\d, v160, 2*16384+2048\n\t"
        ".else\n\t"
        "AG_FRAG \\d, v160, ((\\q) %% 5)*16384\n\t"
        ".endif\n\t"
        ".endm\n\t"
        ".macro AG_WF d, q, i\n\t"
        ".if ((\\q) %% 5) == 4\n\t"
        "AG_FRAG \\d, v167, (\\i/2)*16384+(\\i&1)*1024\n\t"
        ".else\n\t"
        "AG_FRAG \\d, v161, ((\\q) %% 5)*16384+\\i*1024\n\t"
        ".endif\n\t"
        ".endm\n\t"
        ".macro AG_HALF c, q, last, r0, r1, r2\n\t"
        ".set AG_B0, 136+8*(((\\q)+1) %% 3)\n\t"
        ".set AG_B1, 136+8*(((\\q)+2) %% 3)\n\t"
        ".set AG_B2, 136+8*((\\q) %% 3)\n\t"
        "s_waitcnt lgkmcnt(2)\n\t"
        "AG_MM \\c, 128, AG_B0\n\t"
        "AG_WF AG_B0, \\q, 3\n\t"
        "s_add_u32 %[need], %[need], 1\n\t"
        "AG_MIN\n\t"
        "AG_MM \\c+16, 128, AG_B1\n\t"
        ".if \\last == 0\n\t"
        "s_cmp_ge_u32 %[m], %[need]\n\t"
        "s_cbranch_scc1 .LAG_F\\@\n\t"
        ".endif\n\t"
        "AG_STN \\r0, \\r1\n\t"
        "s_waitcnt lgkmcnt(2)\n\t"
        "AG_MM \\c+32, 128, AG_B2\n\t"
        "s_waitcnt lgkmcnt(0)\n\t"
        "AG_MM \\c+48, 128, AG_B0\n\t"
        ".if \\last == 0\n\t"
        "AG_DONE\n\t"
        "AG_SPIN\n\t"
        "AG_WF AG_B1, (\\q)+1, 0\n\t"
        "AG_WF AG_B2, (\\q)+1, 1\n\t"
        "s_branch .LAG_J\\@\n\t"
        ".LAG_F\\@:\n\t"
        "AG_WF AG_B1, (\\q)+1, 0\n\t"
        "AG_STN \\r0, \\r1\n\t"
        "s_waitcnt lgkmcnt(4)\n\t"
        "AG_MM \\c+32, 128, AG_B2\n\t"
        "AG_WF AG_B2, (\\q)+1, 1\n\t"
        "s_waitcnt lgkmcnt(4)\n\t"
        "AG_MM \\c+48, 128, AG_B0\n\t"
        "AG_DONE\n\t"
        ".LAG_J\\@:\n\t"
        "AG_XF 128, (\\q)+1\n\t"
        "ds_read_b64 v[164:165], v162\n\t"
        "AG_WF AG_B0, (\\q)+1, 2\n\t"
        ".endif\n\t"
        "AG_STN \\r1, \\r2\n\t"
        ".endm\n\t"
        ".macro AG_SCALE\n\t"
        "ds_read_b32 v164, v162 offset:12\n\t"
        "s_nop 15\n\t"
        "s_nop 7\n\t"
        "s_waitcnt lgkmcnt(0)\n\t"
        "v_readfirstlane_b32 %[m], v164\n\t"
        "s_cmp_eq_u32 %[m], 0\n\t"
        "s_cselect_b32 %[scl], %[scale], 0x7fc00000\n\t"
        ".endm\n\t"

        "s_waitcnt vmcnt(0) lgkmcnt(0)\n\t"
        "v_mbcnt_lo_u32_b32 v167, -1, 0\n\t"
        "v_mbcnt_hi_u32_b32 v167, -1, v167\n\t"
        "v_lshrrev_b32 v165, 5, v167\n\t"
        "v_and_b32 v167, 31, v167\n\t"
        "v_lshlrev_b32 v167, 4, v167\n\t"
        "v_lshl_add_u32 v166, v165, 16, v167\n\t"
        "v_lshl_add_u32 v167, v165, 13, v167\n\t"
        "v_add_u32 v160, %[sa], v167\n\t"
        "v_add_u32 v161, %[sb], v167\n\t"
        "v_subrev_u32 v167, 65536-2048, v161\n\t"
        "v_mov_b32 v162, %[sc]\n\t"
        "v_mov_b32 v163, 0\n\t"
        ".set AG_I, 0\n\t"
        ".rept 128\n\t"
        "v_mov_b32 v[AG_I], 0\n\t"
        ".set AG_I, AG_I+1\n\t"
        ".endr\n\t"
        "s_mov_b32 vcc_lo, %[olo]\n\t"
        "s_mov_b32 vcc_hi, %[ohi]\n\t"
        "s_mov_b32 %[need], 1\n\t"
        "AG_SPIN\n\t"
        "AG_FRAG 144, v161, 0\n\t"
        "AG_FRAG 128, v160, 0\n\t"
        "AG_FRAG 136, v160, 2048\n\t"
        "AG_FRAG 152, v161, 1024\n\t"
        ".LAG_LOOP:\n\t"
        "AG_FULL 0, 16384\n\t"
        "AG_FULL 16384, 32768\n\t"
        "AG_FULL 32768, 49152\n\t"
        "AG_FULL 49152, 0\n\t"
        "s_cmp_le_u32 %[need], 64-" STR(NSPLIT) "\n\t"
        "s_cbranch_scc1 .LAG_LOOP\n\t"
        "ds_read_b64 v[164:165], v162\n\t"
        "AG_WF 136, 0, 2\n\t"
        ".set AG_I, 0\n\t"
        ".rept " STR(NSPLIT) "\n\t"
        "AG_HALF 0, AG_I, (AG_I==" STR(NSPLIT) "-1), 0, 0, 0\n\t"
        ".set AG_I, AG_I+1\n\t"
        ".endr\n\t"
        "AG_DONE\n\t"
        "AG_SCALE\n\t"
        "AG_STN 0, " STR(NS0) "\n\t"
        "AG_SPIN\n\t"
        "AG_WF AG_B1, AG_I, 0\n\t"
        "AG_WF AG_B2, AG_I, 1\n\t"
        "AG_XF 128, AG_I\n\t"
        "ds_read_b64 v[164:165], v162\n\t"
        "AG_WF AG_B0, AG_I, 2\n\t"
        ".set AG_I, 0\n\t"
        ".rept " STR(NSPLIT) "\n\t"
        ".set AG_R0, " STR(NS0) "+(AG_I*(16-" STR(NS0) ")+" STR(NST) "-1)/" STR(NST) "\n\t"
        ".set AG_R2, " STR(NS0) "+((AG_I+1)*(16-" STR(NS0) ")+" STR(NST) "-1)/" STR(NST) "\n\t"
        ".if AG_I >= " STR(NST) "\n\t"
        ".set AG_R0, 16\n\t"
        ".set AG_R2, 16\n\t"
        ".endif\n\t"
        "AG_HALF 64, AG_I+" STR(NSPLIT) ", (AG_I==" STR(NSPLIT) "-1), AG_R0, (AG_R0+AG_R2+1)/2, AG_R2\n\t"
        ".set AG_I, AG_I+1\n\t"
        ".endr\n\t"
        "AG_SCALE\n\t"
        "s_add_u32 vcc_lo, vcc_lo, 96*16384\n\t"
        "s_addc_u32 vcc_hi, vcc_hi, 0\n\t"
        ".irp r, 0, 1, 2, 3, 4, 5, 6, 7, 8, 9, 10, 11, 12, 13, 14, 15\n\t"
        "AG_ST 64, \\r\n\t"
        ".endr\n\t"
        "s_nop 1\n\t"
        : [scale] "+s"(scale), [need] "=&s"(need), [m] "=&s"(m), [spin] "=&s"(spin), [scl] "=&s"(scl)
        : [sa] "s"(sa), [sb] "s"(sb), [sc] "s"(sc), [olo] "s"(olo), [ohi] "s"(ohi)
        : "memory", "scc", "vcc", V10(), V10(1), V10(2), V10(3), V10(4), V10(5), V10(6), V10(7), V10(8), V10(9),
          V10(10), V10(11), V10(12), V10(13), V10(14), V10(15),
          "v160", "v161", "v162", "v163", "v164", "v165", "v166", "v167");
}

__device__ __forceinline__ char* peer_of_lane(const Params& P) {
    u32 lo = 0, hi = 0;
#pragma unroll
    for (int p = 0; p < NR; ++p) {
        asm volatile("v_writelane_b32 %0, %2, %4\n\tv_writelane_b32 %1, %3, %4"
                     : "+v"(lo), "+v"(hi) : "s"((u32)(u64)P.ws[p]), "s"((u32)((u64)P.ws[p] >> 32)), "i"(p));
    }
    return (char*)(((u64)hi << 32) | lo);
}
__device__ __forceinline__ void arrive(LAS unsigned char* L, int lane) {
    if (lane == 0) __hip_atomic_fetch_add((LAS u32*)(L + LDS_PRO), 1u, __ATOMIC_RELAXED, __HIP_MEMORY_SCOPE_WORKGROUP);
}

template <bool W> __device__ __forceinline__ void prologue_unit(const Params& P, LAS unsigned char* L, char* myws, u32 callv,
                                                                 const u32 (&d)[16], int idx, int xg, int wave, int lane) {
    u32 x[16];
    __builtin_amdgcn_s_setprio(3);
    const int u = xg * 4 + wave;
    const bool half = !W && u >= (int)R0;
    char* peer = peer_of_lane(P);
    WAIT_VM0();
    if (W) {
        order_w(x, d);
    } else {
        LAS u32* xc = (LAS u32*)(L + LDS_XCH);
        if (lane == 0) __hip_atomic_fetch_add(xc, 1u, __ATOMIC_RELAXED, __HIP_MEMORY_SCOPE_WORKGROUP);
        for (u32 spin = 0; spin < SPINS && UNI(*(volatile LAS u32*)xc) < 4; ++spin) {}
        FENCE();
        LAS unsigned char* b = L + LDS_A + 32768 + lane * 64 + (half ? (wave & 1) * 8192 + (wave & 2) * 16 : wave * 16);
        const u32 hi = half ? 16 : 8192;
#pragma unroll
        for (int i = 0; i < 4; ++i) {
            const u32x4 v = *(LAS u32x4*)(b + (i & 1) * 4096 + (i >> 1) * hi);
            x[4 * i] = v[0]; x[4 * i + 1] = v[1]; x[4 * i + 2] = v[2]; x[4 * i + 3] = v[3];
        }
    }
    const u32 par = (UNI(callv) + 1) & 1;
    const bool raw = W && u == (int)ROUNDS - 1;
    u32 len64 = 64;
    if (!W && u == 0) {
        char* dst = myws + WS_CB + par * CB_BYTES + cb_off(idx, u);
#pragma unroll
        for (int i = 0; i < 4; ++i) st_sys_s(lane * 16, dst + i * 1024, piece(x, i));
    } else if (!W || u >= NPW) {
        len64 = encode_unit<W ? 2 : 3>(x, L + LDS_A + wave * 8192, L, myws + WS_CB + par * CB_BYTES + cb_off(idx, u), lane, raw);
    }
    if (lane == 0) *(volatile LAS u32*)(L + LDS_LEN + wave * 4) = len64;
    const char* qa = myws + WS_IMG + par * IMG_BYTES + unit_off(idx, half ? u & ~1 : u) + (half ? (u & 1) * 2048 : 0);
    const char* qb = half ? myws + WS_IMG + par * IMG_BYTES + unit_off(idx, u | 1) + (u & 1) * 2048 : qa + 2048;
    st_sys_s(lane * 16, qa, piece(x, 0)); st_sys_s(lane * 16, qa + 1024, piece(x, 1));
    st_sys_s(lane * 16, qb, piece(x, 2)); st_sys_s(lane * 16, qb + 1024, piece(x, 3));
    WAIT_VM(4);
    if (xg == 0 && (!W || u >= NPW) && lane < NR && lane != P.rank)
        asm volatile("global_store_dword %0, %1, off sc0 sc1" :: "v"(peer + WS_RDY + par * 32768 + (P.rank * 32 + idx) * 128 + u * 4),
                     "v"(((UNI(callv) + 1) << 7) | len64) : "memory");
    if (xg == 0 && (!W || u >= NPW)) WAIT_VM(1); else WAIT_VM(0);
    arrive(L, lane);
    __builtin_amdgcn_s_setprio(0);
}

__global__ void __launch_bounds__(NTHREADS, 3) ag_gemm(Params P) {
    __shared__ __attribute__((aligned(16))) unsigned char lds_raw[LDS_BYTES];
    LAS unsigned char* L = (LAS unsigned char*)lds_raw;
    const int hw = UNI(threadIdx.x >> 6);
    const int wave = hw < 4 ? 11 - hw : hw - 4;
    const int lane = threadIdx.x & 63;
    const int bid = blockIdx.x;
    const unsigned char* px = P.x;
    const unsigned char* pw_ = P.w;
    char* myws = P.myws;
    int rank = P.rank;
    u32 srcmap = P.srcmap;
    asm volatile("" : "+s"(px), "+s"(pw_), "+s"(myws), "+s"(rank), "+s"(srcmap) : "s"(P.ws[0]));
    const int xg = bid & 7, idx = bid >> 3;
    const int rg = (xg + 5) & 7;
    const int tm = (xg >> 1) * 4 + (idx >> 3), tn = (xg & 1) * 8 + (idx & 7);
    GAS u32* tmo = (GAS u32*)(myws + WS_TMO);

    u32 xo = (lane >> 2) * KS + (lane & 3) * 16;
    u32 d[16];
    if (wave < 4 && idx < 16) {
        const unsigned char* g = px + (size_t)(idx * 256 + wave * 64) * KS + rg * 64;
        u32 keep;
        asm volatile("s_mov_b32 %1, m0\n\ts_mov_b32 m0, %3\n\ts_nop 0\n\tglobal_load_lds_dwordx4 %0, %2\n\t"
                     ".rept 3\n\ts_add_u32 m0, m0, 0x400\n\tv_add_u32 %0, 16*512, %0\n\tglobal_load_lds_dwordx4 %0, %2\n\t.endr\n\t"
                     "s_mov_b32 m0, %1"
                     : "+v"(xo), "=&s"(keep) : "s"(g), "s"((u32)(size_t)L + LDS_A + 32768 + wave * 4096) : "memory", "scc");
    }
    FENCE();
    const int pr = (idx >> 4) + 2 * (11 - wave);
    if (wave < 4 ? idx >= 16 : wave >= 10 && pr < NPW) load_w(d, pw_, idx, wave < 4 ? rg * 4 + wave : pr, lane);
    u32 call = __hip_atomic_load((GAS u32*)(myws + WS_CNT + bid * 128), __ATOMIC_RELAXED, __HIP_MEMORY_SCOPE_AGENT);
    if (wave < 4) {
        encode_clear(L + LDS_A + wave * 8192, lane);
    } else if (wave < 8) {
        const u32 f = (wave - 4) * 64 + lane;
        u32 dl = 0, cl = 0x0C0C0C0Cu, k = 0;
#pragma unroll
        for (u32 i = 0; i < 4; ++i) {
            const u32 c = (f >> (2 * i)) & 3u;
            dl |= (c ? 4 + c : k) << (8 * i);
            cl ^= c ? 0u : (12u ^ i) << (8 * k);
            k += c == 0;
        }
        *(LAS u32*)(L + LDS_DL + f * 4) = dl;
        *(LAS u32*)(L + LDS_CL + f * 4) = cl;
    } else if (wave == 8) {
        ((LAS u32*)(L + LDS_CTL))[lane] = 0;
    }
    BAR();

    if (wave < 8) {
        if (wave < 4) {
            if (idx < 16) prologue_unit<false>(P, L, myws, call, d, idx, rg, wave, lane);
            else prologue_unit<true>(P, L, myws, call, d, idx, rg, wave, lane);
        }
        typedef const __attribute__((address_space(4))) float* cf32;
        const u32 scb = UNI(__builtin_bit_cast(u32, *(cf32)(u64)P.sx * *(cf32)(u64)P.sw));
        const int g = wave >> 1, nh = wave & 1;
        const u64 ob = (u64)P.out + ((size_t)(tm * 256 + g * 32) * MN + tn * 256 + nh * 128) * 4;
        u32 olo = (u32)ob, ohi = (u32)(ob >> 32);
        mfma_role((u32)(size_t)L + LDS_A + g * 512, (u32)(size_t)L + LDS_B + nh * 512, (u32)(size_t)L + LDS_CTL + wave * 16,
                  olo, ohi, scb);
    } else if (wave < 10) {
        const u64 tk = rt();
        __builtin_amdgcn_s_setprio(2);
        if (wave == 8) {
            char* peer = peer_of_lane(P);
            call = UNI(call) + 1;
            if (rg == 0 && lane < NR) {
                u32 junk;
                asm volatile("global_load_dword %0, %1, off sc0 sc1\n\ts_waitcnt vmcnt(0)" : "=&v"(junk) : "v"(peer + WS_PAD + bid * 128) : "memory");
            }
            WAIT_VM0();
            for (u32 spin = 0; spin < SPINS && UNI(*(volatile LAS u32*)(L + LDS_PRO)) < 7; ++spin) __builtin_amdgcn_s_sleep(1);
            FENCE();
            const u32x4 ln = *(LAS u32x4*)(L + LDS_LEN);
            if (lane < NR) {
                if (lane != rank) { if (rg) st_sys(peer + WS_RDY + (call & 1) * 32768 + (rank * 32 + idx) * 128 + rg * 16, ln | (call << 7)); }
                else st_sys(peer + WS_UF + idx * 2048 + rg * 16, u32x4{call, call, call, call});
            }
            if (lane == 0) __hip_atomic_store((GAS u32*)(myws + WS_CNT + bid * 128), call, __ATOMIC_RELAXED, __HIP_MEMORY_SCOPE_AGENT);
        } else {
            call = UNI(call) + 1;
            arrive(L, lane);
            u32 job = XCD_ID() * 32 + idx;
#pragma unroll 1
            for (int k = 0; k < NWG; ++k, job = (job + 1) & (NWG - 1)) {
                u32 old = 0;
                if (lane == 0) {
                    __hip_atomic_fetch_max((GAS u32*)(myws + WS_TIK + (job >> 5) * 128), call * 2048, __ATOMIC_RELAXED, __HIP_MEMORY_SCOPE_AGENT);
                    FENCE();
                    old = __hip_atomic_fetch_max((GAS u32*)(myws + WS_JOB + job * 128), call, __ATOMIC_RELAXED, __HIP_MEMORY_SCOPE_AGENT);
                }
                WAIT_VM0();
                if (UNI(old) < call) break;
            }
            *(volatile LAS u32*)(L + LDS_JOB) = job + 1;
        }
        const int isB = wave - 8;
        const int panel = isB ? 16 + tn : tm;
        const char* img = myws + WS_IMG + (call & 1) * IMG_BYTES + panel * 16384;
        const u32 ldsb = (u32)(size_t)L + (isB ? LDS_B : LDS_A);
        volatile LAS u32* ctl = (volatile LAS u32*)(L + LDS_CTL + (lane & 7) * 16);
        const char* uf = myws + WS_UF + panel * 2048;
        const u32 voff = lane * 16 + ((panel) & 3) * UNIT;
        Ld S = {0, 0, 0, 0, 0, ~0u, 0, 0, (u64)img, (u64)(img + (16 + tn - panel) * 16384 + (size_t)S0 * 524288)};
        u32 err = 0;
        const u32 vctl = (u32)(size_t)ctl, vfull = vctl + isB * 4, vf = (lane & 31) * 4;
        const u64 t0 = rt();
        WAIT_VM0();
        asm volatile("v_mov_b32 v40, 0\n\tv_mov_b32 v41, 0\n\tv_mov_b32 v47, 0\n\tv_mov_b32 v44, %0\n\tv_mov_b32 v45, %1\n\t"
                     "v_mov_b32 v46, %2\n\tv_mov_b32 v48, %3"
                     :: "v"((voff + UNIT) & 16383), "v"((voff + 2 * UNIT) & 16383), "v"((voff + 3 * UNIT) & 16383),
                        "v"(lane * 16 + ((tn + 3) & 3) * UNIT) : "v40", "v41", "v44", "v45", "v46", "v47", "v48");
        bool go = true;
        auto late = [&]() {
            if (UNI(__hip_atomic_load(tmo, __ATOMIC_RELAXED, __HIP_MEMORY_SCOPE_AGENT)) == call || rt() - t0 > DEADLINE) {
                if (lane == 0) __hip_atomic_store(tmo, call, __ATOMIC_RELAXED, __HIP_MEMORY_SCOPE_AGENT);
                ctl[3] = 1;
                err = ~0u;
            }
            asm volatile("s_waitcnt vmcnt(0) lgkmcnt(0)" ::: "memory");
            go = rt() - t0 < 3 * DEADLINE;
        };
#define RUN(M, end, l) while (go && load_run<M>(S, end, l, err, call, uf, voff, vf, vctl, vfull)) late();
        while (rt() - tk < LDELAY) __builtin_amdgcn_s_sleep(4);
        while (go && load_run<0, 4>(S, QEASY, ldsb, err, call, uf, voff, vf, vctl, vfull)) late();
        RUN(0, S0, ldsb);
        const u32 l0 = (u32)(size_t)L + LDS_A;
        if (isB) { RUN(2, NSTAGE, l0); S.g -= (size_t)NSPLIT * 524288; RUN(4, NQ, l0); }
        else { RUN(1, NSTAGE, l0); S.g -= (size_t)NSPLIT * 524288 - 2048; RUN(3, NQ, l0); }
        RUN(5, 0, 0);
        ctl[isB] = 0xFFFFu;
        WAIT_VM(0);
    } else {
        __builtin_amdgcn_s_setprio(2);
        const u32 vo = lane * 16, wk = wave - 10;
        {
            u32 x[16];
            const u32 pdst = (srcmap >> (4 * xg)) & 7u;
            const bool push = pr < NPW && pdst != (u32)rank;
            const u32 up = 32 + ((rank - pdst - 1) & 7) + 7 * pr, pp = idx | 16;
            const char* pw = P.ws[pdst];
            const char* q0 = pw + WS_IMG + unit_off(pp, up);
            asm volatile("" : "+s"(q0));
            WAIT_VM0();
            call = UNI(call) + 1;
            if (push) {
                order_w(x, d);
                q0 += (call & 1) * IMG_BYTES;
                st_sys_s(vo, q0, piece(x, 0));
                arrive(L, lane);
                st_sys_s(vo, q0 + 1024, piece(x, 1));
                st_sys_s(vo, q0 + 2048, piece(x, 2));
                st_sys_s(vo, q0 + 3072, piece(x, 3));
                WAIT_VM(0); st_flag(pw + WS_UF + pp * 2048 + up * 4, call, lane);
            } else {
                arrive(L, lane);
            }
            if (wk) *(volatile LAS u32*)(L + LDS_PSH) = 1;
        }
        call = UNI(call);
        u32 job = 0;
        for (u32 spin = 0; spin < SPINS && job == 0; ++spin) { __builtin_amdgcn_s_sleep(1); job = UNI(*(volatile LAS u32*)(L + LDS_JOB)); }
        job = (job - 1) & (NWG - 1);
        const u32 slot = job >> 5, src = (srcmap >> (4 * slot)) & 7u;
        if (src == (u32)rank) return;
        const char* tik = myws + WS_TIK + slot * 128;
        const char* scb = P.ws[src] + WS_CB + (call & 1) * CB_BYTES;
        const char* rdy = myws + WS_RDY + (call & 1) * 32768 + src * 4096;
        const char* dp = myws + WS_IMG + (call & 1) * IMG_BYTES;
        const u32 u0 = 32 + ((src - rank - 1) & 7);
        LAS unsigned char* sl = L + LDS_WRK + wk * WRK_BYTES + 3072;
        const u32 rot = 2 * rank;
        auto unit_of = [rot](u32 t) -> u32 {
            auto px = [rot](u32 k) { k = (k + rot) & 15; return 4 * (k & 3) + (k >> 2); };
            auto pw = [rot](u32 k) { k = (k + rot) & 15; return 16 + 8 * (k & 1) + (k >> 1); };
            if (t < T0) return t < 64 ? px(t & 15) * 32 + (t >> 4) : pw(t & 15) * 32 + NPW + (t >> 4) - 4;
            if (t < T1) { t -= T0; return ((t & 1) ? pw((t >> 1) & 15) : px((t >> 1) & 15)) * 32 + 4 + (t >> 5); }
            if (t < T2) {
                const u32 pr = (t - T1) / 48, i = (t - T1) % 48;
                return ((i & 16) ? px(i & 15) : pw(i & 15)) * 32 + R0 + 2 * pr + (i >> 5);
            }
            return t < NT ? px((t - T2) & 15) * 32 + R0 + 2 * ((t - T2) >> 4) + 1 : 0u;
        };
        const u64 t0 = rt();
        auto await = [&](u32 un) -> u32 {
            for (u32 n = 1;; ++n) {
                u32 rv;
                if (lane == 0) asm volatile("global_atomic_or v162, %0, %0, %1 sc0 sc1\n\ts_waitcnt vmcnt(0)" :: "v"(0u), "s"(rdy + un * 4) : "memory", "v162");
                asm volatile("v_readfirstlane_b32 %0, v162" : "=s"(rv) :: "memory", "v162");
                if ((rv >> 7) >= call) return rv;
                if ((n & 31) == 0 && (UNI(__hip_atomic_load(tmo, __ATOMIC_RELAXED, __HIP_MEMORY_SCOPE_AGENT)) == call || rt() - t0 > DEADLINE)) {
                    if (lane == 0) __hip_atomic_store(tmo, call, __ATOMIC_RELAXED, __HIP_MEMORY_SCOPE_AGENT);
                    return 0;
                }
                __builtin_amdgcn_s_sleep(2);
            }
        };
        auto book = [&](u64 at, u32 cost) {
            const u64 now = at << 10, c = cost;
            if (lane == 0)
                asm volatile("global_atomic_umax_x2 %0, %1, %3 offset:2048\n\tglobal_atomic_add_x2 v[160:161], %0, %2, %3 offset:2048 sc0"
                             :: "v"(0u), "v"(now), "v"(c), "s"(tik) : "memory", "v160", "v161");
        };
        const char* pend = nullptr;
        u32 pn = 0;
        auto flag = [&]() {
            if ((u32)lane <= pn) asm volatile("global_store_dword %0, %1, %2 sc0 sc1" :: "v"(lane * 28), "v"(call), "s"(pend) : "memory");
            pend = nullptr;
        };
        u32 t = (job & 31) + 32 * wk, un = unit_of(t), lat = 0, plat = 0;
        if (lane < 2)
            asm volatile("global_load_dword v163, %0, %1 sc0 sc1" :: "v"((lane * 482 + (job & 31)) * SLOT + UNIT + 128), "s"(scb) : "memory", "v163");
        for (u32 spin = 0; spin < SPINS && UNI(*(volatile LAS u32*)(L + (t < 16 ? LDS_PSH : LDS_PRO))) < (t < 16 ? 1 : 7); ++spin)
            __builtin_amdgcn_s_sleep(2);
        u32 rv = await(un);
        if (!rv) return;
        WAIT_VM0();
        book(rt(), (((rv & 127u) + 1) >> 1) * RATE);
        bool now_ = t < NOWAIT;
        if (!now_) WAIT_VM(0);
        t += 64;
        for (;;) {
            u32 len64 = rv & 127u;
            len64 = len64 < 41 ? 41 : (len64 > 64 ? 64 : len64);
            const u32 len = len64 * 64, panel = un >> 5, r = un & 31;
            const char* q = scb + cb_off(panel, r);
            const bool more = t < NT;
            un = unit_of(t);
            const char* rp = rdy + un * 4;
            asm volatile("" : "+s"(q), "+s"(rp));
            if (!now_) {
                u32 lo, hi;
                asm volatile("v_readfirstlane_b32 %0, v160\n\tv_readfirstlane_b32 %1, v161" : "=s"(lo), "=s"(hi) :: "memory", "v160", "v161");
                const u32 start = (u32)((((u64)hi << 32) | lo) >> 10);
                for (u32 d = start - (u32)rt(); d - 1 < 1000u; d = start - (u32)rt()) {
                    if (pend) { if (vm_idle()) { WAIT_VM(0); flag(); } }
                    else if (d > 12) __builtin_amdgcn_s_sleep(2);
                }
            }
            now_ = false;
            const u32 i0 = (u32)rt();
            PIECE(0, q); PIECE(1, q);
            if (2048 + vo < len) PIECE(2, q);
            if (3072 + vo < len) PIECE(3, q);
            if (lane == 0) asm volatile("global_atomic_or v162, %0, %0, %3 sc0 sc1\n\tglobal_atomic_add v163, %0, %1, %2 sc0"
                                        :: "v"(0u), "v"(1u), "s"(tik), "s"(rp) : "memory", "v162", "v163");
            const bool half = panel < 16 && r >= R0;
            const u32 u = u0 + 7 * (half ? r & ~1u : r);
            const char* qa = dp + unit_off(panel, u) + (half ? (r & 1) * 2048 : 0);
            const char* qb = half ? dp + unit_off(panel, u + 7) + (r & 1) * 2048 : qa + 2048;
            const u32 ho = panel < 16 ? 3072 : 2560;
            const bool late = pend != nullptr;
            if (late) WAIT_VM(5);
            asm volatile("" : "+s"(qa), "+s"(qb) :: "memory");
            if (late) { flag(); WAIT_VM(3); } else { WAIT_VM(2); }
            const u32x4 w = *(LAS u32x4*)(sl + lane * 16);
            const u32 hd = *(LAS u32*)(sl + ho), w2 = *(LAS u32*)(sl + ho + 4 + lane * 4);
            const u64 now = rt();
            if (late) WAIT_VM(1); else WAIT_VM(0);
            asm volatile("v_readfirstlane_b32 %0, v163\n\tv_readfirstlane_b32 %1, v162" : "=s"(t), "=s"(rv) :: "memory", "v162", "v163");
            plat = lat;
            lat = (u32)now - i0;
            const u32 ml = lat < plat ? lat : plat;
            const u32 rate = ml > LATHI ? RATE + RATE / 8 : RATE;
            t += 128 - call * 2048;
            const bool known = (rv >> 7) >= call;
            if (more && known) book(now, (((rv & 127u) + 1) >> 1) * rate);
            FENCE(); SB();
            if (len64 == 64) {
                const u32x4 e1 = *(LAS u32x4*)(sl + 1024 + lane * 16), e2 = *(LAS u32x4*)(sl + 2048 + lane * 16), e3 = *(LAS u32x4*)(sl + 3072 + lane * 16);
                st_sys_s(vo, qa, w); st_sys_s(vo, qa + 1024, e1); st_sys_s(vo, qb, e2); st_sys_s(vo, qb + 1024, e3);
            } else if (panel < 16) {
                decode_unit<3>(qa, qb, UNI(hd), w, w2, sl, L, lane);
            } else {
                decode_unit<2>(qa, qb, UNI(hd), w, w2, sl, L, lane);
            }
            pend = myws + WS_UF + panel * 2048 + (u + (half && (r & 1) ? 4 * NSPLIT : 0)) * 4;
            pn = half;
            if (!more) break;
            if (!known) {
                WAIT_VM(0); flag();
                rv = await(un);
                if (!rv) return;
                WAIT_VM0();
                book(rt(), (((rv & 127u) + 1) >> 1) * rate);
                WAIT_VM(0);
            }
            WAIT_VM(4);
        }
        WAIT_VM(0);
        flag();
    }
}

extern "C" void kernel_launch(int rank, int nranks, ncclComm_t comm,
                              hipStream_t stream,
                              void* const* d_in, const int* in_sizes, int n_in,
                              void* d_out, int out_size,
                              const int* in_dtypes, int out_dtype,
                              void* const* d_out_all,
                              void* const* d_ws_all, size_t ws_bytes) {
    (void)comm; (void)in_sizes; (void)n_in; (void)out_size; (void)in_dtypes; (void)out_dtype; (void)d_out_all;
    if (nranks != NR || (unsigned)rank >= (unsigned)NR || ws_bytes < WS_NEED) {
        fprintf(stderr, "kernel_launch: unexpected configuration\n");
        return;
    }
    Params p;
    p.x = (const unsigned char*)d_in[0];
    p.w = (const unsigned char*)d_in[1];
    p.sx = (const float*)d_in[2];
    p.sw = (const float*)d_in[3];
    p.out = (float*)d_out;
    for (int i = 0; i < NR; ++i) p.ws[i] = (char*)d_ws_all[i];
    p.myws = p.ws[rank];
    p.rank = rank;
    p.pad = 0;
    static const u32 near_link[NR] = {0x74510263u, 0x26301475u, 0x17542036u, 0x74153620u,
                                      0x63024157u, 0x60235741u, 0x51476302u, 0x02367514u};
    p.srcmap = near_link[rank];
    hipLaunchKernelGGL(ag_gemm, dim3(NWG), dim3(NTHREADS), 0, stream, p);
    HIP_CHECK(hipGetLastError());
}
```

```cpp
#include <hip/hip_runtime.h>
#include <rccl/rccl.h>
#include <cstdio>

#define HIP_CHECK(expr) do { hipError_t _e = (expr); if (_e != hipSuccess) { \
    fprintf(stderr, "HIP error %s at %s:%d\n", hipGetErrorString(_e), __FILE__, __LINE__); \
    return; } } while (0)

#define GAS __attribute__((address_space(1)))
#define LAS __attribute__((address_space(3)))
typedef unsigned u32;
typedef unsigned short u16;
typedef unsigned long long u64;
typedef unsigned u32x2 __attribute__((ext_vector_type(2)));
typedef unsigned u32x4 __attribute__((ext_vector_type(4)));

constexpr int NR = 8;
constexpr int MN = 4096;
constexpr int KS = 512;
constexpr int NWG = 256;
constexpr int NTHREADS = 768;
constexpr u32 ROUNDS = 32;
constexpr u32 NSTAGE = 64;
#define NSPLIT 8
#define NS0 2
#define NST 6
constexpr int NPW = 2;
constexpr u32 RATE = 215;
constexpr u32 LATHI = 95;
constexpr u32 NOWAIT = 8;
constexpr u32 QEASY = 32, LDELAY = 500;
constexpr u32 S0 = NSTAGE - NSPLIT;
constexpr u32 NQ = NSTAGE + NSPLIT;
constexpr u32 R0 = ((S0 * 4 - 32 + 13) / 14) * 2;
constexpr u32 US = 32 + 7 * R0;
constexpr u32 NPAIR = (ROUNDS - R0) / 2;
constexpr u32 T0 = 128 - 16 * NPW, T1 = T0 + (R0 - 4) * 32, T2 = T1 + NPAIR * 48, NT = T2 + NPAIR * 16;
constexpr u32 UNIT = 4096;
constexpr size_t IMG_BYTES = 32u << 20;
constexpr size_t CB_BYTES = 5u << 20;
__host__ __device__ constexpr u32 unit_off(u32 panel, u32 u) { return (u >> 2) * 524288u + panel * 16384u + ((u + panel) & 3) * UNIT; }
constexpr u32 SLOT = UNIT + 256;
__host__ __device__ constexpr u32 cb_off(u32 panel, u32 r) { return (r * 32 + panel) * SLOT; }
constexpr size_t WS_CNT = 0;
constexpr size_t WS_TMO = 48 << 10;
constexpr size_t WS_TIK = 52 << 10;
constexpr size_t WS_UF = 64 << 10;
constexpr size_t WS_RDY = 128 << 10;
constexpr size_t WS_JOB = 192 << 10;
constexpr size_t WS_PAD = 224 << 10;
constexpr size_t WS_IMG = 1 << 20;
constexpr size_t WS_CB = WS_IMG + 2 * IMG_BYTES;
constexpr size_t WS_NEED = WS_CB + 2 * CB_BYTES;
constexpr u64 DEADLINE = 50000000ull;
constexpr u32 SPINS = 1u << 25;
constexpr u32 LDS_CTL = 0;
constexpr u32 LDS_PSH = LDS_CTL + 132, LDS_XCH = LDS_CTL + 136, LDS_PRO = LDS_CTL + 140;
constexpr u32 LDS_JOB = LDS_CTL + 160, LDS_LEN = LDS_CTL + 176;
constexpr u32 LDS_DL = 512, LDS_CL = LDS_DL + 1024;
constexpr u32 LDS_WRK = LDS_CL + 1024, WRK_BYTES = 7168;
constexpr u32 LDS_A = LDS_WRK + 2 * WRK_BYTES, LDS_B = LDS_A + 65536;
constexpr u32 LDS_BYTES = LDS_B + 65536;

struct Params {
    const unsigned char* x;
    const unsigned char* w;
    char* myws;
    int rank;
    u32 srcmap;
    const float* sx;
    const float* sw;
    float* out;
    u64 pad;
    char* ws[NR];
};

__device__ __forceinline__ u64 rt() { return __builtin_amdgcn_s_memrealtime(); }
#define WAIT_VM(n) asm volatile("s_waitcnt vmcnt(" #n ")" ::: "memory")
#define WAIT_VM0() do { __builtin_amdgcn_s_waitcnt(0x0F70); WAIT_VM(0); } while (0)
#define FENCE() asm volatile("" ::: "memory")
#define SB() __builtin_amdgcn_sched_barrier(0)
#define XCD_ID() (__builtin_amdgcn_s_getreg((3 << 11) | 20) & 7u)
#define UNI(x) __builtin_amdgcn_readfirstlane(x)

__device__ __forceinline__ void st_sys(char* p, u32x4 v) {
    asm volatile("global_store_dwordx4 %0, %1, off sc0 sc1\n\ts_nop 1" :: "v"(p), "v"(v) : "memory");
}
__device__ __forceinline__ void st_sys_s(u32 voff, const void* base, u32x4 v) {
    asm volatile("global_store_dwordx4 %0, %1, %2 sc0 sc1\n\ts_nop 1" :: "v"(voff), "v"(v), "s"(base) : "memory");
}
__device__ __forceinline__ void st_flag(const void* base, u32 v, int lane) {
    if (lane == 0) asm volatile("global_store_dword %0, %1, %2 sc0 sc1" :: "v"(0u), "v"(v), "s"(base) : "memory");
}

#define STR2(x) #x
#define STR(x) STR2(x)
struct Ld {
    u32 nops, q, rmask, nf;
    u32 pidx, pb, slot;
    u64 fifo, g, gw;
};
#define G2(v, s_, o) "global_load_lds_dwordx4 " v ", " s_ " offset:" #o " sc0 sc1\n\tglobal_load_lds_dwordx4 " v ", " s_ " offset:" #o "+1024" " sc0 sc1\n\t"
#define EASE ".if %c[ease]\n\ts_sleep %c[ease]\n\t.endif\n\t"
#define M0A(x) "s_add_u32 m0, m0, " x "\n\ts_nop 0\n\t"
#define GW(v) G2(v, "%[g]", 0) M0A("%[land]") G2(v, "%[g]", 2048)
template <int MODE, int EASY = 0>
__device__ __forceinline__ u32 load_run(Ld& S, u32 end, u32 lds, u32 err, u32 call, const void* uf,
                                        u32 vo, u32 vf, u32 vctl, u32 vfull) {
    u32 a, b, land, rc, keep;
    asm volatile(
        "s_mov_b32 %[keep], m0\n\t"
        "s_mov_b32 %[rc], 0\n\t"
        "s_branch .LL%=\n\t"
        ".LZ%=:\n\t"
        ".if %c[md] == 0\n\t"
        "s_sleep 1\n\t"
        ".endif\n\t"
        ".LL%=:\n\t"
        "ds_read_b32 v42, %[vctl] offset:8\n\t"
        "s_getreg_b32 %[a], hwreg(HW_REG_IB_STS)\n\t"
        "s_bfe_u32 %[b], %[a], 0x20016\n\t"
        "s_and_b32 %[a], %[a], 15\n\t"
        "s_lshl4_add_u32 %[a], %[b], %[a]\n\t"
        "s_sub_u32 %[land], %[nops], %[a]\n\t"
        "s_mov_b64 vcc, %[fifo]\n\t"
        "s_sub_u32 %[a], %[land], vcc_lo\n\t"
        "s_bitcmp0_b32 %[a], 11\n\t"
        "s_cselect_b32 %[a], %[nf], 0\n\t"
        "s_cmp_eq_u32 %[a], 0\n\t"
        "s_cbranch_scc1 .LP%=\n\t"
        "v_add_u32 v41, 1, v41\n\t"
        "s_lshr_b64 %[fifo], %[fifo], 12\n\t"
        "ds_write_b32 %[vfull], v41\n\t"
        "s_sub_u32 %[nf], %[nf], 1\n\t"
        ".LP%=:\n\t"
        "s_add_u32 %[rc], %[rc], 0x100000\n\t"
        "s_cbranch_scc1 .LT%=\n\t"
        ".if %c[md] == 5\n\t"
        "s_waitcnt lgkmcnt(0)\n\t"
        "s_cmp_lg_u32 %[nf], 0\n\t"
        "s_cbranch_scc1 .LL%=\n\t"
        ".else\n\t"
        ".if %c[md] != 4\n\t"
        "s_or_b32 %[rmask], %[rmask], %[err]\n\t"
        "s_sub_u32 %[a], %[land], %[pidx]\n\t"
        "s_cmp_lt_i32 %[a], 0\n\t"
        "s_cbranch_scc1 .LO%=\n\t"
        "s_cmp_eq_u32 %[pb], -1\n\t"
        "s_cbranch_scc1 .LS%=\n\t"
        "v_cmp_le_u32 vcc, %[call], v40\n\t"
        "s_sub_u32 %[a], %[q], %[pb]\n\t"
        "s_lshl_b32 %[a], %[a], 1\n\t"
        "s_min_u32 %[a], %[a], 16\n\t"
        "s_lshr_b32 %[b], vcc_lo, %[a]\n\t"
        "s_lshr_b32 %[b], %[b], %[a]\n\t"
        "s_or_b32 %[rmask], %[rmask], %[b]\n\t"
        "s_mov_b32 %[pb], -1\n\t"
        "s_andn2_b32 %[b], 15, %[rmask]\n\t"
        "s_cbranch_scc0 .LO%=\n\t"
        ".LS%=:\n\t"
        "s_andn2_b32 %[b], 0xffff, %[rmask]\n\t"
        "s_cbranch_scc0 .LO%=\n\t"
        "s_lshl_b32 %[b], %[q], 4\n\t"
        "v_add_u32 v43, %[b], %[vf]\n\t"
        ".if %c[md] & 1\n\t"
        "v_subrev_u32 v40, 0x400, v43\n\t"
        "v_cmp_gt_u32 vcc, %[whole], v40\n\t"
        "v_subrev_u32 v40, 16*" STR(NSPLIT) ", v43\n\t"
        "s_nop 0\n\t"
        "v_cndmask_b32 v43, v43, v40, vcc\n\t"
        ".endif\n\t"
        "s_add_u32 %[nops], %[nops], 1\n\t"
        "s_mov_b32 exec_hi, 0\n\t"
        "global_atomic_or v40, v43, v47, %[uf] sc0 sc1\n\t"
        "s_mov_b32 exec_hi, -1\n\t"
        "s_mov_b32 %[pidx], %[nops]\n\t"
        "s_mov_b32 %[pb], %[q]\n\t"
        ".LO%=:\n\t"
        ".endif\n\t"
        ".if %c[md] == 0\n\t"
        "s_sub_u32 %[a], %[q], 3\n\t"
        ".else\n\t"
        "s_cmp_ge_u32 %[q], 68-" STR(NSPLIT) "\n\t"
        "s_subb_u32 %[a], %[q], 3\n\t"
        ".endif\n\t"
        "s_waitcnt lgkmcnt(0)\n\t"
        "v_cmp_gt_i32 vcc, %[a], v42\n\t"
        "s_lshl_b32 %[a], %[slot], 14\n\t"
        ".if %c[md]\n\t"
        "s_add_u32 %[b], %[a], 0x10000\n\t"
        "s_cmp_eq_u32 %[slot], 4\n\t"
        "s_cselect_b32 %[a], 2*16384+2048, %[a]\n\t"
        "s_cselect_b32 %[b], 2048, %[b]\n\t"
        "s_cselect_b32 %[land], 16384-2048, 0\n\t"
        ".endif\n\t"
        ".if %c[md] != 4\n\t"
        "s_andn2_b32 m0, 15, %[rmask]\n\t"
        "s_or_b32 m0, m0, vcc_lo\n\t"
        ".else\n\t"
        "s_cmp_lg_u32 vcc_lo, 0\n\t"
        ".endif\n\t"
        "s_cbranch_scc1 .LZ%=\n\t"
        ".if %c[md] == 0\n\t"
        "s_add_u32 m0, %[a], %[lds]\n\t"
        "s_add_u32 %[nops], %[nops], 16\n\t"
        G2("%[vo]", "%[g]", 0) G2("%[vo]", "%[g]", 2048) EASE M0A("0x1000") G2("v44", "%[g]", 0) G2("v44", "%[g]", 2048) EASE M0A("0x1000")
        G2("v45", "%[g]", 0) G2("v45", "%[g]", 2048) EASE M0A("0x1000") G2("v46", "%[g]", 0) G2("v46", "%[g]", 2048)
        ".elseif %c[md] & 1\n\t"
        "s_add_u32 m0, %[a], %[lds]\n\t"
        "s_add_u32 %[nops], %[nops], 6+2*%c[md]\n\t"
        G2("%[vo]", "%[g]", 0) M0A("0x1000") G2("v44", "%[g]", 0) M0A("0x1000") G2("v45", "%[g]", 0) M0A("0x1000") G2("v46", "%[g]", 0)
        ".if %c[md] == 3\n\t"
        "s_add_u32 %[b], %[b], %[lds]\n\t"
        "s_add_u32 m0, %[b], 0x3000\n\t"
        "s_mov_b64 vcc, %[gw]\n\t"
        G2("v48", "vcc", 0) M0A("%[land]") G2("v48", "vcc", 2048)
        "s_add_u32 vcc_lo, vcc_lo, 0x80000\n\t"
        "s_addc_u32 vcc_hi, vcc_hi, 0\n\t"
        "s_mov_b64 %[gw], vcc\n\t"
        ".endif\n\t"
        ".else\n\t"
        "s_add_u32 m0, %[b], %[lds]\n\t"
        "s_sub_u32 %[a], 0x1000, %[land]\n\t"
        "s_add_u32 %[nops], %[nops], 20-2*%c[md]\n\t"
        GW("%[vo]") M0A("%[a]") GW("v44") M0A("%[a]") GW("v45")
        ".if %c[md] == 2\n\t"
        M0A("%[a]") GW("v46")
        ".endif\n\t"
        ".endif\n\t"
        "s_mov_b64 vcc, %[g]\n\t"
        "s_add_u32 vcc_lo, vcc_lo, 0x80000\n\t"
        "s_addc_u32 vcc_hi, vcc_hi, 0\n\t"
        "s_mov_b64 %[g], vcc\n\t"
        "s_and_b32 vcc_lo, %[nops], 0xfff\n\t"
        "s_mov_b32 vcc_hi, 0\n\t"
        "s_mul_i32 %[a], %[nf], 12\n\t"
        "s_lshl_b64 vcc, vcc, %[a]\n\t"
        "s_or_b64 %[fifo], %[fifo], vcc\n\t"
        "s_add_u32 %[nf], %[nf], 1\n\t"
        "s_lshr_b32 %[rmask], %[rmask], 4\n\t"
        "s_add_u32 %[slot], %[slot], 1\n\t"
        "s_cmp_eq_u32 %[slot], 4+(%c[md]+7)/8\n\t"
        "s_cselect_b32 %[slot], 0, %[slot]\n\t"
        "s_add_u32 %[q], %[q], 1\n\t"
        "s_cmp_lt_u32 %[q], %[end]\n\t"
        "s_cbranch_scc1 .LL%=\n\t"
        ".endif\n\t"
        "s_mov_b32 %[rc], 0\n\t"
        "s_branch .LE%=\n\t"
        ".LT%=:\n\t"
        "s_mov_b32 %[rc], 1\n\t"
        "s_waitcnt lgkmcnt(0)\n\t"
        ".LE%=:\n\t"
        "s_mov_b32 m0, %[keep]"
        : [nops] "+s"(S.nops), [q] "+s"(S.q), [rmask] "+s"(S.rmask), [nf] "+s"(S.nf), [pidx] "+s"(S.pidx), [pb] "+s"(S.pb),
          [slot] "+s"(S.slot), [fifo] "+s"(S.fifo), [g] "+s"(S.g), [gw] "+s"(S.gw),
          [a] "=&s"(a), [b] "=&s"(b), [land] "=&s"(land), [rc] "=&s"(rc), [keep] "=&s"(keep)
        : [end] "s"(end), [lds] "s"(lds), [err] "s"(err), [call] "s"(call), [uf] "s"(uf),
          [vo] "v"(vo), [vf] "v"(vf), [vctl] "v"(vctl), [vfull] "v"(vfull), [md] "i"(MODE), [ease] "i"(EASY), [whole] "i"(4 * (US - 4 * S0))
        : "memory", "scc", "vcc", "v40", "v41", "v42", "v43", "v44", "v45", "v46", "v47", "v48");
    return rc;
}
#define PIECE(i, g) do { u32 k_; asm volatile("s_mov_b32 %0, m0\n\ts_mov_b32 m0, %3\n\ts_nop 0\n\t" \
    "global_load_lds_dwordx4 %1, %2 offset:1024*" #i " sc0 sc1\n\ts_mov_b32 m0, %0" : "=&s"(k_) : "v"(vo), "s"(g), "s"((u32)(size_t)sl) : "memory"); } while (0)
__device__ __forceinline__ bool vm_idle() {
    u32 r;
    asm volatile("s_getreg_b32 %0, hwreg(HW_REG_IB_STS)" : "=s"(r) :: "memory");
    return (r & 0xC0000Fu) == 0;
}
__device__ __forceinline__ u32 perm(u32 hi, u32 lo, u32 sel) { return __builtin_amdgcn_perm(hi, lo, sel); }
__device__ __forceinline__ u32 bfi(u32 m, u32 a, u32 b) { return __builtin_amdgcn_bitop3_b32(m, a, b, 0xCA); }
__device__ __forceinline__ u32 and_or(u32 a, u32 m, u32 c) { return __builtin_amdgcn_bitop3_b32(a, m, c, 0xEA); }
#define BAR() asm volatile("s_waitcnt lgkmcnt(0)\n\ts_barrier" ::: "memory")
__device__ __forceinline__ u32 scan64(u32 v) {
    v += __builtin_amdgcn_update_dpp(0u, v, 0x111, 0xf, 0xf, true);
    v += __builtin_amdgcn_update_dpp(0u, v, 0x112, 0xf, 0xf, true);
    v += __builtin_amdgcn_update_dpp(0u, v, 0x114, 0xf, 0xf, true);
    v += __builtin_amdgcn_update_dpp(0u, v, 0x118, 0xf, 0xf, true);
    v += __builtin_amdgcn_update_dpp(0u, v, 0x142, 0xa, 0xf, false);
    v += __builtin_amdgcn_update_dpp(0u, v, 0x143, 0xc, 0xf, false);
    return v;
}
__device__ __forceinline__ void lds_or(LAS unsigned char* p, u32 v) {
    __hip_atomic_fetch_or((LAS u32*)p, v, __ATOMIC_RELAXED, __HIP_MEMORY_SCOPE_WORKGROUP);
}

__device__ __forceinline__ void load_w(u32 (&d)[16], const unsigned char* w, int panel, int r, int lane) {
    const auto rs = __builtin_amdgcn_make_buffer_rsrc((void*)w, 0, MN * KS, 0x00020000);
    const u32 so = r * 16 * MN + (panel & 15) * 256;
#pragma unroll
    for (int k = 0; k < 16; ++k) d[k] = __builtin_amdgcn_raw_buffer_load_b32(rs, lane * 4, so + k * MN, 0);
}
__device__ __forceinline__ void order_w(u32 (&x)[16], const u32 (&d)[16]) {
#pragma unroll
    for (int q = 0; q < 4; ++q) {
        u32 t0 = perm(d[4 * q + 1], d[4 * q], 0x05010400), t1 = perm(d[4 * q + 1], d[4 * q], 0x07030602);
        u32 t2 = perm(d[4 * q + 3], d[4 * q + 2], 0x05010400), t3 = perm(d[4 * q + 3], d[4 * q + 2], 0x07030602);
        x[q] = perm(t2, t0, 0x05040100); x[4 + q] = perm(t2, t0, 0x07060302);
        x[8 + q] = perm(t3, t1, 0x05040100); x[12 + q] = perm(t3, t1, 0x07060302);
    }
}
__device__ __forceinline__ u32x4 piece(const u32 (&x)[16], int i) { return u32x4{x[4 * i], x[4 * i + 1], x[4 * i + 2], x[4 * i + 3]}; }

template <int MB> struct Fmt {
    static constexpr u32 EM = (MB == 3 ? 0x0Fu : 0x1Fu) * 0x01010101u;
    static constexpr u32 HEAD = MB == 3 ? 3072 : 2048;
    static constexpr u32 OFF_H = MB == 3 ? 0 : 512, OFF_2 = OFF_H + 4;
};
__device__ __forceinline__ u32 zero2(u32 w) { return __builtin_amdgcn_bitop3_b32(w, w >> 1, 0x55555555u, 0x02); }
__device__ __forceinline__ u32 code_of(u32 hi, u32 lo, u32 v) {
    const u32 r = perm(hi, lo, v);
    return __builtin_amdgcn_bitop3_b32(r, r >> 2, 0x03030303u, 0x28);
}
__device__ __forceinline__ void lookup4(u32* sel, u32 w, LAS unsigned char* tab) {
#pragma unroll
    for (int j = 0; j < 4; ++j) sel[j] = *(LAS u32*)(tab + ((w >> (8 * j)) & 255u) * 4);
}
__device__ __forceinline__ void gather4(const u32* v, const u32* sel, u32 z, u32 a) {
#pragma unroll
    for (int j = 0; j < 4; ++j) {
        const u32 o = j ? a + __builtin_popcount(z & ((1u << (8 * j)) - 1)) : a;
        const u32 c = perm(0u, v[j], sel[j]);
        LAS unsigned char* p = (LAS unsigned char*)(size_t)(o & ~3u);
        lds_or(p, __builtin_amdgcn_alignbyte(c, 0u, ~o));
        lds_or(p + 4, __builtin_amdgcn_alignbyte(0u, c, ~o));
    }
}
__device__ __forceinline__ void window4(u32* o, u32x2* q, u32 z, u32 a) {
#pragma unroll
    for (int j = 0; j < 4; ++j) {
        o[j] = j ? a + __builtin_popcount(z & ((1u << (8 * j)) - 1)) : a;
        LAS u32* p = (LAS u32*)(size_t)(o[j] & ~3u);
        q[j] = u32x2{p[0], p[1]};
    }
}
__device__ __forceinline__ u32 spread(u32 tab, u32 o, u32x2 q, u32 sel) {
    return perm(tab, __builtin_amdgcn_alignbyte(q[1], q[0], o), sel);
}
__device__ __forceinline__ u32 pack4(const u32* c) {
    u32 w = __builtin_amdgcn_udot4(c[0], 0x40100401u, 0u, false);
#pragma unroll
    for (int j = 1; j < 4; ++j) w |= __builtin_amdgcn_udot4(c[j], 0x40100401u, 0u, false) << (8 * j);
    return w;
}
__device__ __forceinline__ void encode_clear(LAS unsigned char* T, int lane) {
#pragma unroll
    for (int i = 2; i < 5; ++i) *(LAS u32x4*)(T + i * 1024 + lane * 16) = u32x4{0, 0, 0, 0};
}

template <int MB> __device__ __forceinline__ u32 encode_unit(const u32 (&x)[16], LAS unsigned char* T, LAS unsigned char* L,
                                                             char* dst, int lane, bool raw) {
    typedef Fmt<MB> F;
    u32 sum = 0;
#pragma unroll
    for (int j = 0; j < 16; j += 5) sum = __builtin_amdgcn_udot4(x[j] & 0x7F7F7F7Fu, 0x01010101u, sum, false);
    const int tot = __builtin_amdgcn_readlane(scan64(sum), 63);
    const u32 eb = (u32)((tot - (1516 << MB)) >> (10 + MB)) & (F::EM & 31u);
    const u32 B = (eb << MB) * 0x01010101u;
    u32 v[16], c[16];
    u32x4 w, z;
#pragma unroll
    for (int j = 0; j < 16; ++j) {
        v[j] = (((x[j] | 0x80808080u) - B) >> MB) & F::EM;
        c[j] = code_of(0u, 0x03020100u, v[j]);
    }
    u32 cnt[5] = {0};
#pragma unroll
    for (int q = 0; q < 4; ++q) {
        w[q] = pack4(c + 4 * q); z[q] = zero2(w[q]); cnt[q + 1] = cnt[q] + __builtin_popcount(z[q]);
        lookup4(c + 4 * q, w[q], L + LDS_CL);
    }
    FENCE();
    const u32 incl = scan64(cnt[4]), n1 = __builtin_amdgcn_readlane(incl, 63);
    u32 n2 = 0, n3 = 0, len64 = 64;
    raw = raw || n1 > 2040;
    const u32 off3 = F::OFF_2 + ((n1 + 15) >> 4) * 4;
    if (!raw) {
        const u32 a1 = incl - cnt[4] + ((u32)(size_t)T + 2047);
#pragma unroll
        for (int q = 0; q < 4; ++q) gather4(v + 4 * q, c + 4 * q, z[q], a1 + cnt[q]);
        st_sys_s(lane * 16, dst, w);
        if (MB == 3) {
            u32x4 o[2];
#pragma unroll
            for (int p = 0; p < 8; ++p) o[p >> 2][p & 3] = bfi(0x87878787u, x[2 * p], bfi(0xF0F0F0F0u, x[2 * p + 1] << 4, x[2 * p + 1] >> 4));
            st_sys_s(lane * 16, dst + 1024, o[0]);
            st_sys_s(lane * 16, dst + 2048, o[1]);
        } else {
            u32x4 o;
            u32x2 sg;
#pragma unroll
            for (int q = 0; q < 4; ++q)
                o[q] = bfi(0x0F0F0F0Fu, bfi(0x03030303u, x[4 * q], x[4 * q + 1] << 2), bfi(0x03030303u, x[4 * q + 2], x[4 * q + 3] << 2) << 4);
#pragma unroll
            for (int k = 0; k < 2; ++k) {
                u32 t = x[8 * k];
#pragma unroll
                for (int j = 1; j < 8; ++j) t = bfi(0x80808080u >> j, x[8 * k + j] >> j, t);
                sg[k] = t;
            }
            *(LAS u32x2*)(T + lane * 8) = sg;
            st_sys_s(lane * 16, dst + 1024, o);
        }
        FENCE();
        for (u32 b = 0; b < n1; b += 1024) {
            const u32x4 y = *(LAS u32x4*)(T + 2048 + b + lane * 16);
            u32 u[4], c2[4];
#pragma unroll
            for (int j = 0; j < 4; ++j) {
                c2[j] = code_of(0x00000300u, 0x00000102u, (y[j] + 0x01010101u) & F::EM);
                u[j] = (y[j] + 0x05050505u) & F::EM;
            }
            const u32 w2 = pack4(c2);
            lookup4(c2, w2, L + LDS_CL);
            FENCE();
            const u32 z2 = zero2(w2), k2 = __builtin_popcount(z2), i2 = scan64(k2);
            if (b + lane * 16 < n1) *(LAS u32*)(T + F::OFF_2 + (b >> 2) + lane * 4) = w2;
            const u32 all = n2 + __builtin_amdgcn_readlane(i2, 63);
            if (all <= 1016 && k2) gather4(u, c2, z2, n2 + i2 - k2 + ((u32)(size_t)T + 4095));
            n2 = all;
        }
        raw = n2 > 1016;
    }
    if (!raw) {
        FENCE();
        const u32x4 y = *(LAS u32x4*)(T + 4096 + lane * 16);
        u32 k3 = 0, m[4];
#pragma unroll
        for (int j = 0; j < 4; ++j) { m[j] = ((y[j] & 0x1C1C1C1Cu) + 0x1C1C1C1Cu) & 0x20202020u; k3 += __builtin_popcount(m[j]); }
        u32 i3 = 0;
        if (__builtin_amdgcn_ballot_w64(k3 != 0)) { i3 = scan64(k3); n3 = __builtin_amdgcn_readlane(i3, 63); }
        const u32 offp = off3 + ((n2 + 15) >> 4) * 4;
        len64 = (F::HEAD + offp + 2 * n3 + 63) >> 6;
        raw = len64 >= 64 || n3 > 63;
        if (!raw) {
            if (lane * 16 < n2)
                *(LAS u32*)(T + off3 + lane * 4) = bfi(0x0F0F0F0Fu, bfi(0x03030303u, y[0], y[1] << 2), bfi(0x03030303u, y[2], y[3] << 2) << 4);
            if (lane == 0) *(LAS u32*)(T + F::OFF_H) = eb | (n1 << 5) | (n2 << 16) | (n3 << 26);
            if (n3) {
                LAS unsigned char* p = T + offp + 2 * (i3 - k3);
                u32 mm = (m[0] >> 5) | (m[1] >> 4) | (m[2] >> 3) | (m[3] >> 2);
                while (mm) {
                    const u32 b = __builtin_ctz(mm), i = lane * 16 + (b & 3u) * 4 + (b >> 3);
                    mm &= mm - 1;
                    *(LAS u16*)p = (u16)((i << 5) | T[4096 + i]);
                    p += 2;
                }
            }
        }
    }
    if (raw) {
        WAIT_VM(0);
#pragma unroll
        for (int i = 0; i < 4; ++i) st_sys_s(lane * 16, dst + i * 1024, piece(x, i));
        return 64;
    }
    FENCE();
    const u32 tl = len64 * 64 - F::HEAD;
    if (lane * 16 < tl) st_sys_s(lane * 16, dst + F::HEAD, *(LAS u32x4*)(T + lane * 16));
    if (MB == 2 && 1024 + lane * 16 < tl) st_sys_s(lane * 16, dst + F::HEAD + 1024, *(LAS u32x4*)(T + 1024 + lane * 16));
    return len64;
}

template <int MB> __device__ __forceinline__ void decode_unit(const char* qa, const char* qb, u32 hdr, u32x4 w, u32 w2,
                                                              LAS unsigned char* S, LAS unsigned char* L, int lane) {
    typedef Fmt<MB> F;
    LAS unsigned char* T = S + F::HEAD;
    const u32 eb = hdr & 31u, n1 = (hdr >> 5) & 2047u, n2 = (hdr >> 16) & 1023u, n3 = hdr >> 26;
    const u32 off3 = F::OFF_2 + ((n1 + 15) >> 4) * 4, offp = off3 + ((n2 + 15) >> 4) * 4;
    const u32 w3 = *(LAS u32*)(T + off3 + lane * 4);
    const u32 en = *(LAS u16*)(T + offp + lane * 2);
    u32 s1[16], s2[4];
    lookup4(s2, w2, L + LDS_DL);
    const u32x4 o0 = *(LAS u32x4*)(S + 1024 + lane * 16);
    u32x4 o1;
    if (MB == 3) o1 = *(LAS u32x4*)(S + 2048 + lane * 16);
    else { const u32x2 sg = *(LAS u32x2*)(T + lane * 8); o1 = u32x4{sg[0], sg[1], 0, 0}; }
    FENCE(); SB();
    const u32 eb4 = eb * 0x01010101u;
    const u32 t1 = ((eb4 + 0x03020100u) & F::EM) << MB, t2 = ((eb4 + 0x041F0000u) & F::EM) << MB, t3 = ((eb4 + 0x1E1D1C1Bu) & F::EM) << MB;
    u32x4 z;
    u32 cnt[5] = {0};
#pragma unroll
    for (int q = 0; q < 4; ++q) { z[q] = zero2(w[q]); cnt[q + 1] = cnt[q] + __builtin_popcount(z[q]); }
    u32 z2 = zero2(w2), k2 = __builtin_popcount(z2);
    const u32 both = cnt[4] | (k2 << 16), incl = scan64(both);
    const u32 excl = incl - both + ((u32)(size_t)S - 2048) + (((u32)(size_t)S - 3072) << 16);
    SB();
    {
        u32x4 y;
#pragma unroll
        for (int j = 0; j < 4; ++j) y[j] = perm(0u, t3, (w3 >> (2 * j)) & 0x03030303u);
        *(LAS u32x4*)(S - 3072 + lane * 16) = y;
        FENCE();
        if ((u32)lane < n3) *(S - 3072 + ((en >> 5) & 1023u)) = (unsigned char)(((en + eb + 27u) & (F::EM & 31u)) << MB);
        FENCE();
    }
    u32 o[16];
    u32x2 q[16];
    window4(o, q, z2, excl >> 16);
    SB();
#pragma unroll
    for (int i = 0; i < 4; ++i) lookup4(s1 + 4 * i, w[i], L + LDS_DL);
    FENCE(); SB();
    *(LAS u32x4*)(S - 2048 + lane * 16) =
        u32x4{spread(t2, o[0], q[0], s2[0]), spread(t2, o[1], q[1], s2[1]), spread(t2, o[2], q[2], s2[2]), spread(t2, o[3], q[3], s2[3])};
    if (n1 > 1024) {
        w2 = *(LAS u32*)(T + F::OFF_2 + 256 + lane * 4);
        lookup4(s2, w2, L + LDS_DL);
        z2 = zero2(w2); k2 = __builtin_popcount(z2);
        window4(o, q, z2, (__builtin_amdgcn_readlane(incl, 63) >> 16) + (u32)(size_t)S - 3072 + scan64(k2) - k2);
        *(LAS u32x4*)(S - 2048 + 1024 + lane * 16) =
            u32x4{spread(t2, o[0], q[0], s2[0]), spread(t2, o[1], q[1], s2[1]), spread(t2, o[2], q[2], s2[2]), spread(t2, o[3], q[3], s2[3])};
    }
    FENCE(); SB();
#pragma unroll
    for (int i = 0; i < 4; ++i) window4(o + 4 * i, q + 4 * i, z[i], (excl & 0xFFFFu) + cnt[i]);
    SB();
    u32x4 e[4];
#pragma unroll
    for (int j = 0; j < 16; ++j) {
        const u32 ex = spread(t1, o[j], q[j], s1[j]);
        if (MB == 3) {
            const u32 v = j < 8 ? o0[(j >> 1) & 3] : o1[(j >> 1) & 3];
            e[j >> 2][j & 3] = (j & 1) ? and_or(v >> 4, 0x07070707u, and_or(v << 4, 0x80808080u, ex)) : and_or(v, 0x87878787u, ex);
        } else {
            e[j >> 2][j & 3] = and_or(o0[j >> 2] >> (2 * (j & 3)), 0x03030303u, and_or(o1[j >> 3] << (j & 7), 0x80808080u, ex));
        }
        if ((j & 3) == 3) st_sys_s(lane * 16, (j < 8 ? qa : qb) + (j & 4) * 256, e[j >> 2]);
    }
}

#define V10(a) "v" #a "0", "v" #a "1", "v" #a "2", "v" #a "3", "v" #a "4", "v" #a "5", "v" #a "6", "v" #a "7", "v" #a "8", "v" #a "9"
__device__ __forceinline__ void mfma_role(u32 sa, u32 sb, u32 sc, u32 olo, u32 ohi, u32 scale) {
    u32 need, m, spin, scl;
    asm volatile(
        ".macro AG_FRAG d, a, o\n\t"
        "ds_read_b128 v[\\d:\\d+3], \\a offset:\\o\n\t"
        "ds_read_b128 v[\\d+4:\\d+7], \\a offset:\\o+4096\n\t"
        ".endm\n\t"
        ".macro AG_MM c, a, b\n\t"
        "v_mfma_f32_32x32x64_f8f6f4 v[\\c:\\c+15], v[\\a:\\a+7], v[\\b:\\b+7], v[\\c:\\c+15] cbsz:0 blgp:1\n\t"
        ".endm\n\t"
        ".macro AG_MIN\n\t"
        "v_min_u32 v164, v164, v165\n\t"
        "s_nop 0\n\t"
        "v_readfirstlane_b32 %[m], v164\n\t"
        ".endm\n\t"
        ".macro AG_SPIN\n\t"
        "s_mov_b32 %[spin], 0\n\t"
        ".LAG_S\\@:\n\t"
        "ds_read_b64 v[164:165], v162\n\t"
        "s_add_u32 %[spin], %[spin], 1\n\t"
        "s_waitcnt lgkmcnt(0)\n\t"
        "AG_MIN\n\t"
        "s_cmp_ge_u32 %[m], %[need]\n\t"
        "s_cbranch_scc1 .LAG_E\\@\n\t"
        "s_sleep 1\n\t"
        "s_cmp_lt_u32 %[spin], 0x2000000\n\t"
        "s_cbranch_scc1 .LAG_S\\@\n\t"
        "s_mov_b32 %[scale], 0x7fc00000\n\t"
        ".LAG_E\\@:\n\t"
        ".endm\n\t"
        ".macro AG_DONE\n\t"
        "v_add_u32 v163, 1, v163\n\t"
        "ds_write_b32 v162, v163 offset:8\n\t"
        ".endm\n\t"
        ".macro AG_FULL so, sn\n\t"
        "s_waitcnt lgkmcnt(4)\n\t"
        "AG_MM 0, 128, 144\n\t"
        "s_waitcnt lgkmcnt(2)\n\t"
        "AG_MM 64, 136, 144\n\t"
        "ds_read_b64 v[164:165], v162\n\t"
        "AG_FRAG 144, v161, \\so+2048\n\t"
        "s_add_u32 %[need], %[need], 1\n\t"
        "s_waitcnt lgkmcnt(3)\n\t"
        "AG_MM 16, 128, 152\n\t"
        "AG_MM 80, 136, 152\n\t"
        "AG_FRAG 152, v161, \\so+3072\n\t"
        "s_waitcnt lgkmcnt(2)\n\t"
        "AG_MM 32, 128, 144\n\t"
        "AG_MIN\n\t"
        "AG_MM 96, 136, 144\n\t"
        "s_cmp_ge_u32 %[m], %[need]\n\t"
        "s_cbranch_scc1 .LAG_F\\@\n\t"
        "s_waitcnt lgkmcnt(0)\n\t"
        "AG_MM 48, 128, 152\n\t"
        "AG_DONE\n\t"
        "AG_MM 112, 136, 152\n\t"
        "AG_SPIN\n\t"
        "AG_FRAG 144, v161, \\sn\n\t"
        "AG_FRAG 128, v160, \\sn\n\t"
        "s_branch .LAG_J\\@\n\t"
        ".LAG_F\\@:\n\t"
        "AG_FRAG 144, v161, \\sn\n\t"
        "s_waitcnt lgkmcnt(2)\n\t"
        "AG_MM 48, 128, 152\n\t"
        "AG_DONE\n\t"
        "AG_FRAG 128, v160, \\sn\n\t"
        "AG_MM 112, 136, 152\n\t"
        ".LAG_J\\@:\n\t"
        "AG_FRAG 136, v160, \\sn+2048\n\t"
        "AG_FRAG 152, v161, \\sn+1024\n\t"
        ".endm\n\t"
        ".macro AG_PREP c, b\n\t"
        ".irp j, 0, 16, 32, 48\n\t"
        ".irp i, 0, 1, 2, 3\n\t"
        "v_mul_f32 v[\\c+\\j+4*\\b+\\i], %[scl], v[\\c+\\j+4*\\b+\\i]\n\t"
        ".endr\n\t"
        ".endr\n\t"
        "v_swap_b32 v[\\c+4*\\b+1], v[\\c+16+4*\\b]\n\t"
        "v_swap_b32 v[\\c+4*\\b+2], v[\\c+32+4*\\b]\n\t"
        "v_swap_b32 v[\\c+4*\\b+3], v[\\c+48+4*\\b]\n\t"
        "v_swap_b32 v[\\c+16+4*\\b+2], v[\\c+32+4*\\b+1]\n\t"
        "v_swap_b32 v[\\c+16+4*\\b+3], v[\\c+48+4*\\b+1]\n\t"
        "v_swap_b32 v[\\c+32+4*\\b+3], v[\\c+48+4*\\b+2]\n\t"
        ".endm\n\t"
        ".macro AG_ST c, r\n\t"
        ".if ((\\r) & 3) == 0\n\t"
        "AG_PREP \\c, (\\r)/4\n\t"
        ".endif\n\t"
        "global_store_dwordx4 v166, v[\\c+16*((\\r)&3)+((\\r)&12):\\c+16*((\\r)&3)+((\\r)&12)+3], vcc\n\t"
        ".if ((\\r) & 3) == 3\n\t"
        "s_add_u32 vcc_lo, vcc_lo, 5*16384\n\t"
        ".else\n\t"
        "s_add_u32 vcc_lo, vcc_lo, 16384\n\t"
        ".endif\n\t"
        "s_addc_u32 vcc_hi, vcc_hi, 0\n\t"
        ".endm\n\t"
        ".macro AG_STN lo, hi\n\t"
        ".set AG_K, \\lo\n\t"
        ".rept (\\hi)-(\\lo)\n\t"
        "AG_ST 0, AG_K\n\t"
        ".set AG_K, AG_K+1\n\t"
        ".endr\n\t"
        ".endm\n\t"
        ".macro AG_XF d, q\n\t"
        ".if ((\\q) %% 5) == 4\n\t"
        "AG_FRAG \\d, v160, 2*16384+2048\n\t"
        ".else\n\t"
        "AG_FRAG \\d, v160, ((\\q) %% 5)*16384\n\t"
        ".endif\n\t"
        ".endm\n\t"
        ".macro AG_WF d, q, i\n\t"
        ".if ((\\q) %% 5) == 4\n\t"
        "AG_FRAG \\d, v167, (\\i/2)*16384+(\\i&1)*1024\n\t"
        ".else\n\t"
        "AG_FRAG \\d, v161, ((\\q) %% 5)*16384+\\i*1024\n\t"
        ".endif\n\t"
        ".endm\n\t"
        ".macro AG_HALF c, q, last, r0, r1, r2\n\t"
        ".set AG_B0, 136+8*(((\\q)+1) %% 3)\n\t"
        ".set AG_B1, 136+8*(((\\q)+2) %% 3)\n\t"
        ".set AG_B2, 136+8*((\\q) %% 3)\n\t"
        "s_waitcnt lgkmcnt(2)\n\t"
        "AG_MM \\c, 128, AG_B0\n\t"
        "AG_WF AG_B0, \\q, 3\n\t"
        "s_add_u32 %[need], %[need], 1\n\t"
        "AG_MIN\n\t"
        "AG_MM \\c+16, 128, AG_B1\n\t"
        ".if \\last == 0\n\t"
        "s_cmp_ge_u32 %[m], %[need]\n\t"
        "s_cbranch_scc1 .LAG_F\\@\n\t"
        ".endif\n\t"
        "AG_STN \\r0, \\r1\n\t"
        "s_waitcnt lgkmcnt(2)\n\t"
        "AG_MM \\c+32, 128, AG_B2\n\t"
        "s_waitcnt lgkmcnt(0)\n\t"
        "AG_MM \\c+48, 128, AG_B0\n\t"
        ".if \\last == 0\n\t"
        "AG_DONE\n\t"
        "AG_SPIN\n\t"
        "AG_WF AG_B1, (\\q)+1, 0\n\t"
        "AG_WF AG_B2, (\\q)+1, 1\n\t"
        "s_branch .LAG_J\\@\n\t"
        ".LAG_F\\@:\n\t"
        "AG_WF AG_B1, (\\q)+1, 0\n\t"
        "AG_STN \\r0, \\r1\n\t"
        "s_waitcnt lgkmcnt(4)\n\t"
        "AG_MM \\c+32, 128, AG_B2\n\t"
        "AG_WF AG_B2, (\\q)+1, 1\n\t"
        "s_waitcnt lgkmcnt(4)\n\t"
        "AG_MM \\c+48, 128, AG_B0\n\t"
        "AG_DONE\n\t"
        ".LAG_J\\@:\n\t"
        "AG_XF 128, (\\q)+1\n\t"
        "ds_read_b64 v[164:165], v162\n\t"
        "AG_WF AG_B0, (\\q)+1, 2\n\t"
        ".endif\n\t"
        "AG_STN \\r1, \\r2\n\t"
        ".endm\n\t"
        ".macro AG_SCALE\n\t"
        "ds_read_b32 v164, v162 offset:12\n\t"
        "s_nop 15\n\t"
        "s_nop 7\n\t"
        "s_waitcnt lgkmcnt(0)\n\t"
        "v_readfirstlane_b32 %[m], v164\n\t"
        "s_cmp_eq_u32 %[m], 0\n\t"
        "s_cselect_b32 %[scl], %[scale], 0x7fc00000\n\t"
        ".endm\n\t"

        "s_waitcnt vmcnt(0) lgkmcnt(0)\n\t"
        "v_mbcnt_lo_u32_b32 v167, -1, 0\n\t"
        "v_mbcnt_hi_u32_b32 v167, -1, v167\n\t"
        "v_lshrrev_b32 v165, 5, v167\n\t"
        "v_and_b32 v167, 31, v167\n\t"
        "v_lshlrev_b32 v167, 4, v167\n\t"
        "v_lshl_add_u32 v166, v165, 16, v167\n\t"
        "v_lshl_add_u32 v167, v165, 13, v167\n\t"
        "v_add_u32 v160, %[sa], v167\n\t"
        "v_add_u32 v161, %[sb], v167\n\t"
        "v_subrev_u32 v167, 65536-2048, v161\n\t"
        "v_mov_b32 v162, %[sc]\n\t"
        "v_mov_b32 v163, 0\n\t"
        ".set AG_I, 0\n\t"
        ".rept 128\n\t"
        "v_mov_b32 v[AG_I], 0\n\t"
        ".set AG_I, AG_I+1\n\t"
        ".endr\n\t"
        "s_mov_b32 vcc_lo, %[olo]\n\t"
        "s_mov_b32 vcc_hi, %[ohi]\n\t"
        "s_mov_b32 %[need], 1\n\t"
        "AG_SPIN\n\t"
        "AG_FRAG 144, v161, 0\n\t"
        "AG_FRAG 128, v160, 0\n\t"
        "AG_FRAG 136, v160, 2048\n\t"
        "AG_FRAG 152, v161, 1024\n\t"
        ".LAG_LOOP:\n\t"
        "AG_FULL 0, 16384\n\t"
        "AG_FULL 16384, 32768\n\t"
        "AG_FULL 32768, 49152\n\t"
        "AG_FULL 49152, 0\n\t"
        "s_cmp_le_u32 %[need], 64-" STR(NSPLIT) "\n\t"
        "s_cbranch_scc1 .LAG_LOOP\n\t"
        "ds_read_b64 v[164:165], v162\n\t"
        "AG_WF 136, 0, 2\n\t"
        ".set AG_I, 0\n\t"
        ".rept " STR(NSPLIT) "\n\t"
        "AG_HALF 0, AG_I, (AG_I==" STR(NSPLIT) "-1), 0, 0, 0\n\t"
        ".set AG_I, AG_I+1\n\t"
        ".endr\n\t"
        "AG_DONE\n\t"
        "AG_SCALE\n\t"
        "AG_STN 0, " STR(NS0) "\n\t"
        "AG_SPIN\n\t"
        "AG_WF AG_B1, AG_I, 0\n\t"
        "AG_WF AG_B2, AG_I, 1\n\t"
        "AG_XF 128, AG_I\n\t"
        "ds_read_b64 v[164:165], v162\n\t"
        "AG_WF AG_B0, AG_I, 2\n\t"
        ".set AG_I, 0\n\t"
        ".rept " STR(NSPLIT) "\n\t"
        ".set AG_R0, " STR(NS0) "+(AG_I*(16-" STR(NS0) ")+" STR(NST) "-1)/" STR(NST) "\n\t"
        ".set AG_R2, " STR(NS0) "+((AG_I+1)*(16-" STR(NS0) ")+" STR(NST) "-1)/" STR(NST) "\n\t"
        ".if AG_I >= " STR(NST) "\n\t"
        ".set AG_R0, 16\n\t"
        ".set AG_R2, 16\n\t"
        ".endif\n\t"
        "AG_HALF 64, AG_I+" STR(NSPLIT) ", (AG_I==" STR(NSPLIT) "-1), AG_R0, (AG_R0+AG_R2+1)/2, AG_R2\n\t"
        ".set AG_I, AG_I+1\n\t"
        ".endr\n\t"
        "AG_SCALE\n\t"
        "s_add_u32 vcc_lo, vcc_lo, 96*16384\n\t"
        "s_addc_u32 vcc_hi, vcc_hi, 0\n\t"
        ".irp r, 0, 1, 2, 3, 4, 5, 6, 7, 8, 9, 10, 11, 12, 13, 14, 15\n\t"
        "AG_ST 64, \\r\n\t"
        ".endr\n\t"
        "s_nop 1\n\t"
        : [scale] "+s"(scale), [need] "=&s"(need), [m] "=&s"(m), [spin] "=&s"(spin), [scl] "=&s"(scl)
        : [sa] "s"(sa), [sb] "s"(sb), [sc] "s"(sc), [olo] "s"(olo), [ohi] "s"(ohi)
        : "memory", "scc", "vcc", V10(), V10(1), V10(2), V10(3), V10(4), V10(5), V10(6), V10(7), V10(8), V10(9),
          V10(10), V10(11), V10(12), V10(13), V10(14), V10(15),
          "v160", "v161", "v162", "v163", "v164", "v165", "v166", "v167");
}

__device__ __forceinline__ char* peer_of_lane(const Params& P) {
    u32 lo = 0, hi = 0;
#pragma unroll
    for (int p = 0; p < NR; ++p) {
        asm volatile("v_writelane_b32 %0, %2, %4\n\tv_writelane_b32 %1, %3, %4"
                     : "+v"(lo), "+v"(hi) : "s"((u32)(u64)P.ws[p]), "s"((u32)((u64)P.ws[p] >> 32)), "i"(p));
    }
    return (char*)(((u64)hi << 32) | lo);
}
__device__ __forceinline__ void arrive(LAS unsigned char* L, int lane) {
    if (lane == 0) __hip_atomic_fetch_add((LAS u32*)(L + LDS_PRO), 1u, __ATOMIC_RELAXED, __HIP_MEMORY_SCOPE_WORKGROUP);
}

template <bool W> __device__ __forceinline__ void prologue_unit(const Params& P, LAS unsigned char* L, char* myws, u32 callv,
                                                                 const u32 (&d)[16], int idx, int xg, int wave, int lane) {
    u32 x[16];
    __builtin_amdgcn_s_setprio(3);
    const int u = xg * 4 + wave;
    const bool half = !W && u >= (int)R0;
    char* peer = peer_of_lane(P);
    WAIT_VM0();
    if (W) {
        order_w(x, d);
    } else {
        LAS u32* xc = (LAS u32*)(L + LDS_XCH);
        if (lane == 0) __hip_atomic_fetch_add(xc, 1u, __ATOMIC_RELAXED, __HIP_MEMORY_SCOPE_WORKGROUP);
        for (u32 spin = 0; spin < SPINS && UNI(*(volatile LAS u32*)xc) < 4; ++spin) {}
        FENCE();
        LAS unsigned char* b = L + LDS_A + 32768 + lane * 64 + (half ? (wave & 1) * 8192 + (wave & 2) * 16 : wave * 16);
        const u32 hi = half ? 16 : 8192;
#pragma unroll
        for (int i = 0; i < 4; ++i) {
            const u32x4 v = *(LAS u32x4*)(b + (i & 1) * 4096 + (i >> 1) * hi);
            x[4 * i] = v[0]; x[4 * i + 1] = v[1]; x[4 * i + 2] = v[2]; x[4 * i + 3] = v[3];
        }
    }
    const u32 par = (UNI(callv) + 1) & 1;
    const bool raw = u >= (int)ROUNDS - (W ? 1 : 2);
    u32 len64 = 64;
    if (!W && u == 0) {
        char* dst = myws + WS_CB + par * CB_BYTES + cb_off(idx, u);
#pragma unroll
        for (int i = 0; i < 4; ++i) st_sys_s(lane * 16, dst + i * 1024, piece(x, i));
    } else if (!W || u >= NPW) {
        len64 = encode_unit<W ? 2 : 3>(x, L + LDS_A + wave * 8192, L, myws + WS_CB + par * CB_BYTES + cb_off(idx, u), lane, raw);
    }
    if (lane == 0) *(volatile LAS u32*)(L + LDS_LEN + wave * 4) = len64;
    const char* qa = myws + WS_IMG + par * IMG_BYTES + unit_off(idx, half ? u & ~1 : u) + (half ? (u & 1) * 2048 : 0);
    const char* qb = half ? myws + WS_IMG + par * IMG_BYTES + unit_off(idx, u | 1) + (u & 1) * 2048 : qa + 2048;
    st_sys_s(lane * 16, qa, piece(x, 0)); st_sys_s(lane * 16, qa + 1024, piece(x, 1));
    st_sys_s(lane * 16, qb, piece(x, 2)); st_sys_s(lane * 16, qb + 1024, piece(x, 3));
    WAIT_VM(4);
    if (xg == 0 && (!W || u >= NPW) && lane < NR && lane != P.rank)
        asm volatile("global_store_dword %0, %1, off sc0 sc1" :: "v"(peer + WS_RDY + par * 32768 + (P.rank * 32 + idx) * 128 + u * 4),
                     "v"(((UNI(callv) + 1) << 7) | len64) : "memory");
    if (xg == 0 && (!W || u >= NPW)) WAIT_VM(1); else WAIT_VM(0);
    arrive(L, lane);
    __builtin_amdgcn_s_setprio(0);
}

__global__ void __launch_bounds__(NTHREADS, 3) ag_gemm(Params P) {
    __shared__ __attribute__((aligned(16))) unsigned char lds_raw[LDS_BYTES];
    LAS unsigned char* L = (LAS unsigned char*)lds_raw;
    const int hw = UNI(threadIdx.x >> 6);
    const int wave = hw < 4 ? 11 - hw : hw - 4;
    const int lane = threadIdx.x & 63;
    const int bid = blockIdx.x;
    const unsigned char* px = P.x;
    const unsigned char* pw_ = P.w;
    char* myws = P.myws;
    int rank = P.rank;
    u32 srcmap = P.srcmap;
    asm volatile("" : "+s"(px), "+s"(pw_), "+s"(myws), "+s"(rank), "+s"(srcmap) : "s"(P.ws[0]));
    const int xg = bid & 7, idx = bid >> 3;
    const int rg = (xg + 5) & 7;
    const int tm = (xg >> 1) * 4 + (idx >> 3), tn = (xg & 1) * 8 + (idx & 7);
    GAS u32* tmo = (GAS u32*)(myws + WS_TMO);

    u32 xo = (lane >> 2) * KS + (lane & 3) * 16;
    u32 d[16];
    if (wave < 4 && idx < 16) {
        const unsigned char* g = px + (size_t)(idx * 256 + wave * 64) * KS + rg * 64;
        u32 keep;
        asm volatile("s_mov_b32 %1, m0\n\ts_mov_b32 m0, %3\n\ts_nop 0\n\tglobal_load_lds_dwordx4 %0, %2\n\t"
                     ".rept 3\n\ts_add_u32 m0, m0, 0x400\n\tv_add_u32 %0, 16*512, %0\n\tglobal_load_lds_dwordx4 %0, %2\n\t.endr\n\t"
                     "s_mov_b32 m0, %1"
                     : "+v"(xo), "=&s"(keep) : "s"(g), "s"((u32)(size_t)L + LDS_A + 32768 + wave * 4096) : "memory", "scc");
    }
    FENCE();
    const int pr = (idx >> 4) + 2 * (11 - wave);
    if (wave < 4 ? idx >= 16 : wave >= 10 && pr < NPW) load_w(d, pw_, idx, wave < 4 ? rg * 4 + wave : pr, lane);
    u32 call = __hip_atomic_load((GAS u32*)(myws + WS_CNT + bid * 128), __ATOMIC_RELAXED, __HIP_MEMORY_SCOPE_AGENT);
    if (wave < 4) {
        encode_clear(L + LDS_A + wave * 8192, lane);
    } else if (wave < 8) {
        const u32 f = (wave - 4) * 64 + lane;
        u32 dl = 0, cl = 0x0C0C0C0Cu, k = 0;
#pragma unroll
        for (u32 i = 0; i < 4; ++i) {
            const u32 c = (f >> (2 * i)) & 3u;
            dl |= (c ? 4 + c : k) << (8 * i);
            cl ^= c ? 0u : (12u ^ i) << (8 * k);
            k += c == 0;
        }
        *(LAS u32*)(L + LDS_DL + f * 4) = dl;
        *(LAS u32*)(L + LDS_CL + f * 4) = cl;
    } else if (wave == 8) {
        ((LAS u32*)(L + LDS_CTL))[lane] = 0;
    }
    BAR();

    if (wave < 8) {
        if (wave < 4) {
            if (idx < 16) prologue_unit<false>(P, L, myws, call, d, idx, rg, wave, lane);
            else prologue_unit<true>(P, L, myws, call, d, idx, rg, wave, lane);
        }
        typedef const __attribute__((address_space(4))) float* cf32;
        const u32 scb = UNI(__builtin_bit_cast(u32, *(cf32)(u64)P.sx * *(cf32)(u64)P.sw));
        const int g = wave >> 1, nh = wave & 1;
        const u64 ob = (u64)P.out + ((size_t)(tm * 256 + g * 32) * MN + tn * 256 + nh * 128) * 4;
        u32 olo = (u32)ob, ohi = (u32)(ob >> 32);
        mfma_role((u32)(size_t)L + LDS_A + g * 512, (u32)(size_t)L + LDS_B + nh * 512, (u32)(size_t)L + LDS_CTL + wave * 16,
                  olo, ohi, scb);
    } else if (wave < 10) {
        const u64 tk = rt();
        __builtin_amdgcn_s_setprio(2);
        if (wave == 8) {
            char* peer = peer_of_lane(P);
            call = UNI(call) + 1;
            if (rg == 0 && lane < NR) {
                u32 junk;
                asm volatile("global_load_dword %0, %1, off sc0 sc1\n\ts_waitcnt vmcnt(0)" : "=&v"(junk) : "v"(peer + WS_PAD + bid * 128) : "memory");
            }
            WAIT_VM0();
            for (u32 spin = 0; spin < SPINS && UNI(*(volatile LAS u32*)(L + LDS_PRO)) < 7; ++spin) __builtin_amdgcn_s_sleep(1);
            FENCE();
            const u32x4 ln = *(LAS u32x4*)(L + LDS_LEN);
            if (lane < NR) {
                if (lane != rank) { if (rg) st_sys(peer + WS_RDY + (call & 1) * 32768 + (rank * 32 + idx) * 128 + rg * 16, ln | (call << 7)); }
                else st_sys(peer + WS_UF + idx * 2048 + rg * 16, u32x4{call, call, call, call});
            }
            if (lane == 0) __hip_atomic_store((GAS u32*)(myws + WS_CNT + bid * 128), call, __ATOMIC_RELAXED, __HIP_MEMORY_SCOPE_AGENT);
        } else {
            call = UNI(call) + 1;
            arrive(L, lane);
            u32 job = XCD_ID() * 32 + idx;
#pragma unroll 1
            for (int k = 0; k < NWG; ++k, job = (job + 1) & (NWG - 1)) {
                u32 old = 0;
                if (lane == 0) {
                    __hip_atomic_fetch_max((GAS u32*)(myws + WS_TIK + (job >> 5) * 128), call * 2048, __ATOMIC_RELAXED, __HIP_MEMORY_SCOPE_AGENT);
                    FENCE();
                    old = __hip_atomic_fetch_max((GAS u32*)(myws + WS_JOB + job * 128), call, __ATOMIC_RELAXED, __HIP_MEMORY_SCOPE_AGENT);
                }
                WAIT_VM0();
                if (UNI(old) < call) break;
            }
            *(volatile LAS u32*)(L + LDS_JOB) = job + 1;
        }
        const int isB = wave - 8;
        const int panel = isB ? 16 + tn : tm;
        const char* img = myws + WS_IMG + (call & 1) * IMG_BYTES + panel * 16384;
        const u32 ldsb = (u32)(size_t)L + (isB ? LDS_B : LDS_A);
        volatile LAS u32* ctl = (volatile LAS u32*)(L + LDS_CTL + (lane & 7) * 16);
        const char* uf = myws + WS_UF + panel * 2048;
        const u32 voff = lane * 16 + ((panel) & 3) * UNIT;
        Ld S = {0, 0, 0, 0, 0, ~0u, 0, 0, (u64)img, (u64)(img + (16 + tn - panel) * 16384 + (size_t)S0 * 524288)};
        u32 err = 0;
        const u32 vctl = (u32)(size_t)ctl, vfull = vctl + isB * 4, vf = (lane & 31) * 4;
        const u64 t0 = rt();
        WAIT_VM0();
        asm volatile("v_mov_b32 v40, 0\n\tv_mov_b32 v41, 0\n\tv_mov_b32 v47, 0\n\tv_mov_b32 v44, %0\n\tv_mov_b32 v45, %1\n\t"
                     "v_mov_b32 v46, %2\n\tv_mov_b32 v48, %3"
                     :: "v"((voff + UNIT) & 16383), "v"((voff + 2 * UNIT) & 16383), "v"((voff + 3 * UNIT) & 16383),
                        "v"(lane * 16 + ((tn + 3) & 3) * UNIT) : "v40", "v41", "v44", "v45", "v46", "v47", "v48");
        bool go = true;
        auto late = [&]() {
            if (UNI(__hip_atomic_load(tmo, __ATOMIC_RELAXED, __HIP_MEMORY_SCOPE_AGENT)) == call || rt() - t0 > DEADLINE) {
                if (lane == 0) __hip_atomic_store(tmo, call, __ATOMIC_RELAXED, __HIP_MEMORY_SCOPE_AGENT);
                ctl[3] = 1;
                err = ~0u;
            }
            asm volatile("s_waitcnt vmcnt(0) lgkmcnt(0)" ::: "memory");
            go = rt() - t0 < 3 * DEADLINE;
        };
#define RUN(M, end, l) while (go && load_run<M>(S, end, l, err, call, uf, voff, vf, vctl, vfull)) late();
        while (rt() - tk < LDELAY) __builtin_amdgcn_s_sleep(4);
        while (go && load_run<0, 4>(S, QEASY, ldsb, err, call, uf, voff, vf, vctl, vfull)) late();
        RUN(0, S0, ldsb);
        const u32 l0 = (u32)(size_t)L + LDS_A;
        if (isB) { RUN(2, NSTAGE, l0); S.g -= (size_t)NSPLIT * 524288; RUN(4, NQ, l0); }
        else { RUN(1, NSTAGE, l0); S.g -= (size_t)NSPLIT * 524288 - 2048; RUN(3, NQ, l0); }
        RUN(5, 0, 0);
        ctl[isB] = 0xFFFFu;
        WAIT_VM(0);
    } else {
        __builtin_amdgcn_s_setprio(2);
        const u32 vo = lane * 16, wk = wave - 10;
        {
            u32 x[16];
            const u32 pdst = (srcmap >> (4 * xg)) & 7u;
            const bool push = pr < NPW && pdst != (u32)rank;
            const u32 up = 32 + ((rank - pdst - 1) & 7) + 7 * pr, pp = idx | 16;
            const char* pw = P.ws[pdst];
            const char* q0 = pw + WS_IMG + unit_off(pp, up);
            asm volatile("" : "+s"(q0));
            WAIT_VM0();
            call = UNI(call) + 1;
            if (push) {
                order_w(x, d);
                q0 += (call & 1) * IMG_BYTES;
                st_sys_s(vo, q0, piece(x, 0));
                arrive(L, lane);
                st_sys_s(vo, q0 + 1024, piece(x, 1));
                st_sys_s(vo, q0 + 2048, piece(x, 2));
                st_sys_s(vo, q0 + 3072, piece(x, 3));
                WAIT_VM(0); st_flag(pw + WS_UF + pp * 2048 + up * 4, call, lane);
            } else {
                arrive(L, lane);
            }
            if (wk) *(volatile LAS u32*)(L + LDS_PSH) = 1;
        }
        call = UNI(call);
        u32 job = 0;
        for (u32 spin = 0; spin < SPINS && job == 0; ++spin) { __builtin_amdgcn_s_sleep(1); job = UNI(*(volatile LAS u32*)(L + LDS_JOB)); }
        job = (job - 1) & (NWG - 1);
        const u32 slot = job >> 5, src = (srcmap >> (4 * slot)) & 7u;
        if (src == (u32)rank) return;
        const char* tik = myws + WS_TIK + slot * 128;
        const char* scb = P.ws[src] + WS_CB + (call & 1) * CB_BYTES;
        const char* rdy = myws + WS_RDY + (call & 1) * 32768 + src * 4096;
        const char* dp = myws + WS_IMG + (call & 1) * IMG_BYTES;
        const u32 u0 = 32 + ((src - rank - 1) & 7);
        LAS unsigned char* sl = L + LDS_WRK + wk * WRK_BYTES + 3072;
        const u32 rot = 2 * rank;
        auto unit_of = [rot](u32 t) -> u32 {
            auto px = [rot](u32 k) { k = (k + rot) & 15; return 4 * (k & 3) + (k >> 2); };
            auto pw = [rot](u32 k) { k = (k + rot) & 15; return 16 + 8 * (k & 1) + (k >> 1); };
            if (t < T0) return t < 64 ? px(t & 15) * 32 + (t >> 4) : pw(t & 15) * 32 + NPW + (t >> 4) - 4;
            if (t < T1) { t -= T0; return ((t & 1) ? pw((t >> 1) & 15) : px((t >> 1) & 15)) * 32 + 4 + (t >> 5); }
            if (t < T2) {
                const u32 pr = (t - T1) / 48, i = (t - T1) % 48;
                return ((i & 16) ? px(i & 15) : pw(i & 15)) * 32 + R0 + 2 * pr + (i >> 5);
            }
            return t < NT ? px((t - T2) & 15) * 32 + R0 + 2 * ((t - T2) >> 4) + 1 : 0u;
        };
        const u64 t0 = rt();
        auto await = [&](u32 un) -> u32 {
            for (u32 n = 1;; ++n) {
                u32 rv;
                if (lane == 0) asm volatile("global_atomic_or v162, %0, %0, %1 sc0 sc1\n\ts_waitcnt vmcnt(0)" :: "v"(0u), "s"(rdy + un * 4) : "memory", "v162");
                asm volatile("v_readfirstlane_b32 %0, v162" : "=s"(rv) :: "memory", "v162");
                if ((rv >> 7) >= call) return rv;
                if ((n & 31) == 0 && (UNI(__hip_atomic_load(tmo, __ATOMIC_RELAXED, __HIP_MEMORY_SCOPE_AGENT)) == call || rt() - t0 > DEADLINE)) {
                    if (lane == 0) __hip_atomic_store(tmo, call, __ATOMIC_RELAXED, __HIP_MEMORY_SCOPE_AGENT);
                    return 0;
                }
                __builtin_amdgcn_s_sleep(2);
            }
        };
        auto book = [&](u64 at, u32 cost) {
            const u64 now = at << 10, c = cost;
            if (lane == 0)
                asm volatile("global_atomic_umax_x2 %0, %1, %3 offset:2048\n\tglobal_atomic_add_x2 v[160:161], %0, %2, %3 offset:2048 sc0"
                             :: "v"(0u), "v"(now), "v"(c), "s"(tik) : "memory", "v160", "v161");
        };
        const char* pend = nullptr;
        u32 pn = 0;
        auto flag = [&]() {
            if ((u32)lane <= pn) asm volatile("global_store_dword %0, %1, %2 sc0 sc1" :: "v"(lane * 28), "v"(call), "s"(pend) : "memory");
            pend = nullptr;
        };
        u32 t = (job & 31) + 32 * wk, un = unit_of(t), lat = 0, plat = 0;
        if (lane < 2)
            asm volatile("global_load_dword v163, %0, %1 sc0 sc1" :: "v"((lane * 482 + (job & 31)) * SLOT + UNIT + 128), "s"(scb) : "memory", "v163");
        for (u32 spin = 0; spin < SPINS && UNI(*(volatile LAS u32*)(L + (t < 16 ? LDS_PSH : LDS_PRO))) < (t < 16 ? 1 : 7); ++spin)
            __builtin_amdgcn_s_sleep(2);
        u32 rv = await(un);
        if (!rv) return;
        WAIT_VM0();
        book(rt() + (t < NOWAIT ? 0 : 40), (((rv & 127u) + 1) >> 1) * (RATE + 15));
        bool now_ = t < NOWAIT;
        if (!now_) WAIT_VM(0);
        t += 64;
        for (;;) {
            u32 len64 = rv & 127u;
            len64 = len64 < 41 ? 41 : (len64 > 64 ? 64 : len64);
            const u32 len = len64 * 64, panel = un >> 5, r = un & 31;
            const char* q = scb + cb_off(panel, r);
            const bool more = t < NT;
            un = unit_of(t);
            const char* rp = rdy + un * 4;
            asm volatile("" : "+s"(q), "+s"(rp));
            if (!now_) {
                u32 lo, hi;
                asm volatile("v_readfirstlane_b32 %0, v160\n\tv_readfirstlane_b32 %1, v161" : "=s"(lo), "=s"(hi) :: "memory", "v160", "v161");
                const u32 start = (u32)((((u64)hi << 32) | lo) >> 10);
                for (u32 d = start - (u32)rt(); d - 1 < 1000u; d = start - (u32)rt()) {
                    if (pend) { if (vm_idle()) { WAIT_VM(0); flag(); } }
                    else if (d > 12) __builtin_amdgcn_s_sleep(2);
                }
            }
            now_ = false;
            const u32 i0 = (u32)rt();
            PIECE(0, q); PIECE(1, q);
            if (2048 + vo < len) PIECE(2, q);
            if (3072 + vo < len) PIECE(3, q);
            if (lane == 0) asm volatile("global_atomic_or v162, %0, %0, %3 sc0 sc1\n\tglobal_atomic_add v163, %0, %1, %2 sc0"
                                        :: "v"(0u), "v"(1u), "s"(tik), "s"(rp) : "memory", "v162", "v163");
            const bool half = panel < 16 && r >= R0;
            const u32 u = u0 + 7 * (half ? r & ~1u : r);
            const char* qa = dp + unit_off(panel, u) + (half ? (r & 1) * 2048 : 0);
            const char* qb = half ? dp + unit_off(panel, u + 7) + (r & 1) * 2048 : qa + 2048;
            const u32 ho = panel < 16 ? 3072 : 2560;
            const bool late = pend != nullptr;
            if (late) WAIT_VM(5);
            asm volatile("" : "+s"(qa), "+s"(qb) :: "memory");
            if (late) { flag(); WAIT_VM(3); } else { WAIT_VM(2); }
            const u32x4 w = *(LAS u32x4*)(sl + lane * 16);
            const u32 hd = *(LAS u32*)(sl + ho), w2 = *(LAS u32*)(sl + ho + 4 + lane * 4);
            const u64 now = rt();
            if (late) WAIT_VM(1); else WAIT_VM(0);
            asm volatile("v_readfirstlane_b32 %0, v163\n\tv_readfirstlane_b32 %1, v162" : "=s"(t), "=s"(rv) :: "memory", "v162", "v163");
            plat = lat;
            lat = (u32)now - i0;
            const u32 ml = lat < plat ? lat : plat;
            const u32 rate = ml > LATHI ? RATE + RATE / 8 : RATE;
            t += 128 - call * 2048;
            const bool known = (rv >> 7) >= call;
            if (more && known) book(now, (((rv & 127u) + 1) >> 1) * rate);
            FENCE(); SB();
            if (len64 == 64) {
                const u32x4 e1 = *(LAS u32x4*)(sl + 1024 + lane * 16), e2 = *(LAS u32x4*)(sl + 2048 + lane * 16), e3 = *(LAS u32x4*)(sl + 3072 + lane * 16);
                st_sys_s(vo, qa, w); st_sys_s(vo, qa + 1024, e1); st_sys_s(vo, qb, e2); st_sys_s(vo, qb + 1024, e3);
            } else if (panel < 16) {
                decode_unit<3>(qa, qb, UNI(hd), w, w2, sl, L, lane);
            } else {
                decode_unit<2>(qa, qb, UNI(hd), w, w2, sl, L, lane);
            }
            pend = myws + WS_UF + panel * 2048 + (u + (half && (r & 1) ? 4 * NSPLIT : 0)) * 4;
            pn = half;
            if (!more) break;
            if (!known) {
                WAIT_VM(0); flag();
                rv = await(un);
                if (!rv) return;
                WAIT_VM0();
                book(rt(), (((rv & 127u) + 1) >> 1) * rate);
                WAIT_VM(0);
            }
            WAIT_VM(4);
        }
        WAIT_VM(0);
        flag();
    }
}

extern "C" void kernel_launch(int rank, int nranks, ncclComm_t comm,
                              hipStream_t stream,
                              void* const* d_in, const int* in_sizes, int n_in,
                              void* d_out, int out_size,
                              const int* in_dtypes, int out_dtype,
                              void* const* d_out_all,
                              void* const* d_ws_all, size_t ws_bytes) {
    (void)comm; (void)in_sizes; (void)n_in; (void)out_size; (void)in_dtypes; (void)out_dtype; (void)d_out_all;
    if (nranks != NR || (unsigned)rank >= (unsigned)NR || ws_bytes < WS_NEED) {
        fprintf(stderr, "kernel_launch: unexpected configuration\n");
        return;
    }
    Params p;
    p.x = (const unsigned char*)d_in[0];
    p.w = (const unsigned char*)d_in[1];
    p.sx = (const float*)d_in[2];
    p.sw = (const float*)d_in[3];
    p.out = (float*)d_out;
    for (int i = 0; i < NR; ++i) p.ws[i] = (char*)d_ws_all[i];
    p.myws = p.ws[rank];
    p.rank = rank;
    p.pad = 0;
    static const u32 near_link[NR] = {0x74510263u, 0x26301475u, 0x17542036u, 0x74153620u,
                                      0x63024157u, 0x60235741u, 0x51476302u, 0x02367514u};
    p.srcmap = near_link[rank];
    hipLaunchKernelGGL(ag_gemm, dim3(NWG), dim3(NTHREADS), 0, stream, p);
    HIP_CHECK(hipGetLastError());
}
```

```cpp
#include <hip/hip_runtime.h>
#include <rccl/rccl.h>
#include <cstdio>

#define HIP_CHECK(expr) do { hipError_t _e = (expr); if (_e != hipSuccess) { \
    fprintf(stderr, "HIP error %s at %s:%d\n", hipGetErrorString(_e), __FILE__, __LINE__); \
    return; } } while (0)

#define GAS __attribute__((address_space(1)))
#define LAS __attribute__((address_space(3)))
typedef unsigned u32;
typedef unsigned short u16;
typedef unsigned long long u64;
typedef unsigned u32x2 __attribute__((ext_vector_type(2)));
typedef unsigned u32x4 __attribute__((ext_vector_type(4)));

constexpr int NR = 8;
constexpr int MN = 4096;
constexpr int KS = 512;
constexpr int NWG = 256;
constexpr int NTHREADS = 768;
constexpr u32 ROUNDS = 32;
constexpr u32 NSTAGE = 64;
#define NSPLIT 8
#define NS0 2
#define NST 6
constexpr int NPW = 2;
constexpr u32 RATE = 215;
constexpr u32 LATHI = 95;
constexpr u32 NOWAIT = 8;
constexpr u32 QEASY = 32, LDELAY = 500;
constexpr u32 S0 = NSTAGE - NSPLIT;
constexpr u32 NQ = NSTAGE + NSPLIT;
constexpr u32 R0 = ((S0 * 4 - 32 + 13) / 14) * 2;
constexpr u32 US = 32 + 7 * R0;
constexpr u32 NPAIR = (ROUNDS - R0) / 2;
constexpr u32 T0 = 128 - 16 * NPW, T1 = T0 + (R0 - 4) * 32, T2 = T1 + NPAIR * 48, NT = T2 + NPAIR * 16;
constexpr u32 UNIT = 4096;
constexpr size_t IMG_BYTES = 32u << 20;
constexpr size_t CB_BYTES = 5u << 20;
__host__ __device__ constexpr u32 unit_off(u32 panel, u32 u) { return (u >> 2) * 524288u + panel * 16384u + ((u + panel) & 3) * UNIT; }
constexpr u32 SLOT = UNIT + 256;
__host__ __device__ constexpr u32 cb_off(u32 panel, u32 r) { return (r * 32 + panel) * SLOT; }
__host__ __device__ constexpr u32 pos_of(u32 j, u32 r) {
    const u32 u = 32 + 7 * r + j, w = u - 4 * S0;
    return u < 4 * S0 ? u : r < R0 ? 4 * S0 + 16 * (w >> 1) + (w & 1) : 4 * S0 + 2 + 8 * ((r - R0) & ~1u) + 7 * (r & 1) + j;
}
static_assert(US - 4 * S0 == 2 * NPAIR && NSPLIT == 4 * NPAIR, "groups of four stages");
constexpr size_t WS_CNT = 0;
constexpr size_t WS_TMO = 48 << 10;
constexpr size_t WS_TIK = 52 << 10;
constexpr size_t WS_UF = 64 << 10;
constexpr size_t WS_RDY = 128 << 10;
constexpr size_t WS_JOB = 192 << 10;
constexpr size_t WS_PAD = 224 << 10;
constexpr size_t WS_IMG = 1 << 20;
constexpr size_t WS_CB = WS_IMG + 2 * IMG_BYTES;
constexpr size_t WS_NEED = WS_CB + 2 * CB_BYTES;
constexpr u64 DEADLINE = 50000000ull;
constexpr u32 SPINS = 1u << 25;
constexpr u32 LDS_CTL = 0;
constexpr u32 LDS_PSH = LDS_CTL + 132, LDS_XCH = LDS_CTL + 136, LDS_PRO = LDS_CTL + 140;
constexpr u32 LDS_JOB = LDS_CTL + 160, LDS_LEN = LDS_CTL + 176;
constexpr u32 LDS_DL = 512, LDS_CL = LDS_DL + 1024;
constexpr u32 LDS_WRK = LDS_CL + 1024, WRK_BYTES = 7168;
constexpr u32 LDS_A = LDS_WRK + 2 * WRK_BYTES, LDS_B = LDS_A + 65536;
constexpr u32 LDS_BYTES = LDS_B + 65536;

struct Params {
    const unsigned char* x;
    const unsigned char* w;
    char* myws;
    int rank;
    u32 srcmap;
    const float* sx;
    const float* sw;
    float* out;
    u64 pad;
    char* ws[NR];
};

__device__ __forceinline__ u64 rt() { return __builtin_amdgcn_s_memrealtime(); }
#define WAIT_VM(n) asm volatile("s_waitcnt vmcnt(" #n ")" ::: "memory")
#define WAIT_VM0() do { __builtin_amdgcn_s_waitcnt(0x0F70); WAIT_VM(0); } while (0)
#define FENCE() asm volatile("" ::: "memory")
#define SB() __builtin_amdgcn_sched_barrier(0)
#define XCD_ID() (__builtin_amdgcn_s_getreg((3 << 11) | 20) & 7u)
#define UNI(x) __builtin_amdgcn_readfirstlane(x)

__device__ __forceinline__ void st_sys(char* p, u32x4 v) {
    asm volatile("global_store_dwordx4 %0, %1, off sc0 sc1\n\ts_nop 1" :: "v"(p), "v"(v) : "memory");
}
__device__ __forceinline__ void st_sys_s(u32 voff, const void* base, u32x4 v) {
    asm volatile("global_store_dwordx4 %0, %1, %2 sc0 sc1\n\ts_nop 1" :: "v"(voff), "v"(v), "s"(base) : "memory");
}
__device__ __forceinline__ void st_flag(const void* base, u32 v, int lane) {
    if (lane == 0) asm volatile("global_store_dword %0, %1, %2 sc0 sc1" :: "v"(0u), "v"(v), "s"(base) : "memory");
}

#define STR2(x) #x
#define STR(x) STR2(x)
struct Ld {
    u32 nops, q, rmask, nf;
    u32 pidx, pb, slot;
    u64 fifo, g, gw;
};
#define G2(v, s_, o) "global_load_lds_dwordx4 " v ", " s_ " offset:" #o " sc0 sc1\n\tglobal_load_lds_dwordx4 " v ", " s_ " offset:" #o "+1024" " sc0 sc1\n\t"
#define EASE ".if %c[ease]\n\ts_sleep %c[ease]\n\t.endif\n\t"
#define M0A(x) "s_add_u32 m0, m0, " x "\n\ts_nop 0\n\t"
#define GW(v) G2(v, "%[g]", 0) M0A("%[land]") G2(v, "%[g]", 2048)
template <int MODE, int EASY = 0>
__device__ __forceinline__ u32 load_run(Ld& S, u32 end, u32 lds, u32 err, u32 call, const void* uf,
                                        u32 vo, u32 vf, u32 vctl, u32 vfull) {
    u32 a, b, land, rc, keep;
    asm volatile(
        "s_mov_b32 %[keep], m0\n\t"
        "s_mov_b32 %[rc], 0\n\t"
        "s_branch .LL%=\n\t"
        ".LZ%=:\n\t"
        ".if %c[md] == 0\n\t"
        "s_sleep 1\n\t"
        ".endif\n\t"
        ".LL%=:\n\t"
        "ds_read_b32 v42, %[vctl] offset:8\n\t"
        "s_getreg_b32 %[a], hwreg(HW_REG_IB_STS)\n\t"
        "s_bfe_u32 %[b], %[a], 0x20016\n\t"
        "s_and_b32 %[a], %[a], 15\n\t"
        "s_lshl4_add_u32 %[a], %[b], %[a]\n\t"
        "s_sub_u32 %[land], %[nops], %[a]\n\t"
        "s_mov_b64 vcc, %[fifo]\n\t"
        "s_sub_u32 %[a], %[land], vcc_lo\n\t"
        "s_bitcmp0_b32 %[a], 11\n\t"
        "s_cselect_b32 %[a], %[nf], 0\n\t"
        "s_cmp_eq_u32 %[a], 0\n\t"
        "s_cbranch_scc1 .LP%=\n\t"
        "v_add_u32 v41, 1, v41\n\t"
        "s_lshr_b64 %[fifo], %[fifo], 12\n\t"
        ".if %c[md]\n\t"
        "v_lshrrev_b32 v49, 6, v41\n\t"
        "v_mad_u32_u24 v49, v49, v50, v41\n\t"
        "ds_write_b32 %[vfull], v49\n\t"
        ".else\n\t"
        "ds_write_b32 %[vfull], v41\n\t"
        ".endif\n\t"
        "s_sub_u32 %[nf], %[nf], 1\n\t"
        ".LP%=:\n\t"
        "s_add_u32 %[rc], %[rc], 0x100000\n\t"
        "s_cbranch_scc1 .LT%=\n\t"
        ".if %c[md] == 5\n\t"
        "s_waitcnt lgkmcnt(0)\n\t"
        "s_cmp_lg_u32 %[nf], 0\n\t"
        "s_cbranch_scc1 .LL%=\n\t"
        ".else\n\t"
        ".if %c[md] != 4\n\t"
        "s_or_b32 %[rmask], %[rmask], %[err]\n\t"
        "s_sub_u32 %[a], %[land], %[pidx]\n\t"
        "s_cmp_lt_i32 %[a], 0\n\t"
        "s_cbranch_scc1 .LO%=\n\t"
        "s_cmp_eq_u32 %[pb], -1\n\t"
        "s_cbranch_scc1 .LS%=\n\t"
        "v_cmp_le_u32 vcc, %[call], v40\n\t"
        "s_sub_u32 %[a], %[q], %[pb]\n\t"
        "s_lshl_b32 %[a], %[a], 1\n\t"
        "s_min_u32 %[a], %[a], 16\n\t"
        "s_lshr_b32 %[b], vcc_lo, %[a]\n\t"
        "s_lshr_b32 %[b], %[b], %[a]\n\t"
        "s_or_b32 %[rmask], %[rmask], %[b]\n\t"
        "s_mov_b32 %[pb], -1\n\t"
        "s_andn2_b32 %[b], 15, %[rmask]\n\t"
        "s_cbranch_scc0 .LO%=\n\t"
        ".LS%=:\n\t"
        "s_andn2_b32 %[b], 0xffff, %[rmask]\n\t"
        "s_cbranch_scc0 .LO%=\n\t"
        "s_lshl_b32 %[b], %[q], 4\n\t"
        "v_add_u32 v43, %[b], %[vf]\n\t"
        "s_add_u32 %[nops], %[nops], 1\n\t"
        "s_mov_b32 exec_hi, 0\n\t"
        "global_atomic_or v40, v43, v47, %[uf] sc0 sc1\n\t"
        "s_mov_b32 exec_hi, -1\n\t"
        "s_mov_b32 %[pidx], %[nops]\n\t"
        "s_mov_b32 %[pb], %[q]\n\t"
        ".LO%=:\n\t"
        ".endif\n\t"
        ".if %c[md] == 0\n\t"
        "s_sub_u32 %[a], %[q], 3\n\t"
        ".else\n\t"
        ".if %c[md] < 3\n\t"
        "s_and_b32 %[a], %[q], 63\n\t"
        "s_cmp_ge_u32 %[a], 68-" STR(NSPLIT) "\n\t"
        ".else\n\t"
        "s_cmp_eq_u32 0, 0\n\t"
        ".endif\n\t"
        "s_subb_u32 %[a], %[q], 3\n\t"
        ".if %c[md] == 4\n\t"
        "s_cmp_eq_u32 %[q], 68\n\t"
        "s_cselect_b32 %[a], 60, %[a]\n\t"
        ".endif\n\t"
        ".endif\n\t"
        "s_waitcnt lgkmcnt(0)\n\t"
        "v_cmp_gt_i32 vcc, %[a], v42\n\t"
        "s_lshl_b32 %[a], %[slot], 14\n\t"
        ".if %c[md]\n\t"
        "s_add_u32 %[b], %[a], 0x10000\n\t"
        "s_cmp_eq_u32 %[slot], 4\n\t"
        "s_cselect_b32 %[a], 2*16384+2048, %[a]\n\t"
        "s_cselect_b32 %[b], 2048, %[b]\n\t"
        "s_cselect_b32 %[land], 16384-2048, 0\n\t"
        ".endif\n\t"
        ".if %c[md] != 4\n\t"
        "s_andn2_b32 m0, 15, %[rmask]\n\t"
        "s_or_b32 m0, m0, vcc_lo\n\t"
        ".else\n\t"
        "s_cmp_lg_u32 vcc_lo, 0\n\t"
        ".endif\n\t"
        "s_cbranch_scc1 .LZ%=\n\t"
        ".if %c[md] == 0\n\t"
        "s_add_u32 m0, %[a], %[lds]\n\t"
        "s_add_u32 %[nops], %[nops], 16\n\t"
        G2("%[vo]", "%[g]", 0) G2("%[vo]", "%[g]", 2048) EASE M0A("0x1000") G2("v44", "%[g]", 0) G2("v44", "%[g]", 2048) EASE M0A("0x1000")
        G2("v45", "%[g]", 0) G2("v45", "%[g]", 2048) EASE M0A("0x1000") G2("v46", "%[g]", 0) G2("v46", "%[g]", 2048)
        ".elseif %c[md] & 1\n\t"
        "s_add_u32 m0, %[a], %[lds]\n\t"
        "s_add_u32 %[nops], %[nops], 6+2*%c[md]\n\t"
        G2("%[vo]", "%[g]", 0) M0A("0x1000") G2("v44", "%[g]", 0) M0A("0x1000") G2("v45", "%[g]", 0) M0A("0x1000") G2("v46", "%[g]", 0)
        ".if %c[md] == 3\n\t"
        "s_add_u32 %[b], %[b], %[lds]\n\t"
        "s_add_u32 m0, %[b], 0x3000\n\t"
        "s_mov_b64 vcc, %[gw]\n\t"
        G2("v48", "vcc", 0) M0A("%[land]") G2("v48", "vcc", 2048)
        "s_add_u32 vcc_lo, vcc_lo, 0x80000\n\t"
        "s_addc_u32 vcc_hi, vcc_hi, 0\n\t"
        "s_mov_b64 %[gw], vcc\n\t"
        ".endif\n\t"
        ".else\n\t"
        "s_add_u32 m0, %[b], %[lds]\n\t"
        "s_sub_u32 %[a], 0x1000, %[land]\n\t"
        "s_add_u32 %[nops], %[nops], 20-2*%c[md]\n\t"
        GW("%[vo]") M0A("%[a]") GW("v44") M0A("%[a]") GW("v45")
        ".if %c[md] == 2\n\t"
        M0A("%[a]") GW("v46")
        ".endif\n\t"
        ".endif\n\t"
        "s_mov_b64 vcc, %[g]\n\t"
        "s_add_u32 vcc_lo, vcc_lo, 0x80000\n\t"
        "s_addc_u32 vcc_hi, vcc_hi, 0\n\t"
        "s_mov_b64 %[g], vcc\n\t"
        "s_and_b32 vcc_lo, %[nops], 0xfff\n\t"
        "s_mov_b32 vcc_hi, 0\n\t"
        "s_mul_i32 %[a], %[nf], 12\n\t"
        "s_lshl_b64 vcc, vcc, %[a]\n\t"
        "s_or_b64 %[fifo], %[fifo], vcc\n\t"
        "s_add_u32 %[nf], %[nf], 1\n\t"
        "s_lshr_b32 %[rmask], %[rmask], 4\n\t"
        "s_add_u32 %[slot], %[slot], 1\n\t"
        "s_cmp_eq_u32 %[slot], 4+(%c[md]+7)/8\n\t"
        "s_cselect_b32 %[slot], 0, %[slot]\n\t"
        "s_add_u32 %[q], %[q], 1\n\t"
        "s_cmp_lt_u32 %[q], %[end]\n\t"
        "s_cbranch_scc1 .LL%=\n\t"
        ".endif\n\t"
        "s_mov_b32 %[rc], 0\n\t"
        "s_branch .LE%=\n\t"
        ".LT%=:\n\t"
        "s_mov_b32 %[rc], 1\n\t"
        "s_waitcnt lgkmcnt(0)\n\t"
        ".LE%=:\n\t"
        "s_mov_b32 m0, %[keep]"
        : [nops] "+s"(S.nops), [q] "+s"(S.q), [rmask] "+s"(S.rmask), [nf] "+s"(S.nf), [pidx] "+s"(S.pidx), [pb] "+s"(S.pb),
          [slot] "+s"(S.slot), [fifo] "+s"(S.fifo), [g] "+s"(S.g), [gw] "+s"(S.gw),
          [a] "=&s"(a), [b] "=&s"(b), [land] "=&s"(land), [rc] "=&s"(rc), [keep] "=&s"(keep)
        : [end] "s"(end), [lds] "s"(lds), [err] "s"(err), [call] "s"(call), [uf] "s"(uf),
          [vo] "v"(vo), [vf] "v"(vf), [vctl] "v"(vctl), [vfull] "v"(vfull), [md] "i"(MODE), [ease] "i"(EASY)
        : "memory", "scc", "vcc", "v40", "v41", "v42", "v43", "v44", "v45", "v46", "v47", "v48", "v49", "v50");
    return rc;
}
#define PIECE(i, g) do { u32 k_; asm volatile("s_mov_b32 %0, m0\n\ts_mov_b32 m0, %3\n\ts_nop 0\n\t" \
    "global_load_lds_dwordx4 %1, %2 offset:1024*" #i " sc0 sc1\n\ts_mov_b32 m0, %0" : "=&s"(k_) : "v"(vo), "s"(g), "s"((u32)(size_t)sl) : "memory"); } while (0)
__device__ __forceinline__ bool vm_idle() {
    u32 r;
    asm volatile("s_getreg_b32 %0, hwreg(HW_REG_IB_STS)" : "=s"(r) :: "memory");
    return (r & 0xC0000Fu) == 0;
}
__device__ __forceinline__ u32 perm(u32 hi, u32 lo, u32 sel) { return __builtin_amdgcn_perm(hi, lo, sel); }
__device__ __forceinline__ u32 bfi(u32 m, u32 a, u32 b) { return __builtin_amdgcn_bitop3_b32(m, a, b, 0xCA); }
__device__ __forceinline__ u32 and_or(u32 a, u32 m, u32 c) { return __builtin_amdgcn_bitop3_b32(a, m, c, 0xEA); }
#define BAR() asm volatile("s_waitcnt lgkmcnt(0)\n\ts_barrier" ::: "memory")
__device__ __forceinline__ u32 scan64(u32 v) {
    v += __builtin_amdgcn_update_dpp(0u, v, 0x111, 0xf, 0xf, true);
    v += __builtin_amdgcn_update_dpp(0u, v, 0x112, 0xf, 0xf, true);
    v += __builtin_amdgcn_update_dpp(0u, v, 0x114, 0xf, 0xf, true);
    v += __builtin_amdgcn_update_dpp(0u, v, 0x118, 0xf, 0xf, true);
    v += __builtin_amdgcn_update_dpp(0u, v, 0x142, 0xa, 0xf, false);
    v += __builtin_amdgcn_update_dpp(0u, v, 0x143, 0xc, 0xf, false);
    return v;
}
__device__ __forceinline__ void lds_or(LAS unsigned char* p, u32 v) {
    __hip_atomic_fetch_or((LAS u32*)p, v, __ATOMIC_RELAXED, __HIP_MEMORY_SCOPE_WORKGROUP);
}

__device__ __forceinline__ void load_w(u32 (&d)[16], const unsigned char* w, int panel, int r, int lane) {
    const auto rs = __builtin_amdgcn_make_buffer_rsrc((void*)w, 0, MN * KS, 0x00020000);
    const u32 so = r * 16 * MN + (panel & 15) * 256;
#pragma unroll
    for (int k = 0; k < 16; ++k) d[k] = __builtin_amdgcn_raw_buffer_load_b32(rs, lane * 4, so + k * MN, 0);
}
__device__ __forceinline__ void order_w(u32 (&x)[16], const u32 (&d)[16]) {
#pragma unroll
    for (int q = 0; q < 4; ++q) {
        u32 t0 = perm(d[4 * q + 1], d[4 * q], 0x05010400), t1 = perm(d[4 * q + 1], d[4 * q], 0x07030602);
        u32 t2 = perm(d[4 * q + 3], d[4 * q + 2], 0x05010400), t3 = perm(d[4 * q + 3], d[4 * q + 2], 0x07030602);
        x[q] = perm(t2, t0, 0x05040100); x[4 + q] = perm(t2, t0, 0x07060302);
        x[8 + q] = perm(t3, t1, 0x05040100); x[12 + q] = perm(t3, t1, 0x07060302);
    }
}
__device__ __forceinline__ u32x4 piece(const u32 (&x)[16], int i) { return u32x4{x[4 * i], x[4 * i + 1], x[4 * i + 2], x[4 * i + 3]}; }

template <int MB> struct Fmt {
    static constexpr u32 EM = (MB == 3 ? 0x0Fu : 0x1Fu) * 0x01010101u;
    static constexpr u32 HEAD = MB == 3 ? 3072 : 2048;
    static constexpr u32 OFF_H = MB == 3 ? 0 : 512, OFF_2 = OFF_H + 4;
};
__device__ __forceinline__ u32 zero2(u32 w) { return __builtin_amdgcn_bitop3_b32(w, w >> 1, 0x55555555u, 0x02); }
__device__ __forceinline__ u32 code_of(u32 hi, u32 lo, u32 v) {
    const u32 r = perm(hi, lo, v);
    return __builtin_amdgcn_bitop3_b32(r, r >> 2, 0x03030303u, 0x28);
}
__device__ __forceinline__ void lookup4(u32* sel, u32 w, LAS unsigned char* tab) {
#pragma unroll
    for (int j = 0; j < 4; ++j) sel[j] = *(LAS u32*)(tab + ((w >> (8 * j)) & 255u) * 4);
}
__device__ __forceinline__ void gather4(const u32* v, const u32* sel, u32 z, u32 a) {
#pragma unroll
    for (int j = 0; j < 4; ++j) {
        const u32 o = j ? a + __builtin_popcount(z & ((1u << (8 * j)) - 1)) : a;
        const u32 c = perm(0u, v[j], sel[j]);
        LAS unsigned char* p = (LAS unsigned char*)(size_t)(o & ~3u);
        lds_or(p, __builtin_amdgcn_alignbyte(c, 0u, ~o));
        lds_or(p + 4, __builtin_amdgcn_alignbyte(0u, c, ~o));
    }
}
__device__ __forceinline__ void window4(u32* o, u32x2* q, u32 z, u32 a) {
#pragma unroll
    for (int j = 0; j < 4; ++j) {
        o[j] = j ? a + __builtin_popcount(z & ((1u << (8 * j)) - 1)) : a;
        LAS u32* p = (LAS u32*)(size_t)(o[j] & ~3u);
        q[j] = u32x2{p[0], p[1]};
    }
}
__device__ __forceinline__ u32 spread(u32 tab, u32 o, u32x2 q, u32 sel) {
    return perm(tab, __builtin_amdgcn_alignbyte(q[1], q[0], o), sel);
}
__device__ __forceinline__ u32 pack4(const u32* c) {
    u32 w = __builtin_amdgcn_udot4(c[0], 0x40100401u, 0u, false);
#pragma unroll
    for (int j = 1; j < 4; ++j) w |= __builtin_amdgcn_udot4(c[j], 0x40100401u, 0u, false) << (8 * j);
    return w;
}
__device__ __forceinline__ void encode_clear(LAS unsigned char* T, int lane) {
#pragma unroll
    for (int i = 2; i < 5; ++i) *(LAS u32x4*)(T + i * 1024 + lane * 16) = u32x4{0, 0, 0, 0};
}

template <int MB> __device__ __forceinline__ u32 encode_unit(const u32 (&x)[16], LAS unsigned char* T, LAS unsigned char* L,
                                                             char* dst, int lane, bool raw) {
    typedef Fmt<MB> F;
    u32 sum = 0;
#pragma unroll
    for (int j = 0; j < 16; j += 5) sum = __builtin_amdgcn_udot4(x[j] & 0x7F7F7F7Fu, 0x01010101u, sum, false);
    const int tot = __builtin_amdgcn_readlane(scan64(sum), 63);
    const u32 eb = (u32)((tot - (1516 << MB)) >> (10 + MB)) & (F::EM & 31u);
    const u32 B = (eb << MB) * 0x01010101u;
    u32 v[16], c[16];
    u32x4 w, z;
#pragma unroll
    for (int j = 0; j < 16; ++j) {
        v[j] = (((x[j] | 0x80808080u) - B) >> MB) & F::EM;
        c[j] = code_of(0u, 0x03020100u, v[j]);
    }
    u32 cnt[5] = {0};
#pragma unroll
    for (int q = 0; q < 4; ++q) {
        w[q] = pack4(c + 4 * q); z[q] = zero2(w[q]); cnt[q + 1] = cnt[q] + __builtin_popcount(z[q]);
        lookup4(c + 4 * q, w[q], L + LDS_CL);
    }
    FENCE();
    const u32 incl = scan64(cnt[4]), n1 = __builtin_amdgcn_readlane(incl, 63);
    u32 n2 = 0, n3 = 0, len64 = 64;
    raw = raw || n1 > 2040;
    const u32 off3 = F::OFF_2 + ((n1 + 15) >> 4) * 4;
    if (!raw) {
        const u32 a1 = incl - cnt[4] + ((u32)(size_t)T + 2047);
#pragma unroll
        for (int q = 0; q < 4; ++q) gather4(v + 4 * q, c + 4 * q, z[q], a1 + cnt[q]);
        st_sys_s(lane * 16, dst, w);
        if (MB == 3) {
            u32x4 o[2];
#pragma unroll
            for (int p = 0; p < 8; ++p) o[p >> 2][p & 3] = bfi(0x87878787u, x[2 * p], bfi(0xF0F0F0F0u, x[2 * p + 1] << 4, x[2 * p + 1] >> 4));
            st_sys_s(lane * 16, dst + 1024, o[0]);
            st_sys_s(lane * 16, dst + 2048, o[1]);
        } else {
            u32x4 o;
            u32x2 sg;
#pragma unroll
            for (int q = 0; q < 4; ++q)
                o[q] = bfi(0x0F0F0F0Fu, bfi(0x03030303u, x[4 * q], x[4 * q + 1] << 2), bfi(0x03030303u, x[4 * q + 2], x[4 * q + 3] << 2) << 4);
#pragma unroll
            for (int k = 0; k < 2; ++k) {
                u32 t = x[8 * k];
#pragma unroll
                for (int j = 1; j < 8; ++j) t = bfi(0x80808080u >> j, x[8 * k + j] >> j, t);
                sg[k] = t;
            }
            *(LAS u32x2*)(T + lane * 8) = sg;
            st_sys_s(lane * 16, dst + 1024, o);
        }
        FENCE();
        for (u32 b = 0; b < n1; b += 1024) {
            const u32x4 y = *(LAS u32x4*)(T + 2048 + b + lane * 16);
            u32 u[4], c2[4];
#pragma unroll
            for (int j = 0; j < 4; ++j) {
                c2[j] = code_of(0x00000300u, 0x00000102u, (y[j] + 0x01010101u) & F::EM);
                u[j] = (y[j] + 0x05050505u) & F::EM;
            }
            const u32 w2 = pack4(c2);
            lookup4(c2, w2, L + LDS_CL);
            FENCE();
            const u32 z2 = zero2(w2), k2 = __builtin_popcount(z2), i2 = scan64(k2);
            if (b + lane * 16 < n1) *(LAS u32*)(T + F::OFF_2 + (b >> 2) + lane * 4) = w2;
            const u32 all = n2 + __builtin_amdgcn_readlane(i2, 63);
            if (all <= 1016 && k2) gather4(u, c2, z2, n2 + i2 - k2 + ((u32)(size_t)T + 4095));
            n2 = all;
        }
        raw = n2 > 1016;
    }
    if (!raw) {
        FENCE();
        const u32x4 y = *(LAS u32x4*)(T + 4096 + lane * 16);
        u32 k3 = 0, m[4];
#pragma unroll
        for (int j = 0; j < 4; ++j) { m[j] = ((y[j] & 0x1C1C1C1Cu) + 0x1C1C1C1Cu) & 0x20202020u; k3 += __builtin_popcount(m[j]); }
        u32 i3 = 0;
        if (__builtin_amdgcn_ballot_w64(k3 != 0)) { i3 = scan64(k3); n3 = __builtin_amdgcn_readlane(i3, 63); }
        const u32 offp = off3 + ((n2 + 15) >> 4) * 4;
        len64 = (F::HEAD + offp + 2 * n3 + 63) >> 6;
        raw = len64 >= 64 || n3 > 63;
        if (!raw) {
            if (lane * 16 < n2)
                *(LAS u32*)(T + off3 + lane * 4) = bfi(0x0F0F0F0Fu, bfi(0x03030303u, y[0], y[1] << 2), bfi(0x03030303u, y[2], y[3] << 2) << 4);
            if (lane == 0) *(LAS u32*)(T + F::OFF_H) = eb | (n1 << 5) | (n2 << 16) | (n3 << 26);
            if (n3) {
                LAS unsigned char* p = T + offp + 2 * (i3 - k3);
                u32 mm = (m[0] >> 5) | (m[1] >> 4) | (m[2] >> 3) | (m[3] >> 2);
                while (mm) {
                    const u32 b = __builtin_ctz(mm), i = lane * 16 + (b & 3u) * 4 + (b >> 3);
                    mm &= mm - 1;
                    *(LAS u16*)p = (u16)((i << 5) | T[4096 + i]);
                    p += 2;
                }
            }
        }
    }
    if (raw) {
        WAIT_VM(0);
#pragma unroll
        for (int i = 0; i < 4; ++i) st_sys_s(lane * 16, dst + i * 1024, piece(x, i));
        return 64;
    }
    FENCE();
    const u32 tl = len64 * 64 - F::HEAD;
    if (lane * 16 < tl) st_sys_s(lane * 16, dst + F::HEAD, *(LAS u32x4*)(T + lane * 16));
    if (MB == 2 && 1024 + lane * 16 < tl) st_sys_s(lane * 16, dst + F::HEAD + 1024, *(LAS u32x4*)(T + 1024 + lane * 16));
    return len64;
}

template <int MB> __device__ __forceinline__ void decode_unit(const char* qa, const char* qb, u32 hdr, u32x4 w, u32 w2,
                                                              LAS unsigned char* S, LAS unsigned char* L, int lane) {
    typedef Fmt<MB> F;
    LAS unsigned char* T = S + F::HEAD;
    const u32 eb = hdr & 31u, n1 = (hdr >> 5) & 2047u, n2 = (hdr >> 16) & 1023u, n3 = hdr >> 26;
    const u32 off3 = F::OFF_2 + ((n1 + 15) >> 4) * 4, offp = off3 + ((n2 + 15) >> 4) * 4;
    const u32 w3 = *(LAS u32*)(T + off3 + lane * 4);
    const u32 en = *(LAS u16*)(T + offp + lane * 2);
    u32 s1[16], s2[4];
    lookup4(s2, w2, L + LDS_DL);
    const u32x4 o0 = *(LAS u32x4*)(S + 1024 + lane * 16);
    u32x4 o1;
    if (MB == 3) o1 = *(LAS u32x4*)(S + 2048 + lane * 16);
    else { const u32x2 sg = *(LAS u32x2*)(T + lane * 8); o1 = u32x4{sg[0], sg[1], 0, 0}; }
    FENCE(); SB();
    const u32 eb4 = eb * 0x01010101u;
    const u32 t1 = ((eb4 + 0x03020100u) & F::EM) << MB, t2 = ((eb4 + 0x041F0000u) & F::EM) << MB, t3 = ((eb4 + 0x1E1D1C1Bu) & F::EM) << MB;
    u32x4 z;
    u32 cnt[5] = {0};
#pragma unroll
    for (int q = 0; q < 4; ++q) { z[q] = zero2(w[q]); cnt[q + 1] = cnt[q] + __builtin_popcount(z[q]); }
    u32 z2 = zero2(w2), k2 = __builtin_popcount(z2);
    const u32 both = cnt[4] | (k2 << 16), incl = scan64(both);
    const u32 excl = incl - both + ((u32)(size_t)S - 2048) + (((u32)(size_t)S - 3072) << 16);
    SB();
    {
        u32x4 y;
#pragma unroll
        for (int j = 0; j < 4; ++j) y[j] = perm(0u, t3, (w3 >> (2 * j)) & 0x03030303u);
        *(LAS u32x4*)(S - 3072 + lane * 16) = y;
        FENCE();
        if ((u32)lane < n3) *(S - 3072 + ((en >> 5) & 1023u)) = (unsigned char)(((en + eb + 27u) & (F::EM & 31u)) << MB);
        FENCE();
    }
    u32 o[16];
    u32x2 q[16];
    window4(o, q, z2, excl >> 16);
    SB();
#pragma unroll
    for (int i = 0; i < 4; ++i) lookup4(s1 + 4 * i, w[i], L + LDS_DL);
    FENCE(); SB();
    *(LAS u32x4*)(S - 2048 + lane * 16) =
        u32x4{spread(t2, o[0], q[0], s2[0]), spread(t2, o[1], q[1], s2[1]), spread(t2, o[2], q[2], s2[2]), spread(t2, o[3], q[3], s2[3])};
    if (n1 > 1024) {
        w2 = *(LAS u32*)(T + F::OFF_2 + 256 + lane * 4);
        lookup4(s2, w2, L + LDS_DL);
        z2 = zero2(w2); k2 = __builtin_popcount(z2);
        window4(o, q, z2, (__builtin_amdgcn_readlane(incl, 63) >> 16) + (u32)(size_t)S - 3072 + scan64(k2) - k2);
        *(LAS u32x4*)(S - 2048 + 1024 + lane * 16) =
            u32x4{spread(t2, o[0], q[0], s2[0]), spread(t2, o[1], q[1], s2[1]), spread(t2, o[2], q[2], s2[2]), spread(t2, o[3], q[3], s2[3])};
    }
    FENCE(); SB();
#pragma unroll
    for (int i = 0; i < 4; ++i) window4(o + 4 * i, q + 4 * i, z[i], (excl & 0xFFFFu) + cnt[i]);
    SB();
    u32x4 e[4];
#pragma unroll
    for (int j = 0; j < 16; ++j) {
        const u32 ex = spread(t1, o[j], q[j], s1[j]);
        if (MB == 3) {
            const u32 v = j < 8 ? o0[(j >> 1) & 3] : o1[(j >> 1) & 3];
            e[j >> 2][j & 3] = (j & 1) ? and_or(v >> 4, 0x07070707u, and_or(v << 4, 0x80808080u, ex)) : and_or(v, 0x87878787u, ex);
        } else {
            e[j >> 2][j & 3] = and_or(o0[j >> 2] >> (2 * (j & 3)), 0x03030303u, and_or(o1[j >> 3] << (j & 7), 0x80808080u, ex));
        }
        if ((j & 3) == 3) st_sys_s(lane * 16, (j < 8 ? qa : qb) + (j & 4) * 256, e[j >> 2]);
    }
}

#define V10(a) "v" #a "0", "v" #a "1", "v" #a "2", "v" #a "3", "v" #a "4", "v" #a "5", "v" #a "6", "v" #a "7", "v" #a "8", "v" #a "9"
__device__ __forceinline__ void mfma_role(u32 sa, u32 sb, u32 sc, u32 olo, u32 ohi, u32 scale) {
    u32 need, m, spin, scl;
    asm volatile(
        ".macro AG_FRAG d, a, o\n\t"
        "ds_read_b128 v[\\d:\\d+3], \\a offset:\\o\n\t"
        "ds_read_b128 v[\\d+4:\\d+7], \\a offset:\\o+4096\n\t"
        ".endm\n\t"
        ".macro AG_MM c, a, b\n\t"
        "v_mfma_f32_32x32x64_f8f6f4 v[\\c:\\c+15], v[\\a:\\a+7], v[\\b:\\b+7], v[\\c:\\c+15] cbsz:0 blgp:1\n\t"
        ".endm\n\t"
        ".macro AG_MIN\n\t"
        "v_min_u32 v164, v164, v165\n\t"
        "s_nop 0\n\t"
        "v_readfirstlane_b32 %[m], v164\n\t"
        ".endm\n\t"
        ".macro AG_SPIN\n\t"
        "s_mov_b32 %[spin], 0\n\t"
        ".LAG_S\\@:\n\t"
        "ds_read_b64 v[164:165], v162\n\t"
        "s_add_u32 %[spin], %[spin], 1\n\t"
        "s_waitcnt lgkmcnt(0)\n\t"
        "AG_MIN\n\t"
        "s_cmp_ge_u32 %[m], %[need]\n\t"
        "s_cbranch_scc1 .LAG_E\\@\n\t"
        "s_sleep 1\n\t"
        "s_cmp_lt_u32 %[spin], 0x2000000\n\t"
        "s_cbranch_scc1 .LAG_S\\@\n\t"
        "s_mov_b32 %[scale], 0x7fc00000\n\t"
        ".LAG_E\\@:\n\t"
        ".endm\n\t"
        ".macro AG_DONE\n\t"
        "v_add_u32 v163, 1, v163\n\t"
        "ds_write_b32 v162, v163 offset:8\n\t"
        ".endm\n\t"
        ".macro AG_FULL so, sn\n\t"
        "s_waitcnt lgkmcnt(4)\n\t"
        "AG_MM 0, 128, 144\n\t"
        "s_waitcnt lgkmcnt(2)\n\t"
        "AG_MM 64, 136, 144\n\t"
        "ds_read_b64 v[164:165], v162\n\t"
        "AG_FRAG 144, v161, \\so+2048\n\t"
        "s_add_u32 %[need], %[need], 1\n\t"
        "s_waitcnt lgkmcnt(3)\n\t"
        "AG_MM 16, 128, 152\n\t"
        "AG_MM 80, 136, 152\n\t"
        "AG_FRAG 152, v161, \\so+3072\n\t"
        "s_waitcnt lgkmcnt(2)\n\t"
        "AG_MM 32, 128, 144\n\t"
        "AG_MIN\n\t"
        "AG_MM 96, 136, 144\n\t"
        "s_cmp_ge_u32 %[m], %[need]\n\t"
        "s_cbranch_scc1 .LAG_F\\@\n\t"
        "s_waitcnt lgkmcnt(0)\n\t"
        "AG_MM 48, 128, 152\n\t"
        "AG_DONE\n\t"
        "AG_MM 112, 136, 152\n\t"
        "AG_SPIN\n\t"
        "AG_FRAG 144, v161, \\sn\n\t"
        "AG_FRAG 128, v160, \\sn\n\t"
        "s_branch .LAG_J\\@\n\t"
        ".LAG_F\\@:\n\t"
        "AG_FRAG 144, v161, \\sn\n\t"
        "s_waitcnt lgkmcnt(2)\n\t"
        "AG_MM 48, 128, 152\n\t"
        "AG_DONE\n\t"
        "AG_FRAG 128, v160, \\sn\n\t"
        "AG_MM 112, 136, 152\n\t"
        ".LAG_J\\@:\n\t"
        "AG_FRAG 136, v160, \\sn+2048\n\t"
        "AG_FRAG 152, v161, \\sn+1024\n\t"
        ".endm\n\t"
        ".macro AG_PREP c, b\n\t"
        ".irp j, 0, 16, 32, 48\n\t"
        ".irp i, 0, 1, 2, 3\n\t"
        "v_mul_f32 v[\\c+\\j+4*\\b+\\i], %[scl], v[\\c+\\j+4*\\b+\\i]\n\t"
        ".endr\n\t"
        ".endr\n\t"
        "v_swap_b32 v[\\c+4*\\b+1], v[\\c+16+4*\\b]\n\t"
        "v_swap_b32 v[\\c+4*\\b+2], v[\\c+32+4*\\b]\n\t"
        "v_swap_b32 v[\\c+4*\\b+3], v[\\c+48+4*\\b]\n\t"
        "v_swap_b32 v[\\c+16+4*\\b+2], v[\\c+32+4*\\b+1]\n\t"
        "v_swap_b32 v[\\c+16+4*\\b+3], v[\\c+48+4*\\b+1]\n\t"
        "v_swap_b32 v[\\c+32+4*\\b+3], v[\\c+48+4*\\b+2]\n\t"
        ".endm\n\t"
        ".macro AG_ST c, r\n\t"
        ".if ((\\r) & 3) == 0\n\t"
        "AG_PREP \\c, (\\r)/4\n\t"
        ".endif\n\t"
        ".if \\c\n\t"
        "global_store_dwordx4 v166, v[\\c+16*((\\r)&3)+((\\r)&12):\\c+16*((\\r)&3)+((\\r)&12)+3], vcc\n\t"
        ".else\n\t"
        "global_store_dwordx4 v166, v[\\c+16*((\\r)&3)+((\\r)&12):\\c+16*((\\r)&3)+((\\r)&12)+3], vcc nt\n\t"
        ".endif\n\t"
        ".if ((\\r) & 3) == 3\n\t"
        "s_add_u32 vcc_lo, vcc_lo, 5*16384\n\t"
        ".else\n\t"
        "s_add_u32 vcc_lo, vcc_lo, 16384\n\t"
        ".endif\n\t"
        "s_addc_u32 vcc_hi, vcc_hi, 0\n\t"
        ".endm\n\t"
        ".macro AG_STN lo, hi\n\t"
        ".set AG_K, \\lo\n\t"
        ".rept (\\hi)-(\\lo)\n\t"
        "AG_ST 0, AG_K\n\t"
        ".set AG_K, AG_K+1\n\t"
        ".endr\n\t"
        ".endm\n\t"
        ".macro AG_XF d, q\n\t"
        ".if (((\\q)+(\\q)/" STR(NSPLIT) ") %% 5) == 4\n\t"
        "AG_FRAG \\d, v160, 2*16384+2048\n\t"
        ".else\n\t"
        "AG_FRAG \\d, v160, (((\\q)+(\\q)/" STR(NSPLIT) ") %% 5)*16384\n\t"
        ".endif\n\t"
        ".endm\n\t"
        ".macro AG_WF d, q, i\n\t"
        ".if (((\\q)+(\\q)/" STR(NSPLIT) ") %% 5) == 4\n\t"
        "AG_FRAG \\d, v167, (\\i/2)*16384+(\\i&1)*1024\n\t"
        ".else\n\t"
        "AG_FRAG \\d, v161, (((\\q)+(\\q)/" STR(NSPLIT) ") %% 5)*16384+\\i*1024\n\t"
        ".endif\n\t"
        ".endm\n\t"
        ".macro AG_HALF c, q, last, r0, r1, r2\n\t"
        ".set AG_B0, 136+8*(((\\q)+1) %% 3)\n\t"
        ".set AG_B1, 136+8*(((\\q)+2) %% 3)\n\t"
        ".set AG_B2, 136+8*((\\q) %% 3)\n\t"
        "s_waitcnt lgkmcnt(2)\n\t"
        "AG_MM \\c, 128, AG_B0\n\t"
        "AG_WF AG_B0, \\q, 3\n\t"
        "s_add_u32 %[need], %[need], 1\n\t"
        "AG_MIN\n\t"
        "AG_MM \\c+16, 128, AG_B1\n\t"
        ".if \\last == 0\n\t"
        "s_cmp_ge_u32 %[m], %[need]\n\t"
        "s_cbranch_scc1 .LAG_F\\@\n\t"
        ".endif\n\t"
        "AG_STN \\r0, \\r1\n\t"
        "s_waitcnt lgkmcnt(2)\n\t"
        "AG_MM \\c+32, 128, AG_B2\n\t"
        "s_waitcnt lgkmcnt(0)\n\t"
        "AG_MM \\c+48, 128, AG_B0\n\t"
        ".if \\last == 0\n\t"
        "AG_DONE\n\t"
        "AG_SPIN\n\t"
        "AG_WF AG_B1, (\\q)+1, 0\n\t"
        "AG_WF AG_B2, (\\q)+1, 1\n\t"
        "s_branch .LAG_J\\@\n\t"
        ".LAG_F\\@:\n\t"
        "AG_WF AG_B1, (\\q)+1, 0\n\t"
        "AG_STN \\r0, \\r1\n\t"
        "s_waitcnt lgkmcnt(4)\n\t"
        "AG_MM \\c+32, 128, AG_B2\n\t"
        "AG_WF AG_B2, (\\q)+1, 1\n\t"
        "s_waitcnt lgkmcnt(4)\n\t"
        "AG_MM \\c+48, 128, AG_B0\n\t"
        "AG_DONE\n\t"
        ".LAG_J\\@:\n\t"
        "AG_XF 128, (\\q)+1\n\t"
        "ds_read_b64 v[164:165], v162\n\t"
        "AG_WF AG_B0, (\\q)+1, 2\n\t"
        ".endif\n\t"
        "AG_STN \\r1, \\r2\n\t"
        ".endm\n\t"
        ".macro AG_SCALE\n\t"
        "ds_read_b32 v164, v162 offset:12\n\t"
        "s_nop 15\n\t"
        "s_nop 7\n\t"
        "s_waitcnt lgkmcnt(0)\n\t"
        "v_readfirstlane_b32 %[m], v164\n\t"
        "s_cmp_eq_u32 %[m], 0\n\t"
        "s_cselect_b32 %[scl], %[scale], 0x7fc00000\n\t"
        ".endm\n\t"

        "s_waitcnt vmcnt(0) lgkmcnt(0)\n\t"
        "v_mbcnt_lo_u32_b32 v167, -1, 0\n\t"
        "v_mbcnt_hi_u32_b32 v167, -1, v167\n\t"
        "v_lshrrev_b32 v165, 5, v167\n\t"
        "v_and_b32 v167, 31, v167\n\t"
        "v_lshlrev_b32 v167, 4, v167\n\t"
        "v_lshl_add_u32 v166, v165, 16, v167\n\t"
        "v_lshl_add_u32 v167, v165, 13, v167\n\t"
        "v_add_u32 v160, %[sa], v167\n\t"
        "v_add_u32 v161, %[sb], v167\n\t"
        "v_subrev_u32 v167, 65536-2048, v161\n\t"
        "v_mov_b32 v162, %[sc]\n\t"
        "v_mov_b32 v163, 0\n\t"
        ".set AG_I, 0\n\t"
        ".rept 128\n\t"
        "v_mov_b32 v[AG_I], 0\n\t"
        ".set AG_I, AG_I+1\n\t"
        ".endr\n\t"
        "s_mov_b32 vcc_lo, %[olo]\n\t"
        "s_mov_b32 vcc_hi, %[ohi]\n\t"
        "s_mov_b32 %[need], 1\n\t"
        "AG_SPIN\n\t"
        "AG_FRAG 144, v161, 0\n\t"
        "AG_FRAG 128, v160, 0\n\t"
        "AG_FRAG 136, v160, 2048\n\t"
        "AG_FRAG 152, v161, 1024\n\t"
        ".LAG_LOOP:\n\t"
        "AG_FULL 0, 16384\n\t"
        "AG_FULL 16384, 32768\n\t"
        "AG_FULL 32768, 49152\n\t"
        "AG_FULL 49152, 0\n\t"
        "s_cmp_le_u32 %[need], 64-" STR(NSPLIT) "\n\t"
        "s_cbranch_scc1 .LAG_LOOP\n\t"
        "ds_read_b64 v[164:165], v162\n\t"
        "AG_WF 136, 0, 2\n\t"
        ".set AG_I, 0\n\t"
        ".rept " STR(NSPLIT) "\n\t"
        "AG_HALF 0, AG_I, (AG_I==" STR(NSPLIT) "-1), 0, 0, 0\n\t"
        ".set AG_I, AG_I+1\n\t"
        ".endr\n\t"
        "AG_DONE\n\t"
        "AG_SCALE\n\t"
        "AG_STN 0, " STR(NS0) "\n\t"
        "AG_SPIN\n\t"
        "AG_WF AG_B1, AG_I, 0\n\t"
        "AG_WF AG_B2, AG_I, 1\n\t"
        "AG_XF 128, AG_I\n\t"
        "ds_read_b64 v[164:165], v162\n\t"
        "AG_WF AG_B0, AG_I, 2\n\t"
        ".set AG_I, 0\n\t"
        ".rept " STR(NSPLIT) "\n\t"
        ".set AG_R0, " STR(NS0) "+(AG_I*(16-" STR(NS0) ")+" STR(NST) "-1)/" STR(NST) "\n\t"
        ".set AG_R2, " STR(NS0) "+((AG_I+1)*(16-" STR(NS0) ")+" STR(NST) "-1)/" STR(NST) "\n\t"
        ".if AG_I >= " STR(NST) "\n\t"
        ".set AG_R0, 16\n\t"
        ".set AG_R2, 16\n\t"
        ".endif\n\t"
        "AG_HALF 64, AG_I+" STR(NSPLIT) ", (AG_I==" STR(NSPLIT) "-1), AG_R0, (AG_R0+AG_R2+1)/2, AG_R2\n\t"
        ".set AG_I, AG_I+1\n\t"
        ".endr\n\t"
        "AG_SCALE\n\t"
        "s_add_u32 vcc_lo, vcc_lo, 96*16384\n\t"
        "s_addc_u32 vcc_hi, vcc_hi, 0\n\t"
        ".irp r, 0, 1, 2, 3, 4, 5, 6, 7, 8, 9, 10, 11, 12, 13, 14, 15\n\t"
        "AG_ST 64, \\r\n\t"
        ".endr\n\t"
        "s_nop 1\n\t"
        : [scale] "+s"(scale), [need] "=&s"(need), [m] "=&s"(m), [spin] "=&s"(spin), [scl] "=&s"(scl)
        : [sa] "s"(sa), [sb] "s"(sb), [sc] "s"(sc), [olo] "s"(olo), [ohi] "s"(ohi)
        : "memory", "scc", "vcc", V10(), V10(1), V10(2), V10(3), V10(4), V10(5), V10(6), V10(7), V10(8), V10(9),
          V10(10), V10(11), V10(12), V10(13), V10(14), V10(15),
          "v160", "v161", "v162", "v163", "v164", "v165", "v166", "v167");
}

__device__ __forceinline__ char* peer_of_lane(const Params& P) {
    u32 lo = 0, hi = 0;
#pragma unroll
    for (int p = 0; p < NR; ++p) {
        asm volatile("v_writelane_b32 %0, %2, %4\n\tv_writelane_b32 %1, %3, %4"
                     : "+v"(lo), "+v"(hi) : "s"((u32)(u64)P.ws[p]), "s"((u32)((u64)P.ws[p] >> 32)), "i"(p));
    }
    return (char*)(((u64)hi << 32) | lo);
}
__device__ __forceinline__ void arrive(LAS unsigned char* L, int lane) {
    if (lane == 0) __hip_atomic_fetch_add((LAS u32*)(L + LDS_PRO), 1u, __ATOMIC_RELAXED, __HIP_MEMORY_SCOPE_WORKGROUP);
}

template <bool W> __device__ __forceinline__ void prologue_unit(const Params& P, LAS unsigned char* L, char* myws, u32 callv,
                                                                 const u32 (&d)[16], int idx, int xg, int wave, int lane) {
    u32 x[16];
    __builtin_amdgcn_s_setprio(3);
    const int u = xg * 4 + wave;
    const bool half = !W && u >= (int)R0;
    char* peer = peer_of_lane(P);
    WAIT_VM0();
    if (W) {
        order_w(x, d);
    } else {
        LAS u32* xc = (LAS u32*)(L + LDS_XCH);
        if (lane == 0) __hip_atomic_fetch_add(xc, 1u, __ATOMIC_RELAXED, __HIP_MEMORY_SCOPE_WORKGROUP);
        for (u32 spin = 0; spin < SPINS && UNI(*(volatile LAS u32*)xc) < 4; ++spin) {}
        FENCE();
        LAS unsigned char* b = L + LDS_A + 32768 + lane * 64 + (half ? (wave & 1) * 8192 + (wave & 2) * 16 : wave * 16);
        const u32 hi = half ? 16 : 8192;
#pragma unroll
        for (int i = 0; i < 4; ++i) {
            const u32x4 v = *(LAS u32x4*)(b + (i & 1) * 4096 + (i >> 1) * hi);
            x[4 * i] = v[0]; x[4 * i + 1] = v[1]; x[4 * i + 2] = v[2]; x[4 * i + 3] = v[3];
        }
    }
    const u32 par = (UNI(callv) + 1) & 1;
    const bool raw = u >= (int)ROUNDS - (W ? 1 : 3);
    u32 len64 = 64;
    if (!W && u == 0) {
        char* dst = myws + WS_CB + par * CB_BYTES + cb_off(idx, u);
#pragma unroll
        for (int i = 0; i < 4; ++i) st_sys_s(lane * 16, dst + i * 1024, piece(x, i));
    } else if (!W || u >= NPW) {
        len64 = encode_unit<W ? 2 : 3>(x, L + LDS_A + wave * 8192, L, myws + WS_CB + par * CB_BYTES + cb_off(idx, u), lane, raw);
    }
    if (lane == 0) *(volatile LAS u32*)(L + LDS_LEN + wave * 4) = len64;
    const char* qa = myws + WS_IMG + par * IMG_BYTES + unit_off(idx, half ? u & ~1 : u) + (half ? (u & 1) * 2048 : 0);
    const char* qb = half ? myws + WS_IMG + par * IMG_BYTES + unit_off(idx, u | 1) + (u & 1) * 2048 : qa + 2048;
    st_sys_s(lane * 16, qa, piece(x, 0)); st_sys_s(lane * 16, qa + 1024, piece(x, 1));
    st_sys_s(lane * 16, qb, piece(x, 2)); st_sys_s(lane * 16, qb + 1024, piece(x, 3));
    WAIT_VM(4);
    if (xg == 0 && (!W || u >= NPW) && lane < NR && lane != P.rank)
        asm volatile("global_store_dword %0, %1, off sc0 sc1" :: "v"(peer + WS_RDY + par * 32768 + (P.rank * 32 + idx) * 128 + u * 4),
                     "v"(((UNI(callv) + 1) << 7) | len64) : "memory");
    if (xg == 0 && (!W || u >= NPW)) WAIT_VM(1); else WAIT_VM(0);
    arrive(L, lane);
    __builtin_amdgcn_s_setprio(0);
}

__global__ void __launch_bounds__(NTHREADS, 3) ag_gemm(Params P) {
    __shared__ __attribute__((aligned(16))) unsigned char lds_raw[LDS_BYTES];
    LAS unsigned char* L = (LAS unsigned char*)lds_raw;
    const int hw = UNI(threadIdx.x >> 6);
    const int wave = hw < 4 ? 11 - hw : hw - 4;
    const int lane = threadIdx.x & 63;
    const int bid = blockIdx.x;
    const unsigned char* px = P.x;
    const unsigned char* pw_ = P.w;
    char* myws = P.myws;
    int rank = P.rank;
    u32 srcmap = P.srcmap;
    asm volatile("" : "+s"(px), "+s"(pw_), "+s"(myws), "+s"(rank), "+s"(srcmap) : "s"(P.ws[0]));
    const int xg = bid & 7, idx = bid >> 3;
    const int rg = (xg + 5) & 7;
    const int tm = (xg >> 1) * 4 + (idx >> 3), tn = (xg & 1) * 8 + (idx & 7);
    GAS u32* tmo = (GAS u32*)(myws + WS_TMO);

    u32 xo = (lane >> 2) * KS + (lane & 3) * 16;
    u32 d[16];
    if (wave < 4 && idx < 16) {
        const unsigned char* g = px + (size_t)(idx * 256 + wave * 64) * KS + rg * 64;
        u32 keep;
        asm volatile("s_mov_b32 %1, m0\n\ts_mov_b32 m0, %3\n\ts_nop 0\n\tglobal_load_lds_dwordx4 %0, %2\n\t"
                     ".rept 3\n\ts_add_u32 m0, m0, 0x400\n\tv_add_u32 %0, 16*512, %0\n\tglobal_load_lds_dwordx4 %0, %2\n\t.endr\n\t"
                     "s_mov_b32 m0, %1"
                     : "+v"(xo), "=&s"(keep) : "s"(g), "s"((u32)(size_t)L + LDS_A + 32768 + wave * 4096) : "memory", "scc");
    }
    FENCE();
    const int pr = (idx >> 4) + 2 * (11 - wave);
    if (wave < 4 ? idx >= 16 : wave >= 10 && pr < NPW) load_w(d, pw_, idx, wave < 4 ? rg * 4 + wave : pr, lane);
    u32 call = __hip_atomic_load((GAS u32*)(myws + WS_CNT + bid * 128), __ATOMIC_RELAXED, __HIP_MEMORY_SCOPE_AGENT);
    if (wave < 4) {
        encode_clear(L + LDS_A + wave * 8192, lane);
    } else if (wave < 8) {
        const u32 f = (wave - 4) * 64 + lane;
        u32 dl = 0, cl = 0x0C0C0C0Cu, k = 0;
#pragma unroll
        for (u32 i = 0; i < 4; ++i) {
            const u32 c = (f >> (2 * i)) & 3u;
            dl |= (c ? 4 + c : k) << (8 * i);
            cl ^= c ? 0u : (12u ^ i) << (8 * k);
            k += c == 0;
        }
        *(LAS u32*)(L + LDS_DL + f * 4) = dl;
        *(LAS u32*)(L + LDS_CL + f * 4) = cl;
    } else if (wave == 8) {
        ((LAS u32*)(L + LDS_CTL))[lane] = 0;
    }
    BAR();

    if (wave < 8) {
        if (wave < 4) {
            if (idx < 16) prologue_unit<false>(P, L, myws, call, d, idx, rg, wave, lane);
            else prologue_unit<true>(P, L, myws, call, d, idx, rg, wave, lane);
        }
        typedef const __attribute__((address_space(4))) float* cf32;
        const u32 scb = UNI(__builtin_bit_cast(u32, *(cf32)(u64)P.sx * *(cf32)(u64)P.sw));
        const int g = wave >> 1, nh = wave & 1;
        const u64 ob = (u64)P.out + ((size_t)(tm * 256 + g * 32) * MN + tn * 256 + nh * 128) * 4;
        u32 olo = (u32)ob, ohi = (u32)(ob >> 32);
        mfma_role((u32)(size_t)L + LDS_A + g * 512, (u32)(size_t)L + LDS_B + nh * 512, (u32)(size_t)L + LDS_CTL + wave * 16,
                  olo, ohi, scb);
    } else if (wave < 10) {
        const u64 tk = rt();
        __builtin_amdgcn_s_setprio(2);
        if (wave == 8) {
            char* peer = peer_of_lane(P);
            call = UNI(call) + 1;
            if (rg == 0 && lane < NR) {
                u32 junk;
                asm volatile("global_load_dword %0, %1, off sc0 sc1\n\ts_waitcnt vmcnt(0)" : "=&v"(junk) : "v"(peer + WS_PAD + bid * 128) : "memory");
            }
            WAIT_VM0();
            for (u32 spin = 0; spin < SPINS && UNI(*(volatile LAS u32*)(L + LDS_PRO)) < 7; ++spin) __builtin_amdgcn_s_sleep(1);
            FENCE();
            const u32x4 ln = *(LAS u32x4*)(L + LDS_LEN);
            if (lane < NR) {
                if (lane != rank) { if (rg) st_sys(peer + WS_RDY + (call & 1) * 32768 + (rank * 32 + idx) * 128 + rg * 16, ln | (call << 7)); }
                else st_sys(peer + WS_UF + idx * 2048 + rg * 16, u32x4{call, call, call, call});
            }
            if (lane == 0) __hip_atomic_store((GAS u32*)(myws + WS_CNT + bid * 128), call, __ATOMIC_RELAXED, __HIP_MEMORY_SCOPE_AGENT);
        } else {
            call = UNI(call) + 1;
            arrive(L, lane);
            u32 job = XCD_ID() * 32 + idx;
#pragma unroll 1
            for (int k = 0; k < NWG; ++k, job = (job + 1) & (NWG - 1)) {
                u32 old = 0;
                if (lane == 0) {
                    __hip_atomic_fetch_max((GAS u32*)(myws + WS_TIK + (job >> 5) * 128), call * 2048, __ATOMIC_RELAXED, __HIP_MEMORY_SCOPE_AGENT);
                    FENCE();
                    old = __hip_atomic_fetch_max((GAS u32*)(myws + WS_JOB + job * 128), call, __ATOMIC_RELAXED, __HIP_MEMORY_SCOPE_AGENT);
                }
                WAIT_VM0();
                if (UNI(old) < call) break;
            }
            *(volatile LAS u32*)(L + LDS_JOB) = job + 1;
        }
        const int isB = wave - 8;
        const int panel = isB ? 16 + tn : tm;
        const char* img = myws + WS_IMG + (call & 1) * IMG_BYTES + panel * 16384;
        const u32 ldsb = (u32)(size_t)L + (isB ? LDS_B : LDS_A);
        volatile LAS u32* ctl = (volatile LAS u32*)(L + LDS_CTL + (lane & 7) * 16);
        const char* uf = myws + WS_UF + panel * 2048;
        const u32 voff = lane * 16 + ((panel) & 3) * UNIT;
        Ld S = {0, 0, 0, 0, 0, ~0u, 0, 0, (u64)img, (u64)(img + (16 + tn - panel) * 16384 + (size_t)S0 * 524288)};
        u32 err = 0;
        const u32 vctl = (u32)(size_t)ctl, vfull = vctl + isB * 4, vf = (lane & 31) * 4;
        const u64 t0 = rt();
        WAIT_VM0();
        asm volatile("v_mov_b32 v40, 0\n\tv_mov_b32 v41, 0\n\tv_mov_b32 v47, 0\n\tv_mov_b32 v44, %0\n\tv_mov_b32 v45, %1\n\t"
                     "v_mov_b32 v46, %2\n\tv_mov_b32 v48, %3\n\tv_mov_b32 v50, %4"
                     :: "v"((voff + UNIT) & 16383), "v"((voff + 2 * UNIT) & 16383), "v"((voff + 3 * UNIT) & 16383),
                        "v"(lane * 16 + ((tn + 3) & 3) * UNIT), "v"(isB * 4) : "v40", "v41", "v44", "v45", "v46", "v47", "v48", "v50");
        bool go = true;
        auto late = [&]() {
            if (UNI(__hip_atomic_load(tmo, __ATOMIC_RELAXED, __HIP_MEMORY_SCOPE_AGENT)) == call || rt() - t0 > DEADLINE) {
                if (lane == 0) __hip_atomic_store(tmo, call, __ATOMIC_RELAXED, __HIP_MEMORY_SCOPE_AGENT);
                ctl[3] = 1;
                err = ~0u;
            }
            asm volatile("s_waitcnt vmcnt(0) lgkmcnt(0)" ::: "memory");
            go = rt() - t0 < 3 * DEADLINE;
        };
#define RUN(M, end, l) while (go && load_run<M>(S, end, l, err, call, uf, voff, vf, vctl, vfull)) late();
        while (rt() - tk < LDELAY) __builtin_amdgcn_s_sleep(4);
        while (go && load_run<0, 4>(S, QEASY, ldsb, err, call, uf, voff, vf, vctl, vfull)) late();
        RUN(0, S0, ldsb);
        const u32 l0 = (u32)(size_t)L + LDS_A;
        if (isB) { RUN(2, NSTAGE, l0); S.g -= (size_t)NSPLIT * 524288; S.q += 4; RUN(4, NQ, l0); }
        else {
            RUN(1, NSTAGE, l0);
            S.g -= (size_t)4 * 524288 - 2048; S.slot = (NSPLIT + 1) % 5;
            RUN(1, NSTAGE + 4, l0);
            S.g -= (size_t)NSPLIT * 524288;
            RUN(3, NQ, l0);
        }
        RUN(5, 0, 0);
        ctl[isB] = 0xFFFFu;
        WAIT_VM(0);
    } else {
        __builtin_amdgcn_s_setprio(2);
        const u32 vo = lane * 16, wk = wave - 10;
        {
            u32 x[16];
            const u32 pdst = (srcmap >> (4 * xg)) & 7u;
            const bool push = pr < NPW && pdst != (u32)rank;
            const u32 up = 32 + ((rank - pdst - 1) & 7) + 7 * pr, pp = idx | 16;
            const char* pw = P.ws[pdst];
            const char* q0 = pw + WS_IMG + unit_off(pp, up);
            asm volatile("" : "+s"(q0));
            WAIT_VM0();
            call = UNI(call) + 1;
            if (push) {
                order_w(x, d);
                q0 += (call & 1) * IMG_BYTES;
                st_sys_s(vo, q0, piece(x, 0));
                arrive(L, lane);
                st_sys_s(vo, q0 + 1024, piece(x, 1));
                st_sys_s(vo, q0 + 2048, piece(x, 2));
                st_sys_s(vo, q0 + 3072, piece(x, 3));
                WAIT_VM(0); st_flag(pw + WS_UF + pp * 2048 + up * 4, call, lane);
            } else {
                arrive(L, lane);
            }
            if (wk) *(volatile LAS u32*)(L + LDS_PSH) = 1;
        }
        call = UNI(call);
        u32 job = 0;
        for (u32 spin = 0; spin < SPINS && job == 0; ++spin) { __builtin_amdgcn_s_sleep(1); job = UNI(*(volatile LAS u32*)(L + LDS_JOB)); }
        job = (job - 1) & (NWG - 1);
        const u32 slot = job >> 5, src = (srcmap >> (4 * slot)) & 7u;
        if (src == (u32)rank) return;
        const char* tik = myws + WS_TIK + slot * 128;
        const char* scb = P.ws[src] + WS_CB + (call & 1) * CB_BYTES;
        const char* rdy = myws + WS_RDY + (call & 1) * 32768 + src * 4096;
        const char* dp = myws + WS_IMG + (call & 1) * IMG_BYTES;
        const u32 j0 = (src - rank - 1) & 7;
        LAS unsigned char* sl = L + LDS_WRK + wk * WRK_BYTES + 3072;
        const u32 rot = 2 * rank;
        auto unit_of = [rot](u32 t) -> u32 {
            auto px = [rot](u32 k) { k = (k + rot) & 15; return 4 * (k & 3) + (k >> 2); };
            auto pw = [rot](u32 k) { k = (k + rot) & 15; return 16 + 8 * (k & 1) + (k >> 1); };
            if (t < T0) return t < 64 ? px(t & 15) * 32 + (t >> 4) : pw(t & 15) * 32 + NPW + (t >> 4) - 4;
            if (t < T1) { t -= T0; return ((t & 1) ? pw((t >> 1) & 15) : px((t >> 1) & 15)) * 32 + 4 + (t >> 5); }
            if (t < T2) {
                const u32 pr = (t - T1) / 48, i = (t - T1) % 48;
                return ((i & 16) ? px(i & 15) : pw(i & 15)) * 32 + R0 + 2 * pr + (i >> 5);
            }
            return t < NT ? px((t - T2) & 15) * 32 + (t - T2 < 16 ? ROUNDS - 1 : R0 - 1 + 2 * ((t - T2) >> 4)) : 0u;
        };
        const u64 t0 = rt();
        auto await = [&](u32 un) -> u32 {
            for (u32 n = 1;; ++n) {
                u32 rv;
                if (lane == 0) asm volatile("global_atomic_or v162, %0, %0, %1 sc0 sc1\n\ts_waitcnt vmcnt(0)" :: "v"(0u), "s"(rdy + un * 4) : "memory", "v162");
                asm volatile("v_readfirstlane_b32 %0, v162" : "=s"(rv) :: "memory", "v162");
                if ((rv >> 7) >= call) return rv;
                if ((n & 31) == 0 && (UNI(__hip_atomic_load(tmo, __ATOMIC_RELAXED, __HIP_MEMORY_SCOPE_AGENT)) == call || rt() - t0 > DEADLINE)) {
                    if (lane == 0) __hip_atomic_store(tmo, call, __ATOMIC_RELAXED, __HIP_MEMORY_SCOPE_AGENT);
                    return 0;
                }
                __builtin_amdgcn_s_sleep(2);
            }
        };
        auto book = [&](u64 at, u32 cost) {
            const u64 now = at << 10, c = cost;
            if (lane == 0)
                asm volatile("global_atomic_umax_x2 %0, %1, %3 offset:2048\n\tglobal_atomic_add_x2 v[160:161], %0, %2, %3 offset:2048 sc0"
                             :: "v"(0u), "v"(now), "v"(c), "s"(tik) : "memory", "v160", "v161");
        };
        const char* pend = nullptr;
        u32 pn = 0;
        auto flag = [&]() {
            if (lane < 2) asm volatile("global_store_dword %0, %1, %2 sc0 sc1" :: "v"(lane * pn), "v"(call), "s"(pend) : "memory");
            pend = nullptr;
        };
        u32 t = (job & 31) + 32 * wk, un = unit_of(t), lat = 0, plat = 0;
        if (lane < 2)
            asm volatile("global_load_dword v163, %0, %1 sc0 sc1" :: "v"((lane * 482 + (job & 31)) * SLOT + UNIT + 128), "s"(scb) : "memory", "v163");
        for (u32 spin = 0; spin < SPINS && UNI(*(volatile LAS u32*)(L + (t < 16 ? LDS_PSH : LDS_PRO))) < (t < 16 ? 1 : 7); ++spin)
            __builtin_amdgcn_s_sleep(2);
        u32 rv = await(un);
        if (!rv) return;
        WAIT_VM0();
        book(rt() + (t < NOWAIT ? 0 : 40), (((rv & 127u) + 1) >> 1) * (RATE + 15));
        bool now_ = t < NOWAIT;
        if (!now_) WAIT_VM(0);
        t += 64;
        for (;;) {
            u32 len64 = rv & 127u;
            len64 = len64 < 41 ? 41 : (len64 > 64 ? 64 : len64);
            const u32 len = len64 * 64, panel = un >> 5, r = un & 31;
            const char* q = scb + cb_off(panel, r);
            const bool more = t < NT;
            un = unit_of(t);
            const char* rp = rdy + un * 4;
            asm volatile("" : "+s"(q), "+s"(rp));
            if (!now_) {
                u32 lo, hi;
                asm volatile("v_readfirstlane_b32 %0, v160\n\tv_readfirstlane_b32 %1, v161" : "=s"(lo), "=s"(hi) :: "memory", "v160", "v161");
                const u32 start = (u32)((((u64)hi << 32) | lo) >> 10);
                for (u32 d = start - (u32)rt(); d - 1 < 1000u; d = start - (u32)rt()) {
                    if (pend) { if (vm_idle()) { WAIT_VM(0); flag(); } }
                    else if (d > 12) __builtin_amdgcn_s_sleep(2);
                }
            }
            now_ = false;
            const u32 i0 = (u32)rt();
            PIECE(0, q); PIECE(1, q);
            if (2048 + vo < len) PIECE(2, q);
            if (3072 + vo < len) PIECE(3, q);
            if (lane == 0) asm volatile("global_atomic_or v162, %0, %0, %3 sc0 sc1\n\tglobal_atomic_add v163, %0, %1, %2 sc0"
                                        :: "v"(0u), "v"(1u), "s"(tik), "s"(rp) : "memory", "v162", "v163");
            const bool half = panel < 16 && r >= R0;
            const u32 u = pos_of(j0, half ? r & ~1u : r);
            const char* qa = dp + unit_off(panel, u) + (half ? (r & 1) * 2048 : 0);
            const char* qb = half ? dp + unit_off(panel, u + 7) + (r & 1) * 2048 : qa + 2048;
            const u32 ho = panel < 16 ? 3072 : 2560;
            const bool late = pend != nullptr;
            if (late) WAIT_VM(5);
            asm volatile("" : "+s"(qa), "+s"(qb) :: "memory");
            if (late) { flag(); WAIT_VM(3); } else { WAIT_VM(2); }
            const u32x4 w = *(LAS u32x4*)(sl + lane * 16);
            const u32 hd = *(LAS u32*)(sl + ho), w2 = *(LAS u32*)(sl + ho + 4 + lane * 4);
            const u64 now = rt();
            if (late) WAIT_VM(1); else WAIT_VM(0);
            asm volatile("v_readfirstlane_b32 %0, v163\n\tv_readfirstlane_b32 %1, v162" : "=s"(t), "=s"(rv) :: "memory", "v162", "v163");
            plat = lat;
            lat = (u32)now - i0;
            const u32 ml = lat < plat ? lat : plat;
            const u32 rate = ml > LATHI ? RATE + RATE / 8 : RATE;
            t += 128 - call * 2048;
            const bool known = (rv >> 7) >= call;
            if (more && known) book(now, (((rv & 127u) + 1) >> 1) * rate);
            FENCE(); SB();
            if (len64 == 64) {
                const u32x4 e1 = *(LAS u32x4*)(sl + 1024 + lane * 16), e2 = *(LAS u32x4*)(sl + 2048 + lane * 16), e3 = *(LAS u32x4*)(sl + 3072 + lane * 16);
                st_sys_s(vo, qa, w); st_sys_s(vo, qa + 1024, e1); st_sys_s(vo, qb, e2); st_sys_s(vo, qb + 1024, e3);
            } else if (panel < 16) {
                decode_unit<3>(qa, qb, UNI(hd), w, w2, sl, L, lane);
            } else {
                decode_unit<2>(qa, qb, UNI(hd), w, w2, sl, L, lane);
            }
            const u32 sec = u < 240 ? 16 + 4 * NSPLIT : 16;
            pend = myws + WS_UF + panel * 2048 + (u + (half && (r & 1) ? sec : 0)) * 4;
            pn = half ? 28 : panel < 16 && u >= 4 * S0 ? sec * 4 : 0;
            if (!more) break;
            if (!known) {
                WAIT_VM(0); flag();
                rv = await(un);
                if (!rv) return;
                WAIT_VM0();
                book(rt(), (((rv & 127u) + 1) >> 1) * rate);
                WAIT_VM(0);
            }
            WAIT_VM(4);
        }
        WAIT_VM(0);
        flag();
    }
}

extern "C" void kernel_launch(int rank, int nranks, ncclComm_t comm,
                              hipStream_t stream,
                              void* const* d_in, const int* in_sizes, int n_in,
                              void* d_out, int out_size,
                              const int* in_dtypes, int out_dtype,
                              void* const* d_out_all,
                              void* const* d_ws_all, size_t ws_bytes) {
    (void)comm; (void)in_sizes; (void)n_in; (void)out_size; (void)in_dtypes; (void)out_dtype; (void)d_out_all;
    if (nranks != NR || (unsigned)rank >= (unsigned)NR || ws_bytes < WS_NEED) {
        fprintf(stderr, "kernel_launch: unexpected configuration\n");
        return;
    }
    Params p;
    p.x = (const unsigned char*)d_in[0];
    p.w = (const unsigned char*)d_in[1];
    p.sx = (const float*)d_in[2];
    p.sw = (const float*)d_in[3];
    p.out = (float*)d_out;
    for (int i = 0; i < NR; ++i) p.ws[i] = (char*)d_ws_all[i];
    p.myws = p.ws[rank];
    p.rank = rank;
    p.pad = 0;
    static const u32 near_link[NR] = {0x74510263u, 0x26301475u, 0x17542036u, 0x74153620u,
                                      0x63024157u, 0x60235741u, 0x51476302u, 0x02367514u};
    p.srcmap = near_link[rank];
    hipLaunchKernelGGL(ag_gemm, dim3(NWG), dim3(NTHREADS), 0, stream, p);
    HIP_CHECK(hipGetLastError());
}
```

```cpp
#include <hip/hip_runtime.h>
#include <rccl/rccl.h>
#include <cstdio>

#define HIP_CHECK(expr) do { hipError_t _e = (expr); if (_e != hipSuccess) { \
    fprintf(stderr, "HIP error %s at %s:%d\n", hipGetErrorString(_e), __FILE__, __LINE__); \
    return; } } while (0)

#define GAS __attribute__((address_space(1)))
#define LAS __attribute__((address_space(3)))
typedef unsigned u32;
typedef unsigned short u16;
typedef unsigned long long u64;
typedef unsigned u32x2 __attribute__((ext_vector_type(2)));
typedef unsigned u32x4 __attribute__((ext_vector_type(4)));

constexpr int NR = 8;
constexpr int MN = 4096;
constexpr int KS = 512;
constexpr int NWG = 256;
constexpr int NTHREADS = 768;
constexpr u32 ROUNDS = 32;
constexpr u32 NSTAGE = 64;
#define NSPLIT 8
#define NS0 2
#define NST 6
constexpr int NPW = 2;
constexpr u32 RATE = 215;
constexpr u32 LATHI = 95;
constexpr u32 NOWAIT = 8;
constexpr u32 QEASY = 32, LDELAY = 500;
constexpr u32 S0 = NSTAGE - NSPLIT;
constexpr u32 NQ = NSTAGE + NSPLIT;
constexpr u32 R0 = ((S0 * 4 - 32 + 13) / 14) * 2;
constexpr u32 US = 32 + 7 * R0;
constexpr u32 NPAIR = (ROUNDS - R0) / 2;
constexpr u32 T0 = 128 - 16 * NPW, T1 = T0 + (R0 - 4) * 32, T2 = T1 + NPAIR * 48, NT = T2 + NPAIR * 16;
constexpr u32 UNIT = 4096;
constexpr size_t IMG_BYTES = 32u << 20;
constexpr size_t CB_BYTES = 5u << 20;
__host__ __device__ constexpr u32 unit_off(u32 panel, u32 u) { return (u >> 2) * 524288u + panel * 16384u + ((u + panel) & 3) * UNIT; }
constexpr u32 SLOT = UNIT + 256;
__host__ __device__ constexpr u32 cb_off(u32 panel, u32 r) { return (r * 32 + panel) * SLOT; }
__host__ __device__ constexpr u32 pos_of(u32 j, u32 r) {
    const u32 u = 32 + 7 * r + j, w = u - 4 * S0;
    return u < 4 * S0 ? u : r < R0 ? 4 * S0 + 16 * (w >> 1) + (w & 1) : 4 * S0 + 2 + 8 * ((r - R0) & ~1u) + 7 * (r & 1) + j;
}
static_assert(US - 4 * S0 == 2 * NPAIR && NSPLIT == 4 * NPAIR, "groups of four stages");
constexpr size_t WS_CNT = 0;
constexpr size_t WS_TMO = 48 << 10;
constexpr size_t WS_TIK = 52 << 10;
constexpr size_t WS_UF = 64 << 10;
constexpr size_t WS_RDY = 128 << 10;
constexpr size_t WS_JOB = 192 << 10;
constexpr size_t WS_PAD = 224 << 10;
constexpr size_t WS_IMG = 1 << 20;
constexpr size_t WS_CB = WS_IMG + 2 * IMG_BYTES;
constexpr size_t WS_NEED = WS_CB + 2 * CB_BYTES;
constexpr u64 DEADLINE = 50000000ull;
constexpr u32 SPINS = 1u << 25;
constexpr u32 LDS_CTL = 0;
constexpr u32 LDS_PSH = LDS_CTL + 132, LDS_XCH = LDS_CTL + 136, LDS_PRO = LDS_CTL + 140;
constexpr u32 LDS_JOB = LDS_CTL + 160, LDS_LEN = LDS_CTL + 176;
constexpr u32 LDS_DL = 512, LDS_CL = LDS_DL + 1024;
constexpr u32 LDS_WRK = LDS_CL + 1024, WRK_BYTES = 7168;
constexpr u32 LDS_A = LDS_WRK + 2 * WRK_BYTES, LDS_B = LDS_A + 65536;
constexpr u32 LDS_BYTES = LDS_B + 65536;

struct Params {
    const unsigned char* x;
    const unsigned char* w;
    char* myws;
    int rank;
    u32 srcmap;
    const float* sx;
    const float* sw;
    float* out;
    u64 pad;
    char* ws[NR];
};

__device__ __forceinline__ u64 rt() { return __builtin_amdgcn_s_memrealtime(); }
#define WAIT_VM(n) asm volatile("s_waitcnt vmcnt(" #n ")" ::: "memory")
#define WAIT_VM0() do { __builtin_amdgcn_s_waitcnt(0x0F70); WAIT_VM(0); } while (0)
#define FENCE() asm volatile("" ::: "memory")
#define SB() __builtin_amdgcn_sched_barrier(0)
#define XCD_ID() (__builtin_amdgcn_s_getreg((3 << 11) | 20) & 7u)
#define UNI(x) __builtin_amdgcn_readfirstlane(x)

__device__ __forceinline__ void st_sys(char* p, u32x4 v) {
    asm volatile("global_store_dwordx4 %0, %1, off sc0 sc1\n\ts_nop 1" :: "v"(p), "v"(v) : "memory");
}
__device__ __forceinline__ void st_sys_s(u32 voff, const void* base, u32x4 v) {
    asm volatile("global_store_dwordx4 %0, %1, %2 sc0 sc1\n\ts_nop 1" :: "v"(voff), "v"(v), "s"(base) : "memory");
}
__device__ __forceinline__ void st_flag(const void* base, u32 v, int lane) {
    if (lane == 0) asm volatile("global_store_dword %0, %1, %2 sc0 sc1" :: "v"(0u), "v"(v), "s"(base) : "memory");
}

#define STR2(x) #x
#define STR(x) STR2(x)
struct Ld {
    u32 nops, q, rmask, nf;
    u32 pidx, pb, slot;
    u64 fifo, g, gw;
};
#define G2(v, s_, o) "global_load_lds_dwordx4 " v ", " s_ " offset:" #o " sc0 sc1\n\tglobal_load_lds_dwordx4 " v ", " s_ " offset:" #o "+1024" " sc0 sc1\n\t"
#define EASE ".if %c[ease]\n\ts_sleep %c[ease]\n\t.endif\n\t"
#define M0A(x) "s_add_u32 m0, m0, " x "\n\ts_nop 0\n\t"
#define GW(v) G2(v, "%[g]", 0) M0A("%[land]") G2(v, "%[g]", 2048)
template <int MODE, int EASY = 0>
__device__ __forceinline__ u32 load_run(Ld& S, u32 end, u32 lds, u32 err, u32 call, const void* uf,
                                        u32 vo, u32 vf, u32 vctl, u32 vfull) {
    u32 a, b, land, rc, keep;
    asm volatile(
        "s_mov_b32 %[keep], m0\n\t"
        "s_mov_b32 %[rc], 0\n\t"
        "s_branch .LL%=\n\t"
        ".LZ%=:\n\t"
        ".if %c[md] == 0\n\t"
        "s_sleep 1\n\t"
        ".endif\n\t"
        ".LL%=:\n\t"
        "ds_read_b32 v42, %[vctl] offset:8\n\t"
        "s_getreg_b32 %[a], hwreg(HW_REG_IB_STS)\n\t"
        "s_bfe_u32 %[b], %[a], 0x20016\n\t"
        "s_and_b32 %[a], %[a], 15\n\t"
        "s_lshl4_add_u32 %[a], %[b], %[a]\n\t"
        "s_sub_u32 %[land], %[nops], %[a]\n\t"
        "s_mov_b64 vcc, %[fifo]\n\t"
        "s_sub_u32 %[a], %[land], vcc_lo\n\t"
        "s_bitcmp0_b32 %[a], 11\n\t"
        "s_cselect_b32 %[a], %[nf], 0\n\t"
        "s_cmp_eq_u32 %[a], 0\n\t"
        "s_cbranch_scc1 .LP%=\n\t"
        "v_add_u32 v41, 1, v41\n\t"
        "s_lshr_b64 %[fifo], %[fifo], 12\n\t"
        ".if %c[md]\n\t"
        "v_lshrrev_b32 v49, 6, v41\n\t"
        "v_mad_u32_u24 v49, v49, v50, v41\n\t"
        "ds_write_b32 %[vfull], v49\n\t"
        ".else\n\t"
        "ds_write_b32 %[vfull], v41\n\t"
        ".endif\n\t"
        "s_sub_u32 %[nf], %[nf], 1\n\t"
        ".LP%=:\n\t"
        "s_add_u32 %[rc], %[rc], 0x100000\n\t"
        "s_cbranch_scc1 .LT%=\n\t"
        ".if %c[md] == 5\n\t"
        "s_waitcnt lgkmcnt(0)\n\t"
        "s_cmp_lg_u32 %[nf], 0\n\t"
        "s_cbranch_scc1 .LL%=\n\t"
        ".else\n\t"
        ".if %c[md] != 4\n\t"
        "s_or_b32 %[rmask], %[rmask], %[err]\n\t"
        "s_sub_u32 %[a], %[land], %[pidx]\n\t"
        "s_cmp_lt_i32 %[a], 0\n\t"
        "s_cbranch_scc1 .LO%=\n\t"
        "s_cmp_eq_u32 %[pb], -1\n\t"
        "s_cbranch_scc1 .LS%=\n\t"
        "v_cmp_le_u32 vcc, %[call], v40\n\t"
        "s_sub_u32 %[a], %[q], %[pb]\n\t"
        "s_lshl_b32 %[a], %[a], 1\n\t"
        "s_min_u32 %[a], %[a], 16\n\t"
        "s_lshr_b32 %[b], vcc_lo, %[a]\n\t"
        "s_lshr_b32 %[b], %[b], %[a]\n\t"
        "s_or_b32 %[rmask], %[rmask], %[b]\n\t"
        "s_mov_b32 %[pb], -1\n\t"
        "s_andn2_b32 %[b], 15, %[rmask]\n\t"
        "s_cbranch_scc0 .LO%=\n\t"
        ".LS%=:\n\t"
        "s_andn2_b32 %[b], 0xffff, %[rmask]\n\t"
        "s_cbranch_scc0 .LO%=\n\t"
        "s_lshl_b32 %[b], %[q], 4\n\t"
        "v_add_u32 v43, %[b], %[vf]\n\t"
        "s_add_u32 %[nops], %[nops], 1\n\t"
        "s_mov_b32 exec_hi, 0\n\t"
        "global_atomic_or v40, v43, v47, %[uf] sc0 sc1\n\t"
        "s_mov_b32 exec_hi, -1\n\t"
        "s_mov_b32 %[pidx], %[nops]\n\t"
        "s_mov_b32 %[pb], %[q]\n\t"
        ".LO%=:\n\t"
        ".endif\n\t"
        ".if %c[md] == 0\n\t"
        "s_sub_u32 %[a], %[q], 3\n\t"
        ".else\n\t"
        ".if %c[md] < 3\n\t"
        "s_and_b32 %[a], %[q], 63\n\t"
        "s_cmp_ge_u32 %[a], 68-" STR(NSPLIT) "\n\t"
        ".else\n\t"
        "s_cmp_eq_u32 0, 0\n\t"
        ".endif\n\t"
        "s_subb_u32 %[a], %[q], 3\n\t"
        ".if %c[md] == 4\n\t"
        "s_cmp_eq_u32 %[q], 68\n\t"
        "s_cselect_b32 %[a], 60, %[a]\n\t"
        ".endif\n\t"
        ".endif\n\t"
        "s_waitcnt lgkmcnt(0)\n\t"
        "v_cmp_gt_i32 vcc, %[a], v42\n\t"
        "s_lshl_b32 %[a], %[slot], 14\n\t"
        ".if %c[md]\n\t"
        "s_add_u32 %[b], %[a], 0x10000\n\t"
        "s_cmp_eq_u32 %[slot], 4\n\t"
        "s_cselect_b32 %[a], 2*16384+2048, %[a]\n\t"
        "s_cselect_b32 %[b], 2048, %[b]\n\t"
        "s_cselect_b32 %[land], 16384-2048, 0\n\t"
        ".endif\n\t"
        ".if %c[md] != 4\n\t"
        "s_andn2_b32 m0, 15, %[rmask]\n\t"
        "s_or_b32 m0, m0, vcc_lo\n\t"
        ".else\n\t"
        "s_cmp_lg_u32 vcc_lo, 0\n\t"
        ".endif\n\t"
        "s_cbranch_scc1 .LZ%=\n\t"
        ".if %c[md] == 0\n\t"
        "s_add_u32 m0, %[a], %[lds]\n\t"
        "s_add_u32 %[nops], %[nops], 16\n\t"
        G2("%[vo]", "%[g]", 0) G2("%[vo]", "%[g]", 2048) EASE M0A("0x1000") G2("v44", "%[g]", 0) G2("v44", "%[g]", 2048) EASE M0A("0x1000")
        G2("v45", "%[g]", 0) G2("v45", "%[g]", 2048) EASE M0A("0x1000") G2("v46", "%[g]", 0) G2("v46", "%[g]", 2048)
        ".elseif %c[md] & 1\n\t"
        "s_add_u32 m0, %[a], %[lds]\n\t"
        "s_add_u32 %[nops], %[nops], 6+2*%c[md]\n\t"
        G2("%[vo]", "%[g]", 0) M0A("0x1000") G2("v44", "%[g]", 0) M0A("0x1000") G2("v45", "%[g]", 0) M0A("0x1000") G2("v46", "%[g]", 0)
        ".if %c[md] == 3\n\t"
        "s_add_u32 %[b], %[b], %[lds]\n\t"
        "s_add_u32 m0, %[b], 0x3000\n\t"
        "s_mov_b64 vcc, %[gw]\n\t"
        G2("v48", "vcc", 0) M0A("%[land]") G2("v48", "vcc", 2048)
        "s_add_u32 vcc_lo, vcc_lo, 0x80000\n\t"
        "s_addc_u32 vcc_hi, vcc_hi, 0\n\t"
        "s_mov_b64 %[gw], vcc\n\t"
        ".endif\n\t"
        ".else\n\t"
        "s_add_u32 m0, %[b], %[lds]\n\t"
        "s_sub_u32 %[a], 0x1000, %[land]\n\t"
        "s_add_u32 %[nops], %[nops], 20-2*%c[md]\n\t"
        GW("%[vo]") M0A("%[a]") GW("v44") M0A("%[a]") GW("v45")
        ".if %c[md] == 2\n\t"
        M0A("%[a]") GW("v46")
        ".endif\n\t"
        ".endif\n\t"
        "s_mov_b64 vcc, %[g]\n\t"
        "s_add_u32 vcc_lo, vcc_lo, 0x80000\n\t"
        "s_addc_u32 vcc_hi, vcc_hi, 0\n\t"
        "s_mov_b64 %[g], vcc\n\t"
        "s_and_b32 vcc_lo, %[nops], 0xfff\n\t"
        "s_mov_b32 vcc_hi, 0\n\t"
        "s_mul_i32 %[a], %[nf], 12\n\t"
        "s_lshl_b64 vcc, vcc, %[a]\n\t"
        "s_or_b64 %[fifo], %[fifo], vcc\n\t"
        "s_add_u32 %[nf], %[nf], 1\n\t"
        "s_lshr_b32 %[rmask], %[rmask], 4\n\t"
        "s_add_u32 %[slot], %[slot], 1\n\t"
        "s_cmp_eq_u32 %[slot], 4+(%c[md]+7)/8\n\t"
        "s_cselect_b32 %[slot], 0, %[slot]\n\t"
        "s_add_u32 %[q], %[q], 1\n\t"
        "s_cmp_lt_u32 %[q], %[end]\n\t"
        "s_cbranch_scc1 .LL%=\n\t"
        ".endif\n\t"
        "s_mov_b32 %[rc], 0\n\t"
        "s_branch .LE%=\n\t"
        ".LT%=:\n\t"
        "s_mov_b32 %[rc], 1\n\t"
        "s_waitcnt lgkmcnt(0)\n\t"
        ".LE%=:\n\t"
        "s_mov_b32 m0, %[keep]"
        : [nops] "+s"(S.nops), [q] "+s"(S.q), [rmask] "+s"(S.rmask), [nf] "+s"(S.nf), [pidx] "+s"(S.pidx), [pb] "+s"(S.pb),
          [slot] "+s"(S.slot), [fifo] "+s"(S.fifo), [g] "+s"(S.g), [gw] "+s"(S.gw),
          [a] "=&s"(a), [b] "=&s"(b), [land] "=&s"(land), [rc] "=&s"(rc), [keep] "=&s"(keep)
        : [end] "s"(end), [lds] "s"(lds), [err] "s"(err), [call] "s"(call), [uf] "s"(uf),
          [vo] "v"(vo), [vf] "v"(vf), [vctl] "v"(vctl), [vfull] "v"(vfull), [md] "i"(MODE), [ease] "i"(EASY)
        : "memory", "scc", "vcc", "v40", "v41", "v42", "v43", "v44", "v45", "v46", "v47", "v48", "v49", "v50");
    return rc;
}
#define PIECE(i, g) do { u32 k_; asm volatile("s_mov_b32 %0, m0\n\ts_mov_b32 m0, %3\n\ts_nop 0\n\t" \
    "global_load_lds_dwordx4 %1, %2 offset:1024*" #i " sc0 sc1\n\ts_mov_b32 m0, %0" : "=&s"(k_) : "v"(vo), "s"(g), "s"((u32)(size_t)sl) : "memory"); } while (0)
__device__ __forceinline__ bool vm_idle() {
    u32 r;
    asm volatile("s_getreg_b32 %0, hwreg(HW_REG_IB_STS)" : "=s"(r) :: "memory");
    return (r & 0xC0000Fu) == 0;
}
__device__ __forceinline__ u32 perm(u32 hi, u32 lo, u32 sel) { return __builtin_amdgcn_perm(hi, lo, sel); }
__device__ __forceinline__ u32 bfi(u32 m, u32 a, u32 b) { return __builtin_amdgcn_bitop3_b32(m, a, b, 0xCA); }
__device__ __forceinline__ u32 and_or(u32 a, u32 m, u32 c) { return __builtin_amdgcn_bitop3_b32(a, m, c, 0xEA); }
#define BAR() asm volatile("s_waitcnt lgkmcnt(0)\n\ts_barrier" ::: "memory")
__device__ __forceinline__ u32 scan64(u32 v) {
    v += __builtin_amdgcn_update_dpp(0u, v, 0x111, 0xf, 0xf, true);
    v += __builtin_amdgcn_update_dpp(0u, v, 0x112, 0xf, 0xf, true);
    v += __builtin_amdgcn_update_dpp(0u, v, 0x114, 0xf, 0xf, true);
    v += __builtin_amdgcn_update_dpp(0u, v, 0x118, 0xf, 0xf, true);
    v += __builtin_amdgcn_update_dpp(0u, v, 0x142, 0xa, 0xf, false);
    v += __builtin_amdgcn_update_dpp(0u, v, 0x143, 0xc, 0xf, false);
    return v;
}
__device__ __forceinline__ void lds_or(LAS unsigned char* p, u32 v) {
    __hip_atomic_fetch_or((LAS u32*)p, v, __ATOMIC_RELAXED, __HIP_MEMORY_SCOPE_WORKGROUP);
}

__device__ __forceinline__ void load_w(u32 (&d)[16], const unsigned char* w, int panel, int r, int lane) {
    const auto rs = __builtin_amdgcn_make_buffer_rsrc((void*)w, 0, MN * KS, 0x00020000);
    const u32 so = r * 16 * MN + (panel & 15) * 256;
#pragma unroll
    for (int k = 0; k < 16; ++k) d[k] = __builtin_amdgcn_raw_buffer_load_b32(rs, lane * 4, so + k * MN, 0);
}
__device__ __forceinline__ void order_w(u32 (&x)[16], const u32 (&d)[16]) {
#pragma unroll
    for (int q = 0; q < 4; ++q) {
        u32 t0 = perm(d[4 * q + 1], d[4 * q], 0x05010400), t1 = perm(d[4 * q + 1], d[4 * q], 0x07030602);
        u32 t2 = perm(d[4 * q + 3], d[4 * q + 2], 0x05010400), t3 = perm(d[4 * q + 3], d[4 * q + 2], 0x07030602);
        x[q] = perm(t2, t0, 0x05040100); x[4 + q] = perm(t2, t0, 0x07060302);
        x[8 + q] = perm(t3, t1, 0x05040100); x[12 + q] = perm(t3, t1, 0x07060302);
    }
}
__device__ __forceinline__ u32x4 piece(const u32 (&x)[16], int i) { return u32x4{x[4 * i], x[4 * i + 1], x[4 * i + 2], x[4 * i + 3]}; }

template <int MB> struct Fmt {
    static constexpr u32 EM = (MB == 3 ? 0x0Fu : 0x1Fu) * 0x01010101u;
    static constexpr u32 HEAD = MB == 3 ? 3072 : 2048;
    static constexpr u32 OFF_H = MB == 3 ? 0 : 512, OFF_2 = OFF_H + 4;
};
__device__ __forceinline__ u32 zero2(u32 w) { return __builtin_amdgcn_bitop3_b32(w, w >> 1, 0x55555555u, 0x02); }
__device__ __forceinline__ u32 code_of(u32 hi, u32 lo, u32 v) {
    const u32 r = perm(hi, lo, v);
    return __builtin_amdgcn_bitop3_b32(r, r >> 2, 0x03030303u, 0x28);
}
__device__ __forceinline__ void lookup4(u32* sel, u32 w, LAS unsigned char* tab) {
#pragma unroll
    for (int j = 0; j < 4; ++j) sel[j] = *(LAS u32*)(tab + ((w >> (8 * j)) & 255u) * 4);
}
__device__ __forceinline__ void gather4(const u32* v, const u32* sel, u32 z, u32 a) {
#pragma unroll
    for (int j = 0; j < 4; ++j) {
        const u32 o = j ? a + __builtin_popcount(z & ((1u << (8 * j)) - 1)) : a;
        const u32 c = perm(0u, v[j], sel[j]);
        LAS unsigned char* p = (LAS unsigned char*)(size_t)(o & ~3u);
        lds_or(p, __builtin_amdgcn_alignbyte(c, 0u, ~o));
        lds_or(p + 4, __builtin_amdgcn_alignbyte(0u, c, ~o));
    }
}
__device__ __forceinline__ void window4(u32* o, u32x2* q, u32 z, u32 a) {
#pragma unroll
    for (int j = 0; j < 4; ++j) {
        o[j] = j ? a + __builtin_popcount(z & ((1u << (8 * j)) - 1)) : a;
        LAS u32* p = (LAS u32*)(size_t)(o[j] & ~3u);
        q[j] = u32x2{p[0], p[1]};
    }
}
__device__ __forceinline__ u32 spread(u32 tab, u32 o, u32x2 q, u32 sel) {
    return perm(tab, __builtin_amdgcn_alignbyte(q[1], q[0], o), sel);
}
__device__ __forceinline__ u32 pack4(const u32* c) {
    u32 w = __builtin_amdgcn_udot4(c[0], 0x40100401u, 0u, false);
#pragma unroll
    for (int j = 1; j < 4; ++j) w |= __builtin_amdgcn_udot4(c[j], 0x40100401u, 0u, false) << (8 * j);
    return w;
}
__device__ __forceinline__ void encode_clear(LAS unsigned char* T, int lane) {
#pragma unroll
    for (int i = 2; i < 5; ++i) *(LAS u32x4*)(T + i * 1024 + lane * 16) = u32x4{0, 0, 0, 0};
}

template <int MB> __device__ __forceinline__ u32 encode_unit(const u32 (&x)[16], LAS unsigned char* T, LAS unsigned char* L,
                                                             char* dst, int lane, bool raw) {
    typedef Fmt<MB> F;
    u32 sum = 0;
#pragma unroll
    for (int j = 0; j < 16; j += 5) sum = __builtin_amdgcn_udot4(x[j] & 0x7F7F7F7Fu, 0x01010101u, sum, false);
    const int tot = __builtin_amdgcn_readlane(scan64(sum), 63);
    const u32 eb = (u32)((tot - (1516 << MB)) >> (10 + MB)) & (F::EM & 31u);
    const u32 B = (eb << MB) * 0x01010101u;
    u32 v[16], c[16];
    u32x4 w, z;
#pragma unroll
    for (int j = 0; j < 16; ++j) {
        v[j] = (((x[j] | 0x80808080u) - B) >> MB) & F::EM;
        c[j] = code_of(0u, 0x03020100u, v[j]);
    }
    u32 cnt[5] = {0};
#pragma unroll
    for (int q = 0; q < 4; ++q) {
        w[q] = pack4(c + 4 * q); z[q] = zero2(w[q]); cnt[q + 1] = cnt[q] + __builtin_popcount(z[q]);
        lookup4(c + 4 * q, w[q], L + LDS_CL);
    }
    FENCE();
    const u32 incl = scan64(cnt[4]), n1 = __builtin_amdgcn_readlane(incl, 63);
    u32 n2 = 0, n3 = 0, len64 = 64;
    raw = raw || n1 > 2040;
    const u32 off3 = F::OFF_2 + ((n1 + 15) >> 4) * 4;
    if (!raw) {
        const u32 a1 = incl - cnt[4] + ((u32)(size_t)T + 2047);
#pragma unroll
        for (int q = 0; q < 4; ++q) gather4(v + 4 * q, c + 4 * q, z[q], a1 + cnt[q]);
        st_sys_s(lane * 16, dst, w);
        if (MB == 3) {
            u32x4 o[2];
#pragma unroll
            for (int p = 0; p < 8; ++p) o[p >> 2][p & 3] = bfi(0x87878787u, x[2 * p], bfi(0xF0F0F0F0u, x[2 * p + 1] << 4, x[2 * p + 1] >> 4));
            st_sys_s(lane * 16, dst + 1024, o[0]);
            st_sys_s(lane * 16, dst + 2048, o[1]);
        } else {
            u32x4 o;
            u32x2 sg;
#pragma unroll
            for (int q = 0; q < 4; ++q)
                o[q] = bfi(0x0F0F0F0Fu, bfi(0x03030303u, x[4 * q], x[4 * q + 1] << 2), bfi(0x03030303u, x[4 * q + 2], x[4 * q + 3] << 2) << 4);
#pragma unroll
            for (int k = 0; k < 2; ++k) {
                u32 t = x[8 * k];
#pragma unroll
                for (int j = 1; j < 8; ++j) t = bfi(0x80808080u >> j, x[8 * k + j] >> j, t);
                sg[k] = t;
            }
            *(LAS u32x2*)(T + lane * 8) = sg;
            st_sys_s(lane * 16, dst + 1024, o);
        }
        FENCE();
        for (u32 b = 0; b < n1; b += 1024) {
            const u32x4 y = *(LAS u32x4*)(T + 2048 + b + lane * 16);
            u32 u[4], c2[4];
#pragma unroll
            for (int j = 0; j < 4; ++j) {
                c2[j] = code_of(0x00000300u, 0x00000102u, (y[j] + 0x01010101u) & F::EM);
                u[j] = (y[j] + 0x05050505u) & F::EM;
            }
            const u32 w2 = pack4(c2);
            lookup4(c2, w2, L + LDS_CL);
            FENCE();
            const u32 z2 = zero2(w2), k2 = __builtin_popcount(z2), i2 = scan64(k2);
            if (b + lane * 16 < n1) *(LAS u32*)(T + F::OFF_2 + (b >> 2) + lane * 4) = w2;
            const u32 all = n2 + __builtin_amdgcn_readlane(i2, 63);
            if (all <= 1016 && k2) gather4(u, c2, z2, n2 + i2 - k2 + ((u32)(size_t)T + 4095));
            n2 = all;
        }
        raw = n2 > 1016;
    }
    if (!raw) {
        FENCE();
        const u32x4 y = *(LAS u32x4*)(T + 4096 + lane * 16);
        u32 k3 = 0, m[4];
#pragma unroll
        for (int j = 0; j < 4; ++j) { m[j] = ((y[j] & 0x1C1C1C1Cu) + 0x1C1C1C1Cu) & 0x20202020u; k3 += __builtin_popcount(m[j]); }
        u32 i3 = 0;
        if (__builtin_amdgcn_ballot_w64(k3 != 0)) { i3 = scan64(k3); n3 = __builtin_amdgcn_readlane(i3, 63); }
        const u32 offp = off3 + ((n2 + 15) >> 4) * 4;
        len64 = (F::HEAD + offp + 2 * n3 + 63) >> 6;
        raw = len64 >= 64 || n3 > 63;
        if (!raw) {
            if (lane * 16 < n2)
                *(LAS u32*)(T + off3 + lane * 4) = bfi(0x0F0F0F0Fu, bfi(0x03030303u, y[0], y[1] << 2), bfi(0x03030303u, y[2], y[3] << 2) << 4);
            if (lane == 0) *(LAS u32*)(T + F::OFF_H) = eb | (n1 << 5) | (n2 << 16) | (n3 << 26);
            if (n3) {
                LAS unsigned char* p = T + offp + 2 * (i3 - k3);
                u32 mm = (m[0] >> 5) | (m[1] >> 4) | (m[2] >> 3) | (m[3] >> 2);
                while (mm) {
                    const u32 b = __builtin_ctz(mm), i = lane * 16 + (b & 3u) * 4 + (b >> 3);
                    mm &= mm - 1;
                    *(LAS u16*)p = (u16)((i << 5) | T[4096 + i]);
                    p += 2;
                }
            }
        }
    }
    if (raw) {
        WAIT_VM(0);
#pragma unroll
        for (int i = 0; i < 4; ++i) st_sys_s(lane * 16, dst + i * 1024, piece(x, i));
        return 64;
    }
    FENCE();
    const u32 tl = len64 * 64 - F::HEAD;
    if (lane * 16 < tl) st_sys_s(lane * 16, dst + F::HEAD, *(LAS u32x4*)(T + lane * 16));
    if (MB == 2 && 1024 + lane * 16 < tl) st_sys_s(lane * 16, dst + F::HEAD + 1024, *(LAS u32x4*)(T + 1024 + lane * 16));
    return len64;
}

template <int MB> __device__ __forceinline__ void decode_unit(const char* qa, const char* qb, u32 hdr, u32x4 w, u32 w2,
                                                              LAS unsigned char* S, LAS unsigned char* L, int lane) {
    typedef Fmt<MB> F;
    LAS unsigned char* T = S + F::HEAD;
    const u32 eb = hdr & 31u, n1 = (hdr >> 5) & 2047u, n2 = (hdr >> 16) & 1023u, n3 = hdr >> 26;
    const u32 off3 = F::OFF_2 + ((n1 + 15) >> 4) * 4, offp = off3 + ((n2 + 15) >> 4) * 4;
    const u32 w3 = *(LAS u32*)(T + off3 + lane * 4);
    const u32 en = *(LAS u16*)(T + offp + lane * 2);
    u32 s1[16], s2[4];
    lookup4(s2, w2, L + LDS_DL);
    const u32x4 o0 = *(LAS u32x4*)(S + 1024 + lane * 16);
    u32x4 o1;
    if (MB == 3) o1 = *(LAS u32x4*)(S + 2048 + lane * 16);
    else { const u32x2 sg = *(LAS u32x2*)(T + lane * 8); o1 = u32x4{sg[0], sg[1], 0, 0}; }
    FENCE(); SB();
    const u32 eb4 = eb * 0x01010101u;
    const u32 t1 = ((eb4 + 0x03020100u) & F::EM) << MB, t2 = ((eb4 + 0x041F0000u) & F::EM) << MB, t3 = ((eb4 + 0x1E1D1C1Bu) & F::EM) << MB;
    u32x4 z;
    u32 cnt[5] = {0};
#pragma unroll
    for (int q = 0; q < 4; ++q) { z[q] = zero2(w[q]); cnt[q + 1] = cnt[q] + __builtin_popcount(z[q]); }
    u32 z2 = zero2(w2), k2 = __builtin_popcount(z2);
    const u32 both = cnt[4] | (k2 << 16), incl = scan64(both);
    const u32 excl = incl - both + ((u32)(size_t)S - 2048) + (((u32)(size_t)S - 3072) << 16);
    SB();
    {
        u32x4 y;
#pragma unroll
        for (int j = 0; j < 4; ++j) y[j] = perm(0u, t3, (w3 >> (2 * j)) & 0x03030303u);
        *(LAS u32x4*)(S - 3072 + lane * 16) = y;
        FENCE();
        if ((u32)lane < n3) *(S - 3072 + ((en >> 5) & 1023u)) = (unsigned char)(((en + eb + 27u) & (F::EM & 31u)) << MB);
        FENCE();
    }
    u32 o[16];
    u32x2 q[16];
    window4(o, q, z2, excl >> 16);
    SB();
#pragma unroll
    for (int i = 0; i < 4; ++i) lookup4(s1 + 4 * i, w[i], L + LDS_DL);
    FENCE(); SB();
    *(LAS u32x4*)(S - 2048 + lane * 16) =
        u32x4{spread(t2, o[0], q[0], s2[0]), spread(t2, o[1], q[1], s2[1]), spread(t2, o[2], q[2], s2[2]), spread(t2, o[3], q[3], s2[3])};
    if (n1 > 1024) {
        w2 = *(LAS u32*)(T + F::OFF_2 + 256 + lane * 4);
        lookup4(s2, w2, L + LDS_DL);
        z2 = zero2(w2); k2 = __builtin_popcount(z2);
        window4(o, q, z2, (__builtin_amdgcn_readlane(incl, 63) >> 16) + (u32)(size_t)S - 3072 + scan64(k2) - k2);
        *(LAS u32x4*)(S - 2048 + 1024 + lane * 16) =
            u32x4{spread(t2, o[0], q[0], s2[0]), spread(t2, o[1], q[1], s2[1]), spread(t2, o[2], q[2], s2[2]), spread(t2, o[3], q[3], s2[3])};
    }
    FENCE(); SB();
#pragma unroll
    for (int i = 0; i < 4; ++i) window4(o + 4 * i, q + 4 * i, z[i], (excl & 0xFFFFu) + cnt[i]);
    SB();
    u32x4 e[4];
#pragma unroll
    for (int j = 0; j < 16; ++j) {
        const u32 ex = spread(t1, o[j], q[j], s1[j]);
        if (MB == 3) {
            const u32 v = j < 8 ? o0[(j >> 1) & 3] : o1[(j >> 1) & 3];
            e[j >> 2][j & 3] = (j & 1) ? and_or(v >> 4, 0x07070707u, and_or(v << 4, 0x80808080u, ex)) : and_or(v, 0x87878787u, ex);
        } else {
            e[j >> 2][j & 3] = and_or(o0[j >> 2] >> (2 * (j & 3)), 0x03030303u, and_or(o1[j >> 3] << (j & 7), 0x80808080u, ex));
        }
        if ((j & 3) == 3) st_sys_s(lane * 16, (j < 8 ? qa : qb) + (j & 4) * 256, e[j >> 2]);
    }
}

#define V10(a) "v" #a "0", "v" #a "1", "v" #a "2", "v" #a "3", "v" #a "4", "v" #a "5", "v" #a "6", "v" #a "7", "v" #a "8", "v" #a "9"
__device__ __forceinline__ void mfma_role(u32 sa, u32 sb, u32 sc, u32 olo, u32 ohi, u32 scale) {
    u32 need, m, spin, scl;
    asm volatile(
        ".macro AG_FRAG d, a, o\n\t"
        "ds_read_b128 v[\\d:\\d+3], \\a offset:\\o\n\t"
        "ds_read_b128 v[\\d+4:\\d+7], \\a offset:\\o+4096\n\t"
        ".endm\n\t"
        ".macro AG_MM c, a, b\n\t"
        "v_mfma_f32_32x32x64_f8f6f4 v[\\c:\\c+15], v[\\a:\\a+7], v[\\b:\\b+7], v[\\c:\\c+15] cbsz:0 blgp:1\n\t"
        ".endm\n\t"
        ".macro AG_MIN\n\t"
        "v_min_u32 v164, v164, v165\n\t"
        "s_nop 0\n\t"
        "v_readfirstlane_b32 %[m], v164\n\t"
        ".endm\n\t"
        ".macro AG_SPIN\n\t"
        "s_mov_b32 %[spin], 0\n\t"
        ".LAG_S\\@:\n\t"
        "ds_read_b64 v[164:165], v162\n\t"
        "s_add_u32 %[spin], %[spin], 1\n\t"
        "s_waitcnt lgkmcnt(0)\n\t"
        "AG_MIN\n\t"
        "s_cmp_ge_u32 %[m], %[need]\n\t"
        "s_cbranch_scc1 .LAG_E\\@\n\t"
        "s_sleep 1\n\t"
        "s_cmp_lt_u32 %[spin], 0x2000000\n\t"
        "s_cbranch_scc1 .LAG_S\\@\n\t"
        "s_mov_b32 %[scale], 0x7fc00000\n\t"
        ".LAG_E\\@:\n\t"
        ".endm\n\t"
        ".macro AG_DONE\n\t"
        "v_add_u32 v163, 1, v163\n\t"
        "ds_write_b32 v162, v163 offset:8\n\t"
        ".endm\n\t"
        ".macro AG_FULL so, sn\n\t"
        "s_waitcnt lgkmcnt(4)\n\t"
        "AG_MM 0, 128, 144\n\t"
        "s_waitcnt lgkmcnt(2)\n\t"
        "AG_MM 64, 136, 144\n\t"
        "ds_read_b64 v[164:165], v162\n\t"
        "AG_FRAG 144, v161, \\so+2048\n\t"
        "s_add_u32 %[need], %[need], 1\n\t"
        "s_waitcnt lgkmcnt(3)\n\t"
        "AG_MM 16, 128, 152\n\t"
        "AG_MM 80, 136, 152\n\t"
        "AG_FRAG 152, v161, \\so+3072\n\t"
        "s_waitcnt lgkmcnt(2)\n\t"
        "AG_MM 32, 128, 144\n\t"
        "AG_MIN\n\t"
        "AG_MM 96, 136, 144\n\t"
        "s_cmp_ge_u32 %[m], %[need]\n\t"
        "s_cbranch_scc1 .LAG_F\\@\n\t"
        "s_waitcnt lgkmcnt(0)\n\t"
        "AG_MM 48, 128, 152\n\t"
        "AG_DONE\n\t"
        "AG_MM 112, 136, 152\n\t"
        "AG_SPIN\n\t"
        "AG_FRAG 144, v161, \\sn\n\t"
        "AG_FRAG 128, v160, \\sn\n\t"
        "s_branch .LAG_J\\@\n\t"
        ".LAG_F\\@:\n\t"
        "AG_FRAG 144, v161, \\sn\n\t"
        "s_waitcnt lgkmcnt(2)\n\t"
        "AG_MM 48, 128, 152\n\t"
        "AG_DONE\n\t"
        "AG_FRAG 128, v160, \\sn\n\t"
        "AG_MM 112, 136, 152\n\t"
        ".LAG_J\\@:\n\t"
        "AG_FRAG 136, v160, \\sn+2048\n\t"
        "AG_FRAG 152, v161, \\sn+1024\n\t"
        ".endm\n\t"
        ".macro AG_PREP c, b\n\t"
        ".irp j, 0, 16, 32, 48\n\t"
        ".irp i, 0, 1, 2, 3\n\t"
        "v_mul_f32 v[\\c+\\j+4*\\b+\\i], %[scl], v[\\c+\\j+4*\\b+\\i]\n\t"
        ".endr\n\t"
        ".endr\n\t"
        "v_swap_b32 v[\\c+4*\\b+1], v[\\c+16+4*\\b]\n\t"
        "v_swap_b32 v[\\c+4*\\b+2], v[\\c+32+4*\\b]\n\t"
        "v_swap_b32 v[\\c+4*\\b+3], v[\\c+48+4*\\b]\n\t"
        "v_swap_b32 v[\\c+16+4*\\b+2], v[\\c+32+4*\\b+1]\n\t"
        "v_swap_b32 v[\\c+16+4*\\b+3], v[\\c+48+4*\\b+1]\n\t"
        "v_swap_b32 v[\\c+32+4*\\b+3], v[\\c+48+4*\\b+2]\n\t"
        ".endm\n\t"
        ".macro AG_ST c, r\n\t"
        ".if ((\\r) & 3) == 0\n\t"
        "AG_PREP \\c, (\\r)/4\n\t"
        ".endif\n\t"
        ".if \\c\n\t"
        "global_store_dwordx4 v166, v[\\c+16*((\\r)&3)+((\\r)&12):\\c+16*((\\r)&3)+((\\r)&12)+3], vcc\n\t"
        ".else\n\t"
        "global_store_dwordx4 v166, v[\\c+16*((\\r)&3)+((\\r)&12):\\c+16*((\\r)&3)+((\\r)&12)+3], vcc nt\n\t"
        ".endif\n\t"
        ".if ((\\r) & 3) == 3\n\t"
        "s_add_u32 vcc_lo, vcc_lo, 5*16384\n\t"
        ".else\n\t"
        "s_add_u32 vcc_lo, vcc_lo, 16384\n\t"
        ".endif\n\t"
        "s_addc_u32 vcc_hi, vcc_hi, 0\n\t"
        ".endm\n\t"
        ".macro AG_STN lo, hi\n\t"
        ".set AG_K, \\lo\n\t"
        ".rept (\\hi)-(\\lo)\n\t"
        "AG_ST 0, AG_K\n\t"
        ".set AG_K, AG_K+1\n\t"
        ".endr\n\t"
        ".endm\n\t"
        ".macro AG_XF d, q\n\t"
        ".if (((\\q)+(\\q)/" STR(NSPLIT) ") %% 5) == 4\n\t"
        "AG_FRAG \\d, v160, 2*16384+2048\n\t"
        ".else\n\t"
        "AG_FRAG \\d, v160, (((\\q)+(\\q)/" STR(NSPLIT) ") %% 5)*16384\n\t"
        ".endif\n\t"
        ".endm\n\t"
        ".macro AG_WF d, q, i\n\t"
        ".if (((\\q)+(\\q)/" STR(NSPLIT) ") %% 5) == 4\n\t"
        "AG_FRAG \\d, v167, (\\i/2)*16384+(\\i&1)*1024\n\t"
        ".else\n\t"
        "AG_FRAG \\d, v161, (((\\q)+(\\q)/" STR(NSPLIT) ") %% 5)*16384+\\i*1024\n\t"
        ".endif\n\t"
        ".endm\n\t"
        ".macro AG_HALF c, q, last, r0, r1, r2\n\t"
        ".set AG_B0, 136+8*(((\\q)+1) %% 3)\n\t"
        ".set AG_B1, 136+8*(((\\q)+2) %% 3)\n\t"
        ".set AG_B2, 136+8*((\\q) %% 3)\n\t"
        "s_waitcnt lgkmcnt(2)\n\t"
        "AG_MM \\c, 128, AG_B0\n\t"
        "AG_WF AG_B0, \\q, 3\n\t"
        "s_add_u32 %[need], %[need], 1\n\t"
        "AG_MIN\n\t"
        "AG_MM \\c+16, 128, AG_B1\n\t"
        ".if \\last == 0\n\t"
        "s_cmp_ge_u32 %[m], %[need]\n\t"
        "s_cbranch_scc1 .LAG_F\\@\n\t"
        ".endif\n\t"
        "AG_STN \\r0, \\r1\n\t"
        "s_waitcnt lgkmcnt(2)\n\t"
        "AG_MM \\c+32, 128, AG_B2\n\t"
        "s_waitcnt lgkmcnt(0)\n\t"
        "AG_MM \\c+48, 128, AG_B0\n\t"
        ".if \\last == 0\n\t"
        "AG_DONE\n\t"
        "AG_SPIN\n\t"
        "AG_WF AG_B1, (\\q)+1, 0\n\t"
        "AG_WF AG_B2, (\\q)+1, 1\n\t"
        "s_branch .LAG_J\\@\n\t"
        ".LAG_F\\@:\n\t"
        "AG_WF AG_B1, (\\q)+1, 0\n\t"
        "AG_STN \\r0, \\r1\n\t"
        "s_waitcnt lgkmcnt(4)\n\t"
        "AG_MM \\c+32, 128, AG_B2\n\t"
        "AG_WF AG_B2, (\\q)+1, 1\n\t"
        "s_waitcnt lgkmcnt(4)\n\t"
        "AG_MM \\c+48, 128, AG_B0\n\t"
        "AG_DONE\n\t"
        ".LAG_J\\@:\n\t"
        "AG_XF 128, (\\q)+1\n\t"
        "ds_read_b64 v[164:165], v162\n\t"
        "AG_WF AG_B0, (\\q)+1, 2\n\t"
        ".endif\n\t"
        "AG_STN \\r1, \\r2\n\t"
        ".endm\n\t"
        ".macro AG_SCALE\n\t"
        "ds_read_b32 v164, v162 offset:12\n\t"
        "s_nop 15\n\t"
        "s_nop 7\n\t"
        "s_waitcnt lgkmcnt(0)\n\t"
        "v_readfirstlane_b32 %[m], v164\n\t"
        "s_cmp_eq_u32 %[m], 0\n\t"
        "s_cselect_b32 %[scl], %[scale], 0x7fc00000\n\t"
        ".endm\n\t"

        "s_waitcnt vmcnt(0) lgkmcnt(0)\n\t"
        "v_mbcnt_lo_u32_b32 v167, -1, 0\n\t"
        "v_mbcnt_hi_u32_b32 v167, -1, v167\n\t"
        "v_lshrrev_b32 v165, 5, v167\n\t"
        "v_and_b32 v167, 31, v167\n\t"
        "v_lshlrev_b32 v167, 4, v167\n\t"
        "v_lshl_add_u32 v166, v165, 16, v167\n\t"
        "v_lshl_add_u32 v167, v165, 13, v167\n\t"
        "v_add_u32 v160, %[sa], v167\n\t"
        "v_add_u32 v161, %[sb], v167\n\t"
        "v_subrev_u32 v167, 65536-2048, v161\n\t"
        "v_mov_b32 v162, %[sc]\n\t"
        "v_mov_b32 v163, 0\n\t"
        ".set AG_I, 0\n\t"
        ".rept 128\n\t"
        "v_mov_b32 v[AG_I], 0\n\t"
        ".set AG_I, AG_I+1\n\t"
        ".endr\n\t"
        "s_mov_b32 vcc_lo, %[olo]\n\t"
        "s_mov_b32 vcc_hi, %[ohi]\n\t"
        "s_mov_b32 %[need], 1\n\t"
        "AG_SPIN\n\t"
        "AG_FRAG 144, v161, 0\n\t"
        "AG_FRAG 128, v160, 0\n\t"
        "AG_FRAG 136, v160, 2048\n\t"
        "AG_FRAG 152, v161, 1024\n\t"
        ".LAG_LOOP:\n\t"
        "AG_FULL 0, 16384\n\t"
        "AG_FULL 16384, 32768\n\t"
        "AG_FULL 32768, 49152\n\t"
        "AG_FULL 49152, 0\n\t"
        "s_cmp_le_u32 %[need], 64-" STR(NSPLIT) "\n\t"
        "s_cbranch_scc1 .LAG_LOOP\n\t"
        "ds_read_b64 v[164:165], v162\n\t"
        "AG_WF 136, 0, 2\n\t"
        ".set AG_I, 0\n\t"
        ".rept " STR(NSPLIT) "\n\t"
        "AG_HALF 0, AG_I, (AG_I==" STR(NSPLIT) "-1), 0, 0, 0\n\t"
        ".set AG_I, AG_I+1\n\t"
        ".endr\n\t"
        "AG_DONE\n\t"
        "AG_SCALE\n\t"
        "AG_STN 0, " STR(NS0) "\n\t"
        "AG_SPIN\n\t"
        "AG_WF AG_B1, AG_I, 0\n\t"
        "AG_WF AG_B2, AG_I, 1\n\t"
        "AG_XF 128, AG_I\n\t"
        "ds_read_b64 v[164:165], v162\n\t"
        "AG_WF AG_B0, AG_I, 2\n\t"
        ".set AG_I, 0\n\t"
        ".rept " STR(NSPLIT) "\n\t"
        ".set AG_R0, " STR(NS0) "+(AG_I*(16-" STR(NS0) ")+" STR(NST) "-1)/" STR(NST) "\n\t"
        ".set AG_R2, " STR(NS0) "+((AG_I+1)*(16-" STR(NS0) ")+" STR(NST) "-1)/" STR(NST) "\n\t"
        ".if AG_I >= " STR(NST) "\n\t"
        ".set AG_R0, 16\n\t"
        ".set AG_R2, 16\n\t"
        ".endif\n\t"
        "AG_HALF 64, AG_I+" STR(NSPLIT) ", (AG_I==" STR(NSPLIT) "-1), AG_R0, (AG_R0+AG_R2+1)/2, AG_R2\n\t"
        ".set AG_I, AG_I+1\n\t"
        ".endr\n\t"
        "AG_SCALE\n\t"
        "s_add_u32 vcc_lo, vcc_lo, 96*16384\n\t"
        "s_addc_u32 vcc_hi, vcc_hi, 0\n\t"
        ".irp r, 0, 1, 2, 3, 4, 5, 6, 7, 8, 9, 10, 11, 12, 13, 14, 15\n\t"
        "AG_ST 64, \\r\n\t"
        ".endr\n\t"
        "s_nop 1\n\t"
        : [scale] "+s"(scale), [need] "=&s"(need), [m] "=&s"(m), [spin] "=&s"(spin), [scl] "=&s"(scl)
        : [sa] "s"(sa), [sb] "s"(sb), [sc] "s"(sc), [olo] "s"(olo), [ohi] "s"(ohi)
        : "memory", "scc", "vcc", V10(), V10(1), V10(2), V10(3), V10(4), V10(5), V10(6), V10(7), V10(8), V10(9),
          V10(10), V10(11), V10(12), V10(13), V10(14), V10(15),
          "v160", "v161", "v162", "v163", "v164", "v165", "v166", "v167");
}

__device__ __forceinline__ char* peer_of_lane(const Params& P) {
    u32 lo = 0, hi = 0;
#pragma unroll
    for (int p = 0; p < NR; ++p) {
        asm volatile("v_writelane_b32 %0, %2, %4\n\tv_writelane_b32 %1, %3, %4"
                     : "+v"(lo), "+v"(hi) : "s"((u32)(u64)P.ws[p]), "s"((u32)((u64)P.ws[p] >> 32)), "i"(p));
    }
    return (char*)(((u64)hi << 32) | lo);
}
__device__ __forceinline__ void arrive(LAS unsigned char* L, int lane) {
    if (lane == 0) __hip_atomic_fetch_add((LAS u32*)(L + LDS_PRO), 1u, __ATOMIC_RELAXED, __HIP_MEMORY_SCOPE_WORKGROUP);
}

template <bool W> __device__ __forceinline__ void prologue_unit(const Params& P, LAS unsigned char* L, char* myws, u32 callv,
                                                                 const u32 (&d)[16], int idx, int xg, int wave, int lane) {
    u32 x[16];
    __builtin_amdgcn_s_setprio(3);
    const int u = xg * 4 + wave;
    const bool half = !W && u >= (int)R0;
    char* peer = peer_of_lane(P);
    WAIT_VM0();
    if (W) {
        order_w(x, d);
    } else {
        LAS u32* xc = (LAS u32*)(L + LDS_XCH);
        if (lane == 0) __hip_atomic_fetch_add(xc, 1u, __ATOMIC_RELAXED, __HIP_MEMORY_SCOPE_WORKGROUP);
        for (u32 spin = 0; spin < SPINS && UNI(*(volatile LAS u32*)xc) < 4; ++spin) {}
        FENCE();
        LAS unsigned char* b = L + LDS_A + 32768 + lane * 64 + (half ? (wave & 1) * 8192 + (wave & 2) * 16 : wave * 16);
        const u32 hi = half ? 16 : 8192;
#pragma unroll
        for (int i = 0; i < 4; ++i) {
            const u32x4 v = *(LAS u32x4*)(b + (i & 1) * 4096 + (i >> 1) * hi);
            x[4 * i] = v[0]; x[4 * i + 1] = v[1]; x[4 * i + 2] = v[2]; x[4 * i + 3] = v[3];
        }
    }
    const u32 par = (UNI(callv) + 1) & 1;
    const bool raw = u >= (int)ROUNDS - (W ? 1 : 3);
    u32 len64 = 64;
    if (!W && u == 0) {
        char* dst = myws + WS_CB + par * CB_BYTES + cb_off(idx, u);
#pragma unroll
        for (int i = 0; i < 4; ++i) st_sys_s(lane * 16, dst + i * 1024, piece(x, i));
    } else if (!W || u >= NPW) {
        len64 = encode_unit<W ? 2 : 3>(x, L + LDS_A + wave * 8192, L, myws + WS_CB + par * CB_BYTES + cb_off(idx, u), lane, raw);
    }
    if (lane == 0) *(volatile LAS u32*)(L + LDS_LEN + wave * 4) = len64;
    const char* qa = myws + WS_IMG + par * IMG_BYTES + unit_off(idx, half ? u & ~1 : u) + (half ? (u & 1) * 2048 : 0);
    const char* qb = half ? myws + WS_IMG + par * IMG_BYTES + unit_off(idx, u | 1) + (u & 1) * 2048 : qa + 2048;
    st_sys_s(lane * 16, qa, piece(x, 0)); st_sys_s(lane * 16, qa + 1024, piece(x, 1));
    st_sys_s(lane * 16, qb, piece(x, 2)); st_sys_s(lane * 16, qb + 1024, piece(x, 3));
    WAIT_VM(4);
    if (xg == 0 && (!W || u >= NPW) && lane < NR && lane != P.rank)
        asm volatile("global_store_dword %0, %1, off sc0 sc1" :: "v"(peer + WS_RDY + par * 32768 + (P.rank * 32 + idx) * 128 + u * 4),
                     "v"(((UNI(callv) + 1) << 7) | len64) : "memory");
    if (xg == 0 && (!W || u >= NPW)) WAIT_VM(1); else WAIT_VM(0);
    arrive(L, lane);
    __builtin_amdgcn_s_setprio(0);
}

__global__ void __launch_bounds__(NTHREADS, 3) ag_gemm(Params P) {
    __shared__ __attribute__((aligned(16))) unsigned char lds_raw[LDS_BYTES];
    LAS unsigned char* L = (LAS unsigned char*)lds_raw;
    const int hw = UNI(threadIdx.x >> 6);
    const int wave = hw < 4 ? 11 - hw : hw - 4;
    const int lane = threadIdx.x & 63;
    const int bid = blockIdx.x;
    const unsigned char* px = P.x;
    const unsigned char* pw_ = P.w;
    char* myws = P.myws;
    int rank = P.rank;
    u32 srcmap = P.srcmap;
    asm volatile("" : "+s"(px), "+s"(pw_), "+s"(myws), "+s"(rank), "+s"(srcmap) : "s"(P.ws[0]));
    const int xg = bid & 7, idx = bid >> 3;
    const int rg = (xg + 5) & 7;
    const int tm = (xg >> 1) * 4 + (idx >> 3), tn = (xg & 1) * 8 + (idx & 7);
    GAS u32* tmo = (GAS u32*)(myws + WS_TMO);

    u32 xo = (lane >> 2) * KS + (lane & 3) * 16;
    u32 d[16];
    if (wave < 4 && idx < 16) {
        const unsigned char* g = px + (size_t)(idx * 256 + wave * 64) * KS + rg * 64;
        u32 keep;
        asm volatile("s_mov_b32 %1, m0\n\ts_mov_b32 m0, %3\n\ts_nop 0\n\tglobal_load_lds_dwordx4 %0, %2\n\t"
                     ".rept 3\n\ts_add_u32 m0, m0, 0x400\n\tv_add_u32 %0, 16*512, %0\n\tglobal_load_lds_dwordx4 %0, %2\n\t.endr\n\t"
                     "s_mov_b32 m0, %1"
                     : "+v"(xo), "=&s"(keep) : "s"(g), "s"((u32)(size_t)L + LDS_A + 32768 + wave * 4096) : "memory", "scc");
    }
    FENCE();
    const int pr = (idx >> 4) + 2 * (11 - wave);
    if (wave < 4 ? idx >= 16 : wave >= 10 && pr < NPW) load_w(d, pw_, idx, wave < 4 ? rg * 4 + wave : pr, lane);
    u32 call = __hip_atomic_load((GAS u32*)(myws + WS_CNT + bid * 128), __ATOMIC_RELAXED, __HIP_MEMORY_SCOPE_AGENT);
    if (wave < 4) {
        encode_clear(L + LDS_A + wave * 8192, lane);
    } else if (wave < 8) {
        const u32 f = (wave - 4) * 64 + lane;
        u32 dl = 0, cl = 0x0C0C0C0Cu, k = 0;
#pragma unroll
        for (u32 i = 0; i < 4; ++i) {
            const u32 c = (f >> (2 * i)) & 3u;
            dl |= (c ? 4 + c : k) << (8 * i);
            cl ^= c ? 0u : (12u ^ i) << (8 * k);
            k += c == 0;
        }
        *(LAS u32*)(L + LDS_DL + f * 4) = dl;
        *(LAS u32*)(L + LDS_CL + f * 4) = cl;
    } else if (wave == 8) {
        ((LAS u32*)(L + LDS_CTL))[lane] = 0;
    }
    BAR();

    if (wave < 8) {
        if (wave < 4) {
            if (idx < 16) prologue_unit<false>(P, L, myws, call, d, idx, rg, wave, lane);
            else prologue_unit<true>(P, L, myws, call, d, idx, rg, wave, lane);
        }
        typedef const __attribute__((address_space(4))) float* cf32;
        const u32 scb = UNI(__builtin_bit_cast(u32, *(cf32)(u64)P.sx * *(cf32)(u64)P.sw));
        const int g = wave >> 1, nh = wave & 1;
        const u64 ob = (u64)P.out + ((size_t)(tm * 256 + g * 32) * MN + tn * 256 + nh * 128) * 4;
        u32 olo = (u32)ob, ohi = (u32)(ob >> 32);
        mfma_role((u32)(size_t)L + LDS_A + g * 512, (u32)(size_t)L + LDS_B + nh * 512, (u32)(size_t)L + LDS_CTL + wave * 16,
                  olo, ohi, scb);
    } else if (wave < 10) {
        const u64 tk = rt();
        __builtin_amdgcn_s_setprio(2);
        if (wave == 8) {
            char* peer = peer_of_lane(P);
            call = UNI(call) + 1;
            if (rg == 0 && lane < NR) {
                u32 junk;
                asm volatile("global_load_dword %0, %1, off sc0 sc1\n\ts_waitcnt vmcnt(0)" : "=&v"(junk) : "v"(peer + WS_PAD + bid * 128) : "memory");
            }
            WAIT_VM0();
            for (u32 spin = 0; spin < SPINS && UNI(*(volatile LAS u32*)(L + LDS_PRO)) < 7; ++spin) __builtin_amdgcn_s_sleep(1);
            FENCE();
            const u32x4 ln = *(LAS u32x4*)(L + LDS_LEN);
            if (lane < NR) {
                if (lane != rank) { if (rg) st_sys(peer + WS_RDY + (call & 1) * 32768 + (rank * 32 + idx) * 128 + rg * 16, ln | (call << 7)); }
                else st_sys(peer + WS_UF + idx * 2048 + rg * 16, u32x4{call, call, call, call});
            }
            if (lane == 0) __hip_atomic_store((GAS u32*)(myws + WS_CNT + bid * 128), call, __ATOMIC_RELAXED, __HIP_MEMORY_SCOPE_AGENT);
        } else {
            call = UNI(call) + 1;
            arrive(L, lane);
            u32 job = XCD_ID() * 32 + idx;
#pragma unroll 1
            for (int k = 0; k < NWG; ++k, job = (job + 1) & (NWG - 1)) {
                u32 old = 0;
                if (lane == 0) {
                    __hip_atomic_fetch_max((GAS u32*)(myws + WS_TIK + (job >> 5) * 128), call * 2048, __ATOMIC_RELAXED, __HIP_MEMORY_SCOPE_AGENT);
                    FENCE();
                    old = __hip_atomic_fetch_max((GAS u32*)(myws + WS_JOB + job * 128), call, __ATOMIC_RELAXED, __HIP_MEMORY_SCOPE_AGENT);
                }
                WAIT_VM0();
                if (UNI(old) < call) break;
            }
            *(volatile LAS u32*)(L + LDS_JOB) = job + 1;
        }
        const int isB = wave - 8;
        const int panel = isB ? 16 + tn : tm;
        const char* img = myws + WS_IMG + (call & 1) * IMG_BYTES + panel * 16384;
        const u32 ldsb = (u32)(size_t)L + (isB ? LDS_B : LDS_A);
        volatile LAS u32* ctl = (volatile LAS u32*)(L + LDS_CTL + (lane & 7) * 16);
        const char* uf = myws + WS_UF + panel * 2048;
        const u32 voff = lane * 16 + ((panel) & 3) * UNIT;
        Ld S = {0, 0, 0, 0, 0, ~0u, 0, 0, (u64)img, (u64)(img + (16 + tn - panel) * 16384 + (size_t)S0 * 524288)};
        u32 err = 0;
        const u32 vctl = (u32)(size_t)ctl, vfull = vctl + isB * 4, vf = (lane & 31) * 4;
        const u64 t0 = rt();
        WAIT_VM0();
        asm volatile("v_mov_b32 v40, 0\n\tv_mov_b32 v41, 0\n\tv_mov_b32 v47, 0\n\tv_mov_b32 v44, %0\n\tv_mov_b32 v45, %1\n\t"
                     "v_mov_b32 v46, %2\n\tv_mov_b32 v48, %3\n\tv_mov_b32 v50, %4"
                     :: "v"((voff + UNIT) & 16383), "v"((voff + 2 * UNIT) & 16383), "v"((voff + 3 * UNIT) & 16383),
                        "v"(lane * 16 + ((tn + 3) & 3) * UNIT), "v"(isB * 4) : "v40", "v41", "v44", "v45", "v46", "v47", "v48", "v50");
        bool go = true;
        auto late = [&]() {
            if (UNI(__hip_atomic_load(tmo, __ATOMIC_RELAXED, __HIP_MEMORY_SCOPE_AGENT)) == call || rt() - t0 > DEADLINE) {
                if (lane == 0) __hip_atomic_store(tmo, call, __ATOMIC_RELAXED, __HIP_MEMORY_SCOPE_AGENT);
                ctl[3] = 1;
                err = ~0u;
            }
            asm volatile("s_waitcnt vmcnt(0) lgkmcnt(0)" ::: "memory");
            go = rt() - t0 < 3 * DEADLINE;
        };
#define RUN(M, end, l) while (go && load_run<M>(S, end, l, err, call, uf, voff, vf, vctl, vfull)) late();
        while (rt() - tk < LDELAY) __builtin_amdgcn_s_sleep(4);
        while (go && load_run<0, 4>(S, QEASY, ldsb, err, call, uf, voff, vf, vctl, vfull)) late();
        RUN(0, S0, ldsb);
        const u32 l0 = (u32)(size_t)L + LDS_A;
        if (isB) { RUN(2, NSTAGE, l0); S.g -= (size_t)NSPLIT * 524288; S.q += 4; RUN(4, NQ, l0); }
        else {
            RUN(1, NSTAGE, l0);
            S.g -= (size_t)4 * 524288 - 2048; S.slot = (NSPLIT + 1) % 5;
            RUN(1, NSTAGE + 4, l0);
            S.g -= (size_t)NSPLIT * 524288;
            RUN(3, NQ, l0);
        }
        RUN(5, 0, 0);
        ctl[isB] = 0xFFFFu;
        WAIT_VM(0);
    } else {
        __builtin_amdgcn_s_setprio(2);
        const u32 vo = lane * 16, wk = wave - 10;
        {
            u32 x[16];
            const u32 pdst = (srcmap >> (4 * xg)) & 7u;
            const bool push = pr < NPW && pdst != (u32)rank;
            const u32 up = 32 + ((rank - pdst - 1) & 7) + 7 * pr, pp = idx | 16;
            const char* pw = P.ws[pdst];
            const char* q0 = pw + WS_IMG + unit_off(pp, up);
            asm volatile("" : "+s"(q0));
            WAIT_VM0();
            call = UNI(call) + 1;
            if (push) {
                order_w(x, d);
                q0 += (call & 1) * IMG_BYTES;
                st_sys_s(vo, q0, piece(x, 0));
                arrive(L, lane);
                st_sys_s(vo, q0 + 1024, piece(x, 1));
                st_sys_s(vo, q0 + 2048, piece(x, 2));
                st_sys_s(vo, q0 + 3072, piece(x, 3));
                WAIT_VM(0); st_flag(pw + WS_UF + pp * 2048 + up * 4, call, lane);
            } else {
                arrive(L, lane);
            }
            if (wk) *(volatile LAS u32*)(L + LDS_PSH) = 1;
        }
        call = UNI(call);
        u32 job = 0;
        for (u32 spin = 0; spin < SPINS && job == 0; ++spin) { __builtin_amdgcn_s_sleep(1); job = UNI(*(volatile LAS u32*)(L + LDS_JOB)); }
        job = (job - 1) & (NWG - 1);
        const u32 slot = job >> 5, src = (srcmap >> (4 * slot)) & 7u;
        if (src == (u32)rank) return;
        const char* tik = myws + WS_TIK + slot * 128;
        const char* scb = P.ws[src] + WS_CB + (call & 1) * CB_BYTES;
        const char* rdy = myws + WS_RDY + (call & 1) * 32768 + src * 4096;
        const char* dp = myws + WS_IMG + (call & 1) * IMG_BYTES;
        const u32 j0 = (src - rank - 1) & 7;
        LAS unsigned char* sl = L + LDS_WRK + wk * WRK_BYTES + 3072;
        const u32 rot = 2 * rank;
        auto unit_of = [rot](u32 t) -> u32 {
            auto px = [rot](u32 k) { k = (k + rot) & 15; return 4 * (k & 3) + (k >> 2); };
            auto pw = [rot](u32 k) { k = (k + rot) & 15; return 16 + 8 * (k & 1) + (k >> 1); };
            if (t < T0) return t < 64 ? px(t & 15) * 32 + (t >> 4) : pw(t & 15) * 32 + NPW + (t >> 4) - 4;
            if (t < T1) { t -= T0; return ((t & 1) ? pw((t >> 1) & 15) : px((t >> 1) & 15)) * 32 + 4 + (t >> 5); }
            if (t < T2) {
                const u32 pr = (t - T1) / 48, i = (t - T1) % 48;
                return ((i & 16) ? px(i & 15) : pw(i & 15)) * 32 + R0 + 2 * pr + (i >> 5);
            }
            return t < NT ? px((t - T2) & 15) * 32 + (t - T2 < 16 ? ROUNDS - 1 : R0 - 1 + 2 * ((t - T2) >> 4)) : 0u;
        };
        const u64 t0 = rt();
        auto await = [&](u32 un) -> u32 {
            for (u32 n = 1;; ++n) {
                u32 rv;
                if (lane == 0) asm volatile("global_atomic_or v162, %0, %0, %1 sc0 sc1\n\ts_waitcnt vmcnt(0)" :: "v"(0u), "s"(rdy + un * 4) : "memory", "v162");
                asm volatile("v_readfirstlane_b32 %0, v162" : "=s"(rv) :: "memory", "v162");
                if ((rv >> 7) >= call) return rv;
                if ((n & 31) == 0 && (UNI(__hip_atomic_load(tmo, __ATOMIC_RELAXED, __HIP_MEMORY_SCOPE_AGENT)) == call || rt() - t0 > DEADLINE)) {
                    if (lane == 0) __hip_atomic_store(tmo, call, __ATOMIC_RELAXED, __HIP_MEMORY_SCOPE_AGENT);
                    return 0;
                }
                __builtin_amdgcn_s_sleep(2);
            }
        };
        auto book = [&](u64 at, u32 cost) {
            const u64 now = at << 10, c = cost;
            if (lane == 0)
                asm volatile("global_atomic_umax_x2 %0, %1, %3 offset:2048\n\tglobal_atomic_add_x2 v[160:161], %0, %2, %3 offset:2048 sc0"
                             :: "v"(0u), "v"(now), "v"(c), "s"(tik) : "memory", "v160", "v161");
        };
        const char* pend = nullptr;
        u32 pn = 0;
        auto flag = [&]() {
            if (lane < 2) asm volatile("global_store_dword %0, %1, %2 sc0 sc1" :: "v"(lane * pn), "v"(call), "s"(pend) : "memory");
            pend = nullptr;
        };
        u32 t = (job & 31) + 32 * wk, un = unit_of(t), lat = 0, plat = 0;
        if (lane < 2)
            asm volatile("global_load_dword v163, %0, %1 sc0 sc1" :: "v"((lane * 482 + (job & 31)) * SLOT + UNIT + 128), "s"(scb) : "memory", "v163");
        for (u32 spin = 0; spin < SPINS && UNI(*(volatile LAS u32*)(L + (t < 16 ? LDS_PSH : LDS_PRO))) < (t < 16 ? 1 : 7); ++spin)
            __builtin_amdgcn_s_sleep(2);
        u32 rv = await(un);
        if (!rv) return;
        WAIT_VM0();
        book(rt() + (t < NOWAIT ? 0 : 40), (((rv & 127u) + 1) >> 1) * (RATE + 15));
        bool now_ = t < NOWAIT;
        if (!now_) WAIT_VM(0);
        t += 64;
        for (;;) {
            u32 len64 = rv & 127u;
            len64 = len64 < 41 ? 41 : (len64 > 64 ? 64 : len64);
            const u32 len = len64 * 64, panel = un >> 5, r = un & 31;
            const char* q = scb + cb_off(panel, r);
            const bool more = t < NT;
            un = unit_of(t);
            const char* rp = rdy + un * 4;
            asm volatile("" : "+s"(q), "+s"(rp));
            if (!now_) {
                u32 lo, hi;
                asm volatile("v_readfirstlane_b32 %0, v160\n\tv_readfirstlane_b32 %1, v161" : "=s"(lo), "=s"(hi) :: "memory", "v160", "v161");
                const u32 start = (u32)((((u64)hi << 32) | lo) >> 10);
                for (u32 d = start - (u32)rt(); d - 1 < 1000u; d = start - (u32)rt()) {
                    if (pend) { if (vm_idle()) { WAIT_VM(0); flag(); } }
                    else if (d > 12) __builtin_amdgcn_s_sleep(2);
                }
            }
            now_ = false;
            const u32 i0 = (u32)rt();
            PIECE(0, q); PIECE(1, q);
            if (2048 + vo < len) PIECE(2, q);
            if (3072 + vo < len) PIECE(3, q);
            if (lane == 0) asm volatile("global_atomic_or v162, %0, %0, %3 sc0 sc1\n\tglobal_atomic_add v163, %0, %1, %2 sc0"
                                        :: "v"(0u), "v"(1u), "s"(tik), "s"(rp) : "memory", "v162", "v163");
            const bool half = panel < 16 && r >= R0;
            const u32 u = pos_of(j0, half ? r & ~1u : r);
            const char* qa = dp + unit_off(panel, u) + (half ? (r & 1) * 2048 : 0);
            const char* qb = half ? dp + unit_off(panel, u + 7) + (r & 1) * 2048 : qa + 2048;
            const u32 ho = panel < 16 ? 3072 : 2560;
            asm volatile("s_waitcnt vmcnt(2)" : "+s"(qa), "+s"(qb) :: "memory");
            const u32x4 w = *(LAS u32x4*)(sl + lane * 16);
            const u32 hd = *(LAS u32*)(sl + ho), w2 = *(LAS u32*)(sl + ho + 4 + lane * 4);
            const u64 now = rt();
            WAIT_VM(0);
            if (pend) flag();
            asm volatile("v_readfirstlane_b32 %0, v163\n\tv_readfirstlane_b32 %1, v162" : "=s"(t), "=s"(rv) :: "memory", "v162", "v163");
            plat = lat;
            lat = (u32)now - i0;
            const u32 ml = lat < plat ? lat : plat;
            const u32 rate = ml > LATHI ? RATE + RATE / 8 : RATE;
            t += 128 - call * 2048;
            const bool known = (rv >> 7) >= call;
            if (more && known) book(now, (((rv & 127u) + 1) >> 1) * rate);
            FENCE(); SB();
            if (len64 == 64) {
                const u32x4 e1 = *(LAS u32x4*)(sl + 1024 + lane * 16), e2 = *(LAS u32x4*)(sl + 2048 + lane * 16), e3 = *(LAS u32x4*)(sl + 3072 + lane * 16);
                st_sys_s(vo, qa, w); st_sys_s(vo, qa + 1024, e1); st_sys_s(vo, qb, e2); st_sys_s(vo, qb + 1024, e3);
            } else if (panel < 16) {
                decode_unit<3>(qa, qb, UNI(hd), w, w2, sl, L, lane);
            } else {
                decode_unit<2>(qa, qb, UNI(hd), w, w2, sl, L, lane);
            }
            const u32 sec = u < 240 ? 16 + 4 * NSPLIT : 16;
            pend = myws + WS_UF + panel * 2048 + (u + (half && (r & 1) ? sec : 0)) * 4;
            pn = half ? 28 : panel < 16 && u >= 4 * S0 ? sec * 4 : 0;
            if (!more) break;
            if (!known) {
                WAIT_VM(0); flag();
                rv = await(un);
                if (!rv) return;
                WAIT_VM0();
                book(rt(), (((rv & 127u) + 1) >> 1) * rate);
                WAIT_VM(0);
            }
            WAIT_VM(4);
        }
        WAIT_VM(0);
        flag();
    }
}

extern "C" void kernel_launch(int rank, int nranks, ncclComm_t comm,
                              hipStream_t stream,
                              void* const* d_in, const int* in_sizes, int n_in,
                              void* d_out, int out_size,
                              const int* in_dtypes, int out_dtype,
                              void* const* d_out_all,
                              void* const* d_ws_all, size_t ws_bytes) {
    (void)comm; (void)in_sizes; (void)n_in; (void)out_size; (void)in_dtypes; (void)out_dtype; (void)d_out_all;
    if (nranks != NR || (unsigned)rank >= (unsigned)NR || ws_bytes < WS_NEED) {
        fprintf(stderr, "kernel_launch: unexpected configuration\n");
        return;
    }
    Params p;
    p.x = (const unsigned char*)d_in[0];
    p.w = (const unsigned char*)d_in[1];
    p.sx = (const float*)d_in[2];
    p.sw = (const float*)d_in[3];
    p.out = (float*)d_out;
    for (int i = 0; i < NR; ++i) p.ws[i] = (char*)d_ws_all[i];
    p.myws = p.ws[rank];
    p.rank = rank;
    p.pad = 0;
    static const u32 near_link[NR] = {0x74510263u, 0x26301475u, 0x17542036u, 0x74153620u,
                                      0x63024157u, 0x60235741u, 0x51476302u, 0x02367514u};
    p.srcmap = near_link[rank];
    hipLaunchKernelGGL(ag_gemm, dim3(NWG), dim3(NTHREADS), 0, stream, p);
    HIP_CHECK(hipGetLastError());
}
```
